```python
import numpy as np

kernel_name = "gpu_a2a_gemm_m4096_k4096_n32768_f32_silu"


ABSMAX_RTOL = 0.001

COMPUTE_DTYPE = "float32"
EPILOGUE = "silu"
IN_QUANT = None
OUT_QUANT = None
IN_DTYPES = {"x": "float32", "w_mat": "float32"}
WORLD_SIZE = 8

def setup_inputs():

    x = np.random.randn(4096, 4096).astype(np.float32)
    w_mat = (np.random.randn(4096, 32768) / np.sqrt(4096)).astype(np.float32)
    return {"x": x, "w_mat": w_mat}

def _epilogue(y):
    return (y / (1.0 + np.exp(-np.clip(y, -60.0, 60.0)))).astype(np.float32)

def reference(x, w_mat):
    y = (x @ w_mat).astype(np.float32)
    return _epilogue(y)
```

```cpp
#include <hip/hip_runtime.h>
#include <rccl/rccl.h>
#include <cstdio>

typedef _Float16 f16x8 __attribute__((ext_vector_type(8)));
typedef float f32x4 __attribute__((ext_vector_type(4)));
typedef unsigned u32x4 __attribute__((ext_vector_type(4)));

constexpr int NR = 8;
constexpr int M = 512, K = 4096, N = 32768, KS = 512;
constexpr size_t A_OFF = 1u << 20;
constexpr size_t A_BYTES = (size_t)M * K * 2;
constexpr unsigned long long DEADLINE_TICKS = 100000000ull;

struct Ptrs { char* ws[NR]; };

__global__ __launch_bounds__(256, 1) void kern(int phase, int rank, const float* __restrict__ x,
                                               const float* __restrict__ w, float* __restrict__ out, Ptrs P) {
  __shared__ __attribute__((aligned(16))) char lds[2 * 16384 + 16];
  const int tid = threadIdx.x, lane = tid & 63, wv = tid >> 6, b = blockIdx.x;
  char* myws = P.ws[rank];
  unsigned* sh = (unsigned*)(lds + 32768);
  if (tid == 0) {
    unsigned t = __hip_atomic_fetch_add((unsigned*)(myws + 128 * phase), 1u, __ATOMIC_RELAXED, __HIP_MEMORY_SCOPE_AGENT);
    sh[0] = t / 256u + 1u;
  }
  __syncthreads();
  const unsigned call = sh[0];
  const unsigned par = call & 1u;
  if (phase == 0) {
    const int r = b >> 5, mf = b & 31;
    char* dst = P.ws[r] + A_OFF + par * A_BYTES;
    for (int q = 0; q < 4; ++q) {
      const int ksl = wv * 4 + q;
      const float* src = x + (size_t)(r * 512 + mf * 16 + (lane & 15)) * KS + ksl * 32 + (lane >> 4) * 8;
      f32x4 a = *(const f32x4*)src, c = *(const f32x4*)(src + 4);
      f16x8 h;
      for (int j = 0; j < 4; ++j) { h[j] = (_Float16)a[j]; h[4 + j] = (_Float16)c[j]; }
      const int ks = rank * 16 + ksl;
      *(f16x8*)(dst + ((size_t)(ks * 32 + mf) * 64 + lane) * 16) = h;
    }
    asm volatile("s_waitcnt vmcnt(0)" ::: "memory");
    __syncthreads();
    if (tid == 0) {
      __builtin_amdgcn_fence(__ATOMIC_RELEASE, "");
      asm volatile("s_waitcnt vmcnt(0)" ::: "memory");
      __hip_atomic_fetch_add((unsigned*)(P.ws[r] + 128 * (8 + rank)), 1u, __ATOMIC_RELAXED, __HIP_MEMORY_SCOPE_SYSTEM);
    }
    return;
  }
  unsigned* tmo = (unsigned*)(myws + 128 * 2);
  if (tid == 0) {
    unsigned long long t0 = __builtin_amdgcn_s_memrealtime();
    for (int p = 0; p < NR; ++p) {
      if (p == rank) continue;
      unsigned* f = (unsigned*)(myws + 128 * (8 + p));
      while (__hip_atomic_load(f, __ATOMIC_RELAXED, __HIP_MEMORY_SCOPE_SYSTEM) < call * 32u) {
        if (__hip_atomic_load(tmo, __ATOMIC_RELAXED, __HIP_MEMORY_SCOPE_AGENT) == call ||
            __builtin_amdgcn_s_memrealtime() - t0 > DEADLINE_TICKS) {
          __hip_atomic_store(tmo, call, __ATOMIC_RELAXED, __HIP_MEMORY_SCOPE_AGENT);
          break;
        }
        __builtin_amdgcn_s_sleep(2);
      }
    }
    __builtin_amdgcn_fence(__ATOMIC_ACQUIRE, "");
    asm volatile("s_waitcnt vmcnt(0)" ::: "memory");
    sh[1] = (__hip_atomic_load(tmo, __ATOMIC_RELAXED, __HIP_MEMORY_SCOPE_AGENT) == call) ? 1u : 0u;
  }
  __syncthreads();
  const bool bad = sh[1] != 0;
  const char* A = myws + A_OFF + par * A_BYTES;
  const int n0 = b * 128;
  f32x4 acc[8][8];
  for (int i = 0; i < 8; ++i) for (int t = 0; t < 8; ++t) acc[i][t] = f32x4{0, 0, 0, 0};
  const float* wsrc = w + (size_t)(16 * wv + 8 * (lane >> 5)) * N + n0 + 4 * (lane & 31);
  const int g = (2 * wv + (lane >> 5)) & 3;
  const int wr_off = ((wv >> 1) * 8 + 4 * ((lane >> 4) & 1)) * 1024 + (16 * g + (lane & 15)) * 16;
  if (!bad) {
    for (int s = 0; s < K / 64; ++s) {
      f32x4 wr[8];
      for (int i = 0; i < 8; ++i) wr[i] = *(const f32x4*)(wsrc + (size_t)(s * 64 + i) * N);
      char* buf = lds + (s & 1) * 16384;
      for (int t = 0; t < 4; ++t) {
        f16x8 h;
        for (int i = 0; i < 8; ++i) h[i] = (_Float16)wr[i][t];
        *(f16x8*)(buf + wr_off + t * 1024) = h;
      }
      __syncthreads();
      for (int ks = 0; ks < 2; ++ks) {
        f16x8 bf[8], af[8];
        for (int t = 0; t < 8; ++t) bf[t] = *(const f16x8*)(buf + (ks * 8 + t) * 1024 + lane * 16);
        for (int i = 0; i < 8; ++i) af[i] = *(const f16x8*)(A + ((size_t)((s * 2 + ks) * 32 + wv * 8 + i) * 64 + lane) * 16);
        for (int i = 0; i < 8; ++i)
          for (int t = 0; t < 8; ++t)
            acc[i][t] = __builtin_amdgcn_mfma_f32_16x16x32_f16(af[i], bf[t], acc[i][t], 0, 0, 0);
      }
    }
  }
  _Pragma("unroll") for (int i = 0; i < 8; ++i)
    _Pragma("unroll") for (int r = 0; r < 4; ++r)
      _Pragma("unroll") for (int h = 0; h < 2; ++h) {
        f32x4 o;
        for (int t = 0; t < 4; ++t) {
          float y = acc[i][4 * h + t][r];
          float c = fminf(fmaxf(y, -60.f), 60.f);
          o[t] = bad ? __builtin_nanf("") : y * __builtin_amdgcn_rcpf(1.f + __expf(-c));
        }
        const int row = wv * 128 + i * 16 + 4 * (lane >> 4) + r;
        *(f32x4*)(out + (size_t)row * N + n0 + 64 * h + 4 * (lane & 15)) = o;
      }
}

extern "C" void kernel_launch(int rank, int nranks, ncclComm_t comm, hipStream_t stream,
                              void* const* d_in, const int* in_sizes, int n_in, void* d_out, int out_size,
                              const int* in_dtypes, int out_dtype, void* const* d_out_all,
                              void* const* d_ws_all, size_t ws_bytes) {
  if (nranks != NR || (unsigned)rank >= (unsigned)NR || ws_bytes < A_OFF + 2 * A_BYTES) return;
  Ptrs P;
  for (int p = 0; p < NR; ++p) P.ws[p] = (char*)d_ws_all[p];
  kern<<<256, 256, 0, stream>>>(0, rank, (const float*)d_in[0], (const float*)d_in[1], (float*)d_out, P);
  kern<<<256, 256, 0, stream>>>(1, rank, (const float*)d_in[0], (const float*)d_in[1], (float*)d_out, P);
}
```

```cpp
#include <hip/hip_runtime.h>
#include <rccl/rccl.h>
#include <cstdio>

constexpr int NR = 8;
struct Ptrs { char* ws[NR]; };

__global__ __launch_bounds__(256, 1) void a2a_gemm_silu(Ptrs P, char* ws, const float* __restrict__ x, const float* __restrict__ w,
                                               float* __restrict__ out, unsigned* dbg, int rank) {
  __shared__ __attribute__((aligned(16))) char lds[131136];
  const unsigned b = blockIdx.x;
  const unsigned wave = __builtin_amdgcn_readfirstlane(threadIdx.x >> 6);
  asm volatile(
  "KS_%=:\n"
  "s_memrealtime s[54:55]\n"
  "s_lshl_b32 s81, %6, 7\n"
  "s_load_dwordx2 s[52:53], %1, s81\n"
  "s_mov_b32 s35, 0x20000\n"
  "s_lshr_b32 s79, %6, 4\n"
  "s_lshl_b32 s79, s79, 3\n"
  "s_and_b32 s80, %6, 7\n"
  "s_or_b32 s79, s79, s80\n"
  "s_bitcmp1_b32 %6, 3\n"
  "s_cbranch_scc1 T0R_%=\n"
  "s_lshr_b32 s63, s79, 5\n"
  "s_mov_b32 s76, s63\n"
  "s_and_b32 s80, s79, 31\n"
  "s_mov_b32 s64, %5\n"
  "s_lshl_b32 s66, s64, 3\n"
  "s_load_dwordx2 s[56:57], %0, s66\n"
  "s_and_b32 s66, s80, 7\n"
  "s_lshl_b32 s66, s66, 2\n"
  "s_add_u32 s66, s66, %7\n"
  "s_lshl_b32 s88, s64, 20\n"
  "s_lshl_b32 s63, s63, 9\n"
  "s_add_u32 s88, s88, s63\n"
  "s_lshl_b32 s63, s66, 15\n"
  "s_add_u32 s88, s88, s63\n"
  "s_lshr_b32 s80, s80, 3\n"
  "s_lshl_b32 s63, s80, 7\n"
  "s_add_u32 s88, s88, s63\n"
  "s_lshl_b32 s76, s76, 17\n"
  "s_lshl_b32 s63, s66, 9\n"
  "s_add_u32 s76, s76, s63\n"
  "s_lshr_b32 s63, s80, 1\n"
  "s_lshl_b32 s63, s63, 16\n"
  "s_add_u32 s76, s76, s63\n"
  "s_and_b32 s63, s80, 1\n"
  "s_lshl_b32 s63, s63, 14\n"
  "s_add_u32 s76, s76, s63\n"
  "s_branch T0D_%=\n"
  "T0R_%=:\n"
  "s_and_b32 s65, s79, 3\n"
  "s_lshr_b32 s80, s79, 2\n"
  "s_mul_i32 s63, s65, 37\n"
  "s_lshr_b32 s63, s63, 8\n"
  "s_mul_i32 s64, s63, 7\n"
  "s_sub_u32 s64, s65, s64\n"
  "s_sub_u32 s64, %5, s64\n"
  "s_sub_u32 s64, s64, 1\n"
  "s_and_b32 s64, s64, 7\n"
  "s_add_u32 s76, s65, 4\n"
  "s_lshl_b32 s66, s64, 3\n"
  "s_load_dwordx2 s[56:57], %0, s66\n"
  "s_and_b32 s66, s80, 7\n"
  "s_lshl_b32 s66, s66, 2\n"
  "s_add_u32 s66, s66, %7\n"
  "s_lshl_b32 s88, s64, 20\n"
  "s_lshl_b32 s63, s63, 9\n"
  "s_add_u32 s88, s88, s63\n"
  "s_lshl_b32 s63, s66, 15\n"
  "s_add_u32 s88, s88, s63\n"
  "s_lshr_b32 s80, s80, 3\n"
  "s_lshl_b32 s63, s80, 7\n"
  "s_add_u32 s88, s88, s63\n"
  "s_lshl_b32 s76, s76, 17\n"
  "s_lshl_b32 s63, s66, 9\n"
  "s_add_u32 s76, s76, s63\n"
  "s_lshr_b32 s63, s80, 1\n"
  "s_lshl_b32 s63, s63, 16\n"
  "s_add_u32 s76, s76, s63\n"
  "s_and_b32 s63, s80, 1\n"
  "s_lshl_b32 s63, s63, 14\n"
  "s_add_u32 s76, s76, s63\n"
  "T0D_%=:\n"
  "v_mbcnt_lo_u32_b32 v246, -1, 0\n"
  "v_mbcnt_hi_u32_b32 v246, -1, v246\n"
  "v_and_b32 v227, 15, v246\n"
  "v_lshrrev_b32 v226, 4, v246\n"
  "v_lshlrev_b32 v226, 5, v226\n"
  "v_lshl_add_u32 v252, v227, 11, v226\n"
  "s_mov_b64 s[32:33], %4\n"
  "s_mov_b32 s34, 0x800000\n"
  "buffer_load_dwordx4 v[232:235], v252, s[32:35], s88 offen nt\n"
  "buffer_load_dwordx4 v[236:239], v252, s[32:35], s88 offen offset:16 nt\n"
  "s_and_b32 s63, %6, 7\n"
  "s_lshl_b32 s63, s63, 5\n"
  "s_lshr_b32 s64, %6, 3\n"
  "s_add_u32 s63, s63, s64\n"
  "s_lshl_b32 s78, s63, 9\n"
  "s_mov_b64 s[48:49], %2\n"
  "s_add_u32 s48, s48, s78\n"
  "s_addc_u32 s49, s49, 0\n"
  "s_and_b32 s64, s48, 0x7e00\n"
  "s_bcnt1_i32_b32 s64, s64\n"
  "s_andn2_b32 s64, 1, s64\n"
  "s_lshl_b32 s65, s64, 16\n"
  "s_sub_u32 s48, s48, s65\n"
  "s_subb_u32 s49, s49, 0\n"
  "s_sub_u32 s69, 0x800000, s78\n"
  "s_add_u32 s69, s69, s65\n"
  "s_mov_b32 s30, s69\n"
  "s_mov_b32 s31, 0x20000\n"
  "s_lshl_b32 s63, %5, 26\n"
  "s_add_u32 s28, s48, s63\n"
  "s_addc_u32 s29, s49, 0\n"
  "s_mov_b32 s39, 0x20000\n"
  "s_mov_b32 s40, 0x40000\n"
  "s_mov_b32 s41, 0x60000\n"
  "s_mov_b32 s42, 0x80000\n"
  "s_mov_b32 s43, 0xa0000\n"
  "s_mov_b32 s44, 0xc0000\n"
  "s_mov_b32 s45, 0xe0000\n"
  "v_lshrrev_b32 v226, 5, v246\n"
  "v_and_b32 v227, 31, v246\n"
  "v_lshlrev_b32 v227, 4, v227\n"
  "s_lshl_b32 s63, %7, 21\n"
  "s_add_u32 s63, s63, s65\n"
  "v_lshl_add_u32 v242, v226, 20, v227\n"
  "v_add_u32 v242, s63, v242\n"
  "s_bitcmp1_b32 %6, 3\n"
  "s_cbranch_scc0 NOW0_%=\n"
  "buffer_load_dwordx4 v[64:67], v242, s[28:31], 0 offen nt\n"
  "buffer_load_dwordx4 v[68:71], v242, s[28:31], s39 offen nt\n"
  "buffer_load_dwordx4 v[72:75], v242, s[28:31], s40 offen nt\n"
  "buffer_load_dwordx4 v[76:79], v242, s[28:31], s41 offen nt\n"
  "buffer_load_dwordx4 v[80:83], v242, s[28:31], s42 offen nt\n"
  "buffer_load_dwordx4 v[84:87], v242, s[28:31], s43 offen nt\n"
  "buffer_load_dwordx4 v[88:91], v242, s[28:31], s44 offen nt\n"
  "buffer_load_dwordx4 v[92:95], v242, s[28:31], s45 offen nt\n"
  "s_add_u32 s28, s28, 0x800000\n"
  "s_addc_u32 s29, s29, 0\n"
  "buffer_load_dwordx4 v[160:163], v242, s[28:31], 0 offen nt\n"
  "buffer_load_dwordx4 v[164:167], v242, s[28:31], s39 offen nt\n"
  "buffer_load_dwordx4 v[168:171], v242, s[28:31], s40 offen nt\n"
  "buffer_load_dwordx4 v[172:175], v242, s[28:31], s41 offen nt\n"
  "buffer_load_dwordx4 v[176:179], v242, s[28:31], s42 offen nt\n"
  "buffer_load_dwordx4 v[180:183], v242, s[28:31], s43 offen nt\n"
  "buffer_load_dwordx4 v[184:187], v242, s[28:31], s44 offen nt\n"
  "buffer_load_dwordx4 v[188:191], v242, s[28:31], s45 offen nt\n"
  "s_add_u32 s28, s28, 0x800000\n"
  "s_addc_u32 s29, s29, 0\n"
  "NOW0_%=:\n"
  "v_lshl_add_u32 v240, v226, 15, v227\n"
  "s_bcnt1_i32_b32 s63, %7\n"
  "s_and_b32 s63, s63, 1\n"
  "s_lshl_b32 s63, s63, 16\n"
  "s_xor_b32 s64, s63, 0x10000\n"
  "v_add_u32 v241, s63, v240\n"
  "v_add_u32 v240, s64, v240\n"
  "s_lshr_b32 s65, %6, 3\n"
  "s_lshl_b32 s65, s65, 2\n"
  "s_add_u32 s65, s65, %7\n"
  "s_lshl_b32 s65, s65, 9\n"
  "s_add_u32 s65, s65, s64\n"
  "v_lshl_add_u32 v245, v246, 3, s65\n"
  "s_mov_b64 s[36:37], %1\n"
  "s_add_u32 s36, s36, 0xa000\n"
  "s_addc_u32 s37, s37, 0\n"
  "s_sub_u32 s36, s36, s65\n"
  "s_subb_u32 s37, s37, 0\n"
  "s_add_u32 s65, s65, 0x100\n"
  "s_cmp_eq_u32 %7, 0\n"
  "s_cselect_b32 s38, s65, 0\n"
  "s_mov_b32 s39, 0x20000\n"
  "s_waitcnt lgkmcnt(0)\n"
  "s_mov_b32 s89, s54\n"
  "s_sub_u32 s91, s54, s53\n"
  "s_add_u32 s61, s52, 1\n"
  "s_and_b32 s68, s61, 1\n"
  "s_lshl_b32 s68, s68, 22\n"
  "s_add_u32 s68, s68, 0x100000\n"
  "s_mov_b64 s[20:21], %1\n"
  "s_add_u32 s20, s20, s68\n"
  "s_addc_u32 s21, s21, 0\n"
  "s_sub_u32 s24, s20, s63\n"
  "s_subb_u32 s25, s21, 0\n"
  "s_add_u32 s26, s63, 0x400000\n"
  "s_mov_b32 s27, 0x20000\n"
  "s_sub_u32 s20, s20, s64\n"
  "s_subb_u32 s21, s21, 0\n"
  "s_add_u32 s22, s64, 0x400000\n"
  "s_mov_b32 s23, 0x20000\n"
  "v_lshlrev_b32 v243, 4, v246\n"
  "s_lshl_b32 s63, %7, 1\n"
  "v_add_u32 v226, s63, v226\n"
  "v_and_b32 v226, 3, v226\n"
  "v_lshlrev_b32 v226, 8, v226\n"
  "v_and_b32 v227, 15, v246\n"
  "v_lshl_add_u32 v226, v227, 4, v226\n"
  "v_bfe_u32 v227, v246, 4, 1\n"
  "v_lshl_add_u32 v226, v227, 12, v226\n"
  "s_lshr_b32 s63, %7, 1\n"
  "s_lshl_b32 s63, s63, 13\n"
  "v_add_u32 v244, s63, v226\n"
  "v_accvgpr_write_b32 a0, 0\n"
  "v_accvgpr_write_b32 a1, 0\n"
  "v_accvgpr_write_b32 a2, 0\n"
  "v_accvgpr_write_b32 a3, 0\n"
  "v_accvgpr_write_b32 a4, 0\n"
  "v_accvgpr_write_b32 a5, 0\n"
  "v_accvgpr_write_b32 a6, 0\n"
  "v_accvgpr_write_b32 a7, 0\n"
  "v_accvgpr_write_b32 a8, 0\n"
  "v_accvgpr_write_b32 a9, 0\n"
  "v_accvgpr_write_b32 a10, 0\n"
  "v_accvgpr_write_b32 a11, 0\n"
  "v_accvgpr_write_b32 a12, 0\n"
  "v_accvgpr_write_b32 a13, 0\n"
  "v_accvgpr_write_b32 a14, 0\n"
  "v_accvgpr_write_b32 a15, 0\n"
  "v_accvgpr_write_b32 a16, 0\n"
  "v_accvgpr_write_b32 a17, 0\n"
  "v_accvgpr_write_b32 a18, 0\n"
  "v_accvgpr_write_b32 a19, 0\n"
  "v_accvgpr_write_b32 a20, 0\n"
  "v_accvgpr_write_b32 a21, 0\n"
  "v_accvgpr_write_b32 a22, 0\n"
  "v_accvgpr_write_b32 a23, 0\n"
  "v_accvgpr_write_b32 a24, 0\n"
  "v_accvgpr_write_b32 a25, 0\n"
  "v_accvgpr_write_b32 a26, 0\n"
  "v_accvgpr_write_b32 a27, 0\n"
  "v_accvgpr_write_b32 a28, 0\n"
  "v_accvgpr_write_b32 a29, 0\n"
  "v_accvgpr_write_b32 a30, 0\n"
  "v_accvgpr_write_b32 a31, 0\n"
  "v_accvgpr_write_b32 a32, 0\n"
  "v_accvgpr_write_b32 a33, 0\n"
  "v_accvgpr_write_b32 a34, 0\n"
  "v_accvgpr_write_b32 a35, 0\n"
  "v_accvgpr_write_b32 a36, 0\n"
  "v_accvgpr_write_b32 a37, 0\n"
  "v_accvgpr_write_b32 a38, 0\n"
  "v_accvgpr_write_b32 a39, 0\n"
  "v_accvgpr_write_b32 a40, 0\n"
  "v_accvgpr_write_b32 a41, 0\n"
  "v_accvgpr_write_b32 a42, 0\n"
  "v_accvgpr_write_b32 a43, 0\n"
  "v_accvgpr_write_b32 a44, 0\n"
  "v_accvgpr_write_b32 a45, 0\n"
  "v_accvgpr_write_b32 a46, 0\n"
  "v_accvgpr_write_b32 a47, 0\n"
  "v_accvgpr_write_b32 a48, 0\n"
  "v_accvgpr_write_b32 a49, 0\n"
  "v_accvgpr_write_b32 a50, 0\n"
  "v_accvgpr_write_b32 a51, 0\n"
  "v_accvgpr_write_b32 a52, 0\n"
  "v_accvgpr_write_b32 a53, 0\n"
  "v_accvgpr_write_b32 a54, 0\n"
  "v_accvgpr_write_b32 a55, 0\n"
  "v_accvgpr_write_b32 a56, 0\n"
  "v_accvgpr_write_b32 a57, 0\n"
  "v_accvgpr_write_b32 a58, 0\n"
  "v_accvgpr_write_b32 a59, 0\n"
  "v_accvgpr_write_b32 a60, 0\n"
  "v_accvgpr_write_b32 a61, 0\n"
  "v_accvgpr_write_b32 a62, 0\n"
  "v_accvgpr_write_b32 a63, 0\n"
  "v_accvgpr_write_b32 a64, 0\n"
  "v_accvgpr_write_b32 a65, 0\n"
  "v_accvgpr_write_b32 a66, 0\n"
  "v_accvgpr_write_b32 a67, 0\n"
  "v_accvgpr_write_b32 a68, 0\n"
  "v_accvgpr_write_b32 a69, 0\n"
  "v_accvgpr_write_b32 a70, 0\n"
  "v_accvgpr_write_b32 a71, 0\n"
  "v_accvgpr_write_b32 a72, 0\n"
  "v_accvgpr_write_b32 a73, 0\n"
  "v_accvgpr_write_b32 a74, 0\n"
  "v_accvgpr_write_b32 a75, 0\n"
  "v_accvgpr_write_b32 a76, 0\n"
  "v_accvgpr_write_b32 a77, 0\n"
  "v_accvgpr_write_b32 a78, 0\n"
  "v_accvgpr_write_b32 a79, 0\n"
  "v_accvgpr_write_b32 a80, 0\n"
  "v_accvgpr_write_b32 a81, 0\n"
  "v_accvgpr_write_b32 a82, 0\n"
  "v_accvgpr_write_b32 a83, 0\n"
  "v_accvgpr_write_b32 a84, 0\n"
  "v_accvgpr_write_b32 a85, 0\n"
  "v_accvgpr_write_b32 a86, 0\n"
  "v_accvgpr_write_b32 a87, 0\n"
  "v_accvgpr_write_b32 a88, 0\n"
  "v_accvgpr_write_b32 a89, 0\n"
  "v_accvgpr_write_b32 a90, 0\n"
  "v_accvgpr_write_b32 a91, 0\n"
  "v_accvgpr_write_b32 a92, 0\n"
  "v_accvgpr_write_b32 a93, 0\n"
  "v_accvgpr_write_b32 a94, 0\n"
  "v_accvgpr_write_b32 a95, 0\n"
  "v_accvgpr_write_b32 a96, 0\n"
  "v_accvgpr_write_b32 a97, 0\n"
  "v_accvgpr_write_b32 a98, 0\n"
  "v_accvgpr_write_b32 a99, 0\n"
  "v_accvgpr_write_b32 a100, 0\n"
  "v_accvgpr_write_b32 a101, 0\n"
  "v_accvgpr_write_b32 a102, 0\n"
  "v_accvgpr_write_b32 a103, 0\n"
  "v_accvgpr_write_b32 a104, 0\n"
  "v_accvgpr_write_b32 a105, 0\n"
  "v_accvgpr_write_b32 a106, 0\n"
  "v_accvgpr_write_b32 a107, 0\n"
  "v_accvgpr_write_b32 a108, 0\n"
  "v_accvgpr_write_b32 a109, 0\n"
  "v_accvgpr_write_b32 a110, 0\n"
  "v_accvgpr_write_b32 a111, 0\n"
  "v_accvgpr_write_b32 a112, 0\n"
  "v_accvgpr_write_b32 a113, 0\n"
  "v_accvgpr_write_b32 a114, 0\n"
  "v_accvgpr_write_b32 a115, 0\n"
  "v_accvgpr_write_b32 a116, 0\n"
  "v_accvgpr_write_b32 a117, 0\n"
  "v_accvgpr_write_b32 a118, 0\n"
  "v_accvgpr_write_b32 a119, 0\n"
  "v_accvgpr_write_b32 a120, 0\n"
  "v_accvgpr_write_b32 a121, 0\n"
  "v_accvgpr_write_b32 a122, 0\n"
  "v_accvgpr_write_b32 a123, 0\n"
  "v_accvgpr_write_b32 a124, 0\n"
  "v_accvgpr_write_b32 a125, 0\n"
  "v_accvgpr_write_b32 a126, 0\n"
  "v_accvgpr_write_b32 a127, 0\n"
  "v_accvgpr_write_b32 a128, 0\n"
  "v_accvgpr_write_b32 a129, 0\n"
  "v_accvgpr_write_b32 a130, 0\n"
  "v_accvgpr_write_b32 a131, 0\n"
  "v_accvgpr_write_b32 a132, 0\n"
  "v_accvgpr_write_b32 a133, 0\n"
  "v_accvgpr_write_b32 a134, 0\n"
  "v_accvgpr_write_b32 a135, 0\n"
  "v_accvgpr_write_b32 a136, 0\n"
  "v_accvgpr_write_b32 a137, 0\n"
  "v_accvgpr_write_b32 a138, 0\n"
  "v_accvgpr_write_b32 a139, 0\n"
  "v_accvgpr_write_b32 a140, 0\n"
  "v_accvgpr_write_b32 a141, 0\n"
  "v_accvgpr_write_b32 a142, 0\n"
  "v_accvgpr_write_b32 a143, 0\n"
  "v_accvgpr_write_b32 a144, 0\n"
  "v_accvgpr_write_b32 a145, 0\n"
  "v_accvgpr_write_b32 a146, 0\n"
  "v_accvgpr_write_b32 a147, 0\n"
  "v_accvgpr_write_b32 a148, 0\n"
  "v_accvgpr_write_b32 a149, 0\n"
  "v_accvgpr_write_b32 a150, 0\n"
  "v_accvgpr_write_b32 a151, 0\n"
  "v_accvgpr_write_b32 a152, 0\n"
  "v_accvgpr_write_b32 a153, 0\n"
  "v_accvgpr_write_b32 a154, 0\n"
  "v_accvgpr_write_b32 a155, 0\n"
  "v_accvgpr_write_b32 a156, 0\n"
  "v_accvgpr_write_b32 a157, 0\n"
  "v_accvgpr_write_b32 a158, 0\n"
  "v_accvgpr_write_b32 a159, 0\n"
  "s_bitcmp1_b32 %6, 3\n"
  "s_cbranch_scc1 NOF1_%=\n"
  "s_waitcnt vmcnt(0)\n"
  "v_cvt_pk_f16_f32 v232, v232, v233\n"
  "v_cvt_pk_f16_f32 v233, v234, v235\n"
  "v_cvt_pk_f16_f32 v234, v236, v237\n"
  "v_cvt_pk_f16_f32 v235, v238, v239\n"
  "s_mov_b64 s[32:33], s[56:57]\n"
  "s_mov_b32 s34, 0x900000\n"
  "s_add_u32 s63, s76, s68\n"
  "v_and_b32 v224, 0xffff, v240\n"
  "buffer_store_dwordx4 v[232:235], v224, s[32:35], s63 offen sc0 sc1\n"
  "NOF1_%=:\n"
  "v_accvgpr_write_b32 a160, 0\n"
  "v_accvgpr_write_b32 a161, 0\n"
  "v_accvgpr_write_b32 a162, 0\n"
  "v_accvgpr_write_b32 a163, 0\n"
  "v_accvgpr_write_b32 a164, 0\n"
  "v_accvgpr_write_b32 a165, 0\n"
  "v_accvgpr_write_b32 a166, 0\n"
  "v_accvgpr_write_b32 a167, 0\n"
  "v_accvgpr_write_b32 a168, 0\n"
  "v_accvgpr_write_b32 a169, 0\n"
  "v_accvgpr_write_b32 a170, 0\n"
  "v_accvgpr_write_b32 a171, 0\n"
  "v_accvgpr_write_b32 a172, 0\n"
  "v_accvgpr_write_b32 a173, 0\n"
  "v_accvgpr_write_b32 a174, 0\n"
  "v_accvgpr_write_b32 a175, 0\n"
  "v_accvgpr_write_b32 a176, 0\n"
  "v_accvgpr_write_b32 a177, 0\n"
  "v_accvgpr_write_b32 a178, 0\n"
  "v_accvgpr_write_b32 a179, 0\n"
  "v_accvgpr_write_b32 a180, 0\n"
  "v_accvgpr_write_b32 a181, 0\n"
  "v_accvgpr_write_b32 a182, 0\n"
  "v_accvgpr_write_b32 a183, 0\n"
  "v_accvgpr_write_b32 a184, 0\n"
  "v_accvgpr_write_b32 a185, 0\n"
  "v_accvgpr_write_b32 a186, 0\n"
  "v_accvgpr_write_b32 a187, 0\n"
  "v_accvgpr_write_b32 a188, 0\n"
  "v_accvgpr_write_b32 a189, 0\n"
  "v_accvgpr_write_b32 a190, 0\n"
  "v_accvgpr_write_b32 a191, 0\n"
  "v_accvgpr_write_b32 a192, 0\n"
  "v_accvgpr_write_b32 a193, 0\n"
  "v_accvgpr_write_b32 a194, 0\n"
  "v_accvgpr_write_b32 a195, 0\n"
  "v_accvgpr_write_b32 a196, 0\n"
  "v_accvgpr_write_b32 a197, 0\n"
  "v_accvgpr_write_b32 a198, 0\n"
  "v_accvgpr_write_b32 a199, 0\n"
  "v_accvgpr_write_b32 a200, 0\n"
  "v_accvgpr_write_b32 a201, 0\n"
  "v_accvgpr_write_b32 a202, 0\n"
  "v_accvgpr_write_b32 a203, 0\n"
  "v_accvgpr_write_b32 a204, 0\n"
  "v_accvgpr_write_b32 a205, 0\n"
  "v_accvgpr_write_b32 a206, 0\n"
  "v_accvgpr_write_b32 a207, 0\n"
  "v_accvgpr_write_b32 a208, 0\n"
  "v_accvgpr_write_b32 a209, 0\n"
  "v_accvgpr_write_b32 a210, 0\n"
  "v_accvgpr_write_b32 a211, 0\n"
  "v_accvgpr_write_b32 a212, 0\n"
  "v_accvgpr_write_b32 a213, 0\n"
  "v_accvgpr_write_b32 a214, 0\n"
  "v_accvgpr_write_b32 a215, 0\n"
  "v_accvgpr_write_b32 a216, 0\n"
  "v_accvgpr_write_b32 a217, 0\n"
  "v_accvgpr_write_b32 a218, 0\n"
  "v_accvgpr_write_b32 a219, 0\n"
  "v_accvgpr_write_b32 a220, 0\n"
  "v_accvgpr_write_b32 a221, 0\n"
  "v_accvgpr_write_b32 a222, 0\n"
  "v_accvgpr_write_b32 a223, 0\n"
  "v_accvgpr_write_b32 a224, 0\n"
  "v_accvgpr_write_b32 a225, 0\n"
  "v_accvgpr_write_b32 a226, 0\n"
  "v_accvgpr_write_b32 a227, 0\n"
  "v_accvgpr_write_b32 a228, 0\n"
  "v_accvgpr_write_b32 a229, 0\n"
  "v_accvgpr_write_b32 a230, 0\n"
  "v_accvgpr_write_b32 a231, 0\n"
  "v_accvgpr_write_b32 a232, 0\n"
  "v_accvgpr_write_b32 a233, 0\n"
  "v_accvgpr_write_b32 a234, 0\n"
  "v_accvgpr_write_b32 a235, 0\n"
  "v_accvgpr_write_b32 a236, 0\n"
  "v_accvgpr_write_b32 a237, 0\n"
  "v_accvgpr_write_b32 a238, 0\n"
  "v_accvgpr_write_b32 a239, 0\n"
  "v_accvgpr_write_b32 a240, 0\n"
  "v_accvgpr_write_b32 a241, 0\n"
  "v_accvgpr_write_b32 a242, 0\n"
  "v_accvgpr_write_b32 a243, 0\n"
  "v_accvgpr_write_b32 a244, 0\n"
  "v_accvgpr_write_b32 a245, 0\n"
  "v_accvgpr_write_b32 a246, 0\n"
  "v_accvgpr_write_b32 a247, 0\n"
  "v_accvgpr_write_b32 a248, 0\n"
  "v_accvgpr_write_b32 a249, 0\n"
  "v_accvgpr_write_b32 a250, 0\n"
  "v_accvgpr_write_b32 a251, 0\n"
  "v_accvgpr_write_b32 a252, 0\n"
  "v_accvgpr_write_b32 a253, 0\n"
  "v_accvgpr_write_b32 a254, 0\n"
  "v_accvgpr_write_b32 a255, 0\n"
  "s_bitcmp1_b32 %6, 3\n"
  "s_cbranch_scc1 NOFL_%=\n"
  "s_lshl_b32 s63, s79, 2\n"
  "s_add_u32 s63, s63, 0x10000\n"
  "v_mov_b32 v226, s61\n"
  "s_waitcnt vmcnt(0)\n"
  "s_barrier\n"
  "s_cmp_lg_u32 %7, 0\n"
  "s_cbranch_scc1 NOF2_%=\n"
  "s_mov_b64 exec, 1\n"
  "buffer_store_dword v226, off, s[32:35], s63 sc1\n"
  "s_mov_b64 exec, -1\n"
  "NOF2_%=:\n"
  "buffer_load_dwordx4 v[64:67], v242, s[28:31], 0 offen nt\n"
  "buffer_load_dwordx4 v[68:71], v242, s[28:31], s39 offen nt\n"
  "buffer_load_dwordx4 v[72:75], v242, s[28:31], s40 offen nt\n"
  "buffer_load_dwordx4 v[76:79], v242, s[28:31], s41 offen nt\n"
  "buffer_load_dwordx4 v[80:83], v242, s[28:31], s42 offen nt\n"
  "buffer_load_dwordx4 v[84:87], v242, s[28:31], s43 offen nt\n"
  "buffer_load_dwordx4 v[88:91], v242, s[28:31], s44 offen nt\n"
  "buffer_load_dwordx4 v[92:95], v242, s[28:31], s45 offen nt\n"
  "s_add_u32 s28, s28, 0x800000\n"
  "s_addc_u32 s29, s29, 0\n"
  "buffer_load_dwordx4 v[160:163], v242, s[28:31], 0 offen nt\n"
  "buffer_load_dwordx4 v[164:167], v242, s[28:31], s39 offen nt\n"
  "buffer_load_dwordx4 v[168:171], v242, s[28:31], s40 offen nt\n"
  "buffer_load_dwordx4 v[172:175], v242, s[28:31], s41 offen nt\n"
  "buffer_load_dwordx4 v[176:179], v242, s[28:31], s42 offen nt\n"
  "buffer_load_dwordx4 v[180:183], v242, s[28:31], s43 offen nt\n"
  "buffer_load_dwordx4 v[184:187], v242, s[28:31], s44 offen nt\n"
  "buffer_load_dwordx4 v[188:191], v242, s[28:31], s45 offen nt\n"
  "s_add_u32 s28, s28, 0x800000\n"
  "s_addc_u32 s29, s29, 0\n"
  "NOFL_%=:\n"
  "v_subrev_u32 v226, 4, v246\n"
  "v_mul_u32_u24 v227, 37, v226\n"
  "v_bfe_u32 v227, v227, 8, 4\n"
  "v_mul_u32_u24 v228, 7, v227\n"
  "v_sub_u32 v226, v226, v228\n"
  "s_add_u32 s63, %5, 1\n"
  "v_add_u32 v226, s63, v226\n"
  "v_and_b32 v226, 7, v226\n"
  "v_lshl_add_u32 v226, v226, 2, v227\n"
  "s_lshl_b32 s63, %5, 2\n"
  "v_add_u32 v227, s63, v246\n"
  "v_cmp_gt_u32 vcc, 4, v246\n"
  "s_nop 1\n"
  "v_cndmask_b32 v226, v226, v227, vcc\n"
  "v_and_b32 v226, 31, v226\n"
  "v_lshlrev_b32 v248, 24, v226\n"
  "v_and_b32 v249, 31, v246\n"
  "s_cmp_eq_u32 %7, 0\n"
  "s_cselect_b32 s63, 0, 16\n"
  "s_add_u32 s63, s63, 0x20000\n"
  "v_mov_b32 v251, s63\n"
  "v_mov_b32 v247, s89\n"
  "s_lshr_b32 s64, %6, 3\n"
  "s_lshl_b32 s64, s64, 1\n"
  "s_add_u32 s64, s64, 1\n"
  "s_bitcmp1_b32 %6, 0\n"
  "s_cselect_b32 s65, -240, 0\n"
  "s_add_u32 s65, s65, s89\n"
  "s_mul_i32 s63, s64, 166\n"
  "s_lshr_b32 s63, s63, 6\n"
  "s_add_u32 s63, s63, 551\n"
  "s_add_u32 s86, s63, s89\n"
  "s_mul_i32 s63, s64, 165\n"
  "s_lshr_b32 s63, s63, 6\n"
  "s_add_u32 s63, s63, 717\n"
  "s_add_u32 s63, s63, s89\n"
  "v_writelane_b32 v247, s63, 17\n"
  "s_mul_i32 s63, s64, 166\n"
  "s_lshr_b32 s63, s63, 6\n"
  "s_add_u32 s63, s63, 882\n"
  "s_add_u32 s63, s63, s89\n"
  "v_writelane_b32 v247, s63, 18\n"
  "s_mul_i32 s63, s64, 219\n"
  "s_lshr_b32 s63, s63, 6\n"
  "s_add_u32 s63, s63, 1048\n"
  "s_add_u32 s63, s63, s89\n"
  "v_writelane_b32 v247, s63, 19\n"
  "s_mul_i32 s63, s64, 526\n"
  "s_lshr_b32 s63, s63, 6\n"
  "s_add_u32 s63, s63, 1267\n"
  "s_add_u32 s63, s63, s89\n"
  "v_writelane_b32 v247, s63, 20\n"
  "s_mul_i32 s63, s64, 364\n"
  "s_lshr_b32 s63, s63, 6\n"
  "s_add_u32 s63, s63, 1913\n"
  "s_add_u32 s63, s63, s65\n"
  "v_writelane_b32 v247, s63, 21\n"
  "s_mul_i32 s63, s64, 270\n"
  "s_lshr_b32 s63, s63, 6\n"
  "s_add_u32 s63, s63, 2277\n"
  "s_add_u32 s63, s63, s65\n"
  "v_writelane_b32 v247, s63, 22\n"
  "s_mul_i32 s63, s64, 225\n"
  "s_lshr_b32 s63, s63, 6\n"
  "s_add_u32 s63, s63, 2547\n"
  "s_add_u32 s63, s63, s65\n"
  "v_writelane_b32 v247, s63, 23\n"
  "s_mul_i32 s63, s64, 197\n"
  "s_lshr_b32 s63, s63, 6\n"
  "s_add_u32 s63, s63, 2772\n"
  "s_add_u32 s63, s63, s65\n"
  "v_writelane_b32 v247, s63, 24\n"
  "s_mul_i32 s63, s64, 179\n"
  "s_lshr_b32 s63, s63, 6\n"
  "s_add_u32 s63, s63, 2969\n"
  "s_add_u32 s63, s63, s65\n"
  "v_writelane_b32 v247, s63, 25\n"
  "s_mul_i32 s63, s64, 172\n"
  "s_lshr_b32 s63, s63, 6\n"
  "s_add_u32 s63, s63, 3148\n"
  "s_add_u32 s63, s63, s65\n"
  "v_writelane_b32 v247, s63, 26\n"
  "s_mul_i32 s63, s64, 168\n"
  "s_lshr_b32 s63, s63, 6\n"
  "s_add_u32 s63, s63, 3320\n"
  "s_add_u32 s63, s63, s65\n"
  "v_writelane_b32 v247, s63, 27\n"
  "s_mul_i32 s63, s64, 164\n"
  "s_lshr_b32 s63, s63, 6\n"
  "s_add_u32 s63, s63, 3488\n"
  "s_add_u32 s63, s63, s65\n"
  "v_writelane_b32 v247, s63, 28\n"
  "s_mul_i32 s63, s64, 160\n"
  "s_lshr_b32 s63, s63, 6\n"
  "s_add_u32 s63, s63, 3652\n"
  "s_add_u32 s63, s63, s65\n"
  "v_writelane_b32 v247, s63, 29\n"
  "s_mul_i32 s63, s64, 157\n"
  "s_lshr_b32 s63, s63, 6\n"
  "s_add_u32 s63, s63, 3812\n"
  "s_add_u32 s63, s63, s65\n"
  "v_writelane_b32 v247, s63, 30\n"
  "s_mul_i32 s63, s64, 153\n"
  "s_lshr_b32 s63, s63, 6\n"
  "s_add_u32 s63, s63, 3969\n"
  "s_add_u32 s63, s63, s65\n"
  "v_writelane_b32 v247, s63, 31\n"
  "s_mul_i32 s63, s64, 150\n"
  "s_lshr_b32 s63, s63, 6\n"
  "s_add_u32 s63, s63, 4122\n"
  "s_add_u32 s63, s63, s65\n"
  "v_writelane_b32 v247, s63, 32\n"
  "s_mul_i32 s63, s64, 147\n"
  "s_lshr_b32 s63, s63, 6\n"
  "s_add_u32 s63, s63, 4272\n"
  "s_add_u32 s63, s63, s65\n"
  "v_writelane_b32 v247, s63, 33\n"
  "s_mul_i32 s63, s64, 145\n"
  "s_lshr_b32 s63, s63, 6\n"
  "s_add_u32 s63, s63, 4419\n"
  "s_add_u32 s63, s63, s65\n"
  "v_writelane_b32 v247, s63, 34\n"
  "s_mul_i32 s63, s64, 141\n"
  "s_lshr_b32 s63, s63, 6\n"
  "s_add_u32 s63, s63, 4564\n"
  "s_add_u32 s63, s63, s65\n"
  "v_writelane_b32 v247, s63, 35\n"
  "s_mul_i32 s63, s64, 140\n"
  "s_lshr_b32 s63, s63, 6\n"
  "s_add_u32 s63, s63, 4705\n"
  "s_add_u32 s63, s63, s65\n"
  "v_writelane_b32 v247, s63, 36\n"
  "s_mul_i32 s63, s64, 137\n"
  "s_lshr_b32 s63, s63, 6\n"
  "s_add_u32 s63, s63, 4845\n"
  "s_add_u32 s63, s63, s65\n"
  "v_writelane_b32 v247, s63, 37\n"
  "s_mov_b32 s85, 16\n"
  "s_mov_b32 s62, 0\n"
  "v_writelane_b32 v247, s76, 1\n"
  "v_writelane_b32 v247, s56, 2\n"
  "v_writelane_b32 v247, s57, 3\n"
  "s_bfe_u32 s65, %6, 0x10003\n"
  "s_lshl_b32 s65, s65, 2\n"
  "s_and_b32 s80, s79, 3\n"
  "s_add_u32 s65, s65, s80\n"
  "s_add_u32 s65, s65, 4\n"
  "s_lshr_b32 s80, s79, 2\n"
  "s_mul_i32 s63, s65, 37\n"
  "s_lshr_b32 s63, s63, 8\n"
  "s_mul_i32 s64, s63, 7\n"
  "s_sub_u32 s64, s65, s64\n"
  "s_sub_u32 s64, %5, s64\n"
  "s_sub_u32 s64, s64, 1\n"
  "s_and_b32 s64, s64, 7\n"
  "s_add_u32 s76, s65, 4\n"
  "s_lshl_b32 s66, s64, 3\n"
  "s_load_dwordx2 s[56:57], %0, s66\n"
  "s_and_b32 s66, s80, 7\n"
  "s_lshl_b32 s66, s66, 2\n"
  "s_add_u32 s66, s66, %7\n"
  "s_lshl_b32 s88, s64, 20\n"
  "s_lshl_b32 s63, s63, 9\n"
  "s_add_u32 s88, s88, s63\n"
  "s_lshl_b32 s63, s66, 15\n"
  "s_add_u32 s88, s88, s63\n"
  "s_lshr_b32 s80, s80, 3\n"
  "s_lshl_b32 s63, s80, 7\n"
  "s_add_u32 s88, s88, s63\n"
  "s_lshl_b32 s76, s76, 17\n"
  "s_lshl_b32 s63, s66, 9\n"
  "s_add_u32 s76, s76, s63\n"
  "s_lshr_b32 s63, s80, 1\n"
  "s_lshl_b32 s63, s63, 16\n"
  "s_add_u32 s76, s76, s63\n"
  "s_and_b32 s63, s80, 1\n"
  "s_lshl_b32 s63, s63, 14\n"
  "s_add_u32 s76, s76, s63\n"
  "v_writelane_b32 v247, s88, 4\n"
  "v_writelane_b32 v247, s76, 5\n"
  "s_waitcnt lgkmcnt(0)\n"
  "v_writelane_b32 v247, s56, 6\n"
  "v_writelane_b32 v247, s57, 7\n"
  "s_bfe_u32 s65, %6, 0x10003\n"
  "s_lshl_b32 s65, s65, 2\n"
  "s_and_b32 s80, s79, 3\n"
  "s_add_u32 s65, s65, s80\n"
  "s_add_u32 s65, s65, 12\n"
  "s_lshr_b32 s80, s79, 2\n"
  "s_mul_i32 s63, s65, 37\n"
  "s_lshr_b32 s63, s63, 8\n"
  "s_mul_i32 s64, s63, 7\n"
  "s_sub_u32 s64, s65, s64\n"
  "s_sub_u32 s64, %5, s64\n"
  "s_sub_u32 s64, s64, 1\n"
  "s_and_b32 s64, s64, 7\n"
  "s_add_u32 s76, s65, 4\n"
  "s_lshl_b32 s66, s64, 3\n"
  "s_load_dwordx2 s[56:57], %0, s66\n"
  "s_and_b32 s66, s80, 7\n"
  "s_lshl_b32 s66, s66, 2\n"
  "s_add_u32 s66, s66, %7\n"
  "s_lshl_b32 s88, s64, 20\n"
  "s_lshl_b32 s63, s63, 9\n"
  "s_add_u32 s88, s88, s63\n"
  "s_lshl_b32 s63, s66, 15\n"
  "s_add_u32 s88, s88, s63\n"
  "s_lshr_b32 s80, s80, 3\n"
  "s_lshl_b32 s63, s80, 7\n"
  "s_add_u32 s88, s88, s63\n"
  "s_lshl_b32 s76, s76, 17\n"
  "s_lshl_b32 s63, s66, 9\n"
  "s_add_u32 s76, s76, s63\n"
  "s_lshr_b32 s63, s80, 1\n"
  "s_lshl_b32 s63, s63, 16\n"
  "s_add_u32 s76, s76, s63\n"
  "s_and_b32 s63, s80, 1\n"
  "s_lshl_b32 s63, s63, 14\n"
  "s_add_u32 s76, s76, s63\n"
  "v_writelane_b32 v247, s88, 8\n"
  "v_writelane_b32 v247, s76, 9\n"
  "s_waitcnt lgkmcnt(0)\n"
  "v_writelane_b32 v247, s56, 10\n"
  "v_writelane_b32 v247, s57, 11\n"
  "s_bfe_u32 s65, %6, 0x10003\n"
  "s_lshl_b32 s65, s65, 2\n"
  "s_and_b32 s80, s79, 3\n"
  "s_add_u32 s65, s65, s80\n"
  "s_add_u32 s65, s65, 20\n"
  "s_lshr_b32 s80, s79, 2\n"
  "s_mul_i32 s63, s65, 37\n"
  "s_lshr_b32 s63, s63, 8\n"
  "s_mul_i32 s64, s63, 7\n"
  "s_sub_u32 s64, s65, s64\n"
  "s_sub_u32 s64, %5, s64\n"
  "s_sub_u32 s64, s64, 1\n"
  "s_and_b32 s64, s64, 7\n"
  "s_add_u32 s76, s65, 4\n"
  "s_lshl_b32 s66, s64, 3\n"
  "s_load_dwordx2 s[56:57], %0, s66\n"
  "s_and_b32 s66, s80, 7\n"
  "s_lshl_b32 s66, s66, 2\n"
  "s_add_u32 s66, s66, %7\n"
  "s_lshl_b32 s88, s64, 20\n"
  "s_lshl_b32 s63, s63, 9\n"
  "s_add_u32 s88, s88, s63\n"
  "s_lshl_b32 s63, s66, 15\n"
  "s_add_u32 s88, s88, s63\n"
  "s_lshr_b32 s80, s80, 3\n"
  "s_lshl_b32 s63, s80, 7\n"
  "s_add_u32 s88, s88, s63\n"
  "s_lshl_b32 s76, s76, 17\n"
  "s_lshl_b32 s63, s66, 9\n"
  "s_add_u32 s76, s76, s63\n"
  "s_lshr_b32 s63, s80, 1\n"
  "s_lshl_b32 s63, s63, 16\n"
  "s_add_u32 s76, s76, s63\n"
  "s_and_b32 s63, s80, 1\n"
  "s_lshl_b32 s63, s63, 14\n"
  "s_add_u32 s76, s76, s63\n"
  "v_writelane_b32 v247, s88, 12\n"
  "v_writelane_b32 v247, s76, 13\n"
  "s_waitcnt lgkmcnt(0)\n"
  "v_writelane_b32 v247, s56, 14\n"
  "v_writelane_b32 v247, s57, 15\n"
  "s_nop 0\n"
  "v_readlane_b32 s76, v247, 1\n"
  "v_readlane_b32 s56, v247, 2\n"
  "v_readlane_b32 s57, v247, 3\n"
  "s_mov_b32 s84, 0x40000000\n"
  "s_mov_b32 s59, 0\n"
  "s_mov_b32 s58, 3\n"
  "s_mov_b32 s67, 1.0\n"
  "s_mov_b32 s70, 0xf\n"
  "s_mov_b32 s71, 0xf\n"
  "s_mov_b32 s73, 2\n"
  "s_mov_b32 s72, 3\n"
  "s_lshl_b32 s83, %7, 12\n"
  "s_mov_b32 s46, s83\n"
  "s_add_u32 s74, s46, 0x4000\n"
  "s_cmp_eq_u32 %7, 0\n"
  "s_cselect_b32 s75, 0xccc, 0\n"
  "s_cselect_b32 s64, 3, 0\n"
  "s_cselect_b32 s77, 2, 4\n"
  "s_or_b32 s75, s75, 0x333\n"
  "s_and_b32 s75, s75, -1\n"
  "s_bfe_u32 s63, %6, 0x10004\n"
  "s_lshl_b32 s75, s75, s63\n"
  "s_bitcmp1_b32 %6, 3\n"
  "s_cselect_b32 s63, 2, 0\n"
  "s_cselect_b32 s64, s64, 0\n"
  "s_cselect_b32 s77, s77, 4\n"
  "s_lshl_b32 s75, s75, s63\n"
  "s_or_b32 s75, s75, s64\n"
  "s_mov_b32 s87, 0\n"
  "s_waitcnt vmcnt(14)\n"
  "v_cvt_pk_f16_f32 v224, v64, v68\n"
  "s_waitcnt vmcnt(12)\n"
  "v_cvt_pk_f16_f32 v225, v72, v76\n"
  "s_waitcnt vmcnt(10)\n"
  "v_cvt_pk_f16_f32 v226, v80, v84\n"
  "s_waitcnt vmcnt(8)\n"
  "v_cvt_pk_f16_f32 v227, v88, v92\n"
  "ds_write_b128 v244, v[224:227] offset:0\n"
  "v_cvt_pk_f16_f32 v228, v65, v69\n"
  "v_cvt_pk_f16_f32 v229, v73, v77\n"
  "v_cvt_pk_f16_f32 v230, v81, v85\n"
  "v_cvt_pk_f16_f32 v231, v89, v93\n"
  "ds_write_b128 v244, v[228:231] offset:1024\n"
  "s_nop 1\n"
  "v_cvt_pk_f16_f32 v224, v66, v70\n"
  "v_cvt_pk_f16_f32 v225, v74, v78\n"
  "v_cvt_pk_f16_f32 v226, v82, v86\n"
  "v_cvt_pk_f16_f32 v227, v90, v94\n"
  "ds_write_b128 v244, v[224:227] offset:2048\n"
  "v_cvt_pk_f16_f32 v228, v67, v71\n"
  "v_cvt_pk_f16_f32 v229, v75, v79\n"
  "v_cvt_pk_f16_f32 v230, v83, v87\n"
  "v_cvt_pk_f16_f32 v231, v91, v95\n"
  "ds_write_b128 v244, v[228:231] offset:3072\n"
  "v_add_u32 v244, 0x4000, v244\n"
  "v_and_b32 v244, 0x1ffff, v244\n"
  "s_waitcnt vmcnt(6)\n"
  "v_cvt_pk_f16_f32 v224, v160, v164\n"
  "s_waitcnt vmcnt(4)\n"
  "v_cvt_pk_f16_f32 v225, v168, v172\n"
  "s_waitcnt vmcnt(2)\n"
  "v_cvt_pk_f16_f32 v226, v176, v180\n"
  "s_waitcnt vmcnt(0)\n"
  "v_cvt_pk_f16_f32 v227, v184, v188\n"
  "ds_write_b128 v244, v[224:227] offset:0\n"
  "v_cvt_pk_f16_f32 v228, v161, v165\n"
  "v_cvt_pk_f16_f32 v229, v169, v173\n"
  "v_cvt_pk_f16_f32 v230, v177, v181\n"
  "v_cvt_pk_f16_f32 v231, v185, v189\n"
  "ds_write_b128 v244, v[228:231] offset:1024\n"
  "s_nop 1\n"
  "v_cvt_pk_f16_f32 v224, v162, v166\n"
  "v_cvt_pk_f16_f32 v225, v170, v174\n"
  "v_cvt_pk_f16_f32 v226, v178, v182\n"
  "v_cvt_pk_f16_f32 v227, v186, v190\n"
  "ds_write_b128 v244, v[224:227] offset:2048\n"
  "v_cvt_pk_f16_f32 v228, v163, v167\n"
  "v_cvt_pk_f16_f32 v229, v171, v175\n"
  "v_cvt_pk_f16_f32 v230, v179, v183\n"
  "v_cvt_pk_f16_f32 v231, v187, v191\n"
  "ds_write_b128 v244, v[228:231] offset:3072\n"
  "v_add_u32 v244, 0x4000, v244\n"
  "v_and_b32 v244, 0x1ffff, v244\n"
  "s_mov_b64 s[32:33], %1\n"
  "s_mov_b32 s34, 0x900000\n"
  "v_mov_b32 v250, s67\n"
  "s_cmp_lg_u32 %7, 0\n"
  "s_cbranch_scc1 PWD_%=\n"
  "s_memrealtime s[54:55]\n"
  "s_mov_b32 s66, 0x10000\n"
  "v_lshrrev_b32 v253, 1, v243\n"
  "PWL_%=:\n"
  "buffer_load_dwordx2 v[128:129], v253, s[32:35], s66 offen sc1\n"
  "s_memrealtime s[52:53]\n"
  "s_waitcnt vmcnt(0) lgkmcnt(0)\n"
  "v_min_u32 v128, v128, v129\n"
  "v_cmp_gt_u32 vcc, s61, v128\n"
  "s_nop 1\n"
  "s_cmp_eq_u64 vcc, 0\n"
  "s_cbranch_scc1 PWD_%=\n"
  "s_sub_u32 s63, s52, s54\n"
  "s_cmp_gt_u32 s63, 100000000\n"
  "s_cbranch_scc1 PWT_%=\n"
  "s_cmp_lt_u32 s63, 5000\n"
  "s_cbranch_scc1 PWL_%=\n"
  "s_mov_b32 s63, 0x8000\n"
  "buffer_load_dword v246, off, s[32:35], s63 sc1\n"
  "s_waitcnt vmcnt(0)\n"
  "v_readfirstlane_b32 s63, v246\n"
  "s_cmp_eq_u32 s63, s61\n"
  "s_cbranch_scc1 PWT_%=\n"
  "s_sleep 2\n"
  "s_branch PWL_%=\n"
  "PWT_%=:\n"
  "s_mov_b32 s63, 0x8000\n"
  "v_mov_b32 v246, s61\n"
  "buffer_store_dword v246, off, s[32:35], s63 sc1\n"
  "v_mov_b32 v250, 0x7fc00000\n"
  "PWD_%=:\n"
  "v_mov_b32 v253, s91\n"
  "ds_write_b32 v251, v253 offset:8\n"
  "ds_write_b32 v251, v250 offset:12\n"
  "s_waitcnt lgkmcnt(0)\n"
  "s_barrier\n"
  "s_cmp_lg_u32 %7, 0\n"
  "s_cbranch_scc1 NOCTR_%=\n"
  "v_mov_b32 v226, s61\n"
  "s_mov_b64 exec, 1\n"
  "buffer_store_dword v226, off, s[32:35], s81\n"
  "s_mov_b64 exec, -1\n"
  "NOCTR_%=:\n"
  "buffer_load_dwordx4 v[0:3], v240, s[20:23], s46 offen offset:0\n"
  "buffer_load_dwordx4 v[4:7], v241, s[24:27], s74 offen offset:0\n"
  "buffer_load_dwordx4 v[8:11], v241, s[24:27], s46 offen offset:512\n"
  "buffer_load_dwordx4 v[12:15], v240, s[20:23], s74 offen offset:512\n"
  "buffer_load_dwordx4 v[16:19], v241, s[24:27], s46 offen offset:1024\n"
  "buffer_load_dwordx4 v[20:23], v240, s[20:23], s74 offen offset:1024\n"
  "buffer_load_dwordx4 v[24:27], v240, s[20:23], s46 offen offset:1536\n"
  "buffer_load_dwordx4 v[28:31], v241, s[24:27], s74 offen offset:1536\n"
  "buffer_load_dwordx4 v[32:35], v241, s[24:27], s46 offen offset:2048\n"
  "buffer_load_dwordx4 v[36:39], v240, s[20:23], s74 offen offset:2048\n"
  "buffer_load_dwordx4 v[40:43], v240, s[20:23], s46 offen offset:2560\n"
  "buffer_load_dwordx4 v[44:47], v241, s[24:27], s74 offen offset:2560\n"
  "buffer_load_dwordx4 v[48:51], v240, s[20:23], s46 offen offset:3072\n"
  "buffer_load_dwordx4 v[52:55], v241, s[24:27], s74 offen offset:3072\n"
  "buffer_load_dwordx4 v[56:59], v241, s[24:27], s46 offen offset:3584\n"
  "buffer_load_dwordx4 v[60:63], v240, s[20:23], s74 offen offset:3584\n"
  "s_add_u32 s46, s46, 0x10000\n"
  "s_add_u32 s74, s74, 0x10000\n"
  "v_mov_b32 v250, 0x20000\n"
  "ds_read_b32 v253, v250 offset:8\n"
  "ds_read_b32 v250, v250 offset:12\n"
  "ds_read_b128 v[128:131], v243 offset:0\n"
  "ds_read_b128 v[132:135], v243 offset:8192\n"
  "ds_read_b128 v[136:139], v243 offset:1024\n"
  "ds_read_b128 v[140:143], v243 offset:9216\n"
  "ds_read_b128 v[144:147], v243 offset:2048\n"
  "ds_read_b128 v[148:151], v243 offset:10240\n"
  "s_waitcnt lgkmcnt(6)\n"
  "v_readfirstlane_b32 s67, v250\n"
  "v_readfirstlane_b32 s91, v253\n"
  "s_bitcmp1_b32 %6, 3\n"
  "s_cbranch_scc0 NORS_%=\n"
  "v_cvt_pk_f16_f32 v232, v232, v233\n"
  "v_cvt_pk_f16_f32 v233, v234, v235\n"
  "v_cvt_pk_f16_f32 v234, v236, v237\n"
  "v_cvt_pk_f16_f32 v235, v238, v239\n"
  "s_mov_b64 s[32:33], s[56:57]\n"
  "s_mov_b32 s34, 0x900000\n"
  "s_add_u32 s63, s76, s68\n"
  "v_and_b32 v224, 0xffff, v240\n"
  "buffer_store_dwordx4 v[232:235], v224, s[32:35], s63 offen sc0 sc1\n"
  "s_branch NORT_%=\n"
  "NORS_%=:\n"
  "buffer_load_dword v246, off, s[20:23], s84\n"
  "NORT_%=:\n"
  "buffer_load_dword v246, v245, s[36:39], 0 offen sc0 sc1\n"
  "s_mov_b32 s60, 0x20000\n"
  "buffer_load_dword v192, v245, s[20:23], s60 offen\n"
  "buffer_load_dwordx4 v[192:195], v242, s[28:31], 0 offen nt\n"
  "buffer_load_dwordx4 v[196:199], v242, s[28:31], s39 offen nt\n"
  "buffer_load_dwordx4 v[200:203], v242, s[28:31], s40 offen nt\n"
  "buffer_load_dwordx4 v[204:207], v242, s[28:31], s41 offen nt\n"
  "buffer_load_dwordx4 v[208:211], v242, s[28:31], s42 offen nt\n"
  "buffer_load_dwordx4 v[212:215], v242, s[28:31], s43 offen nt\n"
  "buffer_load_dwordx4 v[216:219], v242, s[28:31], s44 offen nt\n"
  "buffer_load_dwordx4 v[220:223], v242, s[28:31], s45 offen nt\n"
  "s_nop 1\n"
  "s_cmp_lt_u32 s91, 10000\n"
  "s_cbranch_scc1 DENT_%=\n"
  "s_memrealtime s[52:53]\n"
  "s_add_u32 s89, s89, 460\n"
  "s_waitcnt lgkmcnt(0)\n"
  "s_sub_u32 s89, s52, s89\n"
  "s_cmp_lt_i32 s89, 150\n"
  "s_cselect_b32 s89, 0, s89\n"
  "s_add_u32 s86, s86, s89\n"
  "s_mov_b32 s90, s86\n"
  "s_bitcmp1_b32 %6, 3\n"
  "s_cselect_b32 s63, 3, 0\n"
  "s_and_b32 s75, s75, s63\n"
  "CLOOP_%=:\n"
  "buffer_load_dwordx4 v[64:67], v240, s[20:23], s46 offen offset:0\n"
  "s_waitcnt vmcnt(12) lgkmcnt(4)\n"
  "v_mfma_f32_16x16x32_f16 a[0:3], v[0:3], v[128:131], a[0:3]\n"
  "s_nop 3\n"
  "v_mfma_f32_16x16x32_f16 a[0:3], v[4:7], v[132:135], a[0:3]\n"
  "ds_read_b128 v[152:155], v243 offset:3072\n"
  "v_mfma_f32_16x16x32_f16 a[32:35], v[12:15], v[132:135], a[32:35]\n"
  "v_mfma_f32_16x16x32_f16 a[32:35], v[8:11], v[128:131], a[32:35]\n"
  "v_mfma_f32_16x16x32_f16 a[64:67], v[16:19], v[128:131], a[64:67]\n"
  "buffer_load_dwordx4 v[68:71], v241, s[24:27], s74 offen offset:0\n"
  "v_mfma_f32_16x16x32_f16 a[64:67], v[20:23], v[132:135], a[64:67]\n"
  "v_mfma_f32_16x16x32_f16 a[96:99], v[28:31], v[132:135], a[96:99]\n"
  "v_mfma_f32_16x16x32_f16 a[96:99], v[24:27], v[128:131], a[96:99]\n"
  "ds_read_b128 v[156:159], v243 offset:11264\n"
  "s_nop 3\n"
  "v_mfma_f32_16x16x32_f16 a[128:131], v[32:35], v[128:131], a[128:131]\n"
  "buffer_load_dwordx4 v[72:75], v241, s[24:27], s46 offen offset:512\n"
  "v_mfma_f32_16x16x32_f16 a[128:131], v[36:39], v[132:135], a[128:131]\n"
  "v_mfma_f32_16x16x32_f16 a[160:163], v[44:47], v[132:135], a[160:163]\n"
  "v_mfma_f32_16x16x32_f16 a[160:163], v[40:43], v[128:131], a[160:163]\n"
  "v_mfma_f32_16x16x32_f16 a[192:195], v[48:51], v[128:131], a[192:195]\n"
  "v_mfma_f32_16x16x32_f16 a[192:195], v[52:55], v[132:135], a[192:195]\n"
  "buffer_load_dwordx4 v[76:79], v240, s[20:23], s74 offen offset:512\n"
  "v_mfma_f32_16x16x32_f16 a[224:227], v[60:63], v[132:135], a[224:227]\n"
  "v_mfma_f32_16x16x32_f16 a[224:227], v[56:59], v[128:131], a[224:227]\n"
  "s_nop 3\n"
  "s_waitcnt lgkmcnt(4)\n"
  "v_mfma_f32_16x16x32_f16 a[4:7], v[0:3], v[136:139], a[4:7]\n"
  "v_mfma_f32_16x16x32_f16 a[4:7], v[4:7], v[140:143], a[4:7]\n"
  "ds_read_b128 v[128:131], v243 offset:4096\n"
  "v_mfma_f32_16x16x32_f16 a[36:39], v[12:15], v[140:143], a[36:39]\n"
  "buffer_load_dwordx4 v[80:83], v241, s[24:27], s46 offen offset:1024\n"
  "v_mfma_f32_16x16x32_f16 a[36:39], v[8:11], v[136:139], a[36:39]\n"
  "v_mfma_f32_16x16x32_f16 a[68:71], v[16:19], v[136:139], a[68:71]\n"
  "s_lshr_b32 s79, s58, 1\n"
  "v_mfma_f32_16x16x32_f16 a[68:71], v[20:23], v[140:143], a[68:71]\n"
  "v_mfma_f32_16x16x32_f16 a[100:103], v[28:31], v[140:143], a[100:103]\n"
  "buffer_load_dwordx4 v[84:87], v240, s[20:23], s74 offen offset:1024\n"
  "v_readlane_b32 s79, v249, s79\n"
  "v_mfma_f32_16x16x32_f16 a[100:103], v[24:27], v[136:139], a[100:103]\n"
  "ds_read_b128 v[132:135], v243 offset:12288\n"
  "s_nop 3\n"
  "v_mfma_f32_16x16x32_f16 a[132:135], v[32:35], v[136:139], a[132:135]\n"
  "s_and_b32 s80, s58, 1\ns_lshl_b32 s80, s80, 23\n"
  "v_mfma_f32_16x16x32_f16 a[132:135], v[36:39], v[140:143], a[132:135]\n"
  "v_mfma_f32_16x16x32_f16 a[164:167], v[44:47], v[140:143], a[164:167]\n"
  "s_mov_b32 s81, s79\n"
  "v_mfma_f32_16x16x32_f16 a[164:167], v[40:43], v[136:139], a[164:167]\n"
  "buffer_load_dwordx4 v[88:91], v240, s[20:23], s46 offen offset:1536\n"
  "v_mfma_f32_16x16x32_f16 a[196:199], v[48:51], v[136:139], a[196:199]\n"
  "v_mfma_f32_16x16x32_f16 a[196:199], v[52:55], v[140:143], a[196:199]\n"
  "v_readlane_b32 s79, v248, s81\n"
  "v_mfma_f32_16x16x32_f16 a[228:231], v[60:63], v[140:143], a[228:231]\n"
  "v_mfma_f32_16x16x32_f16 a[228:231], v[56:59], v[136:139], a[228:231]\n"
  "s_add_u32 s79, s79, s80\n"
  "s_nop 3\n"
  "s_waitcnt lgkmcnt(4)\n"
  "v_mfma_f32_16x16x32_f16 a[8:11], v[0:3], v[144:147], a[8:11]\n"
  "buffer_load_dwordx4 v[92:95], v241, s[24:27], s74 offen offset:1536\n"
  "v_mfma_f32_16x16x32_f16 a[8:11], v[4:7], v[148:151], a[8:11]\n"
  "ds_read_b128 v[136:139], v243 offset:5120\n"
  "s_add_u32 s28, s48, s79\ns_addc_u32 s29, s49, 0\n"
  "v_mfma_f32_16x16x32_f16 a[40:43], v[12:15], v[148:151], a[40:43]\n"
  "v_mfma_f32_16x16x32_f16 a[40:43], v[8:11], v[144:147], a[40:43]\n"
  "s_cmp_lt_u32 s58, 64\ns_cselect_b32 s30, s69, 0\n"
  "v_mfma_f32_16x16x32_f16 a[72:75], v[16:19], v[144:147], a[72:75]\n"
  "buffer_load_dwordx4 v[96:99], v241, s[24:27], s46 offen offset:2048\n"
  "v_mfma_f32_16x16x32_f16 a[72:75], v[20:23], v[148:151], a[72:75]\n"
  "s_add_u32 s58, s58, 1\n"
  "v_mfma_f32_16x16x32_f16 a[104:107], v[28:31], v[148:151], a[104:107]\n"
  "v_mfma_f32_16x16x32_f16 a[104:107], v[24:27], v[144:147], a[104:107]\n"
  "ds_read_b128 v[140:143], v243 offset:13312\n"
  "s_nop 3\n"
  "v_mfma_f32_16x16x32_f16 a[136:139], v[32:35], v[144:147], a[136:139]\n"
  "v_mfma_f32_16x16x32_f16 a[136:139], v[36:39], v[148:151], a[136:139]\n"
  "buffer_load_dwordx4 v[100:103], v240, s[20:23], s74 offen offset:2048\n"
  "v_mfma_f32_16x16x32_f16 a[168:171], v[44:47], v[148:151], a[168:171]\n"
  "v_mfma_f32_16x16x32_f16 a[168:171], v[40:43], v[144:147], a[168:171]\n"
  "v_mfma_f32_16x16x32_f16 a[200:203], v[48:51], v[144:147], a[200:203]\n"
  "s_add_u32 s79, s59, 3\ns_lshr_b32 s80, s79, 1\n"
  "v_mfma_f32_16x16x32_f16 a[200:203], v[52:55], v[148:151], a[200:203]\n"
  "v_mfma_f32_16x16x32_f16 a[232:235], v[60:63], v[148:151], a[232:235]\n"
  "buffer_load_dwordx4 v[104:107], v240, s[20:23], s46 offen offset:2560\n"
  "v_readlane_b32 s80, v249, s80\n"
  "v_mfma_f32_16x16x32_f16 a[232:235], v[56:59], v[144:147], a[232:235]\n"
  "s_nop 3\n"
  "s_waitcnt lgkmcnt(4)\n"
  "v_mfma_f32_16x16x32_f16 a[12:15], v[0:3], v[152:155], a[12:15]\n"
  "s_lshl_b32 s80, s80, 17\ns_add_u32 s80, s80, 0x10000\n"
  "v_mfma_f32_16x16x32_f16 a[12:15], v[4:7], v[156:159], a[12:15]\n"
  "ds_read_b128 v[144:147], v243 offset:6144\n"
  "v_mfma_f32_16x16x32_f16 a[44:47], v[12:15], v[156:159], a[44:47]\n"
  "buffer_load_dwordx4 v[108:111], v241, s[24:27], s74 offen offset:2560\n"
  "s_cmp_lt_u32 s79, 64\ns_cselect_b32 s60, s80, s84\n"
  "v_mfma_f32_16x16x32_f16 a[44:47], v[8:11], v[152:155], a[44:47]\n"
  "v_mfma_f32_16x16x32_f16 a[76:79], v[16:19], v[152:155], a[76:79]\n"
  "v_mfma_f32_16x16x32_f16 a[76:79], v[20:23], v[156:159], a[76:79]\n"
  "v_mfma_f32_16x16x32_f16 a[108:111], v[28:31], v[156:159], a[108:111]\n"
  "s_waitcnt vmcnt(21)\n"
  "v_cmp_le_u32 vcc, s61, v246\n"
  "v_mfma_f32_16x16x32_f16 a[108:111], v[24:27], v[152:155], a[108:111]\n"
  "buffer_load_dwordx4 v[112:115], v240, s[20:23], s46 offen offset:3072\n"
  "ds_read_b128 v[148:151], v243 offset:14336\n"
  "s_nop 3\n"
  "v_mfma_f32_16x16x32_f16 a[140:143], v[32:35], v[152:155], a[140:143]\n"
  "s_or_b32 s71, s71, vcc_lo\n"
  "v_mfma_f32_16x16x32_f16 a[140:143], v[36:39], v[156:159], a[140:143]\n"
  "v_mfma_f32_16x16x32_f16 a[172:175], v[44:47], v[156:159], a[172:175]\n"
  "v_mov_b32 v250, s71\n"
  "v_mfma_f32_16x16x32_f16 a[172:175], v[40:43], v[152:155], a[172:175]\n"
  "v_mfma_f32_16x16x32_f16 a[204:207], v[48:51], v[152:155], a[204:207]\n"
  "buffer_load_dwordx4 v[116:119], v241, s[24:27], s74 offen offset:3072\n"
  "ds_write_b32 v251, v250 offset:0\n"
  "v_mfma_f32_16x16x32_f16 a[204:207], v[52:55], v[156:159], a[204:207]\n"
  "v_mfma_f32_16x16x32_f16 a[236:239], v[60:63], v[156:159], a[236:239]\n"
  "v_mfma_f32_16x16x32_f16 a[236:239], v[56:59], v[152:155], a[236:239]\n"
  "ds_read_b128 v[152:155], v243 offset:7168\n"
  "s_nop 3\n"
  "s_waitcnt lgkmcnt(6)\n"
  "v_mfma_f32_16x16x32_f16 a[16:19], v[0:3], v[128:131], a[16:19]\n"
  "buffer_load_dwordx4 v[120:123], v241, s[24:27], s46 offen offset:3584\n"
  "v_mfma_f32_16x16x32_f16 a[16:19], v[4:7], v[132:135], a[16:19]\n"
  "v_mfma_f32_16x16x32_f16 a[48:51], v[12:15], v[132:135], a[48:51]\n"
  "v_mfma_f32_16x16x32_f16 a[48:51], v[8:11], v[128:131], a[48:51]\n"
  "v_mfma_f32_16x16x32_f16 a[80:83], v[16:19], v[128:131], a[80:83]\n"
  "v_mfma_f32_16x16x32_f16 a[80:83], v[20:23], v[132:135], a[80:83]\n"
  "buffer_load_dwordx4 v[124:127], v240, s[20:23], s74 offen offset:3584\n"
  "v_mfma_f32_16x16x32_f16 a[112:115], v[28:31], v[132:135], a[112:115]\n"
  "v_mfma_f32_16x16x32_f16 a[112:115], v[24:27], v[128:131], a[112:115]\n"
  "s_waitcnt vmcnt(23) lgkmcnt(1)\n"
  "s_barrier\n"
  "s_bitcmp1_b32 s75, s59\ns_cbranch_scc0 ENc0_%=\ns_call_b64 s[50:51], EVENT_%=\ns_branch EVc0_%=\nENc0_%=:\nbuffer_load_dword v246, off, s[20:23], s84\nEVc0_%=:\n"
  "s_nop 3\n"
  "v_mfma_f32_16x16x32_f16 a[144:147], v[32:35], v[128:131], a[144:147]\n"
  "buffer_load_dword v246, v245, s[36:39], 0 offen sc0 sc1\n"
  "v_mfma_f32_16x16x32_f16 a[144:147], v[36:39], v[132:135], a[144:147]\n"
  "buffer_load_dword v160, v245, s[20:23], s60 offen\n"
  "v_mfma_f32_16x16x32_f16 a[176:179], v[44:47], v[132:135], a[176:179]\n"
  "buffer_load_dwordx4 v[160:163], v242, s[28:31], 0 offen nt\n"
  "v_mfma_f32_16x16x32_f16 a[176:179], v[40:43], v[128:131], a[176:179]\n"
  "v_mov_b32 v250, 0x20000\n"
  "v_mfma_f32_16x16x32_f16 a[208:211], v[48:51], v[128:131], a[208:211]\n"
  "ds_read_b32 v250, v250 offset:0\n"
  "v_mfma_f32_16x16x32_f16 a[208:211], v[52:55], v[132:135], a[208:211]\n"
  "ds_read_b128 v[156:159], v243 offset:15360\n"
  "s_waitcnt vmcnt(20)\n"
  "v_cvt_pk_f16_f32 v224, v192, v196\n"
  "v_mfma_f32_16x16x32_f16 a[240:243], v[60:63], v[132:135], a[240:243]\n"
  "v_add_u32 v243, 0x4000, v243\n"
  "v_and_b32 v243, 0x1ffff, v243\n"
  "buffer_load_dwordx4 v[164:167], v242, s[28:31], s39 offen nt\n"
  "v_mfma_f32_16x16x32_f16 a[240:243], v[56:59], v[128:131], a[240:243]\n"
  "v_cvt_pk_f16_f32 v225, v200, v204\n"
  "s_nop 3\n"
  "v_mfma_f32_16x16x32_f16 a[20:23], v[0:3], v[136:139], a[20:23]\n"
  "v_mfma_f32_16x16x32_f16 a[20:23], v[4:7], v[140:143], a[20:23]\n"
  "ds_read_b128 v[128:131], v243 offset:0\n"
  "v_cvt_pk_f16_f32 v226, v208, v212\n"
  "v_mfma_f32_16x16x32_f16 a[52:55], v[12:15], v[140:143], a[52:55]\n"
  "buffer_load_dwordx4 v[168:171], v242, s[28:31], s40 offen nt\n"
  "v_mfma_f32_16x16x32_f16 a[52:55], v[8:11], v[136:139], a[52:55]\n"
  "v_mfma_f32_16x16x32_f16 a[84:87], v[16:19], v[136:139], a[84:87]\n"
  "v_cvt_pk_f16_f32 v227, v216, v220\n"
  "v_mfma_f32_16x16x32_f16 a[84:87], v[20:23], v[140:143], a[84:87]\n"
  "v_mfma_f32_16x16x32_f16 a[116:119], v[28:31], v[140:143], a[116:119]\n"
  "buffer_load_dwordx4 v[172:175], v242, s[28:31], s41 offen nt\n"
  "ds_write_b128 v244, v[224:227] offset:0\n"
  "v_mfma_f32_16x16x32_f16 a[116:119], v[24:27], v[136:139], a[116:119]\n"
  "ds_read_b128 v[132:135], v243 offset:8192\n"
  "s_nop 3\n"
  "v_mfma_f32_16x16x32_f16 a[148:151], v[32:35], v[136:139], a[148:151]\n"
  "v_mfma_f32_16x16x32_f16 a[148:151], v[36:39], v[140:143], a[148:151]\n"
  "v_cvt_pk_f16_f32 v228, v193, v197\n"
  "v_mfma_f32_16x16x32_f16 a[180:183], v[44:47], v[140:143], a[180:183]\n"
  "buffer_load_dwordx4 v[176:179], v242, s[28:31], s42 offen nt\n"
  "v_mfma_f32_16x16x32_f16 a[180:183], v[40:43], v[136:139], a[180:183]\n"
  "v_cvt_pk_f16_f32 v229, v201, v205\n"
  "v_mfma_f32_16x16x32_f16 a[212:215], v[48:51], v[136:139], a[212:215]\n"
  "v_mfma_f32_16x16x32_f16 a[212:215], v[52:55], v[140:143], a[212:215]\n"
  "v_cvt_pk_f16_f32 v230, v209, v213\n"
  "v_mfma_f32_16x16x32_f16 a[244:247], v[60:63], v[140:143], a[244:247]\n"
  "buffer_load_dwordx4 v[180:183], v242, s[28:31], s43 offen nt\n"
  "v_mfma_f32_16x16x32_f16 a[244:247], v[56:59], v[136:139], a[244:247]\n"
  "s_nop 3\n"
  "v_mfma_f32_16x16x32_f16 a[24:27], v[0:3], v[144:147], a[24:27]\n"
  "v_cvt_pk_f16_f32 v231, v217, v221\n"
  "v_mfma_f32_16x16x32_f16 a[24:27], v[4:7], v[148:151], a[24:27]\n"
  "ds_read_b128 v[136:139], v243 offset:1024\n"
  "v_mfma_f32_16x16x32_f16 a[56:59], v[12:15], v[148:151], a[56:59]\n"
  "buffer_load_dwordx4 v[184:187], v242, s[28:31], s44 offen nt\n"
  "ds_write_b128 v244, v[228:231] offset:1024\n"
  "v_mfma_f32_16x16x32_f16 a[56:59], v[8:11], v[144:147], a[56:59]\n"
  "s_waitcnt lgkmcnt(6)\n"
  "v_readfirstlane_b32 s70, v250\n"
  "v_mfma_f32_16x16x32_f16 a[88:91], v[16:19], v[144:147], a[88:91]\n"
  "v_mfma_f32_16x16x32_f16 a[88:91], v[20:23], v[148:151], a[88:91]\n"
  "v_cvt_pk_f16_f32 v224, v194, v198\n"
  "v_mfma_f32_16x16x32_f16 a[120:123], v[28:31], v[148:151], a[120:123]\n"
  "s_memrealtime s[52:53]\n"
  "buffer_load_dwordx4 v[188:191], v242, s[28:31], s45 offen nt\n"
  "v_mfma_f32_16x16x32_f16 a[120:123], v[24:27], v[144:147], a[120:123]\n"
  "ds_read_b128 v[140:143], v243 offset:9216\n"
  "v_cvt_pk_f16_f32 v225, v202, v206\n"
  "s_nop 3\n"
  "v_mfma_f32_16x16x32_f16 a[152:155], v[32:35], v[144:147], a[152:155]\n"
  "v_mfma_f32_16x16x32_f16 a[152:155], v[36:39], v[148:151], a[152:155]\n"
  "v_cvt_pk_f16_f32 v226, v210, v214\n"
  "v_mfma_f32_16x16x32_f16 a[184:187], v[44:47], v[148:151], a[184:187]\n"
  "s_add_u32 s79, s59, 2\ns_lshr_b32 s80, s79, 1\n"
  "v_mfma_f32_16x16x32_f16 a[184:187], v[40:43], v[144:147], a[184:187]\n"
  "s_waitcnt lgkmcnt(0)\n"
  "s_sub_u32 s88, s59, 2\ns_cmp_ge_u32 s88, 6\ns_cbranch_scc1 NPc0_%=\ns_call_b64 s[50:51], PUSH_%=\nNPc0_%=:\ns_sub_u32 s82, s52, s86\ns_cmp_lt_i32 s82, 0\ns_cbranch_scc0 PGc0_%=\nPWc0_%=:\ns_sleep 1\ns_memrealtime s[54:55]\ns_waitcnt lgkmcnt(0)\ns_sub_u32 s88, s54, s86\ns_cmp_lt_i32 s88, 0\ns_cbranch_scc1 PWc0_%=\nPGc0_%=:\n"
  "v_mfma_f32_16x16x32_f16 a[216:219], v[48:51], v[144:147], a[216:219]\n"
  "v_cvt_pk_f16_f32 v227, v218, v222\n"
  "v_readlane_b32 s80, v249, s80\n"
  "v_mfma_f32_16x16x32_f16 a[216:219], v[52:55], v[148:151], a[216:219]\n"
  "s_sub_u32 s82, s82, -131\n"
  "v_mfma_f32_16x16x32_f16 a[248:251], v[60:63], v[148:151], a[248:251]\n"
  "ds_write_b128 v244, v[224:227] offset:2048\n"
  "s_lshl_b32 s80, s80, 17\n"
  "v_mfma_f32_16x16x32_f16 a[248:251], v[56:59], v[144:147], a[248:251]\n"
  "s_nop 3\n"
  "v_mfma_f32_16x16x32_f16 a[28:31], v[0:3], v[152:155], a[28:31]\n"
  "s_cmp_lt_u32 s79, 64\ns_cselect_b32 s46, s80, s84\n"
  "v_mfma_f32_16x16x32_f16 a[28:31], v[4:7], v[156:159], a[28:31]\n"
  "ds_read_b128 v[144:147], v243 offset:2048\n"
  "v_cvt_pk_f16_f32 v228, v195, v199\n"
  "v_mfma_f32_16x16x32_f16 a[60:63], v[12:15], v[156:159], a[60:63]\n"
  "s_max_i32 s82, s82, 151\n"
  "v_mfma_f32_16x16x32_f16 a[60:63], v[8:11], v[152:155], a[60:63]\n"
  "v_cvt_pk_f16_f32 v229, v203, v207\n"
  "v_mfma_f32_16x16x32_f16 a[92:95], v[16:19], v[152:155], a[92:95]\n"
  "s_add_u32 s46, s46, s83\n"
  "v_mfma_f32_16x16x32_f16 a[92:95], v[20:23], v[156:159], a[92:95]\n"
  "v_cvt_pk_f16_f32 v230, v211, v215\n"
  "v_mfma_f32_16x16x32_f16 a[124:127], v[28:31], v[156:159], a[124:127]\n"
  "s_add_u32 s74, s46, 0x4000\n"
  "v_mfma_f32_16x16x32_f16 a[124:127], v[24:27], v[152:155], a[124:127]\n"
  "ds_read_b128 v[148:151], v243 offset:10240\n"
  "s_add_u32 s86, s86, s82\n"
  "s_nop 3\n"
  "v_mfma_f32_16x16x32_f16 a[156:159], v[32:35], v[152:155], a[156:159]\n"
  "v_cvt_pk_f16_f32 v231, v219, v223\n"
  "s_add_u32 s59, s59, 1\n"
  "s_add_u32 s85, s85, 1\n"
  "v_mfma_f32_16x16x32_f16 a[156:159], v[36:39], v[156:159], a[156:159]\n"
  "s_min_u32 s85, s85, 63\n"
  "v_mfma_f32_16x16x32_f16 a[188:191], v[44:47], v[156:159], a[188:191]\n"
  "ds_write_b128 v244, v[228:231] offset:3072\n"
  "v_readlane_b32 s88, v247, s85\n"
  "v_mfma_f32_16x16x32_f16 a[188:191], v[40:43], v[152:155], a[188:191]\n"
  "s_add_u32 s88, s88, s89\n"
  "v_mfma_f32_16x16x32_f16 a[220:223], v[48:51], v[152:155], a[220:223]\n"
  "s_max_u32 s86, s86, s88\n"
  "v_mfma_f32_16x16x32_f16 a[220:223], v[52:55], v[156:159], a[220:223]\n"
  "s_add_u32 s90, s90, 151\n"
  "v_mfma_f32_16x16x32_f16 a[252:255], v[60:63], v[156:159], a[252:255]\n"
  "v_add_u32 v244, 0x4000, v244\n"
  "v_and_b32 v244, 0x1ffff, v244\n"
  "s_max_u32 s90, s90, s88\n"
  "v_mfma_f32_16x16x32_f16 a[252:255], v[56:59], v[152:155], a[252:255]\n"
  "buffer_load_dwordx4 v[0:3], v240, s[20:23], s46 offen offset:0\n"
  "s_waitcnt vmcnt(12)\n"
  "v_mfma_f32_16x16x32_f16 a[0:3], v[64:67], v[128:131], a[0:3]\n"
  "s_nop 3\n"
  "v_mfma_f32_16x16x32_f16 a[0:3], v[68:71], v[132:135], a[0:3]\n"
  "ds_read_b128 v[152:155], v243 offset:3072\n"
  "v_mfma_f32_16x16x32_f16 a[32:35], v[76:79], v[132:135], a[32:35]\n"
  "v_mfma_f32_16x16x32_f16 a[32:35], v[72:75], v[128:131], a[32:35]\n"
  "v_mfma_f32_16x16x32_f16 a[64:67], v[80:83], v[128:131], a[64:67]\n"
  "buffer_load_dwordx4 v[4:7], v241, s[24:27], s74 offen offset:0\n"
  "v_mfma_f32_16x16x32_f16 a[64:67], v[84:87], v[132:135], a[64:67]\n"
  "v_mfma_f32_16x16x32_f16 a[96:99], v[92:95], v[132:135], a[96:99]\n"
  "s_andn2_b32 s79, s70, s72\n"
  "s_cbranch_scc1 PKc1_%=\ns_call_b64 s[50:51], WAITANY_%=\nPKc1_%=:\n"
  "v_mfma_f32_16x16x32_f16 a[96:99], v[88:91], v[128:131], a[96:99]\n"
  "ds_read_b128 v[156:159], v243 offset:11264\n"
  "s_nop 3\n"
  "v_mfma_f32_16x16x32_f16 a[128:131], v[96:99], v[128:131], a[128:131]\n"
  "buffer_load_dwordx4 v[8:11], v241, s[24:27], s46 offen offset:512\n"
  "v_mfma_f32_16x16x32_f16 a[128:131], v[100:103], v[132:135], a[128:131]\n"
  "v_mfma_f32_16x16x32_f16 a[160:163], v[108:111], v[132:135], a[160:163]\n"
  "s_ff1_i32_b32 s79, s79\n"
  "v_mfma_f32_16x16x32_f16 a[160:163], v[104:107], v[128:131], a[160:163]\n"
  "v_mfma_f32_16x16x32_f16 a[192:195], v[112:115], v[128:131], a[192:195]\n"
  "s_bitset1_b32 s72, s79\ns_mov_b32 m0, s73\n"
  "v_mfma_f32_16x16x32_f16 a[192:195], v[116:119], v[132:135], a[192:195]\n"
  "buffer_load_dwordx4 v[12:15], v240, s[20:23], s74 offen offset:512\n"
  "v_mfma_f32_16x16x32_f16 a[224:227], v[124:127], v[132:135], a[224:227]\n"
  "v_writelane_b32 v249, s79, m0\n"
  "v_mfma_f32_16x16x32_f16 a[224:227], v[120:123], v[128:131], a[224:227]\n"
  "s_nop 3\n"
  "v_mfma_f32_16x16x32_f16 a[4:7], v[64:67], v[136:139], a[4:7]\n"
  "s_add_u32 s73, s73, 1\n"
  "v_mfma_f32_16x16x32_f16 a[4:7], v[68:71], v[140:143], a[4:7]\n"
  "ds_read_b128 v[128:131], v243 offset:4096\n"
  "v_mfma_f32_16x16x32_f16 a[36:39], v[76:79], v[140:143], a[36:39]\n"
  "buffer_load_dwordx4 v[16:19], v241, s[24:27], s46 offen offset:1024\n"
  "v_mfma_f32_16x16x32_f16 a[36:39], v[72:75], v[136:139], a[36:39]\n"
  "v_mfma_f32_16x16x32_f16 a[68:71], v[80:83], v[136:139], a[68:71]\n"
  "s_lshr_b32 s79, s58, 1\n"
  "v_mfma_f32_16x16x32_f16 a[68:71], v[84:87], v[140:143], a[68:71]\n"
  "v_mfma_f32_16x16x32_f16 a[100:103], v[92:95], v[140:143], a[100:103]\n"
  "buffer_load_dwordx4 v[20:23], v240, s[20:23], s74 offen offset:1024\n"
  "v_readlane_b32 s79, v249, s79\n"
  "v_mfma_f32_16x16x32_f16 a[100:103], v[88:91], v[136:139], a[100:103]\n"
  "ds_read_b128 v[132:135], v243 offset:12288\n"
  "s_nop 3\n"
  "v_mfma_f32_16x16x32_f16 a[132:135], v[96:99], v[136:139], a[132:135]\n"
  "s_and_b32 s80, s58, 1\ns_lshl_b32 s80, s80, 23\n"
  "v_mfma_f32_16x16x32_f16 a[132:135], v[100:103], v[140:143], a[132:135]\n"
  "v_mfma_f32_16x16x32_f16 a[164:167], v[108:111], v[140:143], a[164:167]\n"
  "s_mov_b32 s81, s79\n"
  "v_mfma_f32_16x16x32_f16 a[164:167], v[104:107], v[136:139], a[164:167]\n"
  "buffer_load_dwordx4 v[24:27], v240, s[20:23], s46 offen offset:1536\n"
  "v_mfma_f32_16x16x32_f16 a[196:199], v[112:115], v[136:139], a[196:199]\n"
  "v_mfma_f32_16x16x32_f16 a[196:199], v[116:119], v[140:143], a[196:199]\n"
  "v_readlane_b32 s79, v248, s81\n"
  "v_mfma_f32_16x16x32_f16 a[228:231], v[124:127], v[140:143], a[228:231]\n"
  "s_cmp_eq_u32 s59, 59\ns_cselect_b32 s66, 0xf800, 0\n"
  "v_mfma_f32_16x16x32_f16 a[228:231], v[120:123], v[136:139], a[228:231]\n"
  "s_add_u32 s79, s79, s80\n"
  "s_nop 3\n"
  "s_waitcnt lgkmcnt(5)\n"
  "v_mfma_f32_16x16x32_f16 a[8:11], v[64:67], v[144:147], a[8:11]\n"
  "buffer_load_dwordx4 v[28:31], v241, s[24:27], s74 offen offset:1536\n"
  "v_mfma_f32_16x16x32_f16 a[8:11], v[68:71], v[148:151], a[8:11]\n"
  "ds_read_b128 v[136:139], v243 offset:5120\n"
  "s_add_u32 s28, s48, s79\ns_addc_u32 s29, s49, 0\n"
  "v_mfma_f32_16x16x32_f16 a[40:43], v[76:79], v[148:151], a[40:43]\n"
  "s_add_u32 s46, s46, s66\ns_add_u32 s74, s74, s66\n"
  "v_mfma_f32_16x16x32_f16 a[40:43], v[72:75], v[144:147], a[40:43]\n"
  "s_cmp_lt_u32 s58, 64\ns_cselect_b32 s30, s69, 0\n"
  "v_mfma_f32_16x16x32_f16 a[72:75], v[80:83], v[144:147], a[72:75]\n"
  "buffer_load_dwordx4 v[32:35], v241, s[24:27], s46 offen offset:2048\n"
  "v_mfma_f32_16x16x32_f16 a[72:75], v[84:87], v[148:151], a[72:75]\n"
  "s_add_u32 s58, s58, 1\n"
  "v_mfma_f32_16x16x32_f16 a[104:107], v[92:95], v[148:151], a[104:107]\n"
  "v_mfma_f32_16x16x32_f16 a[104:107], v[88:91], v[144:147], a[104:107]\n"
  "ds_read_b128 v[140:143], v243 offset:13312\n"
  "s_nop 3\n"
  "v_mfma_f32_16x16x32_f16 a[136:139], v[96:99], v[144:147], a[136:139]\n"
  "v_mfma_f32_16x16x32_f16 a[136:139], v[100:103], v[148:151], a[136:139]\n"
  "buffer_load_dwordx4 v[36:39], v240, s[20:23], s74 offen offset:2048\n"
  "v_mfma_f32_16x16x32_f16 a[168:171], v[108:111], v[148:151], a[168:171]\n"
  "v_mfma_f32_16x16x32_f16 a[168:171], v[104:107], v[144:147], a[168:171]\n"
  "v_mfma_f32_16x16x32_f16 a[200:203], v[112:115], v[144:147], a[200:203]\n"
  "s_add_u32 s79, s59, 3\ns_lshr_b32 s80, s79, 1\n"
  "v_mfma_f32_16x16x32_f16 a[200:203], v[116:119], v[148:151], a[200:203]\n"
  "v_mfma_f32_16x16x32_f16 a[232:235], v[124:127], v[148:151], a[232:235]\n"
  "buffer_load_dwordx4 v[40:43], v240, s[20:23], s46 offen offset:2560\n"
  "v_readlane_b32 s80, v249, s80\n"
  "v_mfma_f32_16x16x32_f16 a[232:235], v[120:123], v[144:147], a[232:235]\n"
  "s_nop 3\n"
  "s_waitcnt lgkmcnt(4)\n"
  "v_mfma_f32_16x16x32_f16 a[12:15], v[64:67], v[152:155], a[12:15]\n"
  "s_lshl_b32 s80, s80, 17\n"
  "v_mfma_f32_16x16x32_f16 a[12:15], v[68:71], v[156:159], a[12:15]\n"
  "ds_read_b128 v[144:147], v243 offset:6144\n"
  "v_mfma_f32_16x16x32_f16 a[44:47], v[76:79], v[156:159], a[44:47]\n"
  "buffer_load_dwordx4 v[44:47], v241, s[24:27], s74 offen offset:2560\n"
  "s_cmp_lt_u32 s79, 64\ns_cselect_b32 s60, s80, s84\n"
  "v_mfma_f32_16x16x32_f16 a[44:47], v[72:75], v[152:155], a[44:47]\n"
  "v_mfma_f32_16x16x32_f16 a[76:79], v[80:83], v[152:155], a[76:79]\n"
  "v_mfma_f32_16x16x32_f16 a[76:79], v[84:87], v[156:159], a[76:79]\n"
  "v_mfma_f32_16x16x32_f16 a[108:111], v[92:95], v[156:159], a[108:111]\n"
  "s_waitcnt vmcnt(21)\n"
  "v_cmp_le_u32 vcc, s61, v246\n"
  "v_mfma_f32_16x16x32_f16 a[108:111], v[88:91], v[152:155], a[108:111]\n"
  "buffer_load_dwordx4 v[48:51], v240, s[20:23], s46 offen offset:3072\n"
  "ds_read_b128 v[148:151], v243 offset:14336\n"
  "s_nop 3\n"
  "v_mfma_f32_16x16x32_f16 a[140:143], v[96:99], v[152:155], a[140:143]\n"
  "s_or_b32 s71, s71, vcc_lo\n"
  "v_mfma_f32_16x16x32_f16 a[140:143], v[100:103], v[156:159], a[140:143]\n"
  "v_mfma_f32_16x16x32_f16 a[172:175], v[108:111], v[156:159], a[172:175]\n"
  "v_mov_b32 v250, s71\n"
  "v_mfma_f32_16x16x32_f16 a[172:175], v[104:107], v[152:155], a[172:175]\n"
  "v_mfma_f32_16x16x32_f16 a[204:207], v[112:115], v[152:155], a[204:207]\n"
  "buffer_load_dwordx4 v[52:55], v241, s[24:27], s74 offen offset:3072\n"
  "ds_write_b32 v251, v250 offset:4\n"
  "v_mfma_f32_16x16x32_f16 a[204:207], v[116:119], v[156:159], a[204:207]\n"
  "v_mfma_f32_16x16x32_f16 a[236:239], v[124:127], v[156:159], a[236:239]\n"
  "v_mfma_f32_16x16x32_f16 a[236:239], v[120:123], v[152:155], a[236:239]\n"
  "ds_read_b128 v[152:155], v243 offset:7168\n"
  "s_nop 3\n"
  "s_waitcnt lgkmcnt(6)\n"
  "v_mfma_f32_16x16x32_f16 a[16:19], v[64:67], v[128:131], a[16:19]\n"
  "buffer_load_dwordx4 v[56:59], v241, s[24:27], s46 offen offset:3584\n"
  "v_mfma_f32_16x16x32_f16 a[16:19], v[68:71], v[132:135], a[16:19]\n"
  "v_mfma_f32_16x16x32_f16 a[48:51], v[76:79], v[132:135], a[48:51]\n"
  "v_mfma_f32_16x16x32_f16 a[48:51], v[72:75], v[128:131], a[48:51]\n"
  "v_mfma_f32_16x16x32_f16 a[80:83], v[80:83], v[128:131], a[80:83]\n"
  "v_mfma_f32_16x16x32_f16 a[80:83], v[84:87], v[132:135], a[80:83]\n"
  "buffer_load_dwordx4 v[60:63], v240, s[20:23], s74 offen offset:3584\n"
  "v_mfma_f32_16x16x32_f16 a[112:115], v[92:95], v[132:135], a[112:115]\n"
  "v_mfma_f32_16x16x32_f16 a[112:115], v[88:91], v[128:131], a[112:115]\n"
  "s_waitcnt vmcnt(23) lgkmcnt(1)\n"
  "s_barrier\n"
  "s_bitcmp1_b32 s75, s59\ns_cbranch_scc0 ENc1_%=\ns_call_b64 s[50:51], EVENT_%=\ns_branch EVc1_%=\nENc1_%=:\nbuffer_load_dword v246, off, s[20:23], s84\nEVc1_%=:\n"
  "s_nop 3\n"
  "v_mfma_f32_16x16x32_f16 a[144:147], v[96:99], v[128:131], a[144:147]\n"
  "buffer_load_dword v246, v245, s[36:39], 0 offen sc0 sc1\n"
  "v_mfma_f32_16x16x32_f16 a[144:147], v[100:103], v[132:135], a[144:147]\n"
  "buffer_load_dword v192, v245, s[20:23], s60 offen\n"
  "v_mfma_f32_16x16x32_f16 a[176:179], v[108:111], v[132:135], a[176:179]\n"
  "buffer_load_dwordx4 v[192:195], v242, s[28:31], 0 offen nt\n"
  "v_mfma_f32_16x16x32_f16 a[176:179], v[104:107], v[128:131], a[176:179]\n"
  "v_mov_b32 v250, 0x20000\n"
  "v_mfma_f32_16x16x32_f16 a[208:211], v[112:115], v[128:131], a[208:211]\n"
  "ds_read_b32 v250, v250 offset:4\n"
  "v_mfma_f32_16x16x32_f16 a[208:211], v[116:119], v[132:135], a[208:211]\n"
  "ds_read_b128 v[156:159], v243 offset:15360\n"
  "s_waitcnt vmcnt(20)\n"
  "v_cvt_pk_f16_f32 v224, v160, v164\n"
  "v_mfma_f32_16x16x32_f16 a[240:243], v[124:127], v[132:135], a[240:243]\n"
  "v_add_u32 v243, 0x4000, v243\n"
  "v_and_b32 v243, 0x1ffff, v243\n"
  "buffer_load_dwordx4 v[196:199], v242, s[28:31], s39 offen nt\n"
  "v_mfma_f32_16x16x32_f16 a[240:243], v[120:123], v[128:131], a[240:243]\n"
  "v_cvt_pk_f16_f32 v225, v168, v172\n"
  "s_nop 3\n"
  "v_mfma_f32_16x16x32_f16 a[20:23], v[64:67], v[136:139], a[20:23]\n"
  "v_mfma_f32_16x16x32_f16 a[20:23], v[68:71], v[140:143], a[20:23]\n"
  "ds_read_b128 v[128:131], v243 offset:0\n"
  "v_cvt_pk_f16_f32 v226, v176, v180\n"
  "v_mfma_f32_16x16x32_f16 a[52:55], v[76:79], v[140:143], a[52:55]\n"
  "buffer_load_dwordx4 v[200:203], v242, s[28:31], s40 offen nt\n"
  "v_mfma_f32_16x16x32_f16 a[52:55], v[72:75], v[136:139], a[52:55]\n"
  "v_mfma_f32_16x16x32_f16 a[84:87], v[80:83], v[136:139], a[84:87]\n"
  "v_cvt_pk_f16_f32 v227, v184, v188\n"
  "v_mfma_f32_16x16x32_f16 a[84:87], v[84:87], v[140:143], a[84:87]\n"
  "v_mfma_f32_16x16x32_f16 a[116:119], v[92:95], v[140:143], a[116:119]\n"
  "buffer_load_dwordx4 v[204:207], v242, s[28:31], s41 offen nt\n"
  "ds_write_b128 v244, v[224:227] offset:0\n"
  "v_mfma_f32_16x16x32_f16 a[116:119], v[88:91], v[136:139], a[116:119]\n"
  "ds_read_b128 v[132:135], v243 offset:8192\n"
  "s_nop 3\n"
  "v_mfma_f32_16x16x32_f16 a[148:151], v[96:99], v[136:139], a[148:151]\n"
  "v_mfma_f32_16x16x32_f16 a[148:151], v[100:103], v[140:143], a[148:151]\n"
  "v_cvt_pk_f16_f32 v228, v161, v165\n"
  "v_mfma_f32_16x16x32_f16 a[180:183], v[108:111], v[140:143], a[180:183]\n"
  "buffer_load_dwordx4 v[208:211], v242, s[28:31], s42 offen nt\n"
  "v_mfma_f32_16x16x32_f16 a[180:183], v[104:107], v[136:139], a[180:183]\n"
  "v_cvt_pk_f16_f32 v229, v169, v173\n"
  "v_mfma_f32_16x16x32_f16 a[212:215], v[112:115], v[136:139], a[212:215]\n"
  "v_mfma_f32_16x16x32_f16 a[212:215], v[116:119], v[140:143], a[212:215]\n"
  "v_cvt_pk_f16_f32 v230, v177, v181\n"
  "v_mfma_f32_16x16x32_f16 a[244:247], v[124:127], v[140:143], a[244:247]\n"
  "buffer_load_dwordx4 v[212:215], v242, s[28:31], s43 offen nt\n"
  "v_mfma_f32_16x16x32_f16 a[244:247], v[120:123], v[136:139], a[244:247]\n"
  "s_nop 3\n"
  "v_mfma_f32_16x16x32_f16 a[24:27], v[64:67], v[144:147], a[24:27]\n"
  "v_cvt_pk_f16_f32 v231, v185, v189\n"
  "v_mfma_f32_16x16x32_f16 a[24:27], v[68:71], v[148:151], a[24:27]\n"
  "ds_read_b128 v[136:139], v243 offset:1024\n"
  "v_mfma_f32_16x16x32_f16 a[56:59], v[76:79], v[148:151], a[56:59]\n"
  "buffer_load_dwordx4 v[216:219], v242, s[28:31], s44 offen nt\n"
  "ds_write_b128 v244, v[228:231] offset:1024\n"
  "v_mfma_f32_16x16x32_f16 a[56:59], v[72:75], v[144:147], a[56:59]\n"
  "s_waitcnt lgkmcnt(6)\n"
  "v_readfirstlane_b32 s70, v250\n"
  "v_mfma_f32_16x16x32_f16 a[88:91], v[80:83], v[144:147], a[88:91]\n"
  "v_mfma_f32_16x16x32_f16 a[88:91], v[84:87], v[148:151], a[88:91]\n"
  "v_cvt_pk_f16_f32 v224, v162, v166\n"
  "v_mfma_f32_16x16x32_f16 a[120:123], v[92:95], v[148:151], a[120:123]\n"
  "s_memrealtime s[52:53]\n"
  "buffer_load_dwordx4 v[220:223], v242, s[28:31], s45 offen nt\n"
  "v_mfma_f32_16x16x32_f16 a[120:123], v[88:91], v[144:147], a[120:123]\n"
  "ds_read_b128 v[140:143], v243 offset:9216\n"
  "v_cvt_pk_f16_f32 v225, v170, v174\n"
  "s_nop 3\n"
  "v_mfma_f32_16x16x32_f16 a[152:155], v[96:99], v[144:147], a[152:155]\n"
  "v_mfma_f32_16x16x32_f16 a[152:155], v[100:103], v[148:151], a[152:155]\n"
  "v_cvt_pk_f16_f32 v226, v178, v182\n"
  "v_mfma_f32_16x16x32_f16 a[184:187], v[108:111], v[148:151], a[184:187]\n"
  "s_add_u32 s79, s59, 2\ns_lshr_b32 s80, s79, 1\n"
  "v_mfma_f32_16x16x32_f16 a[184:187], v[104:107], v[144:147], a[184:187]\n"
  "s_waitcnt lgkmcnt(0)\n"
  "s_sub_u32 s88, s59, 2\ns_cmp_ge_u32 s88, 6\ns_cbranch_scc1 NPc1_%=\ns_call_b64 s[50:51], PUSH_%=\nNPc1_%=:\ns_sub_u32 s82, s52, s86\ns_cmp_lt_i32 s82, 0\ns_cbranch_scc0 PGc1_%=\nPWc1_%=:\ns_sleep 1\ns_memrealtime s[54:55]\ns_waitcnt lgkmcnt(0)\ns_sub_u32 s88, s54, s86\ns_cmp_lt_i32 s88, 0\ns_cbranch_scc1 PWc1_%=\nPGc1_%=:\n"
  "v_mfma_f32_16x16x32_f16 a[216:219], v[112:115], v[144:147], a[216:219]\n"
  "v_cvt_pk_f16_f32 v227, v186, v190\n"
  "v_readlane_b32 s80, v249, s80\n"
  "v_mfma_f32_16x16x32_f16 a[216:219], v[116:119], v[148:151], a[216:219]\n"
  "s_sub_u32 s82, s82, -131\n"
  "v_mfma_f32_16x16x32_f16 a[248:251], v[124:127], v[148:151], a[248:251]\n"
  "ds_write_b128 v244, v[224:227] offset:2048\n"
  "s_lshl_b32 s80, s80, 17\ns_add_u32 s80, s80, 0x10000\n"
  "v_mfma_f32_16x16x32_f16 a[248:251], v[120:123], v[144:147], a[248:251]\n"
  "s_nop 3\n"
  "v_mfma_f32_16x16x32_f16 a[28:31], v[64:67], v[152:155], a[28:31]\n"
  "s_cmp_lt_u32 s79, 64\ns_cselect_b32 s46, s80, s84\n"
  "v_mfma_f32_16x16x32_f16 a[28:31], v[68:71], v[156:159], a[28:31]\n"
  "ds_read_b128 v[144:147], v243 offset:2048\n"
  "v_cvt_pk_f16_f32 v228, v163, v167\n"
  "v_mfma_f32_16x16x32_f16 a[60:63], v[76:79], v[156:159], a[60:63]\n"
  "s_max_i32 s82, s82, 151\n"
  "v_mfma_f32_16x16x32_f16 a[60:63], v[72:75], v[152:155], a[60:63]\n"
  "v_cvt_pk_f16_f32 v229, v171, v175\n"
  "v_mfma_f32_16x16x32_f16 a[92:95], v[80:83], v[152:155], a[92:95]\n"
  "s_add_u32 s46, s46, s83\n"
  "v_mfma_f32_16x16x32_f16 a[92:95], v[84:87], v[156:159], a[92:95]\n"
  "v_cvt_pk_f16_f32 v230, v179, v183\n"
  "v_mfma_f32_16x16x32_f16 a[124:127], v[92:95], v[156:159], a[124:127]\n"
  "s_add_u32 s74, s46, 0x4000\n"
  "v_mfma_f32_16x16x32_f16 a[124:127], v[88:91], v[152:155], a[124:127]\n"
  "ds_read_b128 v[148:151], v243 offset:10240\n"
  "s_add_u32 s86, s86, s82\n"
  "s_nop 3\n"
  "v_mfma_f32_16x16x32_f16 a[156:159], v[96:99], v[152:155], a[156:159]\n"
  "v_cvt_pk_f16_f32 v231, v187, v191\n"
  "s_add_u32 s59, s59, 1\n"
  "s_add_u32 s85, s85, 1\n"
  "v_mfma_f32_16x16x32_f16 a[156:159], v[100:103], v[156:159], a[156:159]\n"
  "s_min_u32 s85, s85, 63\n"
  "v_mfma_f32_16x16x32_f16 a[188:191], v[108:111], v[156:159], a[188:191]\n"
  "ds_write_b128 v244, v[228:231] offset:3072\n"
  "v_readlane_b32 s88, v247, s85\n"
  "v_mfma_f32_16x16x32_f16 a[188:191], v[104:107], v[152:155], a[188:191]\n"
  "s_add_u32 s88, s88, s89\n"
  "v_mfma_f32_16x16x32_f16 a[220:223], v[112:115], v[152:155], a[220:223]\n"
  "s_max_u32 s86, s86, s88\n"
  "v_mfma_f32_16x16x32_f16 a[220:223], v[116:119], v[156:159], a[220:223]\n"
  "s_add_u32 s90, s90, 151\n"
  "v_mfma_f32_16x16x32_f16 a[252:255], v[124:127], v[156:159], a[252:255]\n"
  "v_add_u32 v244, 0x4000, v244\n"
  "v_and_b32 v244, 0x1ffff, v244\n"
  "s_max_u32 s90, s90, s88\n"
  "v_mfma_f32_16x16x32_f16 a[252:255], v[120:123], v[152:155], a[252:255]\n"
  "s_cmp_ge_u32 s59, 60\n"
  "s_cbranch_scc1 DONE_%=\n"
  "s_cmp_lt_u32 s59, 8\n"
  "s_cbranch_scc1 CLOOP_%=\n"
  "s_mov_b32 s75, 0\n"
  "s_cmp_lg_u32 s70, -1\n"
  "s_cbranch_scc1 CLOOP_%=\n"
  "s_sub_u32 s85, s85, 1\n"
  "s_mov_b32 s82, 151\n"
  "s_sub_u32 s90, s90, 151\n"
  "s_mov_b32 s87, 0\n"
  "s_mov_b32 s77, 0\n"
  "s_branch SLOOP_%=\n"
  "LOOP_%=:\n"
  "buffer_load_dwordx4 v[64:67], v240, s[20:23], s46 offen offset:0\n"
  "s_waitcnt vmcnt(10)\n"
  "v_mfma_f32_16x16x32_f16 a[0:3], v[0:3], v[128:131], a[0:3]\n"
  "s_add_u32 s85, s85, 1\n"
  "v_mfma_f32_16x16x32_f16 a[0:3], v[4:7], v[132:135], a[0:3]\n"
  "ds_read_b128 v[152:155], v243 offset:3072\n"
  "v_mfma_f32_16x16x32_f16 a[32:35], v[12:15], v[132:135], a[32:35]\n"
  "s_min_u32 s85, s85, 63\n"
  "v_mfma_f32_16x16x32_f16 a[32:35], v[8:11], v[128:131], a[32:35]\n"
  "v_readlane_b32 s88, v247, s85\n"
  "v_mfma_f32_16x16x32_f16 a[64:67], v[16:19], v[128:131], a[64:67]\n"
  "buffer_load_dwordx4 v[68:71], v241, s[24:27], s74 offen offset:0\n"
  "v_mfma_f32_16x16x32_f16 a[64:67], v[20:23], v[132:135], a[64:67]\n"
  "s_add_u32 s88, s88, s89\n"
  "v_mfma_f32_16x16x32_f16 a[96:99], v[28:31], v[132:135], a[96:99]\n"
  "s_max_u32 s86, s86, s88\n"
  "v_mfma_f32_16x16x32_f16 a[96:99], v[24:27], v[128:131], a[96:99]\n"
  "ds_read_b128 v[156:159], v243 offset:11264\n"
  "v_mfma_f32_16x16x32_f16 a[128:131], v[32:35], v[128:131], a[128:131]\n"
  "buffer_load_dwordx4 v[72:75], v241, s[24:27], s46 offen offset:512\n"
  "v_mfma_f32_16x16x32_f16 a[128:131], v[36:39], v[132:135], a[128:131]\n"
  "s_add_u32 s90, s90, 151\n"
  "v_mfma_f32_16x16x32_f16 a[160:163], v[44:47], v[132:135], a[160:163]\n"
  "s_max_u32 s90, s90, s88\n"
  "v_mfma_f32_16x16x32_f16 a[160:163], v[40:43], v[128:131], a[160:163]\n"
  "s_sub_u32 s65, s86, s90\n"
  "v_mfma_f32_16x16x32_f16 a[192:195], v[48:51], v[128:131], a[192:195]\n"
  "s_cmp_gt_i32 s65, 150\ns_cselect_b32 s65, 5, 0\ns_cselect_b32 s71, 0, 3\n"
  "v_mfma_f32_16x16x32_f16 a[192:195], v[52:55], v[132:135], a[192:195]\n"
  "buffer_load_dwordx4 v[76:79], v240, s[20:23], s74 offen offset:512\n"
  "v_mfma_f32_16x16x32_f16 a[224:227], v[60:63], v[132:135], a[224:227]\n"
  "s_sub_u32 s86, s86, s65\n"
  "v_mfma_f32_16x16x32_f16 a[224:227], v[56:59], v[128:131], a[224:227]\n"
  "v_mfma_f32_16x16x32_f16 a[4:7], v[0:3], v[136:139], a[4:7]\n"
  "s_add_i32 s87, s87, s77\n"
  "v_mfma_f32_16x16x32_f16 a[4:7], v[4:7], v[140:143], a[4:7]\n"
  "ds_read_b128 v[128:131], v243 offset:4096\n"
  "v_mfma_f32_16x16x32_f16 a[36:39], v[12:15], v[140:143], a[36:39]\n"
  "buffer_load_dwordx4 v[80:83], v241, s[24:27], s46 offen offset:1024\n"
  "v_mfma_f32_16x16x32_f16 a[36:39], v[8:11], v[136:139], a[36:39]\n"
  "s_max_i32 s87, s87, 0\n"
  "v_mfma_f32_16x16x32_f16 a[68:71], v[16:19], v[136:139], a[68:71]\n"
  "s_lshr_b32 s79, s58, 1\n"
  "v_mfma_f32_16x16x32_f16 a[68:71], v[20:23], v[140:143], a[68:71]\n"
  "s_min_i32 s87, s87, 6\n"
  "v_mfma_f32_16x16x32_f16 a[100:103], v[28:31], v[140:143], a[100:103]\n"
  "buffer_load_dwordx4 v[84:87], v240, s[20:23], s74 offen offset:1024\n"
  "v_readlane_b32 s79, v249, s79\n"
  "v_mfma_f32_16x16x32_f16 a[100:103], v[24:27], v[136:139], a[100:103]\n"
  "ds_read_b128 v[132:135], v243 offset:12288\n"
  "v_mfma_f32_16x16x32_f16 a[132:135], v[32:35], v[136:139], a[132:135]\n"
  "s_and_b32 s80, s58, 1\ns_lshl_b32 s80, s80, 23\n"
  "v_mfma_f32_16x16x32_f16 a[132:135], v[36:39], v[140:143], a[132:135]\n"
  "v_mfma_f32_16x16x32_f16 a[164:167], v[44:47], v[140:143], a[164:167]\n"
  "s_mov_b32 s81, s79\n"
  "v_mfma_f32_16x16x32_f16 a[164:167], v[40:43], v[136:139], a[164:167]\n"
  "buffer_load_dwordx4 v[88:91], v240, s[20:23], s46 offen offset:1536\n"
  "v_mfma_f32_16x16x32_f16 a[196:199], v[48:51], v[136:139], a[196:199]\n"
  "v_mfma_f32_16x16x32_f16 a[196:199], v[52:55], v[140:143], a[196:199]\n"
  "v_readlane_b32 s79, v248, s81\n"
  "v_mfma_f32_16x16x32_f16 a[228:231], v[60:63], v[140:143], a[228:231]\n"
  "v_mfma_f32_16x16x32_f16 a[228:231], v[56:59], v[136:139], a[228:231]\n"
  "s_add_u32 s79, s79, s80\n"
  "s_waitcnt lgkmcnt(5)\n"
  "v_mfma_f32_16x16x32_f16 a[8:11], v[0:3], v[144:147], a[8:11]\n"
  "buffer_load_dwordx4 v[92:95], v241, s[24:27], s74 offen offset:1536\n"
  "v_mfma_f32_16x16x32_f16 a[8:11], v[4:7], v[148:151], a[8:11]\n"
  "ds_read_b128 v[136:139], v243 offset:5120\n"
  "s_add_u32 s28, s48, s79\ns_addc_u32 s29, s49, 0\n"
  "v_mfma_f32_16x16x32_f16 a[40:43], v[12:15], v[148:151], a[40:43]\n"
  "v_mfma_f32_16x16x32_f16 a[40:43], v[8:11], v[144:147], a[40:43]\n"
  "s_cmp_lt_u32 s58, 64\ns_cselect_b32 s30, s69, 0\n"
  "v_mfma_f32_16x16x32_f16 a[72:75], v[16:19], v[144:147], a[72:75]\n"
  "buffer_load_dwordx4 v[96:99], v241, s[24:27], s46 offen offset:2048\n"
  "v_mfma_f32_16x16x32_f16 a[72:75], v[20:23], v[148:151], a[72:75]\n"
  "s_add_u32 s58, s58, 1\n"
  "v_mfma_f32_16x16x32_f16 a[104:107], v[28:31], v[148:151], a[104:107]\n"
  "v_mfma_f32_16x16x32_f16 a[104:107], v[24:27], v[144:147], a[104:107]\n"
  "ds_read_b128 v[140:143], v243 offset:13312\n"
  "v_mfma_f32_16x16x32_f16 a[136:139], v[32:35], v[144:147], a[136:139]\n"
  "v_mfma_f32_16x16x32_f16 a[136:139], v[36:39], v[148:151], a[136:139]\n"
  "buffer_load_dwordx4 v[100:103], v240, s[20:23], s74 offen offset:2048\n"
  "v_mfma_f32_16x16x32_f16 a[168:171], v[44:47], v[148:151], a[168:171]\n"
  "v_mfma_f32_16x16x32_f16 a[168:171], v[40:43], v[144:147], a[168:171]\n"
  "v_mfma_f32_16x16x32_f16 a[200:203], v[48:51], v[144:147], a[200:203]\n"
  "s_add_u32 s79, s59, 3\ns_lshr_b32 s80, s79, 1\n"
  "v_mfma_f32_16x16x32_f16 a[200:203], v[52:55], v[148:151], a[200:203]\n"
  "v_mfma_f32_16x16x32_f16 a[232:235], v[60:63], v[148:151], a[232:235]\n"
  "buffer_load_dwordx4 v[104:107], v240, s[20:23], s46 offen offset:2560\n"
  "v_readlane_b32 s80, v249, s80\n"
  "v_mfma_f32_16x16x32_f16 a[232:235], v[56:59], v[144:147], a[232:235]\n"
  "s_waitcnt lgkmcnt(4)\n"
  "v_mfma_f32_16x16x32_f16 a[12:15], v[0:3], v[152:155], a[12:15]\n"
  "s_lshl_b32 s80, s80, 17\ns_add_u32 s80, s80, 0x10000\n"
  "v_mfma_f32_16x16x32_f16 a[12:15], v[4:7], v[156:159], a[12:15]\n"
  "ds_read_b128 v[144:147], v243 offset:6144\n"
  "v_mfma_f32_16x16x32_f16 a[44:47], v[12:15], v[156:159], a[44:47]\n"
  "buffer_load_dwordx4 v[108:111], v241, s[24:27], s74 offen offset:2560\n"
  "s_cmp_lt_u32 s79, 64\ns_cselect_b32 s60, s80, s84\n"
  "v_mfma_f32_16x16x32_f16 a[44:47], v[8:11], v[152:155], a[44:47]\n"
  "v_mfma_f32_16x16x32_f16 a[76:79], v[16:19], v[152:155], a[76:79]\n"
  "v_mfma_f32_16x16x32_f16 a[76:79], v[20:23], v[156:159], a[76:79]\n"
  "v_mfma_f32_16x16x32_f16 a[108:111], v[28:31], v[156:159], a[108:111]\n"
  "v_mfma_f32_16x16x32_f16 a[108:111], v[24:27], v[152:155], a[108:111]\n"
  "buffer_load_dwordx4 v[112:115], v240, s[20:23], s46 offen offset:3072\n"
  "ds_read_b128 v[148:151], v243 offset:14336\n"
  "v_mfma_f32_16x16x32_f16 a[140:143], v[32:35], v[152:155], a[140:143]\n"
  "v_mfma_f32_16x16x32_f16 a[140:143], v[36:39], v[156:159], a[140:143]\n"
  "v_mfma_f32_16x16x32_f16 a[172:175], v[44:47], v[156:159], a[172:175]\n"
  "v_mfma_f32_16x16x32_f16 a[172:175], v[40:43], v[152:155], a[172:175]\n"
  "v_mfma_f32_16x16x32_f16 a[204:207], v[48:51], v[152:155], a[204:207]\n"
  "buffer_load_dwordx4 v[116:119], v241, s[24:27], s74 offen offset:3072\n"
  "v_mfma_f32_16x16x32_f16 a[204:207], v[52:55], v[156:159], a[204:207]\n"
  "v_mfma_f32_16x16x32_f16 a[236:239], v[60:63], v[156:159], a[236:239]\n"
  "v_mfma_f32_16x16x32_f16 a[236:239], v[56:59], v[152:155], a[236:239]\n"
  "ds_read_b128 v[152:155], v243 offset:7168\n"
  "s_waitcnt lgkmcnt(5)\n"
  "v_mfma_f32_16x16x32_f16 a[16:19], v[0:3], v[128:131], a[16:19]\n"
  "buffer_load_dwordx4 v[120:123], v241, s[24:27], s46 offen offset:3584\n"
  "v_mfma_f32_16x16x32_f16 a[16:19], v[4:7], v[132:135], a[16:19]\n"
  "v_mfma_f32_16x16x32_f16 a[48:51], v[12:15], v[132:135], a[48:51]\n"
  "v_mfma_f32_16x16x32_f16 a[48:51], v[8:11], v[128:131], a[48:51]\n"
  "v_mfma_f32_16x16x32_f16 a[80:83], v[16:19], v[128:131], a[80:83]\n"
  "v_mfma_f32_16x16x32_f16 a[80:83], v[20:23], v[132:135], a[80:83]\n"
  "buffer_load_dwordx4 v[124:127], v240, s[20:23], s74 offen offset:3584\n"
  "v_mfma_f32_16x16x32_f16 a[112:115], v[28:31], v[132:135], a[112:115]\n"
  "v_mfma_f32_16x16x32_f16 a[112:115], v[24:27], v[128:131], a[112:115]\n"
  "s_barrier\n"
  "buffer_load_dword v160, v245, s[20:23], s60 offen\n"
  "v_mfma_f32_16x16x32_f16 a[144:147], v[32:35], v[128:131], a[144:147]\n"
  "buffer_load_dwordx4 v[160:163], v242, s[28:31], 0 offen nt\n"
  "v_mfma_f32_16x16x32_f16 a[144:147], v[36:39], v[132:135], a[144:147]\n"
  "v_mfma_f32_16x16x32_f16 a[176:179], v[44:47], v[132:135], a[176:179]\n"
  "v_mfma_f32_16x16x32_f16 a[176:179], v[40:43], v[128:131], a[176:179]\n"
  "v_mfma_f32_16x16x32_f16 a[208:211], v[48:51], v[128:131], a[208:211]\n"
  "buffer_load_dwordx4 v[164:167], v242, s[28:31], s39 offen nt\n"
  "v_mfma_f32_16x16x32_f16 a[208:211], v[52:55], v[132:135], a[208:211]\n"
  "ds_read_b128 v[156:159], v243 offset:15360\n"
  "s_waitcnt vmcnt(19)\n"
  "v_cvt_pk_f16_f32 v224, v192, v196\n"
  "v_mfma_f32_16x16x32_f16 a[240:243], v[60:63], v[132:135], a[240:243]\n"
  "v_add_u32 v243, 0x4000, v243\n"
  "v_and_b32 v243, 0x1ffff, v243\n"
  "v_mfma_f32_16x16x32_f16 a[240:243], v[56:59], v[128:131], a[240:243]\n"
  "v_cvt_pk_f16_f32 v225, v200, v204\n"
  "s_waitcnt lgkmcnt(4)\n"
  "v_mfma_f32_16x16x32_f16 a[20:23], v[0:3], v[136:139], a[20:23]\n"
  "buffer_load_dwordx4 v[168:171], v242, s[28:31], s40 offen nt\n"
  "v_mfma_f32_16x16x32_f16 a[20:23], v[4:7], v[140:143], a[20:23]\n"
  "ds_read_b128 v[128:131], v243 offset:0\n"
  "v_cvt_pk_f16_f32 v226, v208, v212\n"
  "v_mfma_f32_16x16x32_f16 a[52:55], v[12:15], v[140:143], a[52:55]\n"
  "v_mfma_f32_16x16x32_f16 a[52:55], v[8:11], v[136:139], a[52:55]\n"
  "v_mfma_f32_16x16x32_f16 a[84:87], v[16:19], v[136:139], a[84:87]\n"
  "buffer_load_dwordx4 v[172:175], v242, s[28:31], s41 offen nt\n"
  "v_cvt_pk_f16_f32 v227, v216, v220\n"
  "v_mfma_f32_16x16x32_f16 a[84:87], v[20:23], v[140:143], a[84:87]\n"
  "v_mfma_f32_16x16x32_f16 a[116:119], v[28:31], v[140:143], a[116:119]\n"
  "ds_write_b128 v244, v[224:227] offset:0\n"
  "v_mfma_f32_16x16x32_f16 a[116:119], v[24:27], v[136:139], a[116:119]\n"
  "ds_read_b128 v[132:135], v243 offset:8192\n"
  "v_mfma_f32_16x16x32_f16 a[148:151], v[32:35], v[136:139], a[148:151]\n"
  "buffer_load_dwordx4 v[176:179], v242, s[28:31], s42 offen nt\n"
  "v_mfma_f32_16x16x32_f16 a[148:151], v[36:39], v[140:143], a[148:151]\n"
  "v_cvt_pk_f16_f32 v228, v193, v197\n"
  "v_mfma_f32_16x16x32_f16 a[180:183], v[44:47], v[140:143], a[180:183]\n"
  "v_mfma_f32_16x16x32_f16 a[180:183], v[40:43], v[136:139], a[180:183]\n"
  "v_cvt_pk_f16_f32 v229, v201, v205\n"
  "v_mfma_f32_16x16x32_f16 a[212:215], v[48:51], v[136:139], a[212:215]\n"
  "buffer_load_dwordx4 v[180:183], v242, s[28:31], s43 offen nt\n"
  "v_mfma_f32_16x16x32_f16 a[212:215], v[52:55], v[140:143], a[212:215]\n"
  "v_cvt_pk_f16_f32 v230, v209, v213\n"
  "v_mfma_f32_16x16x32_f16 a[244:247], v[60:63], v[140:143], a[244:247]\n"
  "v_mfma_f32_16x16x32_f16 a[244:247], v[56:59], v[136:139], a[244:247]\n"
  "s_waitcnt lgkmcnt(5)\n"
  "v_mfma_f32_16x16x32_f16 a[24:27], v[0:3], v[144:147], a[24:27]\n"
  "buffer_load_dwordx4 v[184:187], v242, s[28:31], s44 offen nt\n"
  "v_cvt_pk_f16_f32 v231, v217, v221\n"
  "v_mfma_f32_16x16x32_f16 a[24:27], v[4:7], v[148:151], a[24:27]\n"
  "ds_read_b128 v[136:139], v243 offset:1024\n"
  "v_mfma_f32_16x16x32_f16 a[56:59], v[12:15], v[148:151], a[56:59]\n"
  "ds_write_b128 v244, v[228:231] offset:1024\n"
  "v_mfma_f32_16x16x32_f16 a[56:59], v[8:11], v[144:147], a[56:59]\n"
  "v_mfma_f32_16x16x32_f16 a[88:91], v[16:19], v[144:147], a[88:91]\n"
  "buffer_load_dwordx4 v[188:191], v242, s[28:31], s45 offen nt\n"
  "v_mfma_f32_16x16x32_f16 a[88:91], v[20:23], v[148:151], a[88:91]\n"
  "v_cvt_pk_f16_f32 v224, v194, v198\n"
  "v_mfma_f32_16x16x32_f16 a[120:123], v[28:31], v[148:151], a[120:123]\n"
  "s_memrealtime s[52:53]\n"
  "v_mfma_f32_16x16x32_f16 a[120:123], v[24:27], v[144:147], a[120:123]\n"
  "ds_read_b128 v[140:143], v243 offset:9216\n"
  "v_cvt_pk_f16_f32 v225, v202, v206\n"
  "v_mfma_f32_16x16x32_f16 a[152:155], v[32:35], v[144:147], a[152:155]\n"
  "v_mfma_f32_16x16x32_f16 a[152:155], v[36:39], v[148:151], a[152:155]\n"
  "v_cvt_pk_f16_f32 v226, v210, v214\n"
  "v_mfma_f32_16x16x32_f16 a[184:187], v[44:47], v[148:151], a[184:187]\n"
  "s_add_u32 s79, s59, 2\ns_lshr_b32 s80, s79, 1\n"
  "v_mfma_f32_16x16x32_f16 a[184:187], v[40:43], v[144:147], a[184:187]\n"
  "s_waitcnt lgkmcnt(0)\n"
  "s_sub_u32 s82, s52, s86\ns_cmp_lt_i32 s82, 0\ns_cbranch_scc0 PGl0_%=\nPWl0_%=:\ns_sleep 1\ns_memrealtime s[54:55]\ns_waitcnt lgkmcnt(0)\ns_sub_u32 s88, s54, s86\ns_cmp_lt_i32 s88, 0\ns_cbranch_scc1 PWl0_%=\nPGl0_%=:\n"
  "v_mfma_f32_16x16x32_f16 a[216:219], v[48:51], v[144:147], a[216:219]\n"
  "s_cmp_lt_u32 s82, 44\ns_cselect_b32 s77, 1, -1\n"
  "s_cmp_gt_i32 s82, 125\ns_cselect_b32 s66, 2, 0\ns_max_u32 s75, s75, s66\n"
  "v_cvt_pk_f16_f32 v227, v218, v222\n"
  "v_readlane_b32 s80, v249, s80\n"
  "v_mfma_f32_16x16x32_f16 a[216:219], v[52:55], v[148:151], a[216:219]\n"
  "s_sub_u32 s82, s82, -131\n"
  "v_mfma_f32_16x16x32_f16 a[248:251], v[60:63], v[148:151], a[248:251]\n"
  "ds_write_b128 v244, v[224:227] offset:2048\n"
  "s_lshl_b32 s80, s80, 17\n"
  "v_mfma_f32_16x16x32_f16 a[248:251], v[56:59], v[144:147], a[248:251]\n"
  "v_mfma_f32_16x16x32_f16 a[28:31], v[0:3], v[152:155], a[28:31]\n"
  "s_cmp_lt_u32 s79, 64\ns_cselect_b32 s46, s80, s84\n"
  "v_mfma_f32_16x16x32_f16 a[28:31], v[4:7], v[156:159], a[28:31]\n"
  "ds_read_b128 v[144:147], v243 offset:2048\n"
  "v_cvt_pk_f16_f32 v228, v195, v199\n"
  "v_mfma_f32_16x16x32_f16 a[60:63], v[12:15], v[156:159], a[60:63]\n"
  "s_max_i32 s82, s82, 151\n"
  "v_mfma_f32_16x16x32_f16 a[60:63], v[8:11], v[152:155], a[60:63]\n"
  "v_cvt_pk_f16_f32 v229, v203, v207\n"
  "v_mfma_f32_16x16x32_f16 a[92:95], v[16:19], v[152:155], a[92:95]\n"
  "s_add_u32 s46, s46, s83\n"
  "v_mfma_f32_16x16x32_f16 a[92:95], v[20:23], v[156:159], a[92:95]\n"
  "v_cvt_pk_f16_f32 v230, v211, v215\n"
  "v_mfma_f32_16x16x32_f16 a[124:127], v[28:31], v[156:159], a[124:127]\n"
  "s_add_u32 s74, s46, 0x4000\n"
  "v_mfma_f32_16x16x32_f16 a[124:127], v[24:27], v[152:155], a[124:127]\n"
  "ds_read_b128 v[148:151], v243 offset:10240\n"
  "v_mfma_f32_16x16x32_f16 a[156:159], v[32:35], v[152:155], a[156:159]\n"
  "v_cvt_pk_f16_f32 v231, v219, v223\n"
  "s_add_u32 s59, s59, 1\n"
  "v_mfma_f32_16x16x32_f16 a[156:159], v[36:39], v[156:159], a[156:159]\n"
  "s_add_u32 s86, s86, s82\n"
  "v_mfma_f32_16x16x32_f16 a[188:191], v[44:47], v[156:159], a[188:191]\n"
  "ds_write_b128 v244, v[228:231] offset:3072\n"
  "v_mfma_f32_16x16x32_f16 a[188:191], v[40:43], v[152:155], a[188:191]\n"
  "v_mfma_f32_16x16x32_f16 a[220:223], v[48:51], v[152:155], a[220:223]\n"
  "v_mfma_f32_16x16x32_f16 a[220:223], v[52:55], v[156:159], a[220:223]\n"
  "v_mfma_f32_16x16x32_f16 a[252:255], v[60:63], v[156:159], a[252:255]\n"
  "v_add_u32 v244, 0x4000, v244\n"
  "v_and_b32 v244, 0x1ffff, v244\n"
  "v_mfma_f32_16x16x32_f16 a[252:255], v[56:59], v[152:155], a[252:255]\n"
  "buffer_load_dwordx4 v[0:3], v240, s[20:23], s46 offen offset:0\n"
  "s_waitcnt vmcnt(10)\n"
  "v_mfma_f32_16x16x32_f16 a[0:3], v[64:67], v[128:131], a[0:3]\n"
  "s_add_u32 s85, s85, 1\n"
  "v_mfma_f32_16x16x32_f16 a[0:3], v[68:71], v[132:135], a[0:3]\n"
  "ds_read_b128 v[152:155], v243 offset:3072\n"
  "v_mfma_f32_16x16x32_f16 a[32:35], v[76:79], v[132:135], a[32:35]\n"
  "s_min_u32 s85, s85, 63\n"
  "v_mfma_f32_16x16x32_f16 a[32:35], v[72:75], v[128:131], a[32:35]\n"
  "v_readlane_b32 s88, v247, s85\n"
  "v_mfma_f32_16x16x32_f16 a[64:67], v[80:83], v[128:131], a[64:67]\n"
  "buffer_load_dwordx4 v[4:7], v241, s[24:27], s74 offen offset:0\n"
  "v_mfma_f32_16x16x32_f16 a[64:67], v[84:87], v[132:135], a[64:67]\n"
  "s_add_u32 s88, s88, s89\n"
  "v_mfma_f32_16x16x32_f16 a[96:99], v[92:95], v[132:135], a[96:99]\n"
  "s_andn2_b32 s79, s70, s72\n"
  "v_mfma_f32_16x16x32_f16 a[96:99], v[88:91], v[128:131], a[96:99]\n"
  "ds_read_b128 v[156:159], v243 offset:11264\n"
  "v_mfma_f32_16x16x32_f16 a[128:131], v[96:99], v[128:131], a[128:131]\n"
  "buffer_load_dwordx4 v[8:11], v241, s[24:27], s46 offen offset:512\n"
  "v_mfma_f32_16x16x32_f16 a[128:131], v[100:103], v[132:135], a[128:131]\n"
  "s_max_u32 s86, s86, s88\n"
  "v_mfma_f32_16x16x32_f16 a[160:163], v[108:111], v[132:135], a[160:163]\n"
  "s_ff1_i32_b32 s79, s79\n"
  "v_mfma_f32_16x16x32_f16 a[160:163], v[104:107], v[128:131], a[160:163]\n"
  "s_add_u32 s90, s90, 151\n"
  "v_mfma_f32_16x16x32_f16 a[192:195], v[112:115], v[128:131], a[192:195]\n"
  "s_bitset1_b32 s72, s79\ns_mov_b32 m0, s73\n"
  "v_mfma_f32_16x16x32_f16 a[192:195], v[116:119], v[132:135], a[192:195]\n"
  "buffer_load_dwordx4 v[12:15], v240, s[20:23], s74 offen offset:512\n"
  "v_mfma_f32_16x16x32_f16 a[224:227], v[124:127], v[132:135], a[224:227]\n"
  "v_writelane_b32 v249, s79, m0\n"
  "v_mfma_f32_16x16x32_f16 a[224:227], v[120:123], v[128:131], a[224:227]\n"
  "v_mfma_f32_16x16x32_f16 a[4:7], v[64:67], v[136:139], a[4:7]\n"
  "s_add_u32 s73, s73, 1\n"
  "v_mfma_f32_16x16x32_f16 a[4:7], v[68:71], v[140:143], a[4:7]\n"
  "ds_read_b128 v[128:131], v243 offset:4096\n"
  "v_mfma_f32_16x16x32_f16 a[36:39], v[76:79], v[140:143], a[36:39]\n"
  "buffer_load_dwordx4 v[16:19], v241, s[24:27], s46 offen offset:1024\n"
  "v_mfma_f32_16x16x32_f16 a[36:39], v[72:75], v[136:139], a[36:39]\n"
  "s_max_u32 s90, s90, s88\n"
  "v_mfma_f32_16x16x32_f16 a[68:71], v[80:83], v[136:139], a[68:71]\n"
  "s_lshr_b32 s79, s58, 1\n"
  "v_mfma_f32_16x16x32_f16 a[68:71], v[84:87], v[140:143], a[68:71]\n"
  "s_sub_u32 s65, s86, s90\n"
  "v_mfma_f32_16x16x32_f16 a[100:103], v[92:95], v[140:143], a[100:103]\n"
  "buffer_load_dwordx4 v[20:23], v240, s[20:23], s74 offen offset:1024\n"
  "v_readlane_b32 s79, v249, s79\n"
  "v_mfma_f32_16x16x32_f16 a[100:103], v[88:91], v[136:139], a[100:103]\n"
  "ds_read_b128 v[132:135], v243 offset:12288\n"
  "v_mfma_f32_16x16x32_f16 a[132:135], v[96:99], v[136:139], a[132:135]\n"
  "s_and_b32 s80, s58, 1\ns_lshl_b32 s80, s80, 23\n"
  "v_mfma_f32_16x16x32_f16 a[132:135], v[100:103], v[140:143], a[132:135]\n"
  "s_cmp_gt_i32 s65, 150\ns_cselect_b32 s65, 5, 0\ns_cselect_b32 s71, 0, 3\n"
  "v_mfma_f32_16x16x32_f16 a[164:167], v[108:111], v[140:143], a[164:167]\n"
  "s_mov_b32 s81, s79\n"
  "v_mfma_f32_16x16x32_f16 a[164:167], v[104:107], v[136:139], a[164:167]\n"
  "buffer_load_dwordx4 v[24:27], v240, s[20:23], s46 offen offset:1536\n"
  "v_mfma_f32_16x16x32_f16 a[196:199], v[112:115], v[136:139], a[196:199]\n"
  "s_sub_u32 s86, s86, s65\n"
  "v_mfma_f32_16x16x32_f16 a[196:199], v[116:119], v[140:143], a[196:199]\n"
  "v_readlane_b32 s79, v248, s81\n"
  "v_mfma_f32_16x16x32_f16 a[228:231], v[124:127], v[140:143], a[228:231]\n"
  "s_cmp_eq_u32 s59, 59\ns_cselect_b32 s66, 0xf800, 0\n"
  "v_mfma_f32_16x16x32_f16 a[228:231], v[120:123], v[136:139], a[228:231]\n"
  "s_add_u32 s79, s79, s80\n"
  "s_waitcnt lgkmcnt(5)\n"
  "v_mfma_f32_16x16x32_f16 a[8:11], v[64:67], v[144:147], a[8:11]\n"
  "buffer_load_dwordx4 v[28:31], v241, s[24:27], s74 offen offset:1536\n"
  "v_mfma_f32_16x16x32_f16 a[8:11], v[68:71], v[148:151], a[8:11]\n"
  "ds_read_b128 v[136:139], v243 offset:5120\n"
  "s_add_u32 s28, s48, s79\ns_addc_u32 s29, s49, 0\n"
  "v_mfma_f32_16x16x32_f16 a[40:43], v[76:79], v[148:151], a[40:43]\n"
  "s_add_u32 s46, s46, s66\ns_add_u32 s74, s74, s66\n"
  "v_mfma_f32_16x16x32_f16 a[40:43], v[72:75], v[144:147], a[40:43]\n"
  "s_cmp_lt_u32 s58, 64\ns_cselect_b32 s30, s69, 0\n"
  "v_mfma_f32_16x16x32_f16 a[72:75], v[80:83], v[144:147], a[72:75]\n"
  "buffer_load_dwordx4 v[32:35], v241, s[24:27], s46 offen offset:2048\n"
  "v_mfma_f32_16x16x32_f16 a[72:75], v[84:87], v[148:151], a[72:75]\n"
  "s_add_u32 s58, s58, 1\n"
  "v_mfma_f32_16x16x32_f16 a[104:107], v[92:95], v[148:151], a[104:107]\n"
  "s_add_i32 s87, s87, s77\n"
  "v_mfma_f32_16x16x32_f16 a[104:107], v[88:91], v[144:147], a[104:107]\n"
  "ds_read_b128 v[140:143], v243 offset:13312\n"
  "v_mfma_f32_16x16x32_f16 a[136:139], v[96:99], v[144:147], a[136:139]\n"
  "s_max_i32 s87, s87, 0\n"
  "v_mfma_f32_16x16x32_f16 a[136:139], v[100:103], v[148:151], a[136:139]\n"
  "buffer_load_dwordx4 v[36:39], v240, s[20:23], s74 offen offset:2048\n"
  "v_mfma_f32_16x16x32_f16 a[168:171], v[108:111], v[148:151], a[168:171]\n"
  "s_min_i32 s87, s87, 6\n"
  "v_mfma_f32_16x16x32_f16 a[168:171], v[104:107], v[144:147], a[168:171]\n"
  "v_mfma_f32_16x16x32_f16 a[200:203], v[112:115], v[144:147], a[200:203]\n"
  "s_add_u32 s79, s59, 3\ns_lshr_b32 s80, s79, 1\n"
  "v_mfma_f32_16x16x32_f16 a[200:203], v[116:119], v[148:151], a[200:203]\n"
  "v_mfma_f32_16x16x32_f16 a[232:235], v[124:127], v[148:151], a[232:235]\n"
  "buffer_load_dwordx4 v[40:43], v240, s[20:23], s46 offen offset:2560\n"
  "v_readlane_b32 s80, v249, s80\n"
  "v_mfma_f32_16x16x32_f16 a[232:235], v[120:123], v[144:147], a[232:235]\n"
  "s_waitcnt lgkmcnt(4)\n"
  "v_mfma_f32_16x16x32_f16 a[12:15], v[64:67], v[152:155], a[12:15]\n"
  "s_lshl_b32 s80, s80, 17\n"
  "v_mfma_f32_16x16x32_f16 a[12:15], v[68:71], v[156:159], a[12:15]\n"
  "ds_read_b128 v[144:147], v243 offset:6144\n"
  "v_mfma_f32_16x16x32_f16 a[44:47], v[76:79], v[156:159], a[44:47]\n"
  "buffer_load_dwordx4 v[44:47], v241, s[24:27], s74 offen offset:2560\n"
  "s_cmp_lt_u32 s79, 64\ns_cselect_b32 s60, s80, s84\n"
  "v_mfma_f32_16x16x32_f16 a[44:47], v[72:75], v[152:155], a[44:47]\n"
  "v_mfma_f32_16x16x32_f16 a[76:79], v[80:83], v[152:155], a[76:79]\n"
  "v_mfma_f32_16x16x32_f16 a[76:79], v[84:87], v[156:159], a[76:79]\n"
  "v_mfma_f32_16x16x32_f16 a[108:111], v[92:95], v[156:159], a[108:111]\n"
  "v_mfma_f32_16x16x32_f16 a[108:111], v[88:91], v[152:155], a[108:111]\n"
  "buffer_load_dwordx4 v[48:51], v240, s[20:23], s46 offen offset:3072\n"
  "ds_read_b128 v[148:151], v243 offset:14336\n"
  "v_mfma_f32_16x16x32_f16 a[140:143], v[96:99], v[152:155], a[140:143]\n"
  "v_mfma_f32_16x16x32_f16 a[140:143], v[100:103], v[156:159], a[140:143]\n"
  "v_mfma_f32_16x16x32_f16 a[172:175], v[108:111], v[156:159], a[172:175]\n"
  "v_mfma_f32_16x16x32_f16 a[172:175], v[104:107], v[152:155], a[172:175]\n"
  "v_mfma_f32_16x16x32_f16 a[204:207], v[112:115], v[152:155], a[204:207]\n"
  "buffer_load_dwordx4 v[52:55], v241, s[24:27], s74 offen offset:3072\n"
  "v_mfma_f32_16x16x32_f16 a[204:207], v[116:119], v[156:159], a[204:207]\n"
  "v_mfma_f32_16x16x32_f16 a[236:239], v[124:127], v[156:159], a[236:239]\n"
  "v_mfma_f32_16x16x32_f16 a[236:239], v[120:123], v[152:155], a[236:239]\n"
  "ds_read_b128 v[152:155], v243 offset:7168\n"
  "s_waitcnt lgkmcnt(5)\n"
  "v_mfma_f32_16x16x32_f16 a[16:19], v[64:67], v[128:131], a[16:19]\n"
  "buffer_load_dwordx4 v[56:59], v241, s[24:27], s46 offen offset:3584\n"
  "v_mfma_f32_16x16x32_f16 a[16:19], v[68:71], v[132:135], a[16:19]\n"
  "v_mfma_f32_16x16x32_f16 a[48:51], v[76:79], v[132:135], a[48:51]\n"
  "v_mfma_f32_16x16x32_f16 a[48:51], v[72:75], v[128:131], a[48:51]\n"
  "v_mfma_f32_16x16x32_f16 a[80:83], v[80:83], v[128:131], a[80:83]\n"
  "v_mfma_f32_16x16x32_f16 a[80:83], v[84:87], v[132:135], a[80:83]\n"
  "buffer_load_dwordx4 v[60:63], v240, s[20:23], s74 offen offset:3584\n"
  "v_mfma_f32_16x16x32_f16 a[112:115], v[92:95], v[132:135], a[112:115]\n"
  "v_mfma_f32_16x16x32_f16 a[112:115], v[88:91], v[128:131], a[112:115]\n"
  "s_barrier\n"
  "buffer_load_dword v192, v245, s[20:23], s60 offen\n"
  "v_mfma_f32_16x16x32_f16 a[144:147], v[96:99], v[128:131], a[144:147]\n"
  "buffer_load_dwordx4 v[192:195], v242, s[28:31], 0 offen nt\n"
  "v_mfma_f32_16x16x32_f16 a[144:147], v[100:103], v[132:135], a[144:147]\n"
  "v_mfma_f32_16x16x32_f16 a[176:179], v[108:111], v[132:135], a[176:179]\n"
  "v_mfma_f32_16x16x32_f16 a[176:179], v[104:107], v[128:131], a[176:179]\n"
  "v_mfma_f32_16x16x32_f16 a[208:211], v[112:115], v[128:131], a[208:211]\n"
  "buffer_load_dwordx4 v[196:199], v242, s[28:31], s39 offen nt\n"
  "v_mfma_f32_16x16x32_f16 a[208:211], v[116:119], v[132:135], a[208:211]\n"
  "ds_read_b128 v[156:159], v243 offset:15360\n"
  "s_waitcnt vmcnt(19)\n"
  "v_cvt_pk_f16_f32 v224, v160, v164\n"
  "v_mfma_f32_16x16x32_f16 a[240:243], v[124:127], v[132:135], a[240:243]\n"
  "v_add_u32 v243, 0x4000, v243\n"
  "v_and_b32 v243, 0x1ffff, v243\n"
  "v_mfma_f32_16x16x32_f16 a[240:243], v[120:123], v[128:131], a[240:243]\n"
  "v_cvt_pk_f16_f32 v225, v168, v172\n"
  "s_waitcnt lgkmcnt(4)\n"
  "v_mfma_f32_16x16x32_f16 a[20:23], v[64:67], v[136:139], a[20:23]\n"
  "buffer_load_dwordx4 v[200:203], v242, s[28:31], s40 offen nt\n"
  "v_mfma_f32_16x16x32_f16 a[20:23], v[68:71], v[140:143], a[20:23]\n"
  "ds_read_b128 v[128:131], v243 offset:0\n"
  "v_cvt_pk_f16_f32 v226, v176, v180\n"
  "v_mfma_f32_16x16x32_f16 a[52:55], v[76:79], v[140:143], a[52:55]\n"
  "v_mfma_f32_16x16x32_f16 a[52:55], v[72:75], v[136:139], a[52:55]\n"
  "v_mfma_f32_16x16x32_f16 a[84:87], v[80:83], v[136:139], a[84:87]\n"
  "buffer_load_dwordx4 v[204:207], v242, s[28:31], s41 offen nt\n"
  "v_cvt_pk_f16_f32 v227, v184, v188\n"
  "v_mfma_f32_16x16x32_f16 a[84:87], v[84:87], v[140:143], a[84:87]\n"
  "v_mfma_f32_16x16x32_f16 a[116:119], v[92:95], v[140:143], a[116:119]\n"
  "ds_write_b128 v244, v[224:227] offset:0\n"
  "v_mfma_f32_16x16x32_f16 a[116:119], v[88:91], v[136:139], a[116:119]\n"
  "ds_read_b128 v[132:135], v243 offset:8192\n"
  "v_mfma_f32_16x16x32_f16 a[148:151], v[96:99], v[136:139], a[148:151]\n"
  "buffer_load_dwordx4 v[208:211], v242, s[28:31], s42 offen nt\n"
  "v_mfma_f32_16x16x32_f16 a[148:151], v[100:103], v[140:143], a[148:151]\n"
  "v_cvt_pk_f16_f32 v228, v161, v165\n"
  "v_mfma_f32_16x16x32_f16 a[180:183], v[108:111], v[140:143], a[180:183]\n"
  "v_mfma_f32_16x16x32_f16 a[180:183], v[104:107], v[136:139], a[180:183]\n"
  "v_cvt_pk_f16_f32 v229, v169, v173\n"
  "v_mfma_f32_16x16x32_f16 a[212:215], v[112:115], v[136:139], a[212:215]\n"
  "buffer_load_dwordx4 v[212:215], v242, s[28:31], s43 offen nt\n"
  "v_mfma_f32_16x16x32_f16 a[212:215], v[116:119], v[140:143], a[212:215]\n"
  "v_cvt_pk_f16_f32 v230, v177, v181\n"
  "v_mfma_f32_16x16x32_f16 a[244:247], v[124:127], v[140:143], a[244:247]\n"
  "v_mfma_f32_16x16x32_f16 a[244:247], v[120:123], v[136:139], a[244:247]\n"
  "s_waitcnt lgkmcnt(5)\n"
  "v_mfma_f32_16x16x32_f16 a[24:27], v[64:67], v[144:147], a[24:27]\n"
  "buffer_load_dwordx4 v[216:219], v242, s[28:31], s44 offen nt\n"
  "v_cvt_pk_f16_f32 v231, v185, v189\n"
  "v_mfma_f32_16x16x32_f16 a[24:27], v[68:71], v[148:151], a[24:27]\n"
  "ds_read_b128 v[136:139], v243 offset:1024\n"
  "v_mfma_f32_16x16x32_f16 a[56:59], v[76:79], v[148:151], a[56:59]\n"
  "ds_write_b128 v244, v[228:231] offset:1024\n"
  "v_mfma_f32_16x16x32_f16 a[56:59], v[72:75], v[144:147], a[56:59]\n"
  "v_mfma_f32_16x16x32_f16 a[88:91], v[80:83], v[144:147], a[88:91]\n"
  "buffer_load_dwordx4 v[220:223], v242, s[28:31], s45 offen nt\n"
  "v_mfma_f32_16x16x32_f16 a[88:91], v[84:87], v[148:151], a[88:91]\n"
  "v_cvt_pk_f16_f32 v224, v162, v166\n"
  "v_mfma_f32_16x16x32_f16 a[120:123], v[92:95], v[148:151], a[120:123]\n"
  "s_memrealtime s[52:53]\n"
  "v_mfma_f32_16x16x32_f16 a[120:123], v[88:91], v[144:147], a[120:123]\n"
  "ds_read_b128 v[140:143], v243 offset:9216\n"
  "v_cvt_pk_f16_f32 v225, v170, v174\n"
  "v_mfma_f32_16x16x32_f16 a[152:155], v[96:99], v[144:147], a[152:155]\n"
  "v_mfma_f32_16x16x32_f16 a[152:155], v[100:103], v[148:151], a[152:155]\n"
  "v_cvt_pk_f16_f32 v226, v178, v182\n"
  "v_mfma_f32_16x16x32_f16 a[184:187], v[108:111], v[148:151], a[184:187]\n"
  "s_add_u32 s79, s59, 2\ns_lshr_b32 s80, s79, 1\n"
  "v_mfma_f32_16x16x32_f16 a[184:187], v[104:107], v[144:147], a[184:187]\n"
  "s_waitcnt lgkmcnt(0)\n"
  "s_sub_u32 s82, s52, s86\ns_cmp_lt_i32 s82, 0\ns_cbranch_scc0 PGl1_%=\nPWl1_%=:\ns_sleep 1\ns_memrealtime s[54:55]\ns_waitcnt lgkmcnt(0)\ns_sub_u32 s88, s54, s86\ns_cmp_lt_i32 s88, 0\ns_cbranch_scc1 PWl1_%=\nPGl1_%=:\n"
  "v_mfma_f32_16x16x32_f16 a[216:219], v[112:115], v[144:147], a[216:219]\n"
  "s_cmp_lt_u32 s82, 44\ns_cselect_b32 s77, 1, -1\n"
  "s_cmp_gt_i32 s82, 125\ns_cselect_b32 s66, 2, 0\ns_max_u32 s75, s75, s66\n"
  "v_cvt_pk_f16_f32 v227, v186, v190\n"
  "v_readlane_b32 s80, v249, s80\n"
  "v_mfma_f32_16x16x32_f16 a[216:219], v[116:119], v[148:151], a[216:219]\n"
  "s_sub_u32 s82, s82, -131\n"
  "v_mfma_f32_16x16x32_f16 a[248:251], v[124:127], v[148:151], a[248:251]\n"
  "ds_write_b128 v244, v[224:227] offset:2048\n"
  "s_lshl_b32 s80, s80, 17\ns_add_u32 s80, s80, 0x10000\n"
  "v_mfma_f32_16x16x32_f16 a[248:251], v[120:123], v[144:147], a[248:251]\n"
  "v_mfma_f32_16x16x32_f16 a[28:31], v[64:67], v[152:155], a[28:31]\n"
  "s_cmp_lt_u32 s79, 64\ns_cselect_b32 s46, s80, s84\n"
  "v_mfma_f32_16x16x32_f16 a[28:31], v[68:71], v[156:159], a[28:31]\n"
  "ds_read_b128 v[144:147], v243 offset:2048\n"
  "v_cvt_pk_f16_f32 v228, v163, v167\n"
  "v_mfma_f32_16x16x32_f16 a[60:63], v[76:79], v[156:159], a[60:63]\n"
  "s_max_i32 s82, s82, 151\n"
  "v_mfma_f32_16x16x32_f16 a[60:63], v[72:75], v[152:155], a[60:63]\n"
  "v_cvt_pk_f16_f32 v229, v171, v175\n"
  "v_mfma_f32_16x16x32_f16 a[92:95], v[80:83], v[152:155], a[92:95]\n"
  "s_add_u32 s46, s46, s83\n"
  "v_mfma_f32_16x16x32_f16 a[92:95], v[84:87], v[156:159], a[92:95]\n"
  "v_cvt_pk_f16_f32 v230, v179, v183\n"
  "v_mfma_f32_16x16x32_f16 a[124:127], v[92:95], v[156:159], a[124:127]\n"
  "s_add_u32 s74, s46, 0x4000\n"
  "v_mfma_f32_16x16x32_f16 a[124:127], v[88:91], v[152:155], a[124:127]\n"
  "ds_read_b128 v[148:151], v243 offset:10240\n"
  "v_mfma_f32_16x16x32_f16 a[156:159], v[96:99], v[152:155], a[156:159]\n"
  "v_cvt_pk_f16_f32 v231, v187, v191\n"
  "s_add_u32 s59, s59, 1\n"
  "v_mfma_f32_16x16x32_f16 a[156:159], v[100:103], v[156:159], a[156:159]\n"
  "s_add_u32 s86, s86, s82\n"
  "v_mfma_f32_16x16x32_f16 a[188:191], v[108:111], v[156:159], a[188:191]\n"
  "ds_write_b128 v244, v[228:231] offset:3072\n"
  "v_mfma_f32_16x16x32_f16 a[188:191], v[104:107], v[152:155], a[188:191]\n"
  "v_mfma_f32_16x16x32_f16 a[220:223], v[112:115], v[152:155], a[220:223]\n"
  "v_mfma_f32_16x16x32_f16 a[220:223], v[116:119], v[156:159], a[220:223]\n"
  "v_mfma_f32_16x16x32_f16 a[252:255], v[124:127], v[156:159], a[252:255]\n"
  "v_add_u32 v244, 0x4000, v244\n"
  "v_and_b32 v244, 0x1ffff, v244\n"
  "v_mfma_f32_16x16x32_f16 a[252:255], v[120:123], v[152:155], a[252:255]\n"
  "LSEL_%=:\n"
  "s_cmp_ge_u32 s59, 60\n"
  "s_cbranch_scc1 DONE_%=\n"
  "s_cmp_ge_u32 s59, 52\n"
  "s_cselect_b32 s75, 0, s75\n"
  "s_cmp_lg_u32 s75, 0\n"
  "s_cbranch_scc1 XLOOP_%=\n"
  "s_cmp_ge_u32 s87, s71\n"
  "s_cbranch_scc1 LOOP_%=\n"
  "s_cmp_lt_u32 s59, 52\n"
  "s_cbranch_scc1 SLOOP_%=\n"
  "s_sub_u32 s63, s86, s90\n"
  "s_sub_u32 s63, s63, 371\n"
  "s_cmp_ge_u32 s63, 99780\n"
  "s_cbranch_scc1 XLP2_%=\n"
  "s_branch LOOP_%=\n"
  "s_sub_u32 s63, s86, s90\n"
  "s_cmp_gt_i32 s63, 100151\n"
  "s_cbranch_scc1 XLP2_%=\n"
  "SLOOP_%=:\n"
  "buffer_load_dwordx4 v[64:67], v240, s[20:23], s46 offen offset:0\n"
  "s_waitcnt vmcnt(10)\n"
  "v_mfma_f32_16x16x32_f16 a[0:3], v[0:3], v[128:131], a[0:3]\n"
  "s_add_u32 s85, s85, 1\n"
  "s_nop 3\n"
  "v_mfma_f32_16x16x32_f16 a[0:3], v[4:7], v[132:135], a[0:3]\n"
  "ds_read_b128 v[152:155], v243 offset:3072\n"
  "v_mfma_f32_16x16x32_f16 a[32:35], v[12:15], v[132:135], a[32:35]\n"
  "s_min_u32 s85, s85, 63\n"
  "v_mfma_f32_16x16x32_f16 a[32:35], v[8:11], v[128:131], a[32:35]\n"
  "v_readlane_b32 s88, v247, s85\n"
  "v_mfma_f32_16x16x32_f16 a[64:67], v[16:19], v[128:131], a[64:67]\n"
  "buffer_load_dwordx4 v[68:71], v241, s[24:27], s74 offen offset:0\n"
  "s_nop 3\n"
  "v_mfma_f32_16x16x32_f16 a[64:67], v[20:23], v[132:135], a[64:67]\n"
  "s_add_u32 s88, s88, s89\n"
  "v_mfma_f32_16x16x32_f16 a[96:99], v[28:31], v[132:135], a[96:99]\n"
  "s_max_u32 s86, s86, s88\n"
  "v_mfma_f32_16x16x32_f16 a[96:99], v[24:27], v[128:131], a[96:99]\n"
  "ds_read_b128 v[156:159], v243 offset:11264\n"
  "v_mfma_f32_16x16x32_f16 a[128:131], v[32:35], v[128:131], a[128:131]\n"
  "buffer_load_dwordx4 v[72:75], v241, s[24:27], s46 offen offset:512\n"
  "s_nop 3\n"
  "v_mfma_f32_16x16x32_f16 a[128:131], v[36:39], v[132:135], a[128:131]\n"
  "s_add_u32 s90, s90, 151\n"
  "v_mfma_f32_16x16x32_f16 a[160:163], v[44:47], v[132:135], a[160:163]\n"
  "s_max_u32 s90, s90, s88\n"
  "v_mfma_f32_16x16x32_f16 a[160:163], v[40:43], v[128:131], a[160:163]\n"
  "s_sub_u32 s65, s86, s90\n"
  "v_mfma_f32_16x16x32_f16 a[192:195], v[48:51], v[128:131], a[192:195]\n"
  "s_cmp_gt_i32 s65, 150\ns_cselect_b32 s65, 5, 0\ns_cselect_b32 s71, 0, 3\n"
  "s_nop 3\n"
  "v_mfma_f32_16x16x32_f16 a[192:195], v[52:55], v[132:135], a[192:195]\n"
  "buffer_load_dwordx4 v[76:79], v240, s[20:23], s74 offen offset:512\n"
  "v_mfma_f32_16x16x32_f16 a[224:227], v[60:63], v[132:135], a[224:227]\n"
  "s_sub_u32 s86, s86, s65\n"
  "v_mfma_f32_16x16x32_f16 a[224:227], v[56:59], v[128:131], a[224:227]\n"
  "v_mfma_f32_16x16x32_f16 a[4:7], v[0:3], v[136:139], a[4:7]\n"
  "s_add_i32 s87, s87, s77\n"
  "s_nop 3\n"
  "v_mfma_f32_16x16x32_f16 a[4:7], v[4:7], v[140:143], a[4:7]\n"
  "ds_read_b128 v[128:131], v243 offset:4096\n"
  "v_mfma_f32_16x16x32_f16 a[36:39], v[12:15], v[140:143], a[36:39]\n"
  "buffer_load_dwordx4 v[80:83], v241, s[24:27], s46 offen offset:1024\n"
  "v_mfma_f32_16x16x32_f16 a[36:39], v[8:11], v[136:139], a[36:39]\n"
  "s_max_i32 s87, s87, 0\n"
  "v_mfma_f32_16x16x32_f16 a[68:71], v[16:19], v[136:139], a[68:71]\n"
  "s_lshr_b32 s79, s58, 1\n"
  "v_mfma_f32_16x16x32_f16 a[68:71], v[20:23], v[140:143], a[68:71]\n"
  "s_min_i32 s87, s87, 6\n"
  "s_nop 3\n"
  "v_mfma_f32_16x16x32_f16 a[100:103], v[28:31], v[140:143], a[100:103]\n"
  "buffer_load_dwordx4 v[84:87], v240, s[20:23], s74 offen offset:1024\n"
  "v_readlane_b32 s79, v249, s79\n"
  "v_mfma_f32_16x16x32_f16 a[100:103], v[24:27], v[136:139], a[100:103]\n"
  "ds_read_b128 v[132:135], v243 offset:12288\n"
  "v_mfma_f32_16x16x32_f16 a[132:135], v[32:35], v[136:139], a[132:135]\n"
  "s_and_b32 s80, s58, 1\ns_lshl_b32 s80, s80, 23\n"
  "v_mfma_f32_16x16x32_f16 a[132:135], v[36:39], v[140:143], a[132:135]\n"
  "s_nop 3\n"
  "v_mfma_f32_16x16x32_f16 a[164:167], v[44:47], v[140:143], a[164:167]\n"
  "s_mov_b32 s81, s79\n"
  "v_mfma_f32_16x16x32_f16 a[164:167], v[40:43], v[136:139], a[164:167]\n"
  "buffer_load_dwordx4 v[88:91], v240, s[20:23], s46 offen offset:1536\n"
  "v_mfma_f32_16x16x32_f16 a[196:199], v[48:51], v[136:139], a[196:199]\n"
  "v_mfma_f32_16x16x32_f16 a[196:199], v[52:55], v[140:143], a[196:199]\n"
  "v_readlane_b32 s79, v248, s81\n"
  "s_nop 3\n"
  "v_mfma_f32_16x16x32_f16 a[228:231], v[60:63], v[140:143], a[228:231]\n"
  "v_mfma_f32_16x16x32_f16 a[228:231], v[56:59], v[136:139], a[228:231]\n"
  "s_add_u32 s79, s79, s80\n"
  "s_waitcnt lgkmcnt(5)\n"
  "v_mfma_f32_16x16x32_f16 a[8:11], v[0:3], v[144:147], a[8:11]\n"
  "buffer_load_dwordx4 v[92:95], v241, s[24:27], s74 offen offset:1536\n"
  "v_mfma_f32_16x16x32_f16 a[8:11], v[4:7], v[148:151], a[8:11]\n"
  "ds_read_b128 v[136:139], v243 offset:5120\n"
  "s_add_u32 s28, s48, s79\ns_addc_u32 s29, s49, 0\n"
  "s_nop 3\n"
  "v_mfma_f32_16x16x32_f16 a[40:43], v[12:15], v[148:151], a[40:43]\n"
  "v_mfma_f32_16x16x32_f16 a[40:43], v[8:11], v[144:147], a[40:43]\n"
  "s_cmp_lt_u32 s58, 64\ns_cselect_b32 s30, s69, 0\n"
  "v_mfma_f32_16x16x32_f16 a[72:75], v[16:19], v[144:147], a[72:75]\n"
  "buffer_load_dwordx4 v[96:99], v241, s[24:27], s46 offen offset:2048\n"
  "v_mfma_f32_16x16x32_f16 a[72:75], v[20:23], v[148:151], a[72:75]\n"
  "s_add_u32 s58, s58, 1\n"
  "v_mfma_f32_16x16x32_f16 a[104:107], v[28:31], v[148:151], a[104:107]\n"
  "s_nop 3\n"
  "v_mfma_f32_16x16x32_f16 a[104:107], v[24:27], v[144:147], a[104:107]\n"
  "ds_read_b128 v[140:143], v243 offset:13312\n"
  "v_mfma_f32_16x16x32_f16 a[136:139], v[32:35], v[144:147], a[136:139]\n"
  "v_mfma_f32_16x16x32_f16 a[136:139], v[36:39], v[148:151], a[136:139]\n"
  "buffer_load_dwordx4 v[100:103], v240, s[20:23], s74 offen offset:2048\n"
  "v_mfma_f32_16x16x32_f16 a[168:171], v[44:47], v[148:151], a[168:171]\n"
  "s_nop 3\n"
  "v_mfma_f32_16x16x32_f16 a[168:171], v[40:43], v[144:147], a[168:171]\n"
  "v_mfma_f32_16x16x32_f16 a[200:203], v[48:51], v[144:147], a[200:203]\n"
  "s_add_u32 s79, s59, 3\ns_lshr_b32 s80, s79, 1\n"
  "v_mfma_f32_16x16x32_f16 a[200:203], v[52:55], v[148:151], a[200:203]\n"
  "v_mfma_f32_16x16x32_f16 a[232:235], v[60:63], v[148:151], a[232:235]\n"
  "buffer_load_dwordx4 v[104:107], v240, s[20:23], s46 offen offset:2560\n"
  "v_readlane_b32 s80, v249, s80\n"
  "s_nop 3\n"
  "v_mfma_f32_16x16x32_f16 a[232:235], v[56:59], v[144:147], a[232:235]\n"
  "s_waitcnt lgkmcnt(4)\n"
  "v_mfma_f32_16x16x32_f16 a[12:15], v[0:3], v[152:155], a[12:15]\n"
  "s_lshl_b32 s80, s80, 17\ns_add_u32 s80, s80, 0x10000\n"
  "v_mfma_f32_16x16x32_f16 a[12:15], v[4:7], v[156:159], a[12:15]\n"
  "ds_read_b128 v[144:147], v243 offset:6144\n"
  "v_mfma_f32_16x16x32_f16 a[44:47], v[12:15], v[156:159], a[44:47]\n"
  "buffer_load_dwordx4 v[108:111], v241, s[24:27], s74 offen offset:2560\n"
  "s_cmp_lt_u32 s79, 64\ns_cselect_b32 s60, s80, s84\n"
  "s_nop 3\n"
  "v_mfma_f32_16x16x32_f16 a[44:47], v[8:11], v[152:155], a[44:47]\n"
  "v_mfma_f32_16x16x32_f16 a[76:79], v[16:19], v[152:155], a[76:79]\n"
  "v_mfma_f32_16x16x32_f16 a[76:79], v[20:23], v[156:159], a[76:79]\n"
  "v_mfma_f32_16x16x32_f16 a[108:111], v[28:31], v[156:159], a[108:111]\n"
  "v_mfma_f32_16x16x32_f16 a[108:111], v[24:27], v[152:155], a[108:111]\n"
  "buffer_load_dwordx4 v[112:115], v240, s[20:23], s46 offen offset:3072\n"
  "ds_read_b128 v[148:151], v243 offset:14336\n"
  "s_nop 3\n"
  "v_mfma_f32_16x16x32_f16 a[140:143], v[32:35], v[152:155], a[140:143]\n"
  "v_mfma_f32_16x16x32_f16 a[140:143], v[36:39], v[156:159], a[140:143]\n"
  "v_mfma_f32_16x16x32_f16 a[172:175], v[44:47], v[156:159], a[172:175]\n"
  "v_mfma_f32_16x16x32_f16 a[172:175], v[40:43], v[152:155], a[172:175]\n"
  "s_nop 3\n"
  "v_mfma_f32_16x16x32_f16 a[204:207], v[48:51], v[152:155], a[204:207]\n"
  "buffer_load_dwordx4 v[116:119], v241, s[24:27], s74 offen offset:3072\n"
  "v_mfma_f32_16x16x32_f16 a[204:207], v[52:55], v[156:159], a[204:207]\n"
  "v_mfma_f32_16x16x32_f16 a[236:239], v[60:63], v[156:159], a[236:239]\n"
  "v_mfma_f32_16x16x32_f16 a[236:239], v[56:59], v[152:155], a[236:239]\n"
  "ds_read_b128 v[152:155], v243 offset:7168\n"
  "s_nop 3\n"
  "s_waitcnt lgkmcnt(5)\n"
  "v_mfma_f32_16x16x32_f16 a[16:19], v[0:3], v[128:131], a[16:19]\n"
  "buffer_load_dwordx4 v[120:123], v241, s[24:27], s46 offen offset:3584\n"
  "v_mfma_f32_16x16x32_f16 a[16:19], v[4:7], v[132:135], a[16:19]\n"
  "v_mfma_f32_16x16x32_f16 a[48:51], v[12:15], v[132:135], a[48:51]\n"
  "v_mfma_f32_16x16x32_f16 a[48:51], v[8:11], v[128:131], a[48:51]\n"
  "s_nop 3\n"
  "v_mfma_f32_16x16x32_f16 a[80:83], v[16:19], v[128:131], a[80:83]\n"
  "v_mfma_f32_16x16x32_f16 a[80:83], v[20:23], v[132:135], a[80:83]\n"
  "buffer_load_dwordx4 v[124:127], v240, s[20:23], s74 offen offset:3584\n"
  "v_mfma_f32_16x16x32_f16 a[112:115], v[28:31], v[132:135], a[112:115]\n"
  "v_mfma_f32_16x16x32_f16 a[112:115], v[24:27], v[128:131], a[112:115]\n"
  "s_barrier\n"
  "buffer_load_dword v160, v245, s[20:23], s60 offen\n"
  "s_nop 3\n"
  "v_mfma_f32_16x16x32_f16 a[144:147], v[32:35], v[128:131], a[144:147]\n"
  "buffer_load_dwordx4 v[160:163], v242, s[28:31], 0 offen nt\n"
  "v_mfma_f32_16x16x32_f16 a[144:147], v[36:39], v[132:135], a[144:147]\n"
  "v_mfma_f32_16x16x32_f16 a[176:179], v[44:47], v[132:135], a[176:179]\n"
  "v_mfma_f32_16x16x32_f16 a[176:179], v[40:43], v[128:131], a[176:179]\n"
  "v_mfma_f32_16x16x32_f16 a[208:211], v[48:51], v[128:131], a[208:211]\n"
  "buffer_load_dwordx4 v[164:167], v242, s[28:31], s39 offen nt\n"
  "s_nop 3\n"
  "v_mfma_f32_16x16x32_f16 a[208:211], v[52:55], v[132:135], a[208:211]\n"
  "ds_read_b128 v[156:159], v243 offset:15360\n"
  "s_waitcnt vmcnt(19)\n"
  "v_cvt_pk_f16_f32 v224, v192, v196\n"
  "v_mfma_f32_16x16x32_f16 a[240:243], v[60:63], v[132:135], a[240:243]\n"
  "v_add_u32 v243, 0x4000, v243\n"
  "v_and_b32 v243, 0x1ffff, v243\n"
  "v_mfma_f32_16x16x32_f16 a[240:243], v[56:59], v[128:131], a[240:243]\n"
  "v_cvt_pk_f16_f32 v225, v200, v204\n"
  "s_waitcnt lgkmcnt(4)\n"
  "v_mfma_f32_16x16x32_f16 a[20:23], v[0:3], v[136:139], a[20:23]\n"
  "buffer_load_dwordx4 v[168:171], v242, s[28:31], s40 offen nt\n"
  "s_nop 3\n"
  "v_mfma_f32_16x16x32_f16 a[20:23], v[4:7], v[140:143], a[20:23]\n"
  "ds_read_b128 v[128:131], v243 offset:0\n"
  "v_cvt_pk_f16_f32 v226, v208, v212\n"
  "v_mfma_f32_16x16x32_f16 a[52:55], v[12:15], v[140:143], a[52:55]\n"
  "v_mfma_f32_16x16x32_f16 a[52:55], v[8:11], v[136:139], a[52:55]\n"
  "v_mfma_f32_16x16x32_f16 a[84:87], v[16:19], v[136:139], a[84:87]\n"
  "buffer_load_dwordx4 v[172:175], v242, s[28:31], s41 offen nt\n"
  "v_cvt_pk_f16_f32 v227, v216, v220\n"
  "s_nop 3\n"
  "v_mfma_f32_16x16x32_f16 a[84:87], v[20:23], v[140:143], a[84:87]\n"
  "v_mfma_f32_16x16x32_f16 a[116:119], v[28:31], v[140:143], a[116:119]\n"
  "ds_write_b128 v244, v[224:227] offset:0\n"
  "v_mfma_f32_16x16x32_f16 a[116:119], v[24:27], v[136:139], a[116:119]\n"
  "ds_read_b128 v[132:135], v243 offset:8192\n"
  "v_mfma_f32_16x16x32_f16 a[148:151], v[32:35], v[136:139], a[148:151]\n"
  "buffer_load_dwordx4 v[176:179], v242, s[28:31], s42 offen nt\n"
  "s_nop 3\n"
  "v_mfma_f32_16x16x32_f16 a[148:151], v[36:39], v[140:143], a[148:151]\n"
  "v_cvt_pk_f16_f32 v228, v193, v197\n"
  "v_mfma_f32_16x16x32_f16 a[180:183], v[44:47], v[140:143], a[180:183]\n"
  "v_mfma_f32_16x16x32_f16 a[180:183], v[40:43], v[136:139], a[180:183]\n"
  "v_cvt_pk_f16_f32 v229, v201, v205\n"
  "v_mfma_f32_16x16x32_f16 a[212:215], v[48:51], v[136:139], a[212:215]\n"
  "buffer_load_dwordx4 v[180:183], v242, s[28:31], s43 offen nt\n"
  "v_mfma_f32_16x16x32_f16 a[212:215], v[52:55], v[140:143], a[212:215]\n"
  "v_cvt_pk_f16_f32 v230, v209, v213\n"
  "s_nop 3\n"
  "v_mfma_f32_16x16x32_f16 a[244:247], v[60:63], v[140:143], a[244:247]\n"
  "v_mfma_f32_16x16x32_f16 a[244:247], v[56:59], v[136:139], a[244:247]\n"
  "s_waitcnt lgkmcnt(5)\n"
  "v_mfma_f32_16x16x32_f16 a[24:27], v[0:3], v[144:147], a[24:27]\n"
  "buffer_load_dwordx4 v[184:187], v242, s[28:31], s44 offen nt\n"
  "v_cvt_pk_f16_f32 v231, v217, v221\n"
  "v_mfma_f32_16x16x32_f16 a[24:27], v[4:7], v[148:151], a[24:27]\n"
  "ds_read_b128 v[136:139], v243 offset:1024\n"
  "s_nop 3\n"
  "v_mfma_f32_16x16x32_f16 a[56:59], v[12:15], v[148:151], a[56:59]\n"
  "ds_write_b128 v244, v[228:231] offset:1024\n"
  "v_mfma_f32_16x16x32_f16 a[56:59], v[8:11], v[144:147], a[56:59]\n"
  "v_mfma_f32_16x16x32_f16 a[88:91], v[16:19], v[144:147], a[88:91]\n"
  "buffer_load_dwordx4 v[188:191], v242, s[28:31], s45 offen nt\n"
  "v_mfma_f32_16x16x32_f16 a[88:91], v[20:23], v[148:151], a[88:91]\n"
  "v_cvt_pk_f16_f32 v224, v194, v198\n"
  "s_nop 3\n"
  "v_mfma_f32_16x16x32_f16 a[120:123], v[28:31], v[148:151], a[120:123]\n"
  "s_memrealtime s[52:53]\n"
  "v_mfma_f32_16x16x32_f16 a[120:123], v[24:27], v[144:147], a[120:123]\n"
  "ds_read_b128 v[140:143], v243 offset:9216\n"
  "v_cvt_pk_f16_f32 v225, v202, v206\n"
  "v_mfma_f32_16x16x32_f16 a[152:155], v[32:35], v[144:147], a[152:155]\n"
  "v_mfma_f32_16x16x32_f16 a[152:155], v[36:39], v[148:151], a[152:155]\n"
  "v_cvt_pk_f16_f32 v226, v210, v214\n"
  "s_nop 3\n"
  "v_mfma_f32_16x16x32_f16 a[184:187], v[44:47], v[148:151], a[184:187]\n"
  "s_add_u32 s79, s59, 2\ns_lshr_b32 s80, s79, 1\n"
  "v_mfma_f32_16x16x32_f16 a[184:187], v[40:43], v[144:147], a[184:187]\n"
  "s_waitcnt lgkmcnt(0)\n"
  "s_sub_u32 s82, s52, s86\ns_cmp_lt_i32 s82, 0\ns_cbranch_scc0 PGs0_%=\nPWs0_%=:\ns_sleep 1\ns_memrealtime s[54:55]\ns_waitcnt lgkmcnt(0)\ns_sub_u32 s88, s54, s86\ns_cmp_lt_i32 s88, 0\ns_cbranch_scc1 PWs0_%=\nPGs0_%=:\n"
  "v_mfma_f32_16x16x32_f16 a[216:219], v[48:51], v[144:147], a[216:219]\n"
  "s_cmp_lt_u32 s82, 44\ns_cselect_b32 s77, 1, -1\n"
  "s_cmp_gt_i32 s82, 125\ns_cselect_b32 s66, 2, 0\ns_max_u32 s75, s75, s66\n"
  "v_cvt_pk_f16_f32 v227, v218, v222\n"
  "v_readlane_b32 s80, v249, s80\n"
  "v_mfma_f32_16x16x32_f16 a[216:219], v[52:55], v[148:151], a[216:219]\n"
  "s_sub_u32 s82, s82, -131\n"
  "v_mfma_f32_16x16x32_f16 a[248:251], v[60:63], v[148:151], a[248:251]\n"
  "ds_write_b128 v244, v[224:227] offset:2048\n"
  "s_lshl_b32 s80, s80, 17\n"
  "s_nop 3\n"
  "v_mfma_f32_16x16x32_f16 a[248:251], v[56:59], v[144:147], a[248:251]\n"
  "v_mfma_f32_16x16x32_f16 a[28:31], v[0:3], v[152:155], a[28:31]\n"
  "s_cmp_lt_u32 s79, 64\ns_cselect_b32 s46, s80, s84\n"
  "v_mfma_f32_16x16x32_f16 a[28:31], v[4:7], v[156:159], a[28:31]\n"
  "ds_read_b128 v[144:147], v243 offset:2048\n"
  "v_cvt_pk_f16_f32 v228, v195, v199\n"
  "v_mfma_f32_16x16x32_f16 a[60:63], v[12:15], v[156:159], a[60:63]\n"
  "s_max_i32 s82, s82, 151\n"
  "s_nop 3\n"
  "v_mfma_f32_16x16x32_f16 a[60:63], v[8:11], v[152:155], a[60:63]\n"
  "v_cvt_pk_f16_f32 v229, v203, v207\n"
  "v_mfma_f32_16x16x32_f16 a[92:95], v[16:19], v[152:155], a[92:95]\n"
  "s_add_u32 s46, s46, s83\n"
  "v_mfma_f32_16x16x32_f16 a[92:95], v[20:23], v[156:159], a[92:95]\n"
  "v_cvt_pk_f16_f32 v230, v211, v215\n"
  "v_mfma_f32_16x16x32_f16 a[124:127], v[28:31], v[156:159], a[124:127]\n"
  "s_add_u32 s74, s46, 0x4000\n"
  "s_nop 3\n"
  "v_mfma_f32_16x16x32_f16 a[124:127], v[24:27], v[152:155], a[124:127]\n"
  "ds_read_b128 v[148:151], v243 offset:10240\n"
  "v_mfma_f32_16x16x32_f16 a[156:159], v[32:35], v[152:155], a[156:159]\n"
  "v_cvt_pk_f16_f32 v231, v219, v223\n"
  "s_add_u32 s59, s59, 1\n"
  "v_mfma_f32_16x16x32_f16 a[156:159], v[36:39], v[156:159], a[156:159]\n"
  "s_add_u32 s86, s86, s82\n"
  "v_mfma_f32_16x16x32_f16 a[188:191], v[44:47], v[156:159], a[188:191]\n"
  "ds_write_b128 v244, v[228:231] offset:3072\n"
  "s_nop 3\n"
  "v_mfma_f32_16x16x32_f16 a[188:191], v[40:43], v[152:155], a[188:191]\n"
  "v_mfma_f32_16x16x32_f16 a[220:223], v[48:51], v[152:155], a[220:223]\n"
  "v_mfma_f32_16x16x32_f16 a[220:223], v[52:55], v[156:159], a[220:223]\n"
  "v_mfma_f32_16x16x32_f16 a[252:255], v[60:63], v[156:159], a[252:255]\n"
  "v_add_u32 v244, 0x4000, v244\n"
  "v_and_b32 v244, 0x1ffff, v244\n"
  "v_mfma_f32_16x16x32_f16 a[252:255], v[56:59], v[152:155], a[252:255]\n"
  "buffer_load_dwordx4 v[0:3], v240, s[20:23], s46 offen offset:0\n"
  "s_waitcnt vmcnt(10)\n"
  "v_mfma_f32_16x16x32_f16 a[0:3], v[64:67], v[128:131], a[0:3]\n"
  "s_add_u32 s85, s85, 1\n"
  "s_nop 3\n"
  "v_mfma_f32_16x16x32_f16 a[0:3], v[68:71], v[132:135], a[0:3]\n"
  "ds_read_b128 v[152:155], v243 offset:3072\n"
  "v_mfma_f32_16x16x32_f16 a[32:35], v[76:79], v[132:135], a[32:35]\n"
  "s_min_u32 s85, s85, 63\n"
  "v_mfma_f32_16x16x32_f16 a[32:35], v[72:75], v[128:131], a[32:35]\n"
  "v_readlane_b32 s88, v247, s85\n"
  "v_mfma_f32_16x16x32_f16 a[64:67], v[80:83], v[128:131], a[64:67]\n"
  "buffer_load_dwordx4 v[4:7], v241, s[24:27], s74 offen offset:0\n"
  "s_nop 3\n"
  "v_mfma_f32_16x16x32_f16 a[64:67], v[84:87], v[132:135], a[64:67]\n"
  "s_add_u32 s88, s88, s89\n"
  "v_mfma_f32_16x16x32_f16 a[96:99], v[92:95], v[132:135], a[96:99]\n"
  "s_andn2_b32 s79, s70, s72\n"
  "v_mfma_f32_16x16x32_f16 a[96:99], v[88:91], v[128:131], a[96:99]\n"
  "ds_read_b128 v[156:159], v243 offset:11264\n"
  "v_mfma_f32_16x16x32_f16 a[128:131], v[96:99], v[128:131], a[128:131]\n"
  "buffer_load_dwordx4 v[8:11], v241, s[24:27], s46 offen offset:512\n"
  "s_nop 3\n"
  "v_mfma_f32_16x16x32_f16 a[128:131], v[100:103], v[132:135], a[128:131]\n"
  "s_max_u32 s86, s86, s88\n"
  "v_mfma_f32_16x16x32_f16 a[160:163], v[108:111], v[132:135], a[160:163]\n"
  "s_ff1_i32_b32 s79, s79\n"
  "v_mfma_f32_16x16x32_f16 a[160:163], v[104:107], v[128:131], a[160:163]\n"
  "s_add_u32 s90, s90, 151\n"
  "v_mfma_f32_16x16x32_f16 a[192:195], v[112:115], v[128:131], a[192:195]\n"
  "s_bitset1_b32 s72, s79\ns_mov_b32 m0, s73\n"
  "s_nop 3\n"
  "v_mfma_f32_16x16x32_f16 a[192:195], v[116:119], v[132:135], a[192:195]\n"
  "buffer_load_dwordx4 v[12:15], v240, s[20:23], s74 offen offset:512\n"
  "v_mfma_f32_16x16x32_f16 a[224:227], v[124:127], v[132:135], a[224:227]\n"
  "v_writelane_b32 v249, s79, m0\n"
  "v_mfma_f32_16x16x32_f16 a[224:227], v[120:123], v[128:131], a[224:227]\n"
  "v_mfma_f32_16x16x32_f16 a[4:7], v[64:67], v[136:139], a[4:7]\n"
  "s_add_u32 s73, s73, 1\n"
  "s_nop 3\n"
  "v_mfma_f32_16x16x32_f16 a[4:7], v[68:71], v[140:143], a[4:7]\n"
  "ds_read_b128 v[128:131], v243 offset:4096\n"
  "v_mfma_f32_16x16x32_f16 a[36:39], v[76:79], v[140:143], a[36:39]\n"
  "buffer_load_dwordx4 v[16:19], v241, s[24:27], s46 offen offset:1024\n"
  "v_mfma_f32_16x16x32_f16 a[36:39], v[72:75], v[136:139], a[36:39]\n"
  "s_max_u32 s90, s90, s88\n"
  "v_mfma_f32_16x16x32_f16 a[68:71], v[80:83], v[136:139], a[68:71]\n"
  "s_lshr_b32 s79, s58, 1\n"
  "v_mfma_f32_16x16x32_f16 a[68:71], v[84:87], v[140:143], a[68:71]\n"
  "s_sub_u32 s65, s86, s90\n"
  "s_nop 3\n"
  "v_mfma_f32_16x16x32_f16 a[100:103], v[92:95], v[140:143], a[100:103]\n"
  "buffer_load_dwordx4 v[20:23], v240, s[20:23], s74 offen offset:1024\n"
  "v_readlane_b32 s79, v249, s79\n"
  "v_mfma_f32_16x16x32_f16 a[100:103], v[88:91], v[136:139], a[100:103]\n"
  "ds_read_b128 v[132:135], v243 offset:12288\n"
  "v_mfma_f32_16x16x32_f16 a[132:135], v[96:99], v[136:139], a[132:135]\n"
  "s_and_b32 s80, s58, 1\ns_lshl_b32 s80, s80, 23\n"
  "v_mfma_f32_16x16x32_f16 a[132:135], v[100:103], v[140:143], a[132:135]\n"
  "s_cmp_gt_i32 s65, 150\ns_cselect_b32 s65, 5, 0\ns_cselect_b32 s71, 0, 3\n"
  "s_nop 3\n"
  "v_mfma_f32_16x16x32_f16 a[164:167], v[108:111], v[140:143], a[164:167]\n"
  "s_mov_b32 s81, s79\n"
  "v_mfma_f32_16x16x32_f16 a[164:167], v[104:107], v[136:139], a[164:167]\n"
  "buffer_load_dwordx4 v[24:27], v240, s[20:23], s46 offen offset:1536\n"
  "v_mfma_f32_16x16x32_f16 a[196:199], v[112:115], v[136:139], a[196:199]\n"
  "s_sub_u32 s86, s86, s65\n"
  "v_mfma_f32_16x16x32_f16 a[196:199], v[116:119], v[140:143], a[196:199]\n"
  "v_readlane_b32 s79, v248, s81\n"
  "s_nop 3\n"
  "v_mfma_f32_16x16x32_f16 a[228:231], v[124:127], v[140:143], a[228:231]\n"
  "s_cmp_eq_u32 s59, 59\ns_cselect_b32 s66, 0xf800, 0\n"
  "v_mfma_f32_16x16x32_f16 a[228:231], v[120:123], v[136:139], a[228:231]\n"
  "s_add_u32 s79, s79, s80\n"
  "s_waitcnt lgkmcnt(5)\n"
  "v_mfma_f32_16x16x32_f16 a[8:11], v[64:67], v[144:147], a[8:11]\n"
  "buffer_load_dwordx4 v[28:31], v241, s[24:27], s74 offen offset:1536\n"
  "v_mfma_f32_16x16x32_f16 a[8:11], v[68:71], v[148:151], a[8:11]\n"
  "ds_read_b128 v[136:139], v243 offset:5120\n"
  "s_add_u32 s28, s48, s79\ns_addc_u32 s29, s49, 0\n"
  "s_nop 3\n"
  "v_mfma_f32_16x16x32_f16 a[40:43], v[76:79], v[148:151], a[40:43]\n"
  "s_add_u32 s46, s46, s66\ns_add_u32 s74, s74, s66\n"
  "v_mfma_f32_16x16x32_f16 a[40:43], v[72:75], v[144:147], a[40:43]\n"
  "s_cmp_lt_u32 s58, 64\ns_cselect_b32 s30, s69, 0\n"
  "v_mfma_f32_16x16x32_f16 a[72:75], v[80:83], v[144:147], a[72:75]\n"
  "buffer_load_dwordx4 v[32:35], v241, s[24:27], s46 offen offset:2048\n"
  "v_mfma_f32_16x16x32_f16 a[72:75], v[84:87], v[148:151], a[72:75]\n"
  "s_add_u32 s58, s58, 1\n"
  "v_mfma_f32_16x16x32_f16 a[104:107], v[92:95], v[148:151], a[104:107]\n"
  "s_add_i32 s87, s87, s77\n"
  "s_nop 3\n"
  "v_mfma_f32_16x16x32_f16 a[104:107], v[88:91], v[144:147], a[104:107]\n"
  "ds_read_b128 v[140:143], v243 offset:13312\n"
  "v_mfma_f32_16x16x32_f16 a[136:139], v[96:99], v[144:147], a[136:139]\n"
  "s_max_i32 s87, s87, 0\n"
  "v_mfma_f32_16x16x32_f16 a[136:139], v[100:103], v[148:151], a[136:139]\n"
  "buffer_load_dwordx4 v[36:39], v240, s[20:23], s74 offen offset:2048\n"
  "v_mfma_f32_16x16x32_f16 a[168:171], v[108:111], v[148:151], a[168:171]\n"
  "s_min_i32 s87, s87, 6\n"
  "s_nop 3\n"
  "v_mfma_f32_16x16x32_f16 a[168:171], v[104:107], v[144:147], a[168:171]\n"
  "v_mfma_f32_16x16x32_f16 a[200:203], v[112:115], v[144:147], a[200:203]\n"
  "s_add_u32 s79, s59, 3\ns_lshr_b32 s80, s79, 1\n"
  "v_mfma_f32_16x16x32_f16 a[200:203], v[116:119], v[148:151], a[200:203]\n"
  "v_mfma_f32_16x16x32_f16 a[232:235], v[124:127], v[148:151], a[232:235]\n"
  "buffer_load_dwordx4 v[40:43], v240, s[20:23], s46 offen offset:2560\n"
  "v_readlane_b32 s80, v249, s80\n"
  "s_nop 3\n"
  "v_mfma_f32_16x16x32_f16 a[232:235], v[120:123], v[144:147], a[232:235]\n"
  "s_waitcnt lgkmcnt(4)\n"
  "v_mfma_f32_16x16x32_f16 a[12:15], v[64:67], v[152:155], a[12:15]\n"
  "s_lshl_b32 s80, s80, 17\n"
  "v_mfma_f32_16x16x32_f16 a[12:15], v[68:71], v[156:159], a[12:15]\n"
  "ds_read_b128 v[144:147], v243 offset:6144\n"
  "v_mfma_f32_16x16x32_f16 a[44:47], v[76:79], v[156:159], a[44:47]\n"
  "buffer_load_dwordx4 v[44:47], v241, s[24:27], s74 offen offset:2560\n"
  "s_cmp_lt_u32 s79, 64\ns_cselect_b32 s60, s80, s84\n"
  "s_nop 3\n"
  "v_mfma_f32_16x16x32_f16 a[44:47], v[72:75], v[152:155], a[44:47]\n"
  "v_mfma_f32_16x16x32_f16 a[76:79], v[80:83], v[152:155], a[76:79]\n"
  "v_mfma_f32_16x16x32_f16 a[76:79], v[84:87], v[156:159], a[76:79]\n"
  "v_mfma_f32_16x16x32_f16 a[108:111], v[92:95], v[156:159], a[108:111]\n"
  "v_mfma_f32_16x16x32_f16 a[108:111], v[88:91], v[152:155], a[108:111]\n"
  "buffer_load_dwordx4 v[48:51], v240, s[20:23], s46 offen offset:3072\n"
  "ds_read_b128 v[148:151], v243 offset:14336\n"
  "s_nop 3\n"
  "v_mfma_f32_16x16x32_f16 a[140:143], v[96:99], v[152:155], a[140:143]\n"
  "v_mfma_f32_16x16x32_f16 a[140:143], v[100:103], v[156:159], a[140:143]\n"
  "v_mfma_f32_16x16x32_f16 a[172:175], v[108:111], v[156:159], a[172:175]\n"
  "v_mfma_f32_16x16x32_f16 a[172:175], v[104:107], v[152:155], a[172:175]\n"
  "s_nop 3\n"
  "v_mfma_f32_16x16x32_f16 a[204:207], v[112:115], v[152:155], a[204:207]\n"
  "buffer_load_dwordx4 v[52:55], v241, s[24:27], s74 offen offset:3072\n"
  "v_mfma_f32_16x16x32_f16 a[204:207], v[116:119], v[156:159], a[204:207]\n"
  "v_mfma_f32_16x16x32_f16 a[236:239], v[124:127], v[156:159], a[236:239]\n"
  "v_mfma_f32_16x16x32_f16 a[236:239], v[120:123], v[152:155], a[236:239]\n"
  "ds_read_b128 v[152:155], v243 offset:7168\n"
  "s_nop 3\n"
  "s_waitcnt lgkmcnt(5)\n"
  "v_mfma_f32_16x16x32_f16 a[16:19], v[64:67], v[128:131], a[16:19]\n"
  "buffer_load_dwordx4 v[56:59], v241, s[24:27], s46 offen offset:3584\n"
  "v_mfma_f32_16x16x32_f16 a[16:19], v[68:71], v[132:135], a[16:19]\n"
  "v_mfma_f32_16x16x32_f16 a[48:51], v[76:79], v[132:135], a[48:51]\n"
  "v_mfma_f32_16x16x32_f16 a[48:51], v[72:75], v[128:131], a[48:51]\n"
  "s_nop 3\n"
  "v_mfma_f32_16x16x32_f16 a[80:83], v[80:83], v[128:131], a[80:83]\n"
  "v_mfma_f32_16x16x32_f16 a[80:83], v[84:87], v[132:135], a[80:83]\n"
  "buffer_load_dwordx4 v[60:63], v240, s[20:23], s74 offen offset:3584\n"
  "v_mfma_f32_16x16x32_f16 a[112:115], v[92:95], v[132:135], a[112:115]\n"
  "v_mfma_f32_16x16x32_f16 a[112:115], v[88:91], v[128:131], a[112:115]\n"
  "s_barrier\n"
  "buffer_load_dword v192, v245, s[20:23], s60 offen\n"
  "s_nop 3\n"
  "v_mfma_f32_16x16x32_f16 a[144:147], v[96:99], v[128:131], a[144:147]\n"
  "buffer_load_dwordx4 v[192:195], v242, s[28:31], 0 offen nt\n"
  "v_mfma_f32_16x16x32_f16 a[144:147], v[100:103], v[132:135], a[144:147]\n"
  "v_mfma_f32_16x16x32_f16 a[176:179], v[108:111], v[132:135], a[176:179]\n"
  "v_mfma_f32_16x16x32_f16 a[176:179], v[104:107], v[128:131], a[176:179]\n"
  "v_mfma_f32_16x16x32_f16 a[208:211], v[112:115], v[128:131], a[208:211]\n"
  "buffer_load_dwordx4 v[196:199], v242, s[28:31], s39 offen nt\n"
  "s_nop 3\n"
  "v_mfma_f32_16x16x32_f16 a[208:211], v[116:119], v[132:135], a[208:211]\n"
  "ds_read_b128 v[156:159], v243 offset:15360\n"
  "s_waitcnt vmcnt(19)\n"
  "v_cvt_pk_f16_f32 v224, v160, v164\n"
  "v_mfma_f32_16x16x32_f16 a[240:243], v[124:127], v[132:135], a[240:243]\n"
  "v_add_u32 v243, 0x4000, v243\n"
  "v_and_b32 v243, 0x1ffff, v243\n"
  "v_mfma_f32_16x16x32_f16 a[240:243], v[120:123], v[128:131], a[240:243]\n"
  "v_cvt_pk_f16_f32 v225, v168, v172\n"
  "s_waitcnt lgkmcnt(4)\n"
  "v_mfma_f32_16x16x32_f16 a[20:23], v[64:67], v[136:139], a[20:23]\n"
  "buffer_load_dwordx4 v[200:203], v242, s[28:31], s40 offen nt\n"
  "s_nop 3\n"
  "v_mfma_f32_16x16x32_f16 a[20:23], v[68:71], v[140:143], a[20:23]\n"
  "ds_read_b128 v[128:131], v243 offset:0\n"
  "v_cvt_pk_f16_f32 v226, v176, v180\n"
  "v_mfma_f32_16x16x32_f16 a[52:55], v[76:79], v[140:143], a[52:55]\n"
  "v_mfma_f32_16x16x32_f16 a[52:55], v[72:75], v[136:139], a[52:55]\n"
  "v_mfma_f32_16x16x32_f16 a[84:87], v[80:83], v[136:139], a[84:87]\n"
  "buffer_load_dwordx4 v[204:207], v242, s[28:31], s41 offen nt\n"
  "v_cvt_pk_f16_f32 v227, v184, v188\n"
  "s_nop 3\n"
  "v_mfma_f32_16x16x32_f16 a[84:87], v[84:87], v[140:143], a[84:87]\n"
  "v_mfma_f32_16x16x32_f16 a[116:119], v[92:95], v[140:143], a[116:119]\n"
  "ds_write_b128 v244, v[224:227] offset:0\n"
  "v_mfma_f32_16x16x32_f16 a[116:119], v[88:91], v[136:139], a[116:119]\n"
  "ds_read_b128 v[132:135], v243 offset:8192\n"
  "v_mfma_f32_16x16x32_f16 a[148:151], v[96:99], v[136:139], a[148:151]\n"
  "buffer_load_dwordx4 v[208:211], v242, s[28:31], s42 offen nt\n"
  "s_nop 3\n"
  "v_mfma_f32_16x16x32_f16 a[148:151], v[100:103], v[140:143], a[148:151]\n"
  "v_cvt_pk_f16_f32 v228, v161, v165\n"
  "v_mfma_f32_16x16x32_f16 a[180:183], v[108:111], v[140:143], a[180:183]\n"
  "v_mfma_f32_16x16x32_f16 a[180:183], v[104:107], v[136:139], a[180:183]\n"
  "v_cvt_pk_f16_f32 v229, v169, v173\n"
  "v_mfma_f32_16x16x32_f16 a[212:215], v[112:115], v[136:139], a[212:215]\n"
  "buffer_load_dwordx4 v[212:215], v242, s[28:31], s43 offen nt\n"
  "v_mfma_f32_16x16x32_f16 a[212:215], v[116:119], v[140:143], a[212:215]\n"
  "v_cvt_pk_f16_f32 v230, v177, v181\n"
  "s_nop 3\n"
  "v_mfma_f32_16x16x32_f16 a[244:247], v[124:127], v[140:143], a[244:247]\n"
  "v_mfma_f32_16x16x32_f16 a[244:247], v[120:123], v[136:139], a[244:247]\n"
  "s_waitcnt lgkmcnt(5)\n"
  "v_mfma_f32_16x16x32_f16 a[24:27], v[64:67], v[144:147], a[24:27]\n"
  "buffer_load_dwordx4 v[216:219], v242, s[28:31], s44 offen nt\n"
  "v_cvt_pk_f16_f32 v231, v185, v189\n"
  "v_mfma_f32_16x16x32_f16 a[24:27], v[68:71], v[148:151], a[24:27]\n"
  "ds_read_b128 v[136:139], v243 offset:1024\n"
  "s_nop 3\n"
  "v_mfma_f32_16x16x32_f16 a[56:59], v[76:79], v[148:151], a[56:59]\n"
  "ds_write_b128 v244, v[228:231] offset:1024\n"
  "v_mfma_f32_16x16x32_f16 a[56:59], v[72:75], v[144:147], a[56:59]\n"
  "v_mfma_f32_16x16x32_f16 a[88:91], v[80:83], v[144:147], a[88:91]\n"
  "buffer_load_dwordx4 v[220:223], v242, s[28:31], s45 offen nt\n"
  "v_mfma_f32_16x16x32_f16 a[88:91], v[84:87], v[148:151], a[88:91]\n"
  "v_cvt_pk_f16_f32 v224, v162, v166\n"
  "s_nop 3\n"
  "v_mfma_f32_16x16x32_f16 a[120:123], v[92:95], v[148:151], a[120:123]\n"
  "s_memrealtime s[52:53]\n"
  "v_mfma_f32_16x16x32_f16 a[120:123], v[88:91], v[144:147], a[120:123]\n"
  "ds_read_b128 v[140:143], v243 offset:9216\n"
  "v_cvt_pk_f16_f32 v225, v170, v174\n"
  "v_mfma_f32_16x16x32_f16 a[152:155], v[96:99], v[144:147], a[152:155]\n"
  "v_mfma_f32_16x16x32_f16 a[152:155], v[100:103], v[148:151], a[152:155]\n"
  "v_cvt_pk_f16_f32 v226, v178, v182\n"
  "s_nop 3\n"
  "v_mfma_f32_16x16x32_f16 a[184:187], v[108:111], v[148:151], a[184:187]\n"
  "s_add_u32 s79, s59, 2\ns_lshr_b32 s80, s79, 1\n"
  "v_mfma_f32_16x16x32_f16 a[184:187], v[104:107], v[144:147], a[184:187]\n"
  "s_waitcnt lgkmcnt(0)\n"
  "s_sub_u32 s82, s52, s86\ns_cmp_lt_i32 s82, 0\ns_cbranch_scc0 PGs1_%=\nPWs1_%=:\ns_sleep 1\ns_memrealtime s[54:55]\ns_waitcnt lgkmcnt(0)\ns_sub_u32 s88, s54, s86\ns_cmp_lt_i32 s88, 0\ns_cbranch_scc1 PWs1_%=\nPGs1_%=:\n"
  "v_mfma_f32_16x16x32_f16 a[216:219], v[112:115], v[144:147], a[216:219]\n"
  "s_cmp_lt_u32 s82, 44\ns_cselect_b32 s77, 1, -1\n"
  "s_cmp_gt_i32 s82, 125\ns_cselect_b32 s66, 2, 0\ns_max_u32 s75, s75, s66\n"
  "v_cvt_pk_f16_f32 v227, v186, v190\n"
  "v_readlane_b32 s80, v249, s80\n"
  "v_mfma_f32_16x16x32_f16 a[216:219], v[116:119], v[148:151], a[216:219]\n"
  "s_sub_u32 s82, s82, -131\n"
  "v_mfma_f32_16x16x32_f16 a[248:251], v[124:127], v[148:151], a[248:251]\n"
  "ds_write_b128 v244, v[224:227] offset:2048\n"
  "s_lshl_b32 s80, s80, 17\ns_add_u32 s80, s80, 0x10000\n"
  "s_nop 3\n"
  "v_mfma_f32_16x16x32_f16 a[248:251], v[120:123], v[144:147], a[248:251]\n"
  "v_mfma_f32_16x16x32_f16 a[28:31], v[64:67], v[152:155], a[28:31]\n"
  "s_cmp_lt_u32 s79, 64\ns_cselect_b32 s46, s80, s84\n"
  "v_mfma_f32_16x16x32_f16 a[28:31], v[68:71], v[156:159], a[28:31]\n"
  "ds_read_b128 v[144:147], v243 offset:2048\n"
  "v_cvt_pk_f16_f32 v228, v163, v167\n"
  "v_mfma_f32_16x16x32_f16 a[60:63], v[76:79], v[156:159], a[60:63]\n"
  "s_max_i32 s82, s82, 151\n"
  "s_nop 3\n"
  "v_mfma_f32_16x16x32_f16 a[60:63], v[72:75], v[152:155], a[60:63]\n"
  "v_cvt_pk_f16_f32 v229, v171, v175\n"
  "v_mfma_f32_16x16x32_f16 a[92:95], v[80:83], v[152:155], a[92:95]\n"
  "s_add_u32 s46, s46, s83\n"
  "v_mfma_f32_16x16x32_f16 a[92:95], v[84:87], v[156:159], a[92:95]\n"
  "v_cvt_pk_f16_f32 v230, v179, v183\n"
  "v_mfma_f32_16x16x32_f16 a[124:127], v[92:95], v[156:159], a[124:127]\n"
  "s_add_u32 s74, s46, 0x4000\n"
  "s_nop 3\n"
  "v_mfma_f32_16x16x32_f16 a[124:127], v[88:91], v[152:155], a[124:127]\n"
  "ds_read_b128 v[148:151], v243 offset:10240\n"
  "v_mfma_f32_16x16x32_f16 a[156:159], v[96:99], v[152:155], a[156:159]\n"
  "v_cvt_pk_f16_f32 v231, v187, v191\n"
  "s_add_u32 s59, s59, 1\n"
  "v_mfma_f32_16x16x32_f16 a[156:159], v[100:103], v[156:159], a[156:159]\n"
  "s_add_u32 s86, s86, s82\n"
  "v_mfma_f32_16x16x32_f16 a[188:191], v[108:111], v[156:159], a[188:191]\n"
  "ds_write_b128 v244, v[228:231] offset:3072\n"
  "s_nop 3\n"
  "v_mfma_f32_16x16x32_f16 a[188:191], v[104:107], v[152:155], a[188:191]\n"
  "v_mfma_f32_16x16x32_f16 a[220:223], v[112:115], v[152:155], a[220:223]\n"
  "v_mfma_f32_16x16x32_f16 a[220:223], v[116:119], v[156:159], a[220:223]\n"
  "v_mfma_f32_16x16x32_f16 a[252:255], v[124:127], v[156:159], a[252:255]\n"
  "v_add_u32 v244, 0x4000, v244\n"
  "v_and_b32 v244, 0x1ffff, v244\n"
  "v_mfma_f32_16x16x32_f16 a[252:255], v[120:123], v[152:155], a[252:255]\n"
  "s_branch LSEL_%=\n"
  "XLOOP_%=:\n"
  "s_sub_u32 s75, s75, 1\n"
  "XLP2_%=:\n"
  "buffer_load_dwordx4 v[64:67], v240, s[20:23], s46 offen offset:0\n"
  "s_waitcnt vmcnt(10)\n"
  "v_mfma_f32_16x16x32_f16 a[0:3], v[0:3], v[128:131], a[0:3]\n"
  "s_add_u32 s85, s85, 1\n"
  "s_nop 3\n"
  "v_mfma_f32_16x16x32_f16 a[0:3], v[4:7], v[132:135], a[0:3]\n"
  "ds_read_b128 v[152:155], v243 offset:3072\n"
  "s_nop 3\n"
  "v_mfma_f32_16x16x32_f16 a[32:35], v[12:15], v[132:135], a[32:35]\n"
  "s_min_u32 s85, s85, 63\n"
  "s_nop 3\n"
  "v_mfma_f32_16x16x32_f16 a[32:35], v[8:11], v[128:131], a[32:35]\n"
  "v_readlane_b32 s88, v247, s85\n"
  "v_mfma_f32_16x16x32_f16 a[64:67], v[16:19], v[128:131], a[64:67]\n"
  "buffer_load_dwordx4 v[68:71], v241, s[24:27], s74 offen offset:0\n"
  "s_nop 3\n"
  "v_mfma_f32_16x16x32_f16 a[64:67], v[20:23], v[132:135], a[64:67]\n"
  "s_add_u32 s88, s88, s89\n"
  "s_nop 3\n"
  "v_mfma_f32_16x16x32_f16 a[96:99], v[28:31], v[132:135], a[96:99]\n"
  "s_max_u32 s86, s86, s88\n"
  "v_mfma_f32_16x16x32_f16 a[96:99], v[24:27], v[128:131], a[96:99]\n"
  "ds_read_b128 v[156:159], v243 offset:11264\n"
  "s_nop 3\n"
  "v_mfma_f32_16x16x32_f16 a[128:131], v[32:35], v[128:131], a[128:131]\n"
  "buffer_load_dwordx4 v[72:75], v241, s[24:27], s46 offen offset:512\n"
  "s_nop 3\n"
  "v_mfma_f32_16x16x32_f16 a[128:131], v[36:39], v[132:135], a[128:131]\n"
  "s_add_u32 s90, s90, 151\n"
  "v_mfma_f32_16x16x32_f16 a[160:163], v[44:47], v[132:135], a[160:163]\n"
  "s_max_u32 s90, s90, s88\n"
  "s_nop 3\n"
  "v_mfma_f32_16x16x32_f16 a[160:163], v[40:43], v[128:131], a[160:163]\n"
  "s_sub_u32 s65, s86, s90\n"
  "s_nop 3\n"
  "v_mfma_f32_16x16x32_f16 a[192:195], v[48:51], v[128:131], a[192:195]\n"
  "s_cmp_gt_i32 s65, 150\ns_cselect_b32 s65, 5, 0\ns_cselect_b32 s71, 0, 3\n"
  "s_nop 3\n"
  "v_mfma_f32_16x16x32_f16 a[192:195], v[52:55], v[132:135], a[192:195]\n"
  "buffer_load_dwordx4 v[76:79], v240, s[20:23], s74 offen offset:512\n"
  "v_mfma_f32_16x16x32_f16 a[224:227], v[60:63], v[132:135], a[224:227]\n"
  "s_sub_u32 s86, s86, s65\n"
  "s_nop 3\n"
  "v_mfma_f32_16x16x32_f16 a[224:227], v[56:59], v[128:131], a[224:227]\n"
  "s_nop 3\n"
  "v_mfma_f32_16x16x32_f16 a[4:7], v[0:3], v[136:139], a[4:7]\n"
  "s_add_i32 s87, s87, s77\n"
  "v_mfma_f32_16x16x32_f16 a[4:7], v[4:7], v[140:143], a[4:7]\n"
  "ds_read_b128 v[128:131], v243 offset:4096\n"
  "s_nop 3\n"
  "v_mfma_f32_16x16x32_f16 a[36:39], v[12:15], v[140:143], a[36:39]\n"
  "buffer_load_dwordx4 v[80:83], v241, s[24:27], s46 offen offset:1024\n"
  "s_nop 3\n"
  "v_mfma_f32_16x16x32_f16 a[36:39], v[8:11], v[136:139], a[36:39]\n"
  "s_max_i32 s87, s87, 0\n"
  "v_mfma_f32_16x16x32_f16 a[68:71], v[16:19], v[136:139], a[68:71]\n"
  "s_lshr_b32 s79, s58, 1\n"
  "s_nop 3\n"
  "v_mfma_f32_16x16x32_f16 a[68:71], v[20:23], v[140:143], a[68:71]\n"
  "s_min_i32 s87, s87, 6\n"
  "s_nop 3\n"
  "v_mfma_f32_16x16x32_f16 a[100:103], v[28:31], v[140:143], a[100:103]\n"
  "buffer_load_dwordx4 v[84:87], v240, s[20:23], s74 offen offset:1024\n"
  "v_readlane_b32 s79, v249, s79\n"
  "v_mfma_f32_16x16x32_f16 a[100:103], v[24:27], v[136:139], a[100:103]\n"
  "ds_read_b128 v[132:135], v243 offset:12288\n"
  "s_nop 3\n"
  "v_mfma_f32_16x16x32_f16 a[132:135], v[32:35], v[136:139], a[132:135]\n"
  "s_and_b32 s80, s58, 1\ns_lshl_b32 s80, s80, 23\n"
  "s_nop 3\n"
  "v_mfma_f32_16x16x32_f16 a[132:135], v[36:39], v[140:143], a[132:135]\n"
  "s_nop 3\n"
  "v_mfma_f32_16x16x32_f16 a[164:167], v[44:47], v[140:143], a[164:167]\n"
  "s_mov_b32 s81, s79\n"
  "v_mfma_f32_16x16x32_f16 a[164:167], v[40:43], v[136:139], a[164:167]\n"
  "buffer_load_dwordx4 v[88:91], v240, s[20:23], s46 offen offset:1536\n"
  "s_nop 3\n"
  "v_mfma_f32_16x16x32_f16 a[196:199], v[48:51], v[136:139], a[196:199]\n"
  "s_nop 3\n"
  "v_mfma_f32_16x16x32_f16 a[196:199], v[52:55], v[140:143], a[196:199]\n"
  "v_readlane_b32 s79, v248, s81\n"
  "v_mfma_f32_16x16x32_f16 a[228:231], v[60:63], v[140:143], a[228:231]\n"
  "s_nop 3\n"
  "v_mfma_f32_16x16x32_f16 a[228:231], v[56:59], v[136:139], a[228:231]\n"
  "s_add_u32 s79, s79, s80\n"
  "s_nop 3\n"
  "s_waitcnt lgkmcnt(5)\n"
  "v_mfma_f32_16x16x32_f16 a[8:11], v[0:3], v[144:147], a[8:11]\n"
  "buffer_load_dwordx4 v[92:95], v241, s[24:27], s74 offen offset:1536\n"
  "v_mfma_f32_16x16x32_f16 a[8:11], v[4:7], v[148:151], a[8:11]\n"
  "ds_read_b128 v[136:139], v243 offset:5120\n"
  "s_add_u32 s28, s48, s79\ns_addc_u32 s29, s49, 0\n"
  "s_nop 3\n"
  "v_mfma_f32_16x16x32_f16 a[40:43], v[12:15], v[148:151], a[40:43]\n"
  "s_nop 3\n"
  "v_mfma_f32_16x16x32_f16 a[40:43], v[8:11], v[144:147], a[40:43]\n"
  "s_cmp_lt_u32 s58, 64\ns_cselect_b32 s30, s69, 0\n"
  "v_mfma_f32_16x16x32_f16 a[72:75], v[16:19], v[144:147], a[72:75]\n"
  "buffer_load_dwordx4 v[96:99], v241, s[24:27], s46 offen offset:2048\n"
  "s_nop 3\n"
  "v_mfma_f32_16x16x32_f16 a[72:75], v[20:23], v[148:151], a[72:75]\n"
  "s_add_u32 s58, s58, 1\n"
  "s_nop 3\n"
  "v_mfma_f32_16x16x32_f16 a[104:107], v[28:31], v[148:151], a[104:107]\n"
  "s_nop 3\n"
  "v_mfma_f32_16x16x32_f16 a[104:107], v[24:27], v[144:147], a[104:107]\n"
  "ds_read_b128 v[140:143], v243 offset:13312\n"
  "v_mfma_f32_16x16x32_f16 a[136:139], v[32:35], v[144:147], a[136:139]\n"
  "s_nop 3\n"
  "v_mfma_f32_16x16x32_f16 a[136:139], v[36:39], v[148:151], a[136:139]\n"
  "buffer_load_dwordx4 v[100:103], v240, s[20:23], s74 offen offset:2048\n"
  "s_nop 3\n"
  "v_mfma_f32_16x16x32_f16 a[168:171], v[44:47], v[148:151], a[168:171]\n"
  "v_mfma_f32_16x16x32_f16 a[168:171], v[40:43], v[144:147], a[168:171]\n"
  "s_nop 3\n"
  "v_mfma_f32_16x16x32_f16 a[200:203], v[48:51], v[144:147], a[200:203]\n"
  "s_add_u32 s79, s59, 3\ns_lshr_b32 s80, s79, 1\n"
  "s_nop 3\n"
  "v_mfma_f32_16x16x32_f16 a[200:203], v[52:55], v[148:151], a[200:203]\n"
  "v_mfma_f32_16x16x32_f16 a[232:235], v[60:63], v[148:151], a[232:235]\n"
  "buffer_load_dwordx4 v[104:107], v240, s[20:23], s46 offen offset:2560\n"
  "v_readlane_b32 s80, v249, s80\n"
  "s_nop 3\n"
  "v_mfma_f32_16x16x32_f16 a[232:235], v[56:59], v[144:147], a[232:235]\n"
  "s_nop 3\n"
  "s_waitcnt lgkmcnt(4)\n"
  "v_mfma_f32_16x16x32_f16 a[12:15], v[0:3], v[152:155], a[12:15]\n"
  "s_lshl_b32 s80, s80, 17\ns_add_u32 s80, s80, 0x10000\n"
  "v_mfma_f32_16x16x32_f16 a[12:15], v[4:7], v[156:159], a[12:15]\n"
  "ds_read_b128 v[144:147], v243 offset:6144\n"
  "s_nop 3\n"
  "v_mfma_f32_16x16x32_f16 a[44:47], v[12:15], v[156:159], a[44:47]\n"
  "buffer_load_dwordx4 v[108:111], v241, s[24:27], s74 offen offset:2560\n"
  "s_cmp_lt_u32 s79, 64\ns_cselect_b32 s60, s80, s84\n"
  "s_nop 3\n"
  "v_mfma_f32_16x16x32_f16 a[44:47], v[8:11], v[152:155], a[44:47]\n"
  "s_nop 3\n"
  "v_mfma_f32_16x16x32_f16 a[76:79], v[16:19], v[152:155], a[76:79]\n"
  "v_mfma_f32_16x16x32_f16 a[76:79], v[20:23], v[156:159], a[76:79]\n"
  "s_nop 3\n"
  "v_mfma_f32_16x16x32_f16 a[108:111], v[28:31], v[156:159], a[108:111]\n"
  "s_nop 3\n"
  "v_mfma_f32_16x16x32_f16 a[108:111], v[24:27], v[152:155], a[108:111]\n"
  "buffer_load_dwordx4 v[112:115], v240, s[20:23], s46 offen offset:3072\n"
  "ds_read_b128 v[148:151], v243 offset:14336\n"
  "v_mfma_f32_16x16x32_f16 a[140:143], v[32:35], v[152:155], a[140:143]\n"
  "s_nop 3\n"
  "v_mfma_f32_16x16x32_f16 a[140:143], v[36:39], v[156:159], a[140:143]\n"
  "s_nop 3\n"
  "v_mfma_f32_16x16x32_f16 a[172:175], v[44:47], v[156:159], a[172:175]\n"
  "v_mfma_f32_16x16x32_f16 a[172:175], v[40:43], v[152:155], a[172:175]\n"
  "s_nop 3\n"
  "v_mfma_f32_16x16x32_f16 a[204:207], v[48:51], v[152:155], a[204:207]\n"
  "buffer_load_dwordx4 v[116:119], v241, s[24:27], s74 offen offset:3072\n"
  "s_nop 3\n"
  "v_mfma_f32_16x16x32_f16 a[204:207], v[52:55], v[156:159], a[204:207]\n"
  "v_mfma_f32_16x16x32_f16 a[236:239], v[60:63], v[156:159], a[236:239]\n"
  "s_nop 3\n"
  "v_mfma_f32_16x16x32_f16 a[236:239], v[56:59], v[152:155], a[236:239]\n"
  "ds_read_b128 v[152:155], v243 offset:7168\n"
  "s_nop 3\n"
  "s_waitcnt lgkmcnt(5)\n"
  "v_mfma_f32_16x16x32_f16 a[16:19], v[0:3], v[128:131], a[16:19]\n"
  "buffer_load_dwordx4 v[120:123], v241, s[24:27], s46 offen offset:3584\n"
  "s_nop 3\n"
  "v_mfma_f32_16x16x32_f16 a[16:19], v[4:7], v[132:135], a[16:19]\n"
  "v_mfma_f32_16x16x32_f16 a[48:51], v[12:15], v[132:135], a[48:51]\n"
  "s_nop 3\n"
  "v_mfma_f32_16x16x32_f16 a[48:51], v[8:11], v[128:131], a[48:51]\n"
  "s_nop 3\n"
  "v_mfma_f32_16x16x32_f16 a[80:83], v[16:19], v[128:131], a[80:83]\n"
  "v_mfma_f32_16x16x32_f16 a[80:83], v[20:23], v[132:135], a[80:83]\n"
  "buffer_load_dwordx4 v[124:127], v240, s[20:23], s74 offen offset:3584\n"
  "s_nop 3\n"
  "v_mfma_f32_16x16x32_f16 a[112:115], v[28:31], v[132:135], a[112:115]\n"
  "s_nop 3\n"
  "v_mfma_f32_16x16x32_f16 a[112:115], v[24:27], v[128:131], a[112:115]\n"
  "s_barrier\n"
  "buffer_load_dword v160, v245, s[20:23], s60 offen\n"
  "v_mfma_f32_16x16x32_f16 a[144:147], v[32:35], v[128:131], a[144:147]\n"
  "buffer_load_dwordx4 v[160:163], v242, s[28:31], 0 offen nt\n"
  "s_nop 3\n"
  "v_mfma_f32_16x16x32_f16 a[144:147], v[36:39], v[132:135], a[144:147]\n"
  "s_nop 3\n"
  "v_mfma_f32_16x16x32_f16 a[176:179], v[44:47], v[132:135], a[176:179]\n"
  "v_mfma_f32_16x16x32_f16 a[176:179], v[40:43], v[128:131], a[176:179]\n"
  "s_nop 3\n"
  "v_mfma_f32_16x16x32_f16 a[208:211], v[48:51], v[128:131], a[208:211]\n"
  "buffer_load_dwordx4 v[164:167], v242, s[28:31], s39 offen nt\n"
  "s_nop 3\n"
  "v_mfma_f32_16x16x32_f16 a[208:211], v[52:55], v[132:135], a[208:211]\n"
  "ds_read_b128 v[156:159], v243 offset:15360\n"
  "s_waitcnt vmcnt(19)\n"
  "v_cvt_pk_f16_f32 v224, v192, v196\n"
  "s_nop 3\n"
  "v_mfma_f32_16x16x32_f16 a[240:243], v[60:63], v[132:135], a[240:243]\n"
  "v_add_u32 v243, 0x4000, v243\n"
  "v_and_b32 v243, 0x1ffff, v243\n"
  "v_mfma_f32_16x16x32_f16 a[240:243], v[56:59], v[128:131], a[240:243]\n"
  "v_cvt_pk_f16_f32 v225, v200, v204\n"
  "s_nop 3\n"
  "s_waitcnt lgkmcnt(4)\n"
  "v_mfma_f32_16x16x32_f16 a[20:23], v[0:3], v[136:139], a[20:23]\n"
  "buffer_load_dwordx4 v[168:171], v242, s[28:31], s40 offen nt\n"
  "s_nop 3\n"
  "v_mfma_f32_16x16x32_f16 a[20:23], v[4:7], v[140:143], a[20:23]\n"
  "ds_read_b128 v[128:131], v243 offset:0\n"
  "v_cvt_pk_f16_f32 v226, v208, v212\n"
  "v_mfma_f32_16x16x32_f16 a[52:55], v[12:15], v[140:143], a[52:55]\n"
  "s_nop 3\n"
  "v_mfma_f32_16x16x32_f16 a[52:55], v[8:11], v[136:139], a[52:55]\n"
  "s_nop 3\n"
  "v_mfma_f32_16x16x32_f16 a[84:87], v[16:19], v[136:139], a[84:87]\n"
  "buffer_load_dwordx4 v[172:175], v242, s[28:31], s41 offen nt\n"
  "v_cvt_pk_f16_f32 v227, v216, v220\n"
  "v_mfma_f32_16x16x32_f16 a[84:87], v[20:23], v[140:143], a[84:87]\n"
  "s_nop 3\n"
  "v_mfma_f32_16x16x32_f16 a[116:119], v[28:31], v[140:143], a[116:119]\n"
  "ds_write_b128 v244, v[224:227] offset:0\n"
  "s_nop 3\n"
  "v_mfma_f32_16x16x32_f16 a[116:119], v[24:27], v[136:139], a[116:119]\n"
  "ds_read_b128 v[132:135], v243 offset:8192\n"
  "v_mfma_f32_16x16x32_f16 a[148:151], v[32:35], v[136:139], a[148:151]\n"
  "buffer_load_dwordx4 v[176:179], v242, s[28:31], s42 offen nt\n"
  "s_nop 3\n"
  "v_mfma_f32_16x16x32_f16 a[148:151], v[36:39], v[140:143], a[148:151]\n"
  "v_cvt_pk_f16_f32 v228, v193, v197\n"
  "s_nop 3\n"
  "v_mfma_f32_16x16x32_f16 a[180:183], v[44:47], v[140:143], a[180:183]\n"
  "s_nop 3\n"
  "v_mfma_f32_16x16x32_f16 a[180:183], v[40:43], v[136:139], a[180:183]\n"
  "v_cvt_pk_f16_f32 v229, v201, v205\n"
  "v_mfma_f32_16x16x32_f16 a[212:215], v[48:51], v[136:139], a[212:215]\n"
  "buffer_load_dwordx4 v[180:183], v242, s[28:31], s43 offen nt\n"
  "s_nop 3\n"
  "v_mfma_f32_16x16x32_f16 a[212:215], v[52:55], v[140:143], a[212:215]\n"
  "v_cvt_pk_f16_f32 v230, v209, v213\n"
  "s_nop 3\n"
  "v_mfma_f32_16x16x32_f16 a[244:247], v[60:63], v[140:143], a[244:247]\n"
  "v_mfma_f32_16x16x32_f16 a[244:247], v[56:59], v[136:139], a[244:247]\n"
  "s_nop 3\n"
  "s_waitcnt lgkmcnt(5)\n"
  "v_mfma_f32_16x16x32_f16 a[24:27], v[0:3], v[144:147], a[24:27]\n"
  "buffer_load_dwordx4 v[184:187], v242, s[28:31], s44 offen nt\n"
  "v_cvt_pk_f16_f32 v231, v217, v221\n"
  "s_nop 3\n"
  "v_mfma_f32_16x16x32_f16 a[24:27], v[4:7], v[148:151], a[24:27]\n"
  "ds_read_b128 v[136:139], v243 offset:1024\n"
  "v_mfma_f32_16x16x32_f16 a[56:59], v[12:15], v[148:151], a[56:59]\n"
  "ds_write_b128 v244, v[228:231] offset:1024\n"
  "s_nop 3\n"
  "v_mfma_f32_16x16x32_f16 a[56:59], v[8:11], v[144:147], a[56:59]\n"
  "s_nop 3\n"
  "v_mfma_f32_16x16x32_f16 a[88:91], v[16:19], v[144:147], a[88:91]\n"
  "buffer_load_dwordx4 v[188:191], v242, s[28:31], s45 offen nt\n"
  "v_mfma_f32_16x16x32_f16 a[88:91], v[20:23], v[148:151], a[88:91]\n"
  "v_cvt_pk_f16_f32 v224, v194, v198\n"
  "s_nop 3\n"
  "v_mfma_f32_16x16x32_f16 a[120:123], v[28:31], v[148:151], a[120:123]\n"
  "s_memrealtime s[52:53]\n"
  "s_nop 3\n"
  "v_mfma_f32_16x16x32_f16 a[120:123], v[24:27], v[144:147], a[120:123]\n"
  "ds_read_b128 v[140:143], v243 offset:9216\n"
  "v_cvt_pk_f16_f32 v225, v202, v206\n"
  "s_nop 3\n"
  "v_mfma_f32_16x16x32_f16 a[152:155], v[32:35], v[144:147], a[152:155]\n"
  "v_mfma_f32_16x16x32_f16 a[152:155], v[36:39], v[148:151], a[152:155]\n"
  "v_cvt_pk_f16_f32 v226, v210, v214\n"
  "s_nop 3\n"
  "v_mfma_f32_16x16x32_f16 a[184:187], v[44:47], v[148:151], a[184:187]\n"
  "s_add_u32 s79, s59, 2\ns_lshr_b32 s80, s79, 1\n"
  "s_nop 3\n"
  "v_mfma_f32_16x16x32_f16 a[184:187], v[40:43], v[144:147], a[184:187]\n"
  "s_waitcnt lgkmcnt(0)\n"
  "s_sub_u32 s82, s52, s86\ns_cmp_lt_i32 s82, 0\ns_cbranch_scc0 PGx0_%=\nPWx0_%=:\ns_sleep 1\ns_memrealtime s[54:55]\ns_waitcnt lgkmcnt(0)\ns_sub_u32 s88, s54, s86\ns_cmp_lt_i32 s88, 0\ns_cbranch_scc1 PWx0_%=\nPGx0_%=:\n"
  "v_mfma_f32_16x16x32_f16 a[216:219], v[48:51], v[144:147], a[216:219]\n"
  "s_cmp_lt_u32 s82, 44\ns_cselect_b32 s77, 0, -1\n"
  "v_cvt_pk_f16_f32 v227, v218, v222\n"
  "v_readlane_b32 s80, v249, s80\n"
  "s_nop 3\n"
  "v_mfma_f32_16x16x32_f16 a[216:219], v[52:55], v[148:151], a[216:219]\n"
  "s_sub_u32 s82, s82, -131\n"
  "s_nop 3\n"
  "v_mfma_f32_16x16x32_f16 a[248:251], v[60:63], v[148:151], a[248:251]\n"
  "ds_write_b128 v244, v[224:227] offset:2048\n"
  "s_lshl_b32 s80, s80, 17\n"
  "v_mfma_f32_16x16x32_f16 a[248:251], v[56:59], v[144:147], a[248:251]\n"
  "s_nop 3\n"
  "v_mfma_f32_16x16x32_f16 a[28:31], v[0:3], v[152:155], a[28:31]\n"
  "s_cmp_lt_u32 s79, 64\ns_cselect_b32 s46, s80, s84\n"
  "s_nop 3\n"
  "v_mfma_f32_16x16x32_f16 a[28:31], v[4:7], v[156:159], a[28:31]\n"
  "ds_read_b128 v[144:147], v243 offset:2048\n"
  "v_cvt_pk_f16_f32 v228, v195, v199\n"
  "v_mfma_f32_16x16x32_f16 a[60:63], v[12:15], v[156:159], a[60:63]\n"
  "s_max_i32 s82, s82, 151\n"
  "s_nop 3\n"
  "v_mfma_f32_16x16x32_f16 a[60:63], v[8:11], v[152:155], a[60:63]\n"
  "v_cvt_pk_f16_f32 v229, v203, v207\n"
  "s_nop 3\n"
  "v_mfma_f32_16x16x32_f16 a[92:95], v[16:19], v[152:155], a[92:95]\n"
  "s_add_u32 s46, s46, s83\n"
  "s_nop 3\n"
  "v_mfma_f32_16x16x32_f16 a[92:95], v[20:23], v[156:159], a[92:95]\n"
  "v_cvt_pk_f16_f32 v230, v211, v215\n"
  "v_mfma_f32_16x16x32_f16 a[124:127], v[28:31], v[156:159], a[124:127]\n"
  "s_add_u32 s74, s46, 0x4000\n"
  "s_nop 3\n"
  "v_mfma_f32_16x16x32_f16 a[124:127], v[24:27], v[152:155], a[124:127]\n"
  "ds_read_b128 v[148:151], v243 offset:10240\n"
  "s_nop 3\n"
  "v_mfma_f32_16x16x32_f16 a[156:159], v[32:35], v[152:155], a[156:159]\n"
  "v_cvt_pk_f16_f32 v231, v219, v223\n"
  "s_add_u32 s59, s59, 1\n"
  "v_mfma_f32_16x16x32_f16 a[156:159], v[36:39], v[156:159], a[156:159]\n"
  "s_add_u32 s86, s86, s82\n"
  "s_nop 3\n"
  "v_mfma_f32_16x16x32_f16 a[188:191], v[44:47], v[156:159], a[188:191]\n"
  "ds_write_b128 v244, v[228:231] offset:3072\n"
  "s_nop 3\n"
  "v_mfma_f32_16x16x32_f16 a[188:191], v[40:43], v[152:155], a[188:191]\n"
  "v_mfma_f32_16x16x32_f16 a[220:223], v[48:51], v[152:155], a[220:223]\n"
  "s_nop 3\n"
  "v_mfma_f32_16x16x32_f16 a[220:223], v[52:55], v[156:159], a[220:223]\n"
  "s_nop 3\n"
  "v_mfma_f32_16x16x32_f16 a[252:255], v[60:63], v[156:159], a[252:255]\n"
  "v_add_u32 v244, 0x4000, v244\n"
  "v_and_b32 v244, 0x1ffff, v244\n"
  "v_mfma_f32_16x16x32_f16 a[252:255], v[56:59], v[152:155], a[252:255]\n"
  "buffer_load_dwordx4 v[0:3], v240, s[20:23], s46 offen offset:0\n"
  "s_waitcnt vmcnt(10)\n"
  "v_mfma_f32_16x16x32_f16 a[0:3], v[64:67], v[128:131], a[0:3]\n"
  "s_add_u32 s85, s85, 1\n"
  "s_nop 3\n"
  "v_mfma_f32_16x16x32_f16 a[0:3], v[68:71], v[132:135], a[0:3]\n"
  "ds_read_b128 v[152:155], v243 offset:3072\n"
  "s_nop 3\n"
  "v_mfma_f32_16x16x32_f16 a[32:35], v[76:79], v[132:135], a[32:35]\n"
  "s_min_u32 s85, s85, 63\n"
  "s_nop 3\n"
  "v_mfma_f32_16x16x32_f16 a[32:35], v[72:75], v[128:131], a[32:35]\n"
  "v_readlane_b32 s88, v247, s85\n"
  "v_mfma_f32_16x16x32_f16 a[64:67], v[80:83], v[128:131], a[64:67]\n"
  "buffer_load_dwordx4 v[4:7], v241, s[24:27], s74 offen offset:0\n"
  "s_nop 3\n"
  "v_mfma_f32_16x16x32_f16 a[64:67], v[84:87], v[132:135], a[64:67]\n"
  "s_add_u32 s88, s88, s89\n"
  "s_nop 3\n"
  "v_mfma_f32_16x16x32_f16 a[96:99], v[92:95], v[132:135], a[96:99]\n"
  "s_andn2_b32 s79, s70, s72\n"
  "v_mfma_f32_16x16x32_f16 a[96:99], v[88:91], v[128:131], a[96:99]\n"
  "ds_read_b128 v[156:159], v243 offset:11264\n"
  "s_nop 3\n"
  "v_mfma_f32_16x16x32_f16 a[128:131], v[96:99], v[128:131], a[128:131]\n"
  "buffer_load_dwordx4 v[8:11], v241, s[24:27], s46 offen offset:512\n"
  "s_nop 3\n"
  "v_mfma_f32_16x16x32_f16 a[128:131], v[100:103], v[132:135], a[128:131]\n"
  "s_max_u32 s86, s86, s88\n"
  "v_mfma_f32_16x16x32_f16 a[160:163], v[108:111], v[132:135], a[160:163]\n"
  "s_ff1_i32_b32 s79, s79\n"
  "s_nop 3\n"
  "v_mfma_f32_16x16x32_f16 a[160:163], v[104:107], v[128:131], a[160:163]\n"
  "s_add_u32 s90, s90, 151\n"
  "s_nop 3\n"
  "v_mfma_f32_16x16x32_f16 a[192:195], v[112:115], v[128:131], a[192:195]\n"
  "s_bitset1_b32 s72, s79\ns_mov_b32 m0, s73\n"
  "s_nop 3\n"
  "v_mfma_f32_16x16x32_f16 a[192:195], v[116:119], v[132:135], a[192:195]\n"
  "buffer_load_dwordx4 v[12:15], v240, s[20:23], s74 offen offset:512\n"
  "v_mfma_f32_16x16x32_f16 a[224:227], v[124:127], v[132:135], a[224:227]\n"
  "v_writelane_b32 v249, s79, m0\n"
  "s_nop 3\n"
  "v_mfma_f32_16x16x32_f16 a[224:227], v[120:123], v[128:131], a[224:227]\n"
  "s_nop 3\n"
  "v_mfma_f32_16x16x32_f16 a[4:7], v[64:67], v[136:139], a[4:7]\n"
  "s_add_u32 s73, s73, 1\n"
  "v_mfma_f32_16x16x32_f16 a[4:7], v[68:71], v[140:143], a[4:7]\n"
  "ds_read_b128 v[128:131], v243 offset:4096\n"
  "s_nop 3\n"
  "v_mfma_f32_16x16x32_f16 a[36:39], v[76:79], v[140:143], a[36:39]\n"
  "buffer_load_dwordx4 v[16:19], v241, s[24:27], s46 offen offset:1024\n"
  "s_nop 3\n"
  "v_mfma_f32_16x16x32_f16 a[36:39], v[72:75], v[136:139], a[36:39]\n"
  "s_max_u32 s90, s90, s88\n"
  "v_mfma_f32_16x16x32_f16 a[68:71], v[80:83], v[136:139], a[68:71]\n"
  "s_lshr_b32 s79, s58, 1\n"
  "s_nop 3\n"
  "v_mfma_f32_16x16x32_f16 a[68:71], v[84:87], v[140:143], a[68:71]\n"
  "s_sub_u32 s65, s86, s90\n"
  "s_nop 3\n"
  "v_mfma_f32_16x16x32_f16 a[100:103], v[92:95], v[140:143], a[100:103]\n"
  "buffer_load_dwordx4 v[20:23], v240, s[20:23], s74 offen offset:1024\n"
  "v_readlane_b32 s79, v249, s79\n"
  "v_mfma_f32_16x16x32_f16 a[100:103], v[88:91], v[136:139], a[100:103]\n"
  "ds_read_b128 v[132:135], v243 offset:12288\n"
  "s_nop 3\n"
  "v_mfma_f32_16x16x32_f16 a[132:135], v[96:99], v[136:139], a[132:135]\n"
  "s_and_b32 s80, s58, 1\ns_lshl_b32 s80, s80, 23\n"
  "s_nop 3\n"
  "v_mfma_f32_16x16x32_f16 a[132:135], v[100:103], v[140:143], a[132:135]\n"
  "s_cmp_gt_i32 s65, 150\ns_cselect_b32 s65, 5, 0\ns_cselect_b32 s71, 0, 3\n"
  "s_nop 3\n"
  "v_mfma_f32_16x16x32_f16 a[164:167], v[108:111], v[140:143], a[164:167]\n"
  "s_mov_b32 s81, s79\n"
  "v_mfma_f32_16x16x32_f16 a[164:167], v[104:107], v[136:139], a[164:167]\n"
  "buffer_load_dwordx4 v[24:27], v240, s[20:23], s46 offen offset:1536\n"
  "s_nop 3\n"
  "v_mfma_f32_16x16x32_f16 a[196:199], v[112:115], v[136:139], a[196:199]\n"
  "s_sub_u32 s86, s86, s65\n"
  "s_nop 3\n"
  "v_mfma_f32_16x16x32_f16 a[196:199], v[116:119], v[140:143], a[196:199]\n"
  "v_readlane_b32 s79, v248, s81\n"
  "v_mfma_f32_16x16x32_f16 a[228:231], v[124:127], v[140:143], a[228:231]\n"
  "s_cmp_eq_u32 s59, 59\ns_cselect_b32 s66, 0xf800, 0\n"
  "s_nop 3\n"
  "v_mfma_f32_16x16x32_f16 a[228:231], v[120:123], v[136:139], a[228:231]\n"
  "s_add_u32 s79, s79, s80\n"
  "s_nop 3\n"
  "s_waitcnt lgkmcnt(5)\n"
  "v_mfma_f32_16x16x32_f16 a[8:11], v[64:67], v[144:147], a[8:11]\n"
  "buffer_load_dwordx4 v[28:31], v241, s[24:27], s74 offen offset:1536\n"
  "v_mfma_f32_16x16x32_f16 a[8:11], v[68:71], v[148:151], a[8:11]\n"
  "ds_read_b128 v[136:139], v243 offset:5120\n"
  "s_add_u32 s28, s48, s79\ns_addc_u32 s29, s49, 0\n"
  "s_nop 3\n"
  "v_mfma_f32_16x16x32_f16 a[40:43], v[76:79], v[148:151], a[40:43]\n"
  "s_add_u32 s46, s46, s66\ns_add_u32 s74, s74, s66\n"
  "s_nop 3\n"
  "v_mfma_f32_16x16x32_f16 a[40:43], v[72:75], v[144:147], a[40:43]\n"
  "s_cmp_lt_u32 s58, 64\ns_cselect_b32 s30, s69, 0\n"
  "v_mfma_f32_16x16x32_f16 a[72:75], v[80:83], v[144:147], a[72:75]\n"
  "buffer_load_dwordx4 v[32:35], v241, s[24:27], s46 offen offset:2048\n"
  "s_nop 3\n"
  "v_mfma_f32_16x16x32_f16 a[72:75], v[84:87], v[148:151], a[72:75]\n"
  "s_add_u32 s58, s58, 1\n"
  "s_nop 3\n"
  "v_mfma_f32_16x16x32_f16 a[104:107], v[92:95], v[148:151], a[104:107]\n"
  "s_add_i32 s87, s87, s77\n"
  "s_nop 3\n"
  "v_mfma_f32_16x16x32_f16 a[104:107], v[88:91], v[144:147], a[104:107]\n"
  "ds_read_b128 v[140:143], v243 offset:13312\n"
  "v_mfma_f32_16x16x32_f16 a[136:139], v[96:99], v[144:147], a[136:139]\n"
  "s_max_i32 s87, s87, 0\n"
  "s_nop 3\n"
  "v_mfma_f32_16x16x32_f16 a[136:139], v[100:103], v[148:151], a[136:139]\n"
  "buffer_load_dwordx4 v[36:39], v240, s[20:23], s74 offen offset:2048\n"
  "s_nop 3\n"
  "v_mfma_f32_16x16x32_f16 a[168:171], v[108:111], v[148:151], a[168:171]\n"
  "s_min_i32 s87, s87, 6\n"
  "v_mfma_f32_16x16x32_f16 a[168:171], v[104:107], v[144:147], a[168:171]\n"
  "s_nop 3\n"
  "v_mfma_f32_16x16x32_f16 a[200:203], v[112:115], v[144:147], a[200:203]\n"
  "s_add_u32 s79, s59, 3\ns_lshr_b32 s80, s79, 1\n"
  "s_nop 3\n"
  "v_mfma_f32_16x16x32_f16 a[200:203], v[116:119], v[148:151], a[200:203]\n"
  "v_mfma_f32_16x16x32_f16 a[232:235], v[124:127], v[148:151], a[232:235]\n"
  "buffer_load_dwordx4 v[40:43], v240, s[20:23], s46 offen offset:2560\n"
  "v_readlane_b32 s80, v249, s80\n"
  "s_nop 3\n"
  "v_mfma_f32_16x16x32_f16 a[232:235], v[120:123], v[144:147], a[232:235]\n"
  "s_nop 3\n"
  "s_waitcnt lgkmcnt(4)\n"
  "v_mfma_f32_16x16x32_f16 a[12:15], v[64:67], v[152:155], a[12:15]\n"
  "s_lshl_b32 s80, s80, 17\n"
  "v_mfma_f32_16x16x32_f16 a[12:15], v[68:71], v[156:159], a[12:15]\n"
  "ds_read_b128 v[144:147], v243 offset:6144\n"
  "s_nop 3\n"
  "v_mfma_f32_16x16x32_f16 a[44:47], v[76:79], v[156:159], a[44:47]\n"
  "buffer_load_dwordx4 v[44:47], v241, s[24:27], s74 offen offset:2560\n"
  "s_cmp_lt_u32 s79, 64\ns_cselect_b32 s60, s80, s84\n"
  "s_nop 3\n"
  "v_mfma_f32_16x16x32_f16 a[44:47], v[72:75], v[152:155], a[44:47]\n"
  "s_nop 3\n"
  "v_mfma_f32_16x16x32_f16 a[76:79], v[80:83], v[152:155], a[76:79]\n"
  "v_mfma_f32_16x16x32_f16 a[76:79], v[84:87], v[156:159], a[76:79]\n"
  "s_nop 3\n"
  "v_mfma_f32_16x16x32_f16 a[108:111], v[92:95], v[156:159], a[108:111]\n"
  "s_nop 3\n"
  "v_mfma_f32_16x16x32_f16 a[108:111], v[88:91], v[152:155], a[108:111]\n"
  "buffer_load_dwordx4 v[48:51], v240, s[20:23], s46 offen offset:3072\n"
  "ds_read_b128 v[148:151], v243 offset:14336\n"
  "v_mfma_f32_16x16x32_f16 a[140:143], v[96:99], v[152:155], a[140:143]\n"
  "s_nop 3\n"
  "v_mfma_f32_16x16x32_f16 a[140:143], v[100:103], v[156:159], a[140:143]\n"
  "s_nop 3\n"
  "v_mfma_f32_16x16x32_f16 a[172:175], v[108:111], v[156:159], a[172:175]\n"
  "v_mfma_f32_16x16x32_f16 a[172:175], v[104:107], v[152:155], a[172:175]\n"
  "s_nop 3\n"
  "v_mfma_f32_16x16x32_f16 a[204:207], v[112:115], v[152:155], a[204:207]\n"
  "buffer_load_dwordx4 v[52:55], v241, s[24:27], s74 offen offset:3072\n"
  "s_nop 3\n"
  "v_mfma_f32_16x16x32_f16 a[204:207], v[116:119], v[156:159], a[204:207]\n"
  "v_mfma_f32_16x16x32_f16 a[236:239], v[124:127], v[156:159], a[236:239]\n"
  "s_nop 3\n"
  "v_mfma_f32_16x16x32_f16 a[236:239], v[120:123], v[152:155], a[236:239]\n"
  "ds_read_b128 v[152:155], v243 offset:7168\n"
  "s_nop 3\n"
  "s_waitcnt lgkmcnt(5)\n"
  "v_mfma_f32_16x16x32_f16 a[16:19], v[64:67], v[128:131], a[16:19]\n"
  "buffer_load_dwordx4 v[56:59], v241, s[24:27], s46 offen offset:3584\n"
  "s_nop 3\n"
  "v_mfma_f32_16x16x32_f16 a[16:19], v[68:71], v[132:135], a[16:19]\n"
  "v_mfma_f32_16x16x32_f16 a[48:51], v[76:79], v[132:135], a[48:51]\n"
  "s_nop 3\n"
  "v_mfma_f32_16x16x32_f16 a[48:51], v[72:75], v[128:131], a[48:51]\n"
  "s_nop 3\n"
  "v_mfma_f32_16x16x32_f16 a[80:83], v[80:83], v[128:131], a[80:83]\n"
  "v_mfma_f32_16x16x32_f16 a[80:83], v[84:87], v[132:135], a[80:83]\n"
  "buffer_load_dwordx4 v[60:63], v240, s[20:23], s74 offen offset:3584\n"
  "s_nop 3\n"
  "v_mfma_f32_16x16x32_f16 a[112:115], v[92:95], v[132:135], a[112:115]\n"
  "s_nop 3\n"
  "v_mfma_f32_16x16x32_f16 a[112:115], v[88:91], v[128:131], a[112:115]\n"
  "s_barrier\n"
  "buffer_load_dword v192, v245, s[20:23], s60 offen\n"
  "v_mfma_f32_16x16x32_f16 a[144:147], v[96:99], v[128:131], a[144:147]\n"
  "buffer_load_dwordx4 v[192:195], v242, s[28:31], 0 offen nt\n"
  "s_nop 3\n"
  "v_mfma_f32_16x16x32_f16 a[144:147], v[100:103], v[132:135], a[144:147]\n"
  "s_nop 3\n"
  "v_mfma_f32_16x16x32_f16 a[176:179], v[108:111], v[132:135], a[176:179]\n"
  "v_mfma_f32_16x16x32_f16 a[176:179], v[104:107], v[128:131], a[176:179]\n"
  "s_nop 3\n"
  "v_mfma_f32_16x16x32_f16 a[208:211], v[112:115], v[128:131], a[208:211]\n"
  "buffer_load_dwordx4 v[196:199], v242, s[28:31], s39 offen nt\n"
  "s_nop 3\n"
  "v_mfma_f32_16x16x32_f16 a[208:211], v[116:119], v[132:135], a[208:211]\n"
  "ds_read_b128 v[156:159], v243 offset:15360\n"
  "s_waitcnt vmcnt(19)\n"
  "v_cvt_pk_f16_f32 v224, v160, v164\n"
  "s_nop 3\n"
  "v_mfma_f32_16x16x32_f16 a[240:243], v[124:127], v[132:135], a[240:243]\n"
  "v_add_u32 v243, 0x4000, v243\n"
  "v_and_b32 v243, 0x1ffff, v243\n"
  "v_mfma_f32_16x16x32_f16 a[240:243], v[120:123], v[128:131], a[240:243]\n"
  "v_cvt_pk_f16_f32 v225, v168, v172\n"
  "s_nop 3\n"
  "s_waitcnt lgkmcnt(4)\n"
  "v_mfma_f32_16x16x32_f16 a[20:23], v[64:67], v[136:139], a[20:23]\n"
  "buffer_load_dwordx4 v[200:203], v242, s[28:31], s40 offen nt\n"
  "s_nop 3\n"
  "v_mfma_f32_16x16x32_f16 a[20:23], v[68:71], v[140:143], a[20:23]\n"
  "ds_read_b128 v[128:131], v243 offset:0\n"
  "v_cvt_pk_f16_f32 v226, v176, v180\n"
  "v_mfma_f32_16x16x32_f16 a[52:55], v[76:79], v[140:143], a[52:55]\n"
  "s_nop 3\n"
  "v_mfma_f32_16x16x32_f16 a[52:55], v[72:75], v[136:139], a[52:55]\n"
  "s_nop 3\n"
  "v_mfma_f32_16x16x32_f16 a[84:87], v[80:83], v[136:139], a[84:87]\n"
  "buffer_load_dwordx4 v[204:207], v242, s[28:31], s41 offen nt\n"
  "v_cvt_pk_f16_f32 v227, v184, v188\n"
  "v_mfma_f32_16x16x32_f16 a[84:87], v[84:87], v[140:143], a[84:87]\n"
  "s_nop 3\n"
  "v_mfma_f32_16x16x32_f16 a[116:119], v[92:95], v[140:143], a[116:119]\n"
  "ds_write_b128 v244, v[224:227] offset:0\n"
  "s_nop 3\n"
  "v_mfma_f32_16x16x32_f16 a[116:119], v[88:91], v[136:139], a[116:119]\n"
  "ds_read_b128 v[132:135], v243 offset:8192\n"
  "v_mfma_f32_16x16x32_f16 a[148:151], v[96:99], v[136:139], a[148:151]\n"
  "buffer_load_dwordx4 v[208:211], v242, s[28:31], s42 offen nt\n"
  "s_nop 3\n"
  "v_mfma_f32_16x16x32_f16 a[148:151], v[100:103], v[140:143], a[148:151]\n"
  "v_cvt_pk_f16_f32 v228, v161, v165\n"
  "s_nop 3\n"
  "v_mfma_f32_16x16x32_f16 a[180:183], v[108:111], v[140:143], a[180:183]\n"
  "s_nop 3\n"
  "v_mfma_f32_16x16x32_f16 a[180:183], v[104:107], v[136:139], a[180:183]\n"
  "v_cvt_pk_f16_f32 v229, v169, v173\n"
  "v_mfma_f32_16x16x32_f16 a[212:215], v[112:115], v[136:139], a[212:215]\n"
  "buffer_load_dwordx4 v[212:215], v242, s[28:31], s43 offen nt\n"
  "s_nop 3\n"
  "v_mfma_f32_16x16x32_f16 a[212:215], v[116:119], v[140:143], a[212:215]\n"
  "v_cvt_pk_f16_f32 v230, v177, v181\n"
  "s_nop 3\n"
  "v_mfma_f32_16x16x32_f16 a[244:247], v[124:127], v[140:143], a[244:247]\n"
  "v_mfma_f32_16x16x32_f16 a[244:247], v[120:123], v[136:139], a[244:247]\n"
  "s_nop 3\n"
  "s_waitcnt lgkmcnt(5)\n"
  "v_mfma_f32_16x16x32_f16 a[24:27], v[64:67], v[144:147], a[24:27]\n"
  "buffer_load_dwordx4 v[216:219], v242, s[28:31], s44 offen nt\n"
  "v_cvt_pk_f16_f32 v231, v185, v189\n"
  "s_nop 3\n"
  "v_mfma_f32_16x16x32_f16 a[24:27], v[68:71], v[148:151], a[24:27]\n"
  "ds_read_b128 v[136:139], v243 offset:1024\n"
  "v_mfma_f32_16x16x32_f16 a[56:59], v[76:79], v[148:151], a[56:59]\n"
  "ds_write_b128 v244, v[228:231] offset:1024\n"
  "s_nop 3\n"
  "v_mfma_f32_16x16x32_f16 a[56:59], v[72:75], v[144:147], a[56:59]\n"
  "s_nop 3\n"
  "v_mfma_f32_16x16x32_f16 a[88:91], v[80:83], v[144:147], a[88:91]\n"
  "buffer_load_dwordx4 v[220:223], v242, s[28:31], s45 offen nt\n"
  "v_mfma_f32_16x16x32_f16 a[88:91], v[84:87], v[148:151], a[88:91]\n"
  "v_cvt_pk_f16_f32 v224, v162, v166\n"
  "s_nop 3\n"
  "v_mfma_f32_16x16x32_f16 a[120:123], v[92:95], v[148:151], a[120:123]\n"
  "s_memrealtime s[52:53]\n"
  "s_nop 3\n"
  "v_mfma_f32_16x16x32_f16 a[120:123], v[88:91], v[144:147], a[120:123]\n"
  "ds_read_b128 v[140:143], v243 offset:9216\n"
  "v_cvt_pk_f16_f32 v225, v170, v174\n"
  "s_nop 3\n"
  "v_mfma_f32_16x16x32_f16 a[152:155], v[96:99], v[144:147], a[152:155]\n"
  "v_mfma_f32_16x16x32_f16 a[152:155], v[100:103], v[148:151], a[152:155]\n"
  "v_cvt_pk_f16_f32 v226, v178, v182\n"
  "s_nop 3\n"
  "v_mfma_f32_16x16x32_f16 a[184:187], v[108:111], v[148:151], a[184:187]\n"
  "s_add_u32 s79, s59, 2\ns_lshr_b32 s80, s79, 1\n"
  "s_nop 3\n"
  "v_mfma_f32_16x16x32_f16 a[184:187], v[104:107], v[144:147], a[184:187]\n"
  "s_waitcnt lgkmcnt(0)\n"
  "s_sub_u32 s82, s52, s86\ns_cmp_lt_i32 s82, 0\ns_cbranch_scc0 PGx1_%=\nPWx1_%=:\ns_sleep 1\ns_memrealtime s[54:55]\ns_waitcnt lgkmcnt(0)\ns_sub_u32 s88, s54, s86\ns_cmp_lt_i32 s88, 0\ns_cbranch_scc1 PWx1_%=\nPGx1_%=:\n"
  "v_mfma_f32_16x16x32_f16 a[216:219], v[112:115], v[144:147], a[216:219]\n"
  "s_cmp_lt_u32 s82, 44\ns_cselect_b32 s77, 0, -1\n"
  "v_cvt_pk_f16_f32 v227, v186, v190\n"
  "v_readlane_b32 s80, v249, s80\n"
  "s_nop 3\n"
  "v_mfma_f32_16x16x32_f16 a[216:219], v[116:119], v[148:151], a[216:219]\n"
  "s_sub_u32 s82, s82, -131\n"
  "s_nop 3\n"
  "v_mfma_f32_16x16x32_f16 a[248:251], v[124:127], v[148:151], a[248:251]\n"
  "ds_write_b128 v244, v[224:227] offset:2048\n"
  "s_lshl_b32 s80, s80, 17\ns_add_u32 s80, s80, 0x10000\n"
  "v_mfma_f32_16x16x32_f16 a[248:251], v[120:123], v[144:147], a[248:251]\n"
  "s_nop 3\n"
  "v_mfma_f32_16x16x32_f16 a[28:31], v[64:67], v[152:155], a[28:31]\n"
  "s_cmp_lt_u32 s79, 64\ns_cselect_b32 s46, s80, s84\n"
  "s_nop 3\n"
  "v_mfma_f32_16x16x32_f16 a[28:31], v[68:71], v[156:159], a[28:31]\n"
  "ds_read_b128 v[144:147], v243 offset:2048\n"
  "v_cvt_pk_f16_f32 v228, v163, v167\n"
  "v_mfma_f32_16x16x32_f16 a[60:63], v[76:79], v[156:159], a[60:63]\n"
  "s_max_i32 s82, s82, 151\n"
  "s_nop 3\n"
  "v_mfma_f32_16x16x32_f16 a[60:63], v[72:75], v[152:155], a[60:63]\n"
  "v_cvt_pk_f16_f32 v229, v171, v175\n"
  "s_nop 3\n"
  "v_mfma_f32_16x16x32_f16 a[92:95], v[80:83], v[152:155], a[92:95]\n"
  "s_add_u32 s46, s46, s83\n"
  "s_nop 3\n"
  "v_mfma_f32_16x16x32_f16 a[92:95], v[84:87], v[156:159], a[92:95]\n"
  "v_cvt_pk_f16_f32 v230, v179, v183\n"
  "v_mfma_f32_16x16x32_f16 a[124:127], v[92:95], v[156:159], a[124:127]\n"
  "s_add_u32 s74, s46, 0x4000\n"
  "s_nop 3\n"
  "v_mfma_f32_16x16x32_f16 a[124:127], v[88:91], v[152:155], a[124:127]\n"
  "ds_read_b128 v[148:151], v243 offset:10240\n"
  "s_nop 3\n"
  "v_mfma_f32_16x16x32_f16 a[156:159], v[96:99], v[152:155], a[156:159]\n"
  "v_cvt_pk_f16_f32 v231, v187, v191\n"
  "s_add_u32 s59, s59, 1\n"
  "v_mfma_f32_16x16x32_f16 a[156:159], v[100:103], v[156:159], a[156:159]\n"
  "s_add_u32 s86, s86, s82\n"
  "s_nop 3\n"
  "v_mfma_f32_16x16x32_f16 a[188:191], v[108:111], v[156:159], a[188:191]\n"
  "ds_write_b128 v244, v[228:231] offset:3072\n"
  "s_nop 3\n"
  "v_mfma_f32_16x16x32_f16 a[188:191], v[104:107], v[152:155], a[188:191]\n"
  "v_mfma_f32_16x16x32_f16 a[220:223], v[112:115], v[152:155], a[220:223]\n"
  "s_nop 3\n"
  "v_mfma_f32_16x16x32_f16 a[220:223], v[116:119], v[156:159], a[220:223]\n"
  "s_nop 3\n"
  "v_mfma_f32_16x16x32_f16 a[252:255], v[124:127], v[156:159], a[252:255]\n"
  "v_add_u32 v244, 0x4000, v244\n"
  "v_and_b32 v244, 0x1ffff, v244\n"
  "v_mfma_f32_16x16x32_f16 a[252:255], v[120:123], v[152:155], a[252:255]\n"
  "s_branch LSEL_%=\n"
  "DENT_%=:\n"
  "s_mov_b32 s89, -100000\n"
  "DLOOP_%=:\n"
  "buffer_load_dwordx4 v[64:67], v240, s[20:23], s46 offen offset:0\n"
  "s_waitcnt vmcnt(12) lgkmcnt(4)\n"
  "v_mfma_f32_16x16x32_f16 a[0:3], v[0:3], v[128:131], a[0:3]\n"
  "v_mfma_f32_16x16x32_f16 a[0:3], v[4:7], v[132:135], a[0:3]\n"
  "ds_read_b128 v[152:155], v243 offset:3072\n"
  "v_mfma_f32_16x16x32_f16 a[32:35], v[12:15], v[132:135], a[32:35]\n"
  "v_mfma_f32_16x16x32_f16 a[32:35], v[8:11], v[128:131], a[32:35]\n"
  "v_mfma_f32_16x16x32_f16 a[64:67], v[16:19], v[128:131], a[64:67]\n"
  "buffer_load_dwordx4 v[68:71], v241, s[24:27], s74 offen offset:0\n"
  "v_mfma_f32_16x16x32_f16 a[64:67], v[20:23], v[132:135], a[64:67]\n"
  "v_mfma_f32_16x16x32_f16 a[96:99], v[28:31], v[132:135], a[96:99]\n"
  "v_mfma_f32_16x16x32_f16 a[96:99], v[24:27], v[128:131], a[96:99]\n"
  "ds_read_b128 v[156:159], v243 offset:11264\n"
  "v_mfma_f32_16x16x32_f16 a[128:131], v[32:35], v[128:131], a[128:131]\n"
  "buffer_load_dwordx4 v[72:75], v241, s[24:27], s46 offen offset:512\n"
  "v_mfma_f32_16x16x32_f16 a[128:131], v[36:39], v[132:135], a[128:131]\n"
  "v_mfma_f32_16x16x32_f16 a[160:163], v[44:47], v[132:135], a[160:163]\n"
  "v_mfma_f32_16x16x32_f16 a[160:163], v[40:43], v[128:131], a[160:163]\n"
  "v_mfma_f32_16x16x32_f16 a[192:195], v[48:51], v[128:131], a[192:195]\n"
  "v_mfma_f32_16x16x32_f16 a[192:195], v[52:55], v[132:135], a[192:195]\n"
  "buffer_load_dwordx4 v[76:79], v240, s[20:23], s74 offen offset:512\n"
  "v_mfma_f32_16x16x32_f16 a[224:227], v[60:63], v[132:135], a[224:227]\n"
  "v_mfma_f32_16x16x32_f16 a[224:227], v[56:59], v[128:131], a[224:227]\n"
  "s_waitcnt lgkmcnt(4)\n"
  "v_mfma_f32_16x16x32_f16 a[4:7], v[0:3], v[136:139], a[4:7]\n"
  "v_mfma_f32_16x16x32_f16 a[4:7], v[4:7], v[140:143], a[4:7]\n"
  "ds_read_b128 v[128:131], v243 offset:4096\n"
  "v_mfma_f32_16x16x32_f16 a[36:39], v[12:15], v[140:143], a[36:39]\n"
  "buffer_load_dwordx4 v[80:83], v241, s[24:27], s46 offen offset:1024\n"
  "v_mfma_f32_16x16x32_f16 a[36:39], v[8:11], v[136:139], a[36:39]\n"
  "v_mfma_f32_16x16x32_f16 a[68:71], v[16:19], v[136:139], a[68:71]\n"
  "s_lshr_b32 s79, s58, 1\n"
  "v_mfma_f32_16x16x32_f16 a[68:71], v[20:23], v[140:143], a[68:71]\n"
  "v_mfma_f32_16x16x32_f16 a[100:103], v[28:31], v[140:143], a[100:103]\n"
  "buffer_load_dwordx4 v[84:87], v240, s[20:23], s74 offen offset:1024\n"
  "v_readlane_b32 s79, v249, s79\n"
  "v_mfma_f32_16x16x32_f16 a[100:103], v[24:27], v[136:139], a[100:103]\n"
  "ds_read_b128 v[132:135], v243 offset:12288\n"
  "v_mfma_f32_16x16x32_f16 a[132:135], v[32:35], v[136:139], a[132:135]\n"
  "s_and_b32 s80, s58, 1\ns_lshl_b32 s80, s80, 23\n"
  "v_mfma_f32_16x16x32_f16 a[132:135], v[36:39], v[140:143], a[132:135]\n"
  "v_mfma_f32_16x16x32_f16 a[164:167], v[44:47], v[140:143], a[164:167]\n"
  "s_mov_b32 s81, s79\n"
  "v_mfma_f32_16x16x32_f16 a[164:167], v[40:43], v[136:139], a[164:167]\n"
  "buffer_load_dwordx4 v[88:91], v240, s[20:23], s46 offen offset:1536\n"
  "v_mfma_f32_16x16x32_f16 a[196:199], v[48:51], v[136:139], a[196:199]\n"
  "v_mfma_f32_16x16x32_f16 a[196:199], v[52:55], v[140:143], a[196:199]\n"
  "v_readlane_b32 s79, v248, s81\n"
  "v_mfma_f32_16x16x32_f16 a[228:231], v[60:63], v[140:143], a[228:231]\n"
  "v_mfma_f32_16x16x32_f16 a[228:231], v[56:59], v[136:139], a[228:231]\n"
  "s_add_u32 s79, s79, s80\n"
  "s_waitcnt lgkmcnt(4)\n"
  "v_mfma_f32_16x16x32_f16 a[8:11], v[0:3], v[144:147], a[8:11]\n"
  "buffer_load_dwordx4 v[92:95], v241, s[24:27], s74 offen offset:1536\n"
  "v_mfma_f32_16x16x32_f16 a[8:11], v[4:7], v[148:151], a[8:11]\n"
  "ds_read_b128 v[136:139], v243 offset:5120\n"
  "s_add_u32 s28, s48, s79\ns_addc_u32 s29, s49, 0\n"
  "v_mfma_f32_16x16x32_f16 a[40:43], v[12:15], v[148:151], a[40:43]\n"
  "v_mfma_f32_16x16x32_f16 a[40:43], v[8:11], v[144:147], a[40:43]\n"
  "s_cmp_lt_u32 s58, 64\ns_cselect_b32 s30, s69, 0\n"
  "v_mfma_f32_16x16x32_f16 a[72:75], v[16:19], v[144:147], a[72:75]\n"
  "buffer_load_dwordx4 v[96:99], v241, s[24:27], s46 offen offset:2048\n"
  "v_mfma_f32_16x16x32_f16 a[72:75], v[20:23], v[148:151], a[72:75]\n"
  "s_add_u32 s58, s58, 1\n"
  "v_mfma_f32_16x16x32_f16 a[104:107], v[28:31], v[148:151], a[104:107]\n"
  "v_mfma_f32_16x16x32_f16 a[104:107], v[24:27], v[144:147], a[104:107]\n"
  "ds_read_b128 v[140:143], v243 offset:13312\n"
  "v_mfma_f32_16x16x32_f16 a[136:139], v[32:35], v[144:147], a[136:139]\n"
  "v_mfma_f32_16x16x32_f16 a[136:139], v[36:39], v[148:151], a[136:139]\n"
  "buffer_load_dwordx4 v[100:103], v240, s[20:23], s74 offen offset:2048\n"
  "v_mfma_f32_16x16x32_f16 a[168:171], v[44:47], v[148:151], a[168:171]\n"
  "v_mfma_f32_16x16x32_f16 a[168:171], v[40:43], v[144:147], a[168:171]\n"
  "v_mfma_f32_16x16x32_f16 a[200:203], v[48:51], v[144:147], a[200:203]\n"
  "s_add_u32 s79, s59, 3\ns_lshr_b32 s80, s79, 1\n"
  "v_mfma_f32_16x16x32_f16 a[200:203], v[52:55], v[148:151], a[200:203]\n"
  "v_mfma_f32_16x16x32_f16 a[232:235], v[60:63], v[148:151], a[232:235]\n"
  "buffer_load_dwordx4 v[104:107], v240, s[20:23], s46 offen offset:2560\n"
  "v_readlane_b32 s80, v249, s80\n"
  "v_mfma_f32_16x16x32_f16 a[232:235], v[56:59], v[144:147], a[232:235]\n"
  "s_waitcnt lgkmcnt(4)\n"
  "v_mfma_f32_16x16x32_f16 a[12:15], v[0:3], v[152:155], a[12:15]\n"
  "s_lshl_b32 s80, s80, 17\ns_add_u32 s80, s80, 0x10000\n"
  "v_mfma_f32_16x16x32_f16 a[12:15], v[4:7], v[156:159], a[12:15]\n"
  "ds_read_b128 v[144:147], v243 offset:6144\n"
  "v_mfma_f32_16x16x32_f16 a[44:47], v[12:15], v[156:159], a[44:47]\n"
  "buffer_load_dwordx4 v[108:111], v241, s[24:27], s74 offen offset:2560\n"
  "s_cmp_lt_u32 s79, 64\ns_cselect_b32 s60, s80, s84\n"
  "v_mfma_f32_16x16x32_f16 a[44:47], v[8:11], v[152:155], a[44:47]\n"
  "v_mfma_f32_16x16x32_f16 a[76:79], v[16:19], v[152:155], a[76:79]\n"
  "v_mfma_f32_16x16x32_f16 a[76:79], v[20:23], v[156:159], a[76:79]\n"
  "v_mfma_f32_16x16x32_f16 a[108:111], v[28:31], v[156:159], a[108:111]\n"
  "s_waitcnt vmcnt(21)\n"
  "v_cmp_le_u32 vcc, s61, v246\n"
  "v_mfma_f32_16x16x32_f16 a[108:111], v[24:27], v[152:155], a[108:111]\n"
  "buffer_load_dwordx4 v[112:115], v240, s[20:23], s46 offen offset:3072\n"
  "ds_read_b128 v[148:151], v243 offset:14336\n"
  "v_mfma_f32_16x16x32_f16 a[140:143], v[32:35], v[152:155], a[140:143]\n"
  "s_or_b32 s71, s71, vcc_lo\n"
  "v_mfma_f32_16x16x32_f16 a[140:143], v[36:39], v[156:159], a[140:143]\n"
  "v_mfma_f32_16x16x32_f16 a[172:175], v[44:47], v[156:159], a[172:175]\n"
  "v_mov_b32 v250, s71\n"
  "v_mfma_f32_16x16x32_f16 a[172:175], v[40:43], v[152:155], a[172:175]\n"
  "v_mfma_f32_16x16x32_f16 a[204:207], v[48:51], v[152:155], a[204:207]\n"
  "buffer_load_dwordx4 v[116:119], v241, s[24:27], s74 offen offset:3072\n"
  "ds_write_b32 v251, v250 offset:0\n"
  "v_mfma_f32_16x16x32_f16 a[204:207], v[52:55], v[156:159], a[204:207]\n"
  "v_mfma_f32_16x16x32_f16 a[236:239], v[60:63], v[156:159], a[236:239]\n"
  "v_mfma_f32_16x16x32_f16 a[236:239], v[56:59], v[152:155], a[236:239]\n"
  "ds_read_b128 v[152:155], v243 offset:7168\n"
  "s_waitcnt lgkmcnt(6)\n"
  "v_mfma_f32_16x16x32_f16 a[16:19], v[0:3], v[128:131], a[16:19]\n"
  "buffer_load_dwordx4 v[120:123], v241, s[24:27], s46 offen offset:3584\n"
  "v_mfma_f32_16x16x32_f16 a[16:19], v[4:7], v[132:135], a[16:19]\n"
  "v_mfma_f32_16x16x32_f16 a[48:51], v[12:15], v[132:135], a[48:51]\n"
  "v_mfma_f32_16x16x32_f16 a[48:51], v[8:11], v[128:131], a[48:51]\n"
  "v_mfma_f32_16x16x32_f16 a[80:83], v[16:19], v[128:131], a[80:83]\n"
  "v_mfma_f32_16x16x32_f16 a[80:83], v[20:23], v[132:135], a[80:83]\n"
  "buffer_load_dwordx4 v[124:127], v240, s[20:23], s74 offen offset:3584\n"
  "v_mfma_f32_16x16x32_f16 a[112:115], v[28:31], v[132:135], a[112:115]\n"
  "v_mfma_f32_16x16x32_f16 a[112:115], v[24:27], v[128:131], a[112:115]\n"
  "s_waitcnt vmcnt(23) lgkmcnt(1)\n"
  "s_barrier\n"
  "s_bitcmp1_b32 s75, s59\ns_cbranch_scc0 ENd0_%=\ns_call_b64 s[50:51], EVENT_%=\ns_branch EVd0_%=\nENd0_%=:\nbuffer_load_dword v246, off, s[20:23], s84\nEVd0_%=:\n"
  "v_mfma_f32_16x16x32_f16 a[144:147], v[32:35], v[128:131], a[144:147]\n"
  "buffer_load_dword v246, v245, s[36:39], 0 offen sc0 sc1\n"
  "v_mfma_f32_16x16x32_f16 a[144:147], v[36:39], v[132:135], a[144:147]\n"
  "buffer_load_dword v160, v245, s[20:23], s60 offen\n"
  "v_mfma_f32_16x16x32_f16 a[176:179], v[44:47], v[132:135], a[176:179]\n"
  "buffer_load_dwordx4 v[160:163], v242, s[28:31], 0 offen nt\n"
  "v_mfma_f32_16x16x32_f16 a[176:179], v[40:43], v[128:131], a[176:179]\n"
  "v_mov_b32 v250, 0x20000\n"
  "v_mfma_f32_16x16x32_f16 a[208:211], v[48:51], v[128:131], a[208:211]\n"
  "ds_read_b32 v250, v250 offset:0\n"
  "v_mfma_f32_16x16x32_f16 a[208:211], v[52:55], v[132:135], a[208:211]\n"
  "ds_read_b128 v[156:159], v243 offset:15360\n"
  "s_waitcnt vmcnt(20)\n"
  "v_cvt_pk_f16_f32 v224, v192, v196\n"
  "v_mfma_f32_16x16x32_f16 a[240:243], v[60:63], v[132:135], a[240:243]\n"
  "v_add_u32 v243, 0x4000, v243\n"
  "v_and_b32 v243, 0x1ffff, v243\n"
  "buffer_load_dwordx4 v[164:167], v242, s[28:31], s39 offen nt\n"
  "v_mfma_f32_16x16x32_f16 a[240:243], v[56:59], v[128:131], a[240:243]\n"
  "v_cvt_pk_f16_f32 v225, v200, v204\n"
  "v_mfma_f32_16x16x32_f16 a[20:23], v[0:3], v[136:139], a[20:23]\n"
  "v_mfma_f32_16x16x32_f16 a[20:23], v[4:7], v[140:143], a[20:23]\n"
  "ds_read_b128 v[128:131], v243 offset:0\n"
  "v_cvt_pk_f16_f32 v226, v208, v212\n"
  "v_mfma_f32_16x16x32_f16 a[52:55], v[12:15], v[140:143], a[52:55]\n"
  "buffer_load_dwordx4 v[168:171], v242, s[28:31], s40 offen nt\n"
  "v_mfma_f32_16x16x32_f16 a[52:55], v[8:11], v[136:139], a[52:55]\n"
  "v_mfma_f32_16x16x32_f16 a[84:87], v[16:19], v[136:139], a[84:87]\n"
  "v_cvt_pk_f16_f32 v227, v216, v220\n"
  "v_mfma_f32_16x16x32_f16 a[84:87], v[20:23], v[140:143], a[84:87]\n"
  "v_mfma_f32_16x16x32_f16 a[116:119], v[28:31], v[140:143], a[116:119]\n"
  "buffer_load_dwordx4 v[172:175], v242, s[28:31], s41 offen nt\n"
  "ds_write_b128 v244, v[224:227] offset:0\n"
  "v_mfma_f32_16x16x32_f16 a[116:119], v[24:27], v[136:139], a[116:119]\n"
  "ds_read_b128 v[132:135], v243 offset:8192\n"
  "v_mfma_f32_16x16x32_f16 a[148:151], v[32:35], v[136:139], a[148:151]\n"
  "v_mfma_f32_16x16x32_f16 a[148:151], v[36:39], v[140:143], a[148:151]\n"
  "v_cvt_pk_f16_f32 v228, v193, v197\n"
  "v_mfma_f32_16x16x32_f16 a[180:183], v[44:47], v[140:143], a[180:183]\n"
  "buffer_load_dwordx4 v[176:179], v242, s[28:31], s42 offen\n"
  "v_mfma_f32_16x16x32_f16 a[180:183], v[40:43], v[136:139], a[180:183]\n"
  "v_cvt_pk_f16_f32 v229, v201, v205\n"
  "v_mfma_f32_16x16x32_f16 a[212:215], v[48:51], v[136:139], a[212:215]\n"
  "v_mfma_f32_16x16x32_f16 a[212:215], v[52:55], v[140:143], a[212:215]\n"
  "v_cvt_pk_f16_f32 v230, v209, v213\n"
  "v_mfma_f32_16x16x32_f16 a[244:247], v[60:63], v[140:143], a[244:247]\n"
  "buffer_load_dwordx4 v[180:183], v242, s[28:31], s43 offen\n"
  "v_mfma_f32_16x16x32_f16 a[244:247], v[56:59], v[136:139], a[244:247]\n"
  "v_mfma_f32_16x16x32_f16 a[24:27], v[0:3], v[144:147], a[24:27]\n"
  "v_cvt_pk_f16_f32 v231, v217, v221\n"
  "v_mfma_f32_16x16x32_f16 a[24:27], v[4:7], v[148:151], a[24:27]\n"
  "ds_read_b128 v[136:139], v243 offset:1024\n"
  "v_mfma_f32_16x16x32_f16 a[56:59], v[12:15], v[148:151], a[56:59]\n"
  "buffer_load_dwordx4 v[184:187], v242, s[28:31], s44 offen\n"
  "ds_write_b128 v244, v[228:231] offset:1024\n"
  "v_mfma_f32_16x16x32_f16 a[56:59], v[8:11], v[144:147], a[56:59]\n"
  "s_waitcnt lgkmcnt(6)\n"
  "v_readfirstlane_b32 s70, v250\n"
  "v_mfma_f32_16x16x32_f16 a[88:91], v[16:19], v[144:147], a[88:91]\n"
  "v_mfma_f32_16x16x32_f16 a[88:91], v[20:23], v[148:151], a[88:91]\n"
  "v_cvt_pk_f16_f32 v224, v194, v198\n"
  "v_mfma_f32_16x16x32_f16 a[120:123], v[28:31], v[148:151], a[120:123]\n"
  "buffer_load_dwordx4 v[188:191], v242, s[28:31], s45 offen\n"
  "v_mfma_f32_16x16x32_f16 a[120:123], v[24:27], v[144:147], a[120:123]\n"
  "ds_read_b128 v[140:143], v243 offset:9216\n"
  "v_cvt_pk_f16_f32 v225, v202, v206\n"
  "v_mfma_f32_16x16x32_f16 a[152:155], v[32:35], v[144:147], a[152:155]\n"
  "v_mfma_f32_16x16x32_f16 a[152:155], v[36:39], v[148:151], a[152:155]\n"
  "v_cvt_pk_f16_f32 v226, v210, v214\n"
  "v_mfma_f32_16x16x32_f16 a[184:187], v[44:47], v[148:151], a[184:187]\n"
  "s_add_u32 s79, s59, 2\ns_lshr_b32 s80, s79, 1\n"
  "v_mfma_f32_16x16x32_f16 a[184:187], v[40:43], v[144:147], a[184:187]\n"
  "v_mfma_f32_16x16x32_f16 a[216:219], v[48:51], v[144:147], a[216:219]\n"
  "v_cvt_pk_f16_f32 v227, v218, v222\n"
  "v_readlane_b32 s80, v249, s80\n"
  "v_mfma_f32_16x16x32_f16 a[216:219], v[52:55], v[148:151], a[216:219]\n"
  "v_mfma_f32_16x16x32_f16 a[248:251], v[60:63], v[148:151], a[248:251]\n"
  "ds_write_b128 v244, v[224:227] offset:2048\n"
  "s_lshl_b32 s80, s80, 17\n"
  "v_mfma_f32_16x16x32_f16 a[248:251], v[56:59], v[144:147], a[248:251]\n"
  "s_waitcnt lgkmcnt(7)\n"
  "v_mfma_f32_16x16x32_f16 a[28:31], v[0:3], v[152:155], a[28:31]\n"
  "s_cmp_lt_u32 s79, 64\ns_cselect_b32 s46, s80, s84\n"
  "v_mfma_f32_16x16x32_f16 a[28:31], v[4:7], v[156:159], a[28:31]\n"
  "ds_read_b128 v[144:147], v243 offset:2048\n"
  "v_cvt_pk_f16_f32 v228, v195, v199\n"
  "v_mfma_f32_16x16x32_f16 a[60:63], v[12:15], v[156:159], a[60:63]\n"
  "v_mfma_f32_16x16x32_f16 a[60:63], v[8:11], v[152:155], a[60:63]\n"
  "v_cvt_pk_f16_f32 v229, v203, v207\n"
  "v_mfma_f32_16x16x32_f16 a[92:95], v[16:19], v[152:155], a[92:95]\n"
  "s_add_u32 s46, s46, s83\n"
  "v_mfma_f32_16x16x32_f16 a[92:95], v[20:23], v[156:159], a[92:95]\n"
  "v_cvt_pk_f16_f32 v230, v211, v215\n"
  "v_mfma_f32_16x16x32_f16 a[124:127], v[28:31], v[156:159], a[124:127]\n"
  "s_add_u32 s74, s46, 0x4000\n"
  "v_mfma_f32_16x16x32_f16 a[124:127], v[24:27], v[152:155], a[124:127]\n"
  "ds_read_b128 v[148:151], v243 offset:10240\n"
  "v_mfma_f32_16x16x32_f16 a[156:159], v[32:35], v[152:155], a[156:159]\n"
  "v_cvt_pk_f16_f32 v231, v219, v223\n"
  "s_add_u32 s59, s59, 1\n"
  "v_mfma_f32_16x16x32_f16 a[156:159], v[36:39], v[156:159], a[156:159]\n"
  "v_mfma_f32_16x16x32_f16 a[188:191], v[44:47], v[156:159], a[188:191]\n"
  "ds_write_b128 v244, v[228:231] offset:3072\n"
  "v_mfma_f32_16x16x32_f16 a[188:191], v[40:43], v[152:155], a[188:191]\n"
  "v_mfma_f32_16x16x32_f16 a[220:223], v[48:51], v[152:155], a[220:223]\n"
  "v_mfma_f32_16x16x32_f16 a[220:223], v[52:55], v[156:159], a[220:223]\n"
  "v_mfma_f32_16x16x32_f16 a[252:255], v[60:63], v[156:159], a[252:255]\n"
  "v_add_u32 v244, 0x4000, v244\n"
  "v_and_b32 v244, 0x1ffff, v244\n"
  "v_mfma_f32_16x16x32_f16 a[252:255], v[56:59], v[152:155], a[252:255]\n"
  "buffer_load_dwordx4 v[0:3], v240, s[20:23], s46 offen offset:0\n"
  "s_waitcnt vmcnt(12) lgkmcnt(7)\n"
  "v_mfma_f32_16x16x32_f16 a[0:3], v[64:67], v[128:131], a[0:3]\n"
  "v_mfma_f32_16x16x32_f16 a[0:3], v[68:71], v[132:135], a[0:3]\n"
  "ds_read_b128 v[152:155], v243 offset:3072\n"
  "v_mfma_f32_16x16x32_f16 a[32:35], v[76:79], v[132:135], a[32:35]\n"
  "v_mfma_f32_16x16x32_f16 a[32:35], v[72:75], v[128:131], a[32:35]\n"
  "v_mfma_f32_16x16x32_f16 a[64:67], v[80:83], v[128:131], a[64:67]\n"
  "buffer_load_dwordx4 v[4:7], v241, s[24:27], s74 offen offset:0\n"
  "v_mfma_f32_16x16x32_f16 a[64:67], v[84:87], v[132:135], a[64:67]\n"
  "v_mfma_f32_16x16x32_f16 a[96:99], v[92:95], v[132:135], a[96:99]\n"
  "s_andn2_b32 s79, s70, s72\n"
  "s_cbranch_scc1 PKd1_%=\ns_call_b64 s[50:51], WAITANY_%=\nPKd1_%=:\n"
  "v_mfma_f32_16x16x32_f16 a[96:99], v[88:91], v[128:131], a[96:99]\n"
  "ds_read_b128 v[156:159], v243 offset:11264\n"
  "v_mfma_f32_16x16x32_f16 a[128:131], v[96:99], v[128:131], a[128:131]\n"
  "buffer_load_dwordx4 v[8:11], v241, s[24:27], s46 offen offset:512\n"
  "v_mfma_f32_16x16x32_f16 a[128:131], v[100:103], v[132:135], a[128:131]\n"
  "v_mfma_f32_16x16x32_f16 a[160:163], v[108:111], v[132:135], a[160:163]\n"
  "s_ff1_i32_b32 s79, s79\n"
  "v_mfma_f32_16x16x32_f16 a[160:163], v[104:107], v[128:131], a[160:163]\n"
  "v_mfma_f32_16x16x32_f16 a[192:195], v[112:115], v[128:131], a[192:195]\n"
  "s_bitset1_b32 s72, s79\ns_mov_b32 m0, s73\n"
  "v_mfma_f32_16x16x32_f16 a[192:195], v[116:119], v[132:135], a[192:195]\n"
  "buffer_load_dwordx4 v[12:15], v240, s[20:23], s74 offen offset:512\n"
  "v_mfma_f32_16x16x32_f16 a[224:227], v[124:127], v[132:135], a[224:227]\n"
  "v_writelane_b32 v249, s79, m0\n"
  "v_mfma_f32_16x16x32_f16 a[224:227], v[120:123], v[128:131], a[224:227]\n"
  "s_waitcnt lgkmcnt(6)\n"
  "v_mfma_f32_16x16x32_f16 a[4:7], v[64:67], v[136:139], a[4:7]\n"
  "s_add_u32 s73, s73, 1\n"
  "v_mfma_f32_16x16x32_f16 a[4:7], v[68:71], v[140:143], a[4:7]\n"
  "ds_read_b128 v[128:131], v243 offset:4096\n"
  "v_mfma_f32_16x16x32_f16 a[36:39], v[76:79], v[140:143], a[36:39]\n"
  "buffer_load_dwordx4 v[16:19], v241, s[24:27], s46 offen offset:1024\n"
  "v_mfma_f32_16x16x32_f16 a[36:39], v[72:75], v[136:139], a[36:39]\n"
  "v_mfma_f32_16x16x32_f16 a[68:71], v[80:83], v[136:139], a[68:71]\n"
  "s_lshr_b32 s79, s58, 1\n"
  "v_mfma_f32_16x16x32_f16 a[68:71], v[84:87], v[140:143], a[68:71]\n"
  "v_mfma_f32_16x16x32_f16 a[100:103], v[92:95], v[140:143], a[100:103]\n"
  "buffer_load_dwordx4 v[20:23], v240, s[20:23], s74 offen offset:1024\n"
  "v_readlane_b32 s79, v249, s79\n"
  "v_mfma_f32_16x16x32_f16 a[100:103], v[88:91], v[136:139], a[100:103]\n"
  "ds_read_b128 v[132:135], v243 offset:12288\n"
  "v_mfma_f32_16x16x32_f16 a[132:135], v[96:99], v[136:139], a[132:135]\n"
  "s_and_b32 s80, s58, 1\ns_lshl_b32 s80, s80, 23\n"
  "v_mfma_f32_16x16x32_f16 a[132:135], v[100:103], v[140:143], a[132:135]\n"
  "v_mfma_f32_16x16x32_f16 a[164:167], v[108:111], v[140:143], a[164:167]\n"
  "s_mov_b32 s81, s79\n"
  "v_mfma_f32_16x16x32_f16 a[164:167], v[104:107], v[136:139], a[164:167]\n"
  "buffer_load_dwordx4 v[24:27], v240, s[20:23], s46 offen offset:1536\n"
  "v_mfma_f32_16x16x32_f16 a[196:199], v[112:115], v[136:139], a[196:199]\n"
  "v_mfma_f32_16x16x32_f16 a[196:199], v[116:119], v[140:143], a[196:199]\n"
  "v_readlane_b32 s79, v248, s81\n"
  "v_mfma_f32_16x16x32_f16 a[228:231], v[124:127], v[140:143], a[228:231]\n"
  "s_cmp_eq_u32 s59, 59\ns_cselect_b32 s66, 0xf800, 0\n"
  "v_mfma_f32_16x16x32_f16 a[228:231], v[120:123], v[136:139], a[228:231]\n"
  "s_add_u32 s79, s79, s80\n"
  "s_waitcnt lgkmcnt(5)\n"
  "v_mfma_f32_16x16x32_f16 a[8:11], v[64:67], v[144:147], a[8:11]\n"
  "buffer_load_dwordx4 v[28:31], v241, s[24:27], s74 offen offset:1536\n"
  "v_mfma_f32_16x16x32_f16 a[8:11], v[68:71], v[148:151], a[8:11]\n"
  "ds_read_b128 v[136:139], v243 offset:5120\n"
  "s_add_u32 s28, s48, s79\ns_addc_u32 s29, s49, 0\n"
  "v_mfma_f32_16x16x32_f16 a[40:43], v[76:79], v[148:151], a[40:43]\n"
  "s_add_u32 s46, s46, s66\ns_add_u32 s74, s74, s66\n"
  "v_mfma_f32_16x16x32_f16 a[40:43], v[72:75], v[144:147], a[40:43]\n"
  "s_cmp_lt_u32 s58, 64\ns_cselect_b32 s30, s69, 0\n"
  "v_mfma_f32_16x16x32_f16 a[72:75], v[80:83], v[144:147], a[72:75]\n"
  "buffer_load_dwordx4 v[32:35], v241, s[24:27], s46 offen offset:2048\n"
  "v_mfma_f32_16x16x32_f16 a[72:75], v[84:87], v[148:151], a[72:75]\n"
  "s_add_u32 s58, s58, 1\n"
  "v_mfma_f32_16x16x32_f16 a[104:107], v[92:95], v[148:151], a[104:107]\n"
  "v_mfma_f32_16x16x32_f16 a[104:107], v[88:91], v[144:147], a[104:107]\n"
  "ds_read_b128 v[140:143], v243 offset:13312\n"
  "v_mfma_f32_16x16x32_f16 a[136:139], v[96:99], v[144:147], a[136:139]\n"
  "v_mfma_f32_16x16x32_f16 a[136:139], v[100:103], v[148:151], a[136:139]\n"
  "buffer_load_dwordx4 v[36:39], v240, s[20:23], s74 offen offset:2048\n"
  "v_mfma_f32_16x16x32_f16 a[168:171], v[108:111], v[148:151], a[168:171]\n"
  "v_mfma_f32_16x16x32_f16 a[168:171], v[104:107], v[144:147], a[168:171]\n"
  "v_mfma_f32_16x16x32_f16 a[200:203], v[112:115], v[144:147], a[200:203]\n"
  "s_add_u32 s79, s59, 3\ns_lshr_b32 s80, s79, 1\n"
  "v_mfma_f32_16x16x32_f16 a[200:203], v[116:119], v[148:151], a[200:203]\n"
  "v_mfma_f32_16x16x32_f16 a[232:235], v[124:127], v[148:151], a[232:235]\n"
  "buffer_load_dwordx4 v[40:43], v240, s[20:23], s46 offen offset:2560\n"
  "v_readlane_b32 s80, v249, s80\n"
  "v_mfma_f32_16x16x32_f16 a[232:235], v[120:123], v[144:147], a[232:235]\n"
  "s_waitcnt lgkmcnt(4)\n"
  "v_mfma_f32_16x16x32_f16 a[12:15], v[64:67], v[152:155], a[12:15]\n"
  "s_lshl_b32 s80, s80, 17\n"
  "v_mfma_f32_16x16x32_f16 a[12:15], v[68:71], v[156:159], a[12:15]\n"
  "ds_read_b128 v[144:147], v243 offset:6144\n"
  "v_mfma_f32_16x16x32_f16 a[44:47], v[76:79], v[156:159], a[44:47]\n"
  "buffer_load_dwordx4 v[44:47], v241, s[24:27], s74 offen offset:2560\n"
  "s_cmp_lt_u32 s79, 64\ns_cselect_b32 s60, s80, s84\n"
  "v_mfma_f32_16x16x32_f16 a[44:47], v[72:75], v[152:155], a[44:47]\n"
  "v_mfma_f32_16x16x32_f16 a[76:79], v[80:83], v[152:155], a[76:79]\n"
  "v_mfma_f32_16x16x32_f16 a[76:79], v[84:87], v[156:159], a[76:79]\n"
  "v_mfma_f32_16x16x32_f16 a[108:111], v[92:95], v[156:159], a[108:111]\n"
  "s_waitcnt vmcnt(21)\n"
  "v_cmp_le_u32 vcc, s61, v246\n"
  "v_mfma_f32_16x16x32_f16 a[108:111], v[88:91], v[152:155], a[108:111]\n"
  "buffer_load_dwordx4 v[48:51], v240, s[20:23], s46 offen offset:3072\n"
  "ds_read_b128 v[148:151], v243 offset:14336\n"
  "v_mfma_f32_16x16x32_f16 a[140:143], v[96:99], v[152:155], a[140:143]\n"
  "s_or_b32 s71, s71, vcc_lo\n"
  "v_mfma_f32_16x16x32_f16 a[140:143], v[100:103], v[156:159], a[140:143]\n"
  "v_mfma_f32_16x16x32_f16 a[172:175], v[108:111], v[156:159], a[172:175]\n"
  "v_mov_b32 v250, s71\n"
  "v_mfma_f32_16x16x32_f16 a[172:175], v[104:107], v[152:155], a[172:175]\n"
  "v_mfma_f32_16x16x32_f16 a[204:207], v[112:115], v[152:155], a[204:207]\n"
  "buffer_load_dwordx4 v[52:55], v241, s[24:27], s74 offen offset:3072\n"
  "ds_write_b32 v251, v250 offset:4\n"
  "v_mfma_f32_16x16x32_f16 a[204:207], v[116:119], v[156:159], a[204:207]\n"
  "v_mfma_f32_16x16x32_f16 a[236:239], v[124:127], v[156:159], a[236:239]\n"
  "v_mfma_f32_16x16x32_f16 a[236:239], v[120:123], v[152:155], a[236:239]\n"
  "ds_read_b128 v[152:155], v243 offset:7168\n"
  "s_waitcnt lgkmcnt(6)\n"
  "v_mfma_f32_16x16x32_f16 a[16:19], v[64:67], v[128:131], a[16:19]\n"
  "buffer_load_dwordx4 v[56:59], v241, s[24:27], s46 offen offset:3584\n"
  "v_mfma_f32_16x16x32_f16 a[16:19], v[68:71], v[132:135], a[16:19]\n"
  "v_mfma_f32_16x16x32_f16 a[48:51], v[76:79], v[132:135], a[48:51]\n"
  "v_mfma_f32_16x16x32_f16 a[48:51], v[72:75], v[128:131], a[48:51]\n"
  "v_mfma_f32_16x16x32_f16 a[80:83], v[80:83], v[128:131], a[80:83]\n"
  "v_mfma_f32_16x16x32_f16 a[80:83], v[84:87], v[132:135], a[80:83]\n"
  "buffer_load_dwordx4 v[60:63], v240, s[20:23], s74 offen offset:3584\n"
  "v_mfma_f32_16x16x32_f16 a[112:115], v[92:95], v[132:135], a[112:115]\n"
  "v_mfma_f32_16x16x32_f16 a[112:115], v[88:91], v[128:131], a[112:115]\n"
  "s_waitcnt vmcnt(23) lgkmcnt(1)\n"
  "s_barrier\n"
  "s_bitcmp1_b32 s75, s59\ns_cbranch_scc0 ENd1_%=\ns_call_b64 s[50:51], EVENT_%=\ns_branch EVd1_%=\nENd1_%=:\nbuffer_load_dword v246, off, s[20:23], s84\nEVd1_%=:\n"
  "v_mfma_f32_16x16x32_f16 a[144:147], v[96:99], v[128:131], a[144:147]\n"
  "buffer_load_dword v246, v245, s[36:39], 0 offen sc0 sc1\n"
  "v_mfma_f32_16x16x32_f16 a[144:147], v[100:103], v[132:135], a[144:147]\n"
  "buffer_load_dword v192, v245, s[20:23], s60 offen\n"
  "v_mfma_f32_16x16x32_f16 a[176:179], v[108:111], v[132:135], a[176:179]\n"
  "buffer_load_dwordx4 v[192:195], v242, s[28:31], 0 offen nt\n"
  "v_mfma_f32_16x16x32_f16 a[176:179], v[104:107], v[128:131], a[176:179]\n"
  "v_mov_b32 v250, 0x20000\n"
  "v_mfma_f32_16x16x32_f16 a[208:211], v[112:115], v[128:131], a[208:211]\n"
  "ds_read_b32 v250, v250 offset:4\n"
  "v_mfma_f32_16x16x32_f16 a[208:211], v[116:119], v[132:135], a[208:211]\n"
  "ds_read_b128 v[156:159], v243 offset:15360\n"
  "s_waitcnt vmcnt(20)\n"
  "v_cvt_pk_f16_f32 v224, v160, v164\n"
  "v_mfma_f32_16x16x32_f16 a[240:243], v[124:127], v[132:135], a[240:243]\n"
  "v_add_u32 v243, 0x4000, v243\n"
  "v_and_b32 v243, 0x1ffff, v243\n"
  "buffer_load_dwordx4 v[196:199], v242, s[28:31], s39 offen nt\n"
  "v_mfma_f32_16x16x32_f16 a[240:243], v[120:123], v[128:131], a[240:243]\n"
  "v_cvt_pk_f16_f32 v225, v168, v172\n"
  "v_mfma_f32_16x16x32_f16 a[20:23], v[64:67], v[136:139], a[20:23]\n"
  "v_mfma_f32_16x16x32_f16 a[20:23], v[68:71], v[140:143], a[20:23]\n"
  "ds_read_b128 v[128:131], v243 offset:0\n"
  "v_cvt_pk_f16_f32 v226, v176, v180\n"
  "v_mfma_f32_16x16x32_f16 a[52:55], v[76:79], v[140:143], a[52:55]\n"
  "buffer_load_dwordx4 v[200:203], v242, s[28:31], s40 offen nt\n"
  "v_mfma_f32_16x16x32_f16 a[52:55], v[72:75], v[136:139], a[52:55]\n"
  "v_mfma_f32_16x16x32_f16 a[84:87], v[80:83], v[136:139], a[84:87]\n"
  "v_cvt_pk_f16_f32 v227, v184, v188\n"
  "v_mfma_f32_16x16x32_f16 a[84:87], v[84:87], v[140:143], a[84:87]\n"
  "v_mfma_f32_16x16x32_f16 a[116:119], v[92:95], v[140:143], a[116:119]\n"
  "buffer_load_dwordx4 v[204:207], v242, s[28:31], s41 offen nt\n"
  "ds_write_b128 v244, v[224:227] offset:0\n"
  "v_mfma_f32_16x16x32_f16 a[116:119], v[88:91], v[136:139], a[116:119]\n"
  "ds_read_b128 v[132:135], v243 offset:8192\n"
  "v_mfma_f32_16x16x32_f16 a[148:151], v[96:99], v[136:139], a[148:151]\n"
  "v_mfma_f32_16x16x32_f16 a[148:151], v[100:103], v[140:143], a[148:151]\n"
  "v_cvt_pk_f16_f32 v228, v161, v165\n"
  "v_mfma_f32_16x16x32_f16 a[180:183], v[108:111], v[140:143], a[180:183]\n"
  "buffer_load_dwordx4 v[208:211], v242, s[28:31], s42 offen\n"
  "v_mfma_f32_16x16x32_f16 a[180:183], v[104:107], v[136:139], a[180:183]\n"
  "v_cvt_pk_f16_f32 v229, v169, v173\n"
  "v_mfma_f32_16x16x32_f16 a[212:215], v[112:115], v[136:139], a[212:215]\n"
  "v_mfma_f32_16x16x32_f16 a[212:215], v[116:119], v[140:143], a[212:215]\n"
  "v_cvt_pk_f16_f32 v230, v177, v181\n"
  "v_mfma_f32_16x16x32_f16 a[244:247], v[124:127], v[140:143], a[244:247]\n"
  "buffer_load_dwordx4 v[212:215], v242, s[28:31], s43 offen\n"
  "v_mfma_f32_16x16x32_f16 a[244:247], v[120:123], v[136:139], a[244:247]\n"
  "v_mfma_f32_16x16x32_f16 a[24:27], v[64:67], v[144:147], a[24:27]\n"
  "v_cvt_pk_f16_f32 v231, v185, v189\n"
  "v_mfma_f32_16x16x32_f16 a[24:27], v[68:71], v[148:151], a[24:27]\n"
  "ds_read_b128 v[136:139], v243 offset:1024\n"
  "v_mfma_f32_16x16x32_f16 a[56:59], v[76:79], v[148:151], a[56:59]\n"
  "buffer_load_dwordx4 v[216:219], v242, s[28:31], s44 offen\n"
  "ds_write_b128 v244, v[228:231] offset:1024\n"
  "v_mfma_f32_16x16x32_f16 a[56:59], v[72:75], v[144:147], a[56:59]\n"
  "s_waitcnt lgkmcnt(6)\n"
  "v_readfirstlane_b32 s70, v250\n"
  "v_mfma_f32_16x16x32_f16 a[88:91], v[80:83], v[144:147], a[88:91]\n"
  "v_mfma_f32_16x16x32_f16 a[88:91], v[84:87], v[148:151], a[88:91]\n"
  "v_cvt_pk_f16_f32 v224, v162, v166\n"
  "v_mfma_f32_16x16x32_f16 a[120:123], v[92:95], v[148:151], a[120:123]\n"
  "buffer_load_dwordx4 v[220:223], v242, s[28:31], s45 offen\n"
  "v_mfma_f32_16x16x32_f16 a[120:123], v[88:91], v[144:147], a[120:123]\n"
  "ds_read_b128 v[140:143], v243 offset:9216\n"
  "v_cvt_pk_f16_f32 v225, v170, v174\n"
  "v_mfma_f32_16x16x32_f16 a[152:155], v[96:99], v[144:147], a[152:155]\n"
  "v_mfma_f32_16x16x32_f16 a[152:155], v[100:103], v[148:151], a[152:155]\n"
  "v_cvt_pk_f16_f32 v226, v178, v182\n"
  "v_mfma_f32_16x16x32_f16 a[184:187], v[108:111], v[148:151], a[184:187]\n"
  "s_add_u32 s79, s59, 2\ns_lshr_b32 s80, s79, 1\n"
  "v_mfma_f32_16x16x32_f16 a[184:187], v[104:107], v[144:147], a[184:187]\n"
  "v_mfma_f32_16x16x32_f16 a[216:219], v[112:115], v[144:147], a[216:219]\n"
  "v_cvt_pk_f16_f32 v227, v186, v190\n"
  "v_readlane_b32 s80, v249, s80\n"
  "v_mfma_f32_16x16x32_f16 a[216:219], v[116:119], v[148:151], a[216:219]\n"
  "v_mfma_f32_16x16x32_f16 a[248:251], v[124:127], v[148:151], a[248:251]\n"
  "ds_write_b128 v244, v[224:227] offset:2048\n"
  "s_lshl_b32 s80, s80, 17\ns_add_u32 s80, s80, 0x10000\n"
  "v_mfma_f32_16x16x32_f16 a[248:251], v[120:123], v[144:147], a[248:251]\n"
  "s_waitcnt lgkmcnt(7)\n"
  "v_mfma_f32_16x16x32_f16 a[28:31], v[64:67], v[152:155], a[28:31]\n"
  "s_cmp_lt_u32 s79, 64\ns_cselect_b32 s46, s80, s84\n"
  "v_mfma_f32_16x16x32_f16 a[28:31], v[68:71], v[156:159], a[28:31]\n"
  "ds_read_b128 v[144:147], v243 offset:2048\n"
  "v_cvt_pk_f16_f32 v228, v163, v167\n"
  "v_mfma_f32_16x16x32_f16 a[60:63], v[76:79], v[156:159], a[60:63]\n"
  "v_mfma_f32_16x16x32_f16 a[60:63], v[72:75], v[152:155], a[60:63]\n"
  "v_cvt_pk_f16_f32 v229, v171, v175\n"
  "v_mfma_f32_16x16x32_f16 a[92:95], v[80:83], v[152:155], a[92:95]\n"
  "s_add_u32 s46, s46, s83\n"
  "v_mfma_f32_16x16x32_f16 a[92:95], v[84:87], v[156:159], a[92:95]\n"
  "v_cvt_pk_f16_f32 v230, v179, v183\n"
  "v_mfma_f32_16x16x32_f16 a[124:127], v[92:95], v[156:159], a[124:127]\n"
  "s_add_u32 s74, s46, 0x4000\n"
  "v_mfma_f32_16x16x32_f16 a[124:127], v[88:91], v[152:155], a[124:127]\n"
  "ds_read_b128 v[148:151], v243 offset:10240\n"
  "v_mfma_f32_16x16x32_f16 a[156:159], v[96:99], v[152:155], a[156:159]\n"
  "v_cvt_pk_f16_f32 v231, v187, v191\n"
  "s_add_u32 s59, s59, 1\n"
  "v_mfma_f32_16x16x32_f16 a[156:159], v[100:103], v[156:159], a[156:159]\n"
  "v_mfma_f32_16x16x32_f16 a[188:191], v[108:111], v[156:159], a[188:191]\n"
  "ds_write_b128 v244, v[228:231] offset:3072\n"
  "v_mfma_f32_16x16x32_f16 a[188:191], v[104:107], v[152:155], a[188:191]\n"
  "v_mfma_f32_16x16x32_f16 a[220:223], v[112:115], v[152:155], a[220:223]\n"
  "v_mfma_f32_16x16x32_f16 a[220:223], v[116:119], v[156:159], a[220:223]\n"
  "v_mfma_f32_16x16x32_f16 a[252:255], v[124:127], v[156:159], a[252:255]\n"
  "v_add_u32 v244, 0x4000, v244\n"
  "v_and_b32 v244, 0x1ffff, v244\n"
  "v_mfma_f32_16x16x32_f16 a[252:255], v[120:123], v[152:155], a[252:255]\n"
  "s_cmp_ge_u32 s59, 60\n"
  "s_cbranch_scc1 DONE_%=\n"
  "s_cmp_lt_u32 s59, 15\n"
  "s_cbranch_scc1 DLOOP_%=\n"
  "s_mov_b32 s75, 0\n"
  "s_cmp_lg_u32 s70, -1\n"
  "s_cbranch_scc1 DLOOP_%=\n"
  "HLOOP_%=:\n"
  "buffer_load_dwordx4 v[64:67], v240, s[20:23], s46 offen offset:0\n"
  "s_waitcnt vmcnt(10) lgkmcnt(7)\n"
  "v_mfma_f32_16x16x32_f16 a[0:3], v[0:3], v[128:131], a[0:3]\n"
  "v_mfma_f32_16x16x32_f16 a[0:3], v[4:7], v[132:135], a[0:3]\n"
  "ds_read_b128 v[152:155], v243 offset:3072\n"
  "v_mfma_f32_16x16x32_f16 a[32:35], v[12:15], v[132:135], a[32:35]\n"
  "v_mfma_f32_16x16x32_f16 a[32:35], v[8:11], v[128:131], a[32:35]\n"
  "v_mfma_f32_16x16x32_f16 a[64:67], v[16:19], v[128:131], a[64:67]\n"
  "buffer_load_dwordx4 v[68:71], v241, s[24:27], s74 offen offset:0\n"
  "v_mfma_f32_16x16x32_f16 a[64:67], v[20:23], v[132:135], a[64:67]\n"
  "v_mfma_f32_16x16x32_f16 a[96:99], v[28:31], v[132:135], a[96:99]\n"
  "v_mfma_f32_16x16x32_f16 a[96:99], v[24:27], v[128:131], a[96:99]\n"
  "ds_read_b128 v[156:159], v243 offset:11264\n"
  "v_mfma_f32_16x16x32_f16 a[128:131], v[32:35], v[128:131], a[128:131]\n"
  "buffer_load_dwordx4 v[72:75], v241, s[24:27], s46 offen offset:512\n"
  "v_mfma_f32_16x16x32_f16 a[128:131], v[36:39], v[132:135], a[128:131]\n"
  "v_mfma_f32_16x16x32_f16 a[160:163], v[44:47], v[132:135], a[160:163]\n"
  "v_mfma_f32_16x16x32_f16 a[160:163], v[40:43], v[128:131], a[160:163]\n"
  "v_mfma_f32_16x16x32_f16 a[192:195], v[48:51], v[128:131], a[192:195]\n"
  "v_mfma_f32_16x16x32_f16 a[192:195], v[52:55], v[132:135], a[192:195]\n"
  "buffer_load_dwordx4 v[76:79], v240, s[20:23], s74 offen offset:512\n"
  "v_mfma_f32_16x16x32_f16 a[224:227], v[60:63], v[132:135], a[224:227]\n"
  "v_mfma_f32_16x16x32_f16 a[224:227], v[56:59], v[128:131], a[224:227]\n"
  "s_waitcnt lgkmcnt(6)\n"
  "v_mfma_f32_16x16x32_f16 a[4:7], v[0:3], v[136:139], a[4:7]\n"
  "v_mfma_f32_16x16x32_f16 a[4:7], v[4:7], v[140:143], a[4:7]\n"
  "ds_read_b128 v[128:131], v243 offset:4096\n"
  "v_mfma_f32_16x16x32_f16 a[36:39], v[12:15], v[140:143], a[36:39]\n"
  "buffer_load_dwordx4 v[80:83], v241, s[24:27], s46 offen offset:1024\n"
  "v_mfma_f32_16x16x32_f16 a[36:39], v[8:11], v[136:139], a[36:39]\n"
  "v_mfma_f32_16x16x32_f16 a[68:71], v[16:19], v[136:139], a[68:71]\n"
  "s_lshr_b32 s79, s58, 1\n"
  "v_mfma_f32_16x16x32_f16 a[68:71], v[20:23], v[140:143], a[68:71]\n"
  "v_mfma_f32_16x16x32_f16 a[100:103], v[28:31], v[140:143], a[100:103]\n"
  "buffer_load_dwordx4 v[84:87], v240, s[20:23], s74 offen offset:1024\n"
  "v_readlane_b32 s79, v249, s79\n"
  "v_mfma_f32_16x16x32_f16 a[100:103], v[24:27], v[136:139], a[100:103]\n"
  "ds_read_b128 v[132:135], v243 offset:12288\n"
  "v_mfma_f32_16x16x32_f16 a[132:135], v[32:35], v[136:139], a[132:135]\n"
  "s_and_b32 s80, s58, 1\ns_lshl_b32 s80, s80, 23\n"
  "v_mfma_f32_16x16x32_f16 a[132:135], v[36:39], v[140:143], a[132:135]\n"
  "v_mfma_f32_16x16x32_f16 a[164:167], v[44:47], v[140:143], a[164:167]\n"
  "s_mov_b32 s81, s79\n"
  "v_mfma_f32_16x16x32_f16 a[164:167], v[40:43], v[136:139], a[164:167]\n"
  "buffer_load_dwordx4 v[88:91], v240, s[20:23], s46 offen offset:1536\n"
  "v_mfma_f32_16x16x32_f16 a[196:199], v[48:51], v[136:139], a[196:199]\n"
  "v_mfma_f32_16x16x32_f16 a[196:199], v[52:55], v[140:143], a[196:199]\n"
  "v_readlane_b32 s79, v248, s81\n"
  "v_mfma_f32_16x16x32_f16 a[228:231], v[60:63], v[140:143], a[228:231]\n"
  "v_mfma_f32_16x16x32_f16 a[228:231], v[56:59], v[136:139], a[228:231]\n"
  "s_add_u32 s79, s79, s80\n"
  "s_waitcnt lgkmcnt(5)\n"
  "v_mfma_f32_16x16x32_f16 a[8:11], v[0:3], v[144:147], a[8:11]\n"
  "buffer_load_dwordx4 v[92:95], v241, s[24:27], s74 offen offset:1536\n"
  "v_mfma_f32_16x16x32_f16 a[8:11], v[4:7], v[148:151], a[8:11]\n"
  "ds_read_b128 v[136:139], v243 offset:5120\n"
  "s_add_u32 s28, s48, s79\ns_addc_u32 s29, s49, 0\n"
  "v_mfma_f32_16x16x32_f16 a[40:43], v[12:15], v[148:151], a[40:43]\n"
  "v_mfma_f32_16x16x32_f16 a[40:43], v[8:11], v[144:147], a[40:43]\n"
  "s_cmp_lt_u32 s58, 64\ns_cselect_b32 s30, s69, 0\n"
  "v_mfma_f32_16x16x32_f16 a[72:75], v[16:19], v[144:147], a[72:75]\n"
  "buffer_load_dwordx4 v[96:99], v241, s[24:27], s46 offen offset:2048\n"
  "v_mfma_f32_16x16x32_f16 a[72:75], v[20:23], v[148:151], a[72:75]\n"
  "s_add_u32 s58, s58, 1\n"
  "v_mfma_f32_16x16x32_f16 a[104:107], v[28:31], v[148:151], a[104:107]\n"
  "v_mfma_f32_16x16x32_f16 a[104:107], v[24:27], v[144:147], a[104:107]\n"
  "ds_read_b128 v[140:143], v243 offset:13312\n"
  "v_mfma_f32_16x16x32_f16 a[136:139], v[32:35], v[144:147], a[136:139]\n"
  "v_mfma_f32_16x16x32_f16 a[136:139], v[36:39], v[148:151], a[136:139]\n"
  "buffer_load_dwordx4 v[100:103], v240, s[20:23], s74 offen offset:2048\n"
  "v_mfma_f32_16x16x32_f16 a[168:171], v[44:47], v[148:151], a[168:171]\n"
  "v_mfma_f32_16x16x32_f16 a[168:171], v[40:43], v[144:147], a[168:171]\n"
  "v_mfma_f32_16x16x32_f16 a[200:203], v[48:51], v[144:147], a[200:203]\n"
  "s_add_u32 s79, s59, 3\ns_lshr_b32 s80, s79, 1\n"
  "v_mfma_f32_16x16x32_f16 a[200:203], v[52:55], v[148:151], a[200:203]\n"
  "v_mfma_f32_16x16x32_f16 a[232:235], v[60:63], v[148:151], a[232:235]\n"
  "buffer_load_dwordx4 v[104:107], v240, s[20:23], s46 offen offset:2560\n"
  "v_readlane_b32 s80, v249, s80\n"
  "v_mfma_f32_16x16x32_f16 a[232:235], v[56:59], v[144:147], a[232:235]\n"
  "s_waitcnt lgkmcnt(4)\n"
  "v_mfma_f32_16x16x32_f16 a[12:15], v[0:3], v[152:155], a[12:15]\n"
  "s_lshl_b32 s80, s80, 17\ns_add_u32 s80, s80, 0x10000\n"
  "v_mfma_f32_16x16x32_f16 a[12:15], v[4:7], v[156:159], a[12:15]\n"
  "ds_read_b128 v[144:147], v243 offset:6144\n"
  "v_mfma_f32_16x16x32_f16 a[44:47], v[12:15], v[156:159], a[44:47]\n"
  "buffer_load_dwordx4 v[108:111], v241, s[24:27], s74 offen offset:2560\n"
  "s_cmp_lt_u32 s79, 64\ns_cselect_b32 s60, s80, s84\n"
  "v_mfma_f32_16x16x32_f16 a[44:47], v[8:11], v[152:155], a[44:47]\n"
  "v_mfma_f32_16x16x32_f16 a[76:79], v[16:19], v[152:155], a[76:79]\n"
  "v_mfma_f32_16x16x32_f16 a[76:79], v[20:23], v[156:159], a[76:79]\n"
  "v_mfma_f32_16x16x32_f16 a[108:111], v[28:31], v[156:159], a[108:111]\n"
  "v_mfma_f32_16x16x32_f16 a[108:111], v[24:27], v[152:155], a[108:111]\n"
  "buffer_load_dwordx4 v[112:115], v240, s[20:23], s46 offen offset:3072\n"
  "ds_read_b128 v[148:151], v243 offset:14336\n"
  "v_mfma_f32_16x16x32_f16 a[140:143], v[32:35], v[152:155], a[140:143]\n"
  "v_mfma_f32_16x16x32_f16 a[140:143], v[36:39], v[156:159], a[140:143]\n"
  "v_mfma_f32_16x16x32_f16 a[172:175], v[44:47], v[156:159], a[172:175]\n"
  "v_mfma_f32_16x16x32_f16 a[172:175], v[40:43], v[152:155], a[172:175]\n"
  "v_mfma_f32_16x16x32_f16 a[204:207], v[48:51], v[152:155], a[204:207]\n"
  "buffer_load_dwordx4 v[116:119], v241, s[24:27], s74 offen offset:3072\n"
  "v_mfma_f32_16x16x32_f16 a[204:207], v[52:55], v[156:159], a[204:207]\n"
  "v_mfma_f32_16x16x32_f16 a[236:239], v[60:63], v[156:159], a[236:239]\n"
  "v_mfma_f32_16x16x32_f16 a[236:239], v[56:59], v[152:155], a[236:239]\n"
  "ds_read_b128 v[152:155], v243 offset:7168\n"
  "s_waitcnt lgkmcnt(5)\n"
  "v_mfma_f32_16x16x32_f16 a[16:19], v[0:3], v[128:131], a[16:19]\n"
  "buffer_load_dwordx4 v[120:123], v241, s[24:27], s46 offen offset:3584\n"
  "v_mfma_f32_16x16x32_f16 a[16:19], v[4:7], v[132:135], a[16:19]\n"
  "v_mfma_f32_16x16x32_f16 a[48:51], v[12:15], v[132:135], a[48:51]\n"
  "v_mfma_f32_16x16x32_f16 a[48:51], v[8:11], v[128:131], a[48:51]\n"
  "v_mfma_f32_16x16x32_f16 a[80:83], v[16:19], v[128:131], a[80:83]\n"
  "v_mfma_f32_16x16x32_f16 a[80:83], v[20:23], v[132:135], a[80:83]\n"
  "buffer_load_dwordx4 v[124:127], v240, s[20:23], s74 offen offset:3584\n"
  "v_mfma_f32_16x16x32_f16 a[112:115], v[28:31], v[132:135], a[112:115]\n"
  "v_mfma_f32_16x16x32_f16 a[112:115], v[24:27], v[128:131], a[112:115]\n"
  "s_barrier\n"
  "buffer_load_dword v160, v245, s[20:23], s60 offen\n"
  "v_mfma_f32_16x16x32_f16 a[144:147], v[32:35], v[128:131], a[144:147]\n"
  "buffer_load_dwordx4 v[160:163], v242, s[28:31], 0 offen nt\n"
  "v_mfma_f32_16x16x32_f16 a[144:147], v[36:39], v[132:135], a[144:147]\n"
  "v_mfma_f32_16x16x32_f16 a[176:179], v[44:47], v[132:135], a[176:179]\n"
  "v_mfma_f32_16x16x32_f16 a[176:179], v[40:43], v[128:131], a[176:179]\n"
  "v_mfma_f32_16x16x32_f16 a[208:211], v[48:51], v[128:131], a[208:211]\n"
  "buffer_load_dwordx4 v[164:167], v242, s[28:31], s39 offen nt\n"
  "v_mfma_f32_16x16x32_f16 a[208:211], v[52:55], v[132:135], a[208:211]\n"
  "ds_read_b128 v[156:159], v243 offset:15360\n"
  "s_waitcnt vmcnt(19)\n"
  "v_cvt_pk_f16_f32 v224, v192, v196\n"
  "v_mfma_f32_16x16x32_f16 a[240:243], v[60:63], v[132:135], a[240:243]\n"
  "v_add_u32 v243, 0x4000, v243\n"
  "v_and_b32 v243, 0x1ffff, v243\n"
  "v_mfma_f32_16x16x32_f16 a[240:243], v[56:59], v[128:131], a[240:243]\n"
  "v_cvt_pk_f16_f32 v225, v200, v204\n"
  "s_waitcnt lgkmcnt(4)\n"
  "v_mfma_f32_16x16x32_f16 a[20:23], v[0:3], v[136:139], a[20:23]\n"
  "buffer_load_dwordx4 v[168:171], v242, s[28:31], s40 offen nt\n"
  "v_mfma_f32_16x16x32_f16 a[20:23], v[4:7], v[140:143], a[20:23]\n"
  "ds_read_b128 v[128:131], v243 offset:0\n"
  "v_cvt_pk_f16_f32 v226, v208, v212\n"
  "v_mfma_f32_16x16x32_f16 a[52:55], v[12:15], v[140:143], a[52:55]\n"
  "v_mfma_f32_16x16x32_f16 a[52:55], v[8:11], v[136:139], a[52:55]\n"
  "v_mfma_f32_16x16x32_f16 a[84:87], v[16:19], v[136:139], a[84:87]\n"
  "buffer_load_dwordx4 v[172:175], v242, s[28:31], s41 offen nt\n"
  "v_cvt_pk_f16_f32 v227, v216, v220\n"
  "v_mfma_f32_16x16x32_f16 a[84:87], v[20:23], v[140:143], a[84:87]\n"
  "v_mfma_f32_16x16x32_f16 a[116:119], v[28:31], v[140:143], a[116:119]\n"
  "ds_write_b128 v244, v[224:227] offset:0\n"
  "v_mfma_f32_16x16x32_f16 a[116:119], v[24:27], v[136:139], a[116:119]\n"
  "ds_read_b128 v[132:135], v243 offset:8192\n"
  "v_mfma_f32_16x16x32_f16 a[148:151], v[32:35], v[136:139], a[148:151]\n"
  "buffer_load_dwordx4 v[176:179], v242, s[28:31], s42 offen\n"
  "v_mfma_f32_16x16x32_f16 a[148:151], v[36:39], v[140:143], a[148:151]\n"
  "v_cvt_pk_f16_f32 v228, v193, v197\n"
  "v_mfma_f32_16x16x32_f16 a[180:183], v[44:47], v[140:143], a[180:183]\n"
  "v_mfma_f32_16x16x32_f16 a[180:183], v[40:43], v[136:139], a[180:183]\n"
  "v_cvt_pk_f16_f32 v229, v201, v205\n"
  "v_mfma_f32_16x16x32_f16 a[212:215], v[48:51], v[136:139], a[212:215]\n"
  "buffer_load_dwordx4 v[180:183], v242, s[28:31], s43 offen\n"
  "v_mfma_f32_16x16x32_f16 a[212:215], v[52:55], v[140:143], a[212:215]\n"
  "v_cvt_pk_f16_f32 v230, v209, v213\n"
  "v_mfma_f32_16x16x32_f16 a[244:247], v[60:63], v[140:143], a[244:247]\n"
  "v_mfma_f32_16x16x32_f16 a[244:247], v[56:59], v[136:139], a[244:247]\n"
  "s_waitcnt lgkmcnt(5)\n"
  "v_mfma_f32_16x16x32_f16 a[24:27], v[0:3], v[144:147], a[24:27]\n"
  "buffer_load_dwordx4 v[184:187], v242, s[28:31], s44 offen\n"
  "v_cvt_pk_f16_f32 v231, v217, v221\n"
  "v_mfma_f32_16x16x32_f16 a[24:27], v[4:7], v[148:151], a[24:27]\n"
  "ds_read_b128 v[136:139], v243 offset:1024\n"
  "v_mfma_f32_16x16x32_f16 a[56:59], v[12:15], v[148:151], a[56:59]\n"
  "ds_write_b128 v244, v[228:231] offset:1024\n"
  "v_mfma_f32_16x16x32_f16 a[56:59], v[8:11], v[144:147], a[56:59]\n"
  "v_mfma_f32_16x16x32_f16 a[88:91], v[16:19], v[144:147], a[88:91]\n"
  "buffer_load_dwordx4 v[188:191], v242, s[28:31], s45 offen\n"
  "v_mfma_f32_16x16x32_f16 a[88:91], v[20:23], v[148:151], a[88:91]\n"
  "v_cvt_pk_f16_f32 v224, v194, v198\n"
  "v_mfma_f32_16x16x32_f16 a[120:123], v[28:31], v[148:151], a[120:123]\n"
  "v_mfma_f32_16x16x32_f16 a[120:123], v[24:27], v[144:147], a[120:123]\n"
  "ds_read_b128 v[140:143], v243 offset:9216\n"
  "v_cvt_pk_f16_f32 v225, v202, v206\n"
  "v_mfma_f32_16x16x32_f16 a[152:155], v[32:35], v[144:147], a[152:155]\n"
  "v_mfma_f32_16x16x32_f16 a[152:155], v[36:39], v[148:151], a[152:155]\n"
  "v_cvt_pk_f16_f32 v226, v210, v214\n"
  "v_mfma_f32_16x16x32_f16 a[184:187], v[44:47], v[148:151], a[184:187]\n"
  "s_add_u32 s79, s59, 2\ns_lshr_b32 s80, s79, 1\n"
  "v_mfma_f32_16x16x32_f16 a[184:187], v[40:43], v[144:147], a[184:187]\n"
  "v_mfma_f32_16x16x32_f16 a[216:219], v[48:51], v[144:147], a[216:219]\n"
  "v_cvt_pk_f16_f32 v227, v218, v222\n"
  "v_readlane_b32 s80, v249, s80\n"
  "v_mfma_f32_16x16x32_f16 a[216:219], v[52:55], v[148:151], a[216:219]\n"
  "v_mfma_f32_16x16x32_f16 a[248:251], v[60:63], v[148:151], a[248:251]\n"
  "ds_write_b128 v244, v[224:227] offset:2048\n"
  "s_lshl_b32 s80, s80, 17\n"
  "v_mfma_f32_16x16x32_f16 a[248:251], v[56:59], v[144:147], a[248:251]\n"
  "s_waitcnt lgkmcnt(7)\n"
  "v_mfma_f32_16x16x32_f16 a[28:31], v[0:3], v[152:155], a[28:31]\n"
  "s_cmp_lt_u32 s79, 64\ns_cselect_b32 s46, s80, s84\n"
  "v_mfma_f32_16x16x32_f16 a[28:31], v[4:7], v[156:159], a[28:31]\n"
  "ds_read_b128 v[144:147], v243 offset:2048\n"
  "v_cvt_pk_f16_f32 v228, v195, v199\n"
  "v_mfma_f32_16x16x32_f16 a[60:63], v[12:15], v[156:159], a[60:63]\n"
  "v_mfma_f32_16x16x32_f16 a[60:63], v[8:11], v[152:155], a[60:63]\n"
  "v_cvt_pk_f16_f32 v229, v203, v207\n"
  "v_mfma_f32_16x16x32_f16 a[92:95], v[16:19], v[152:155], a[92:95]\n"
  "s_add_u32 s46, s46, s83\n"
  "v_mfma_f32_16x16x32_f16 a[92:95], v[20:23], v[156:159], a[92:95]\n"
  "v_cvt_pk_f16_f32 v230, v211, v215\n"
  "v_mfma_f32_16x16x32_f16 a[124:127], v[28:31], v[156:159], a[124:127]\n"
  "s_add_u32 s74, s46, 0x4000\n"
  "v_mfma_f32_16x16x32_f16 a[124:127], v[24:27], v[152:155], a[124:127]\n"
  "ds_read_b128 v[148:151], v243 offset:10240\n"
  "v_mfma_f32_16x16x32_f16 a[156:159], v[32:35], v[152:155], a[156:159]\n"
  "v_cvt_pk_f16_f32 v231, v219, v223\n"
  "s_add_u32 s59, s59, 1\n"
  "v_mfma_f32_16x16x32_f16 a[156:159], v[36:39], v[156:159], a[156:159]\n"
  "v_mfma_f32_16x16x32_f16 a[188:191], v[44:47], v[156:159], a[188:191]\n"
  "ds_write_b128 v244, v[228:231] offset:3072\n"
  "v_mfma_f32_16x16x32_f16 a[188:191], v[40:43], v[152:155], a[188:191]\n"
  "v_mfma_f32_16x16x32_f16 a[220:223], v[48:51], v[152:155], a[220:223]\n"
  "v_mfma_f32_16x16x32_f16 a[220:223], v[52:55], v[156:159], a[220:223]\n"
  "v_mfma_f32_16x16x32_f16 a[252:255], v[60:63], v[156:159], a[252:255]\n"
  "v_add_u32 v244, 0x4000, v244\n"
  "v_and_b32 v244, 0x1ffff, v244\n"
  "v_mfma_f32_16x16x32_f16 a[252:255], v[56:59], v[152:155], a[252:255]\n"
  "buffer_load_dwordx4 v[0:3], v240, s[20:23], s46 offen offset:0\n"
  "s_waitcnt vmcnt(10) lgkmcnt(7)\n"
  "v_mfma_f32_16x16x32_f16 a[0:3], v[64:67], v[128:131], a[0:3]\n"
  "v_mfma_f32_16x16x32_f16 a[0:3], v[68:71], v[132:135], a[0:3]\n"
  "ds_read_b128 v[152:155], v243 offset:3072\n"
  "v_mfma_f32_16x16x32_f16 a[32:35], v[76:79], v[132:135], a[32:35]\n"
  "v_mfma_f32_16x16x32_f16 a[32:35], v[72:75], v[128:131], a[32:35]\n"
  "v_mfma_f32_16x16x32_f16 a[64:67], v[80:83], v[128:131], a[64:67]\n"
  "buffer_load_dwordx4 v[4:7], v241, s[24:27], s74 offen offset:0\n"
  "v_mfma_f32_16x16x32_f16 a[64:67], v[84:87], v[132:135], a[64:67]\n"
  "v_mfma_f32_16x16x32_f16 a[96:99], v[92:95], v[132:135], a[96:99]\n"
  "s_andn2_b32 s79, s70, s72\n"
  "v_mfma_f32_16x16x32_f16 a[96:99], v[88:91], v[128:131], a[96:99]\n"
  "ds_read_b128 v[156:159], v243 offset:11264\n"
  "v_mfma_f32_16x16x32_f16 a[128:131], v[96:99], v[128:131], a[128:131]\n"
  "buffer_load_dwordx4 v[8:11], v241, s[24:27], s46 offen offset:512\n"
  "v_mfma_f32_16x16x32_f16 a[128:131], v[100:103], v[132:135], a[128:131]\n"
  "v_mfma_f32_16x16x32_f16 a[160:163], v[108:111], v[132:135], a[160:163]\n"
  "s_ff1_i32_b32 s79, s79\n"
  "v_mfma_f32_16x16x32_f16 a[160:163], v[104:107], v[128:131], a[160:163]\n"
  "v_mfma_f32_16x16x32_f16 a[192:195], v[112:115], v[128:131], a[192:195]\n"
  "s_bitset1_b32 s72, s79\ns_mov_b32 m0, s73\n"
  "v_mfma_f32_16x16x32_f16 a[192:195], v[116:119], v[132:135], a[192:195]\n"
  "buffer_load_dwordx4 v[12:15], v240, s[20:23], s74 offen offset:512\n"
  "v_mfma_f32_16x16x32_f16 a[224:227], v[124:127], v[132:135], a[224:227]\n"
  "v_writelane_b32 v249, s79, m0\n"
  "v_mfma_f32_16x16x32_f16 a[224:227], v[120:123], v[128:131], a[224:227]\n"
  "s_waitcnt lgkmcnt(6)\n"
  "v_mfma_f32_16x16x32_f16 a[4:7], v[64:67], v[136:139], a[4:7]\n"
  "s_add_u32 s73, s73, 1\n"
  "v_mfma_f32_16x16x32_f16 a[4:7], v[68:71], v[140:143], a[4:7]\n"
  "ds_read_b128 v[128:131], v243 offset:4096\n"
  "v_mfma_f32_16x16x32_f16 a[36:39], v[76:79], v[140:143], a[36:39]\n"
  "buffer_load_dwordx4 v[16:19], v241, s[24:27], s46 offen offset:1024\n"
  "v_mfma_f32_16x16x32_f16 a[36:39], v[72:75], v[136:139], a[36:39]\n"
  "v_mfma_f32_16x16x32_f16 a[68:71], v[80:83], v[136:139], a[68:71]\n"
  "s_lshr_b32 s79, s58, 1\n"
  "v_mfma_f32_16x16x32_f16 a[68:71], v[84:87], v[140:143], a[68:71]\n"
  "v_mfma_f32_16x16x32_f16 a[100:103], v[92:95], v[140:143], a[100:103]\n"
  "buffer_load_dwordx4 v[20:23], v240, s[20:23], s74 offen offset:1024\n"
  "v_readlane_b32 s79, v249, s79\n"
  "v_mfma_f32_16x16x32_f16 a[100:103], v[88:91], v[136:139], a[100:103]\n"
  "ds_read_b128 v[132:135], v243 offset:12288\n"
  "v_mfma_f32_16x16x32_f16 a[132:135], v[96:99], v[136:139], a[132:135]\n"
  "s_and_b32 s80, s58, 1\ns_lshl_b32 s80, s80, 23\n"
  "v_mfma_f32_16x16x32_f16 a[132:135], v[100:103], v[140:143], a[132:135]\n"
  "v_mfma_f32_16x16x32_f16 a[164:167], v[108:111], v[140:143], a[164:167]\n"
  "s_mov_b32 s81, s79\n"
  "v_mfma_f32_16x16x32_f16 a[164:167], v[104:107], v[136:139], a[164:167]\n"
  "buffer_load_dwordx4 v[24:27], v240, s[20:23], s46 offen offset:1536\n"
  "v_mfma_f32_16x16x32_f16 a[196:199], v[112:115], v[136:139], a[196:199]\n"
  "v_mfma_f32_16x16x32_f16 a[196:199], v[116:119], v[140:143], a[196:199]\n"
  "v_readlane_b32 s79, v248, s81\n"
  "v_mfma_f32_16x16x32_f16 a[228:231], v[124:127], v[140:143], a[228:231]\n"
  "s_cmp_eq_u32 s59, 59\ns_cselect_b32 s66, 0xf800, 0\n"
  "v_mfma_f32_16x16x32_f16 a[228:231], v[120:123], v[136:139], a[228:231]\n"
  "s_add_u32 s79, s79, s80\n"
  "s_waitcnt lgkmcnt(5)\n"
  "v_mfma_f32_16x16x32_f16 a[8:11], v[64:67], v[144:147], a[8:11]\n"
  "buffer_load_dwordx4 v[28:31], v241, s[24:27], s74 offen offset:1536\n"
  "v_mfma_f32_16x16x32_f16 a[8:11], v[68:71], v[148:151], a[8:11]\n"
  "ds_read_b128 v[136:139], v243 offset:5120\n"
  "s_add_u32 s28, s48, s79\ns_addc_u32 s29, s49, 0\n"
  "v_mfma_f32_16x16x32_f16 a[40:43], v[76:79], v[148:151], a[40:43]\n"
  "s_add_u32 s46, s46, s66\ns_add_u32 s74, s74, s66\n"
  "v_mfma_f32_16x16x32_f16 a[40:43], v[72:75], v[144:147], a[40:43]\n"
  "s_cmp_lt_u32 s58, 64\ns_cselect_b32 s30, s69, 0\n"
  "v_mfma_f32_16x16x32_f16 a[72:75], v[80:83], v[144:147], a[72:75]\n"
  "buffer_load_dwordx4 v[32:35], v241, s[24:27], s46 offen offset:2048\n"
  "v_mfma_f32_16x16x32_f16 a[72:75], v[84:87], v[148:151], a[72:75]\n"
  "s_add_u32 s58, s58, 1\n"
  "v_mfma_f32_16x16x32_f16 a[104:107], v[92:95], v[148:151], a[104:107]\n"
  "v_mfma_f32_16x16x32_f16 a[104:107], v[88:91], v[144:147], a[104:107]\n"
  "ds_read_b128 v[140:143], v243 offset:13312\n"
  "v_mfma_f32_16x16x32_f16 a[136:139], v[96:99], v[144:147], a[136:139]\n"
  "v_mfma_f32_16x16x32_f16 a[136:139], v[100:103], v[148:151], a[136:139]\n"
  "buffer_load_dwordx4 v[36:39], v240, s[20:23], s74 offen offset:2048\n"
  "v_mfma_f32_16x16x32_f16 a[168:171], v[108:111], v[148:151], a[168:171]\n"
  "v_mfma_f32_16x16x32_f16 a[168:171], v[104:107], v[144:147], a[168:171]\n"
  "v_mfma_f32_16x16x32_f16 a[200:203], v[112:115], v[144:147], a[200:203]\n"
  "s_add_u32 s79, s59, 3\ns_lshr_b32 s80, s79, 1\n"
  "v_mfma_f32_16x16x32_f16 a[200:203], v[116:119], v[148:151], a[200:203]\n"
  "v_mfma_f32_16x16x32_f16 a[232:235], v[124:127], v[148:151], a[232:235]\n"
  "buffer_load_dwordx4 v[40:43], v240, s[20:23], s46 offen offset:2560\n"
  "v_readlane_b32 s80, v249, s80\n"
  "v_mfma_f32_16x16x32_f16 a[232:235], v[120:123], v[144:147], a[232:235]\n"
  "s_waitcnt lgkmcnt(4)\n"
  "v_mfma_f32_16x16x32_f16 a[12:15], v[64:67], v[152:155], a[12:15]\n"
  "s_lshl_b32 s80, s80, 17\n"
  "v_mfma_f32_16x16x32_f16 a[12:15], v[68:71], v[156:159], a[12:15]\n"
  "ds_read_b128 v[144:147], v243 offset:6144\n"
  "v_mfma_f32_16x16x32_f16 a[44:47], v[76:79], v[156:159], a[44:47]\n"
  "buffer_load_dwordx4 v[44:47], v241, s[24:27], s74 offen offset:2560\n"
  "s_cmp_lt_u32 s79, 64\ns_cselect_b32 s60, s80, s84\n"
  "v_mfma_f32_16x16x32_f16 a[44:47], v[72:75], v[152:155], a[44:47]\n"
  "v_mfma_f32_16x16x32_f16 a[76:79], v[80:83], v[152:155], a[76:79]\n"
  "v_mfma_f32_16x16x32_f16 a[76:79], v[84:87], v[156:159], a[76:79]\n"
  "v_mfma_f32_16x16x32_f16 a[108:111], v[92:95], v[156:159], a[108:111]\n"
  "v_mfma_f32_16x16x32_f16 a[108:111], v[88:91], v[152:155], a[108:111]\n"
  "buffer_load_dwordx4 v[48:51], v240, s[20:23], s46 offen offset:3072\n"
  "ds_read_b128 v[148:151], v243 offset:14336\n"
  "v_mfma_f32_16x16x32_f16 a[140:143], v[96:99], v[152:155], a[140:143]\n"
  "v_mfma_f32_16x16x32_f16 a[140:143], v[100:103], v[156:159], a[140:143]\n"
  "v_mfma_f32_16x16x32_f16 a[172:175], v[108:111], v[156:159], a[172:175]\n"
  "v_mfma_f32_16x16x32_f16 a[172:175], v[104:107], v[152:155], a[172:175]\n"
  "v_mfma_f32_16x16x32_f16 a[204:207], v[112:115], v[152:155], a[204:207]\n"
  "buffer_load_dwordx4 v[52:55], v241, s[24:27], s74 offen offset:3072\n"
  "v_mfma_f32_16x16x32_f16 a[204:207], v[116:119], v[156:159], a[204:207]\n"
  "v_mfma_f32_16x16x32_f16 a[236:239], v[124:127], v[156:159], a[236:239]\n"
  "v_mfma_f32_16x16x32_f16 a[236:239], v[120:123], v[152:155], a[236:239]\n"
  "ds_read_b128 v[152:155], v243 offset:7168\n"
  "s_waitcnt lgkmcnt(5)\n"
  "v_mfma_f32_16x16x32_f16 a[16:19], v[64:67], v[128:131], a[16:19]\n"
  "buffer_load_dwordx4 v[56:59], v241, s[24:27], s46 offen offset:3584\n"
  "v_mfma_f32_16x16x32_f16 a[16:19], v[68:71], v[132:135], a[16:19]\n"
  "v_mfma_f32_16x16x32_f16 a[48:51], v[76:79], v[132:135], a[48:51]\n"
  "v_mfma_f32_16x16x32_f16 a[48:51], v[72:75], v[128:131], a[48:51]\n"
  "v_mfma_f32_16x16x32_f16 a[80:83], v[80:83], v[128:131], a[80:83]\n"
  "v_mfma_f32_16x16x32_f16 a[80:83], v[84:87], v[132:135], a[80:83]\n"
  "buffer_load_dwordx4 v[60:63], v240, s[20:23], s74 offen offset:3584\n"
  "v_mfma_f32_16x16x32_f16 a[112:115], v[92:95], v[132:135], a[112:115]\n"
  "v_mfma_f32_16x16x32_f16 a[112:115], v[88:91], v[128:131], a[112:115]\n"
  "s_barrier\n"
  "buffer_load_dword v192, v245, s[20:23], s60 offen\n"
  "v_mfma_f32_16x16x32_f16 a[144:147], v[96:99], v[128:131], a[144:147]\n"
  "buffer_load_dwordx4 v[192:195], v242, s[28:31], 0 offen nt\n"
  "v_mfma_f32_16x16x32_f16 a[144:147], v[100:103], v[132:135], a[144:147]\n"
  "v_mfma_f32_16x16x32_f16 a[176:179], v[108:111], v[132:135], a[176:179]\n"
  "v_mfma_f32_16x16x32_f16 a[176:179], v[104:107], v[128:131], a[176:179]\n"
  "v_mfma_f32_16x16x32_f16 a[208:211], v[112:115], v[128:131], a[208:211]\n"
  "buffer_load_dwordx4 v[196:199], v242, s[28:31], s39 offen nt\n"
  "v_mfma_f32_16x16x32_f16 a[208:211], v[116:119], v[132:135], a[208:211]\n"
  "ds_read_b128 v[156:159], v243 offset:15360\n"
  "s_waitcnt vmcnt(19)\n"
  "v_cvt_pk_f16_f32 v224, v160, v164\n"
  "v_mfma_f32_16x16x32_f16 a[240:243], v[124:127], v[132:135], a[240:243]\n"
  "v_add_u32 v243, 0x4000, v243\n"
  "v_and_b32 v243, 0x1ffff, v243\n"
  "v_mfma_f32_16x16x32_f16 a[240:243], v[120:123], v[128:131], a[240:243]\n"
  "v_cvt_pk_f16_f32 v225, v168, v172\n"
  "s_waitcnt lgkmcnt(4)\n"
  "v_mfma_f32_16x16x32_f16 a[20:23], v[64:67], v[136:139], a[20:23]\n"
  "buffer_load_dwordx4 v[200:203], v242, s[28:31], s40 offen nt\n"
  "v_mfma_f32_16x16x32_f16 a[20:23], v[68:71], v[140:143], a[20:23]\n"
  "ds_read_b128 v[128:131], v243 offset:0\n"
  "v_cvt_pk_f16_f32 v226, v176, v180\n"
  "v_mfma_f32_16x16x32_f16 a[52:55], v[76:79], v[140:143], a[52:55]\n"
  "v_mfma_f32_16x16x32_f16 a[52:55], v[72:75], v[136:139], a[52:55]\n"
  "v_mfma_f32_16x16x32_f16 a[84:87], v[80:83], v[136:139], a[84:87]\n"
  "buffer_load_dwordx4 v[204:207], v242, s[28:31], s41 offen nt\n"
  "v_cvt_pk_f16_f32 v227, v184, v188\n"
  "v_mfma_f32_16x16x32_f16 a[84:87], v[84:87], v[140:143], a[84:87]\n"
  "v_mfma_f32_16x16x32_f16 a[116:119], v[92:95], v[140:143], a[116:119]\n"
  "ds_write_b128 v244, v[224:227] offset:0\n"
  "v_mfma_f32_16x16x32_f16 a[116:119], v[88:91], v[136:139], a[116:119]\n"
  "ds_read_b128 v[132:135], v243 offset:8192\n"
  "v_mfma_f32_16x16x32_f16 a[148:151], v[96:99], v[136:139], a[148:151]\n"
  "buffer_load_dwordx4 v[208:211], v242, s[28:31], s42 offen\n"
  "v_mfma_f32_16x16x32_f16 a[148:151], v[100:103], v[140:143], a[148:151]\n"
  "v_cvt_pk_f16_f32 v228, v161, v165\n"
  "v_mfma_f32_16x16x32_f16 a[180:183], v[108:111], v[140:143], a[180:183]\n"
  "v_mfma_f32_16x16x32_f16 a[180:183], v[104:107], v[136:139], a[180:183]\n"
  "v_cvt_pk_f16_f32 v229, v169, v173\n"
  "v_mfma_f32_16x16x32_f16 a[212:215], v[112:115], v[136:139], a[212:215]\n"
  "buffer_load_dwordx4 v[212:215], v242, s[28:31], s43 offen\n"
  "v_mfma_f32_16x16x32_f16 a[212:215], v[116:119], v[140:143], a[212:215]\n"
  "v_cvt_pk_f16_f32 v230, v177, v181\n"
  "v_mfma_f32_16x16x32_f16 a[244:247], v[124:127], v[140:143], a[244:247]\n"
  "v_mfma_f32_16x16x32_f16 a[244:247], v[120:123], v[136:139], a[244:247]\n"
  "s_waitcnt lgkmcnt(5)\n"
  "v_mfma_f32_16x16x32_f16 a[24:27], v[64:67], v[144:147], a[24:27]\n"
  "buffer_load_dwordx4 v[216:219], v242, s[28:31], s44 offen\n"
  "v_cvt_pk_f16_f32 v231, v185, v189\n"
  "v_mfma_f32_16x16x32_f16 a[24:27], v[68:71], v[148:151], a[24:27]\n"
  "ds_read_b128 v[136:139], v243 offset:1024\n"
  "v_mfma_f32_16x16x32_f16 a[56:59], v[76:79], v[148:151], a[56:59]\n"
  "ds_write_b128 v244, v[228:231] offset:1024\n"
  "v_mfma_f32_16x16x32_f16 a[56:59], v[72:75], v[144:147], a[56:59]\n"
  "v_mfma_f32_16x16x32_f16 a[88:91], v[80:83], v[144:147], a[88:91]\n"
  "buffer_load_dwordx4 v[220:223], v242, s[28:31], s45 offen\n"
  "v_mfma_f32_16x16x32_f16 a[88:91], v[84:87], v[148:151], a[88:91]\n"
  "v_cvt_pk_f16_f32 v224, v162, v166\n"
  "v_mfma_f32_16x16x32_f16 a[120:123], v[92:95], v[148:151], a[120:123]\n"
  "v_mfma_f32_16x16x32_f16 a[120:123], v[88:91], v[144:147], a[120:123]\n"
  "ds_read_b128 v[140:143], v243 offset:9216\n"
  "v_cvt_pk_f16_f32 v225, v170, v174\n"
  "v_mfma_f32_16x16x32_f16 a[152:155], v[96:99], v[144:147], a[152:155]\n"
  "v_mfma_f32_16x16x32_f16 a[152:155], v[100:103], v[148:151], a[152:155]\n"
  "v_cvt_pk_f16_f32 v226, v178, v182\n"
  "v_mfma_f32_16x16x32_f16 a[184:187], v[108:111], v[148:151], a[184:187]\n"
  "s_add_u32 s79, s59, 2\ns_lshr_b32 s80, s79, 1\n"
  "v_mfma_f32_16x16x32_f16 a[184:187], v[104:107], v[144:147], a[184:187]\n"
  "v_mfma_f32_16x16x32_f16 a[216:219], v[112:115], v[144:147], a[216:219]\n"
  "v_cvt_pk_f16_f32 v227, v186, v190\n"
  "v_readlane_b32 s80, v249, s80\n"
  "v_mfma_f32_16x16x32_f16 a[216:219], v[116:119], v[148:151], a[216:219]\n"
  "v_mfma_f32_16x16x32_f16 a[248:251], v[124:127], v[148:151], a[248:251]\n"
  "ds_write_b128 v244, v[224:227] offset:2048\n"
  "s_lshl_b32 s80, s80, 17\ns_add_u32 s80, s80, 0x10000\n"
  "v_mfma_f32_16x16x32_f16 a[248:251], v[120:123], v[144:147], a[248:251]\n"
  "s_waitcnt lgkmcnt(7)\n"
  "v_mfma_f32_16x16x32_f16 a[28:31], v[64:67], v[152:155], a[28:31]\n"
  "s_cmp_lt_u32 s79, 64\ns_cselect_b32 s46, s80, s84\n"
  "v_mfma_f32_16x16x32_f16 a[28:31], v[68:71], v[156:159], a[28:31]\n"
  "ds_read_b128 v[144:147], v243 offset:2048\n"
  "v_cvt_pk_f16_f32 v228, v163, v167\n"
  "v_mfma_f32_16x16x32_f16 a[60:63], v[76:79], v[156:159], a[60:63]\n"
  "v_mfma_f32_16x16x32_f16 a[60:63], v[72:75], v[152:155], a[60:63]\n"
  "v_cvt_pk_f16_f32 v229, v171, v175\n"
  "v_mfma_f32_16x16x32_f16 a[92:95], v[80:83], v[152:155], a[92:95]\n"
  "s_add_u32 s46, s46, s83\n"
  "v_mfma_f32_16x16x32_f16 a[92:95], v[84:87], v[156:159], a[92:95]\n"
  "v_cvt_pk_f16_f32 v230, v179, v183\n"
  "v_mfma_f32_16x16x32_f16 a[124:127], v[92:95], v[156:159], a[124:127]\n"
  "s_add_u32 s74, s46, 0x4000\n"
  "v_mfma_f32_16x16x32_f16 a[124:127], v[88:91], v[152:155], a[124:127]\n"
  "ds_read_b128 v[148:151], v243 offset:10240\n"
  "v_mfma_f32_16x16x32_f16 a[156:159], v[96:99], v[152:155], a[156:159]\n"
  "v_cvt_pk_f16_f32 v231, v187, v191\n"
  "s_add_u32 s59, s59, 1\n"
  "v_mfma_f32_16x16x32_f16 a[156:159], v[100:103], v[156:159], a[156:159]\n"
  "v_mfma_f32_16x16x32_f16 a[188:191], v[108:111], v[156:159], a[188:191]\n"
  "ds_write_b128 v244, v[228:231] offset:3072\n"
  "v_mfma_f32_16x16x32_f16 a[188:191], v[104:107], v[152:155], a[188:191]\n"
  "v_mfma_f32_16x16x32_f16 a[220:223], v[112:115], v[152:155], a[220:223]\n"
  "v_mfma_f32_16x16x32_f16 a[220:223], v[116:119], v[156:159], a[220:223]\n"
  "v_mfma_f32_16x16x32_f16 a[252:255], v[124:127], v[156:159], a[252:255]\n"
  "v_add_u32 v244, 0x4000, v244\n"
  "v_and_b32 v244, 0x1ffff, v244\n"
  "v_mfma_f32_16x16x32_f16 a[252:255], v[120:123], v[152:155], a[252:255]\n"
  "s_cmp_lt_u32 s59, 60\n"
  "s_cbranch_scc1 HLOOP_%=\n"
  "s_branch DONE_%=\n"
  "EVENT_%=:\n"
  "s_and_b32 s63, s77, 3\n"
  "s_add_u32 s77, s77, 1\n"
  "s_cmp_eq_u32 s63, 1\n"
  "s_cbranch_scc1 EVS_%=\n"
  "s_cmp_gt_u32 s63, 1\n"
  "s_cbranch_scc1 EVA_%=\n"
  "s_sub_u32 s63, s77, 1\n"
  "s_add_u32 s64, s77, 1\n"
  "s_add_u32 s65, s77, 2\n"
  "v_readlane_b32 s88, v247, s63\n"
  "v_readlane_b32 s76, v247, s77\n"
  "v_readlane_b32 s56, v247, s64\n"
  "v_readlane_b32 s57, v247, s65\n"
  "s_nop 0\n"
  "s_mov_b64 s[32:33], %4\n"
  "s_mov_b32 s34, 0x800000\n"
  "buffer_load_dwordx4 v[232:235], v252, s[32:35], s88 offen nt\n"
  "buffer_load_dwordx4 v[236:239], v252, s[32:35], s88 offen offset:16 nt\n"
  "s_branch EVX_%=\n"
  "EVS_%=:\n"
  "v_cvt_pk_f16_f32 v232, v232, v233\n"
  "v_cvt_pk_f16_f32 v233, v234, v235\n"
  "v_cvt_pk_f16_f32 v234, v236, v237\n"
  "v_cvt_pk_f16_f32 v235, v238, v239\n"
  "s_mov_b64 s[32:33], s[56:57]\n"
  "s_mov_b32 s34, 0x900000\n"
  "s_add_u32 s63, s76, s68\n"
  "v_and_b32 v224, 0xffff, v240\n"
  "buffer_store_dwordx4 v[232:235], v224, s[32:35], s63 offen sc0 sc1\n"
  "s_cmp_lg_u32 %7, 0\n"
  "s_cselect_b32 s63, 2, 0\n"
  "s_add_u32 s77, s77, s63\n"
  "s_branch EVX_%=\n"
  "EVA_%=:\n"
  "s_lshr_b32 s64, s76, 17\n"
  "s_mov_b32 s34, 0x900000\n"
  "s_cmp_eq_u32 s63, 3\n"
  "s_cbranch_scc1 EVF_%=\n"
  "s_mov_b64 s[32:33], %1\n"
  "s_lshl_b32 s64, s64, 7\n"
  "s_add_u32 s64, s64, 0x8e00\n"
  "v_mov_b32 v253, 4\n"
  "s_mov_b64 exec, 1\n"
  "buffer_atomic_add v253, off, s[32:35], s64 sc0\n"
  "s_mov_b64 exec, -1\n"
  "s_branch EVX_%=\n"
  "EVF_%=:\n"
  "v_readfirstlane_b32 s63, v253\n"
  "s_lshl_b32 s65, s61, 7\n"
  "s_add_u32 s63, s63, 4\n"
  "s_cmp_lg_u32 s63, s65\n"
  "s_cbranch_scc1 EVD2_%=\n"
  "s_mov_b64 s[32:33], s[56:57]\n"
  "s_lshl_b32 s64, s64, 3\n"
  "s_add_u32 s64, s64, 0xa000\n"
  "v_mov_b32 v253, s61\n"
  "s_mov_b64 exec, 1\n"
  "buffer_store_dword v253, off, s[32:35], s64 sc0 sc1\n"
  "s_mov_b64 exec, -1\n"
  "s_branch EVX_%=\n"
  "EVD2_%=:\n"
  "buffer_load_dword v246, off, s[20:23], s84\n"
  "EVX_%=:\n"
  "s_setpc_b64 s[50:51]\n"
  "PUSH_%=:\n"
  "s_lshl_b32 s66, s88, 2\n"
  "s_mov_b32 s34, 0x900000\n"
  "s_waitcnt vmcnt(27)\n"
  "s_cmp_lt_u32 s66, 12\n"
  "s_cbranch_scc1 PU1_%=\n"
  "v_readfirstlane_b32 s63, v253\n"
  "s_lshl_b32 s65, s61, 7\n"
  "s_add_u32 s63, s63, 1\n"
  "s_cmp_lg_u32 s63, s65\n"
  "s_cbranch_scc1 PU1_%=\n"
  "s_sub_u32 s63, s66, 7\n"
  "s_sub_u32 s64, s66, 6\n"
  "s_sub_u32 s65, s66, 5\n"
  "v_readlane_b32 s63, v247, s63\n"
  "v_readlane_b32 s32, v247, s64\n"
  "v_readlane_b32 s33, v247, s65\n"
  "s_lshr_b32 s63, s63, 17\n"
  "s_lshl_b32 s63, s63, 3\n"
  "s_add_u32 s63, s63, 0xa000\n"
  "v_mov_b32 v250, s61\n"
  "s_mov_b64 exec, 1\n"
  "s_nop 0\n"
  "buffer_store_dword v250, off, s[32:35], s63 sc0 sc1\n"
  "s_mov_b64 exec, -1\n"
  "PU1_%=:\n"
  "s_sub_u32 s63, s66, 8\n"
  "s_cmp_ge_u32 s63, 12\n"
  "s_cbranch_scc1 PU2_%=\n"
  "s_sub_u32 s63, s66, 3\n"
  "v_readlane_b32 s63, v247, s63\n"
  "s_mov_b64 s[32:33], %1\n"
  "s_lshr_b32 s63, s63, 17\n"
  "s_lshl_b32 s63, s63, 7\n"
  "s_add_u32 s63, s63, 0x8e00\n"
  "v_mov_b32 v253, 1\n"
  "s_mov_b64 exec, 1\n"
  "buffer_atomic_add v253, off, s[32:35], s63 sc0\n"
  "s_mov_b64 exec, -1\n"
  "PU2_%=:\n"
  "s_sub_u32 s63, s66, 4\n"
  "s_cmp_ge_u32 s63, 12\n"
  "s_cbranch_scc1 PU3_%=\n"
  "s_add_u32 s63, s66, 1\n"
  "s_add_u32 s64, s66, 2\n"
  "s_add_u32 s65, s66, 3\n"
  "v_readlane_b32 s63, v247, s63\n"
  "v_readlane_b32 s32, v247, s64\n"
  "v_readlane_b32 s33, v247, s65\n"
  "v_cvt_pk_f16_f32 v232, v232, v233\n"
  "v_cvt_pk_f16_f32 v233, v234, v235\n"
  "v_cvt_pk_f16_f32 v234, v236, v237\n"
  "v_cvt_pk_f16_f32 v235, v238, v239\n"
  "s_add_u32 s63, s63, s68\n"
  "v_and_b32 v236, 0xffff, v240\n"
  "buffer_store_dwordx4 v[232:235], v236, s[32:35], s63 offen sc0 sc1\n"
  "PU3_%=:\n"
  "s_cmp_ge_u32 s66, 12\n"
  "s_cbranch_scc1 PU4_%=\n"
  "s_add_u32 s63, s66, 4\n"
  "v_readlane_b32 s88, v247, s63\n"
  "s_nop 2\n"
  "s_mov_b64 s[32:33], %4\n"
  "s_mov_b32 s34, 0x800000\n"
  "buffer_load_dwordx4 v[232:235], v252, s[32:35], s88 offen nt\n"
  "buffer_load_dwordx4 v[236:239], v252, s[32:35], s88 offen offset:16 nt\n"
  "PU4_%=:\n"
  "s_cmp_lg_u32 s66, 12\n"
  "s_cbranch_scc1 CQN_%=\n"
  "s_getpc_b64 s[54:55]\n"
  "CPN_%=:\n"
  "s_add_u32 s54, s54, KS_%=-CPN_%=\n"
  "s_addc_u32 s55, s55, -1\n"
  "v_and_b32 v237, 0xffff, v245\n"
  "v_min_u32 v237, KE_%=-KS_%=-4, v237\n"
  "global_load_dword v237, v237, s[54:55]\n"
  "CQN_%=:\n"
  "s_setpc_b64 s[50:51]\n"
  "WAITANY_%=:\n"
  "s_cmp_eq_u32 s72, -1\n"
  "s_cbranch_scc1 WAR_%=\n"
  "s_cmp_lg_u32 %7, 0\n"
  "s_cbranch_scc1 WAB_%=\n"
  "s_memrealtime s[54:55]\n"
  "s_mov_b64 s[32:33], %1\n"
  "s_mov_b32 s34, 0x900000\n"
  "WAL_%=:\n"
  "buffer_load_dword v246, v245, s[36:39], 0 offen sc0 sc1\n"
  "s_memrealtime s[52:53]\n"
  "s_waitcnt vmcnt(0) lgkmcnt(0)\n"
  "v_cmp_le_u32 vcc, s61, v246\n"
  "s_nop 1\n"
  "s_or_b32 s71, s71, vcc_lo\n"
  "s_andn2_b32 s79, s71, s72\n"
  "s_cbranch_scc1 WAD_%=\n"
  "s_sub_u32 s63, s52, s54\n"
  "s_cmp_gt_u32 s63, 100000000\n"
  "s_cbranch_scc1 WAT_%=\n"
  "s_cmp_lt_u32 s63, 5000\n"
  "s_cbranch_scc1 WAL_%=\n"
  "s_mov_b32 s63, 0x8000\n"
  "buffer_load_dword v246, off, s[32:35], s63 sc1\n"
  "s_waitcnt vmcnt(0)\n"
  "v_readfirstlane_b32 s63, v246\n"
  "s_cmp_eq_u32 s63, s61\n"
  "s_cbranch_scc1 WAT_%=\n"
  "s_sleep 2\n"
  "s_branch WAL_%=\n"
  "WAT_%=:\n"
  "s_mov_b32 s63, 0x8000\n"
  "v_mov_b32 v246, s61\n"
  "buffer_store_dword v246, off, s[32:35], s63 sc1\n"
  "s_mov_b32 s71, 0\n"
  "WAD_%=:\n"
  "v_mov_b32 v246, 0\n"
  "v_mov_b32 v250, s71\n"
  "ds_write_b32 v251, v250 offset:8\n"
  "s_waitcnt lgkmcnt(0)\n"
  "WAB_%=:\n"
  "s_barrier\n"
  "v_mov_b32 v250, 0x20000\n"
  "ds_read_b32 v250, v250 offset:8\n"
  "s_waitcnt lgkmcnt(0)\n"
  "v_readfirstlane_b32 s70, v250\n"
  "s_cmp_lg_u32 s70, 0\n"
  "s_cbranch_scc1 WAO_%=\n"
  "s_mov_b32 s67, 0x7fc00000\n"
  "s_mov_b32 s70, -1\n"
  "s_mov_b32 s71, -1\n"
  "WAO_%=:\n"
  "s_andn2_b32 s79, s70, s72\n"
  "WAR_%=:\n"
  "s_setpc_b64 s[50:51]\n"
  "DONE_%=:\n"
  "s_lshr_b32 s79, s58, 1\n"
  "s_waitcnt vmcnt(17) lgkmcnt(7)\n"
  "v_mfma_f32_16x16x32_f16 a[0:3], v[0:3], v[128:131], a[0:3]\n"
  "ds_read_b128 v[152:155], v243 offset:3072\n"
  "v_readlane_b32 s79, v249, s79\n"
  "s_nop 3\n"
  "v_mfma_f32_16x16x32_f16 a[0:3], v[4:7], v[132:135], a[0:3]\n"
  "s_and_b32 s80, s58, 1\ns_lshl_b32 s80, s80, 23\n"
  "s_nop 3\n"
  "v_mfma_f32_16x16x32_f16 a[32:35], v[12:15], v[132:135], a[32:35]\n"
  "s_mov_b32 s81, s79\n"
  "s_nop 3\n"
  "v_mfma_f32_16x16x32_f16 a[32:35], v[8:11], v[128:131], a[32:35]\n"
  "ds_read_b128 v[156:159], v243 offset:11264\n"
  "v_readlane_b32 s79, v248, s81\n"
  "v_mfma_f32_16x16x32_f16 a[64:67], v[16:19], v[128:131], a[64:67]\n"
  "s_add_u32 s79, s79, s80\n"
  "s_nop 3\n"
  "v_mfma_f32_16x16x32_f16 a[64:67], v[20:23], v[132:135], a[64:67]\n"
  "s_add_u32 s28, s48, s79\ns_addc_u32 s29, s49, 0\n"
  "s_nop 3\n"
  "v_mfma_f32_16x16x32_f16 a[96:99], v[28:31], v[132:135], a[96:99]\n"
  "s_cmp_lt_u32 s58, 64\ns_cselect_b32 s30, s69, 0\n"
  "v_mfma_f32_16x16x32_f16 a[96:99], v[24:27], v[128:131], a[96:99]\n"
  "s_add_u32 s58, s58, 1\n"
  "s_nop 3\n"
  "s_waitcnt lgkmcnt(6)\n"
  "v_mfma_f32_16x16x32_f16 a[4:7], v[0:3], v[136:139], a[4:7]\n"
  "ds_read_b128 v[128:131], v243 offset:4096\n"
  "v_readlane_b32 s80, v249, 31\n"
  "s_nop 3\n"
  "v_mfma_f32_16x16x32_f16 a[4:7], v[4:7], v[140:143], a[4:7]\n"
  "s_lshl_b32 s80, s80, 17\n"
  "v_mfma_f32_16x16x32_f16 a[36:39], v[12:15], v[140:143], a[36:39]\n"
  "s_add_u32 s60, s80, 0x10000\n"
  "s_nop 3\n"
  "v_mfma_f32_16x16x32_f16 a[36:39], v[8:11], v[136:139], a[36:39]\n"
  "ds_read_b128 v[132:135], v243 offset:12288\n"
  "s_nop 3\n"
  "v_mfma_f32_16x16x32_f16 a[68:71], v[16:19], v[136:139], a[68:71]\n"
  "s_nop 3\n"
  "v_mfma_f32_16x16x32_f16 a[68:71], v[20:23], v[140:143], a[68:71]\n"
  "s_mov_b64 s[32:33], %1\ns_mov_b32 s34, 0x900000\ns_mov_b32 s66, 0xb000\ns_cmp_lg_u32 %7, 0\ns_cbranch_scc1 GA_%=\nv_mov_b32 v253, 1\ns_mov_b64 exec, 1\nbuffer_atomic_add v253, off, s[32:35], s66 sc0\ns_mov_b64 exec, -1\nGA_%=:\n"
  "v_mfma_f32_16x16x32_f16 a[100:103], v[28:31], v[140:143], a[100:103]\n"
  "buffer_load_dword v160, v245, s[20:23], s60 offen\n"
  "s_nop 3\n"
  "v_mfma_f32_16x16x32_f16 a[100:103], v[24:27], v[136:139], a[100:103]\n"
  "s_nop 3\n"
  "s_waitcnt lgkmcnt(5)\n"
  "v_mfma_f32_16x16x32_f16 a[8:11], v[0:3], v[144:147], a[8:11]\n"
  "ds_read_b128 v[136:139], v243 offset:5120\n"
  "v_mfma_f32_16x16x32_f16 a[8:11], v[4:7], v[148:151], a[8:11]\n"
  "buffer_load_dwordx4 v[160:163], v242, s[28:31], 0 offen nt\n"
  "s_nop 3\n"
  "v_mfma_f32_16x16x32_f16 a[40:43], v[12:15], v[148:151], a[40:43]\n"
  "s_nop 3\n"
  "v_mfma_f32_16x16x32_f16 a[40:43], v[8:11], v[144:147], a[40:43]\n"
  "ds_read_b128 v[140:143], v243 offset:13312\n"
  "v_mfma_f32_16x16x32_f16 a[72:75], v[16:19], v[144:147], a[72:75]\n"
  "s_nop 3\n"
  "v_mfma_f32_16x16x32_f16 a[72:75], v[20:23], v[148:151], a[72:75]\n"
  "buffer_load_dwordx4 v[164:167], v242, s[28:31], s39 offen nt\n"
  "s_nop 3\n"
  "v_mfma_f32_16x16x32_f16 a[104:107], v[28:31], v[148:151], a[104:107]\n"
  "v_mfma_f32_16x16x32_f16 a[104:107], v[24:27], v[144:147], a[104:107]\n"
  "s_nop 3\n"
  "s_waitcnt lgkmcnt(4)\n"
  "v_mfma_f32_16x16x32_f16 a[12:15], v[0:3], v[152:155], a[12:15]\n"
  "ds_read_b128 v[144:147], v243 offset:6144\n"
  "s_nop 3\n"
  "v_mfma_f32_16x16x32_f16 a[12:15], v[4:7], v[156:159], a[12:15]\n"
  "buffer_load_dwordx4 v[168:171], v242, s[28:31], s40 offen nt\n"
  "s_nop 3\n"
  "v_mfma_f32_16x16x32_f16 a[44:47], v[12:15], v[156:159], a[44:47]\n"
  "v_mfma_f32_16x16x32_f16 a[44:47], v[8:11], v[152:155], a[44:47]\n"
  "ds_read_b128 v[148:151], v243 offset:14336\n"
  "s_nop 3\n"
  "v_mfma_f32_16x16x32_f16 a[76:79], v[16:19], v[152:155], a[76:79]\n"
  "s_nop 3\n"
  "v_mfma_f32_16x16x32_f16 a[76:79], v[20:23], v[156:159], a[76:79]\n"
  "buffer_load_dwordx4 v[172:175], v242, s[28:31], s41 offen nt\n"
  "v_mfma_f32_16x16x32_f16 a[108:111], v[28:31], v[156:159], a[108:111]\n"
  "s_nop 3\n"
  "v_mfma_f32_16x16x32_f16 a[108:111], v[24:27], v[152:155], a[108:111]\n"
  "s_nop 3\n"
  "s_waitcnt lgkmcnt(4)\n"
  "v_mfma_f32_16x16x32_f16 a[16:19], v[0:3], v[128:131], a[16:19]\n"
  "ds_read_b128 v[152:155], v243 offset:7168\n"
  "v_mfma_f32_16x16x32_f16 a[16:19], v[4:7], v[132:135], a[16:19]\n"
  "buffer_load_dwordx4 v[176:179], v242, s[28:31], s42 offen nt\n"
  "s_nop 3\n"
  "v_mfma_f32_16x16x32_f16 a[48:51], v[12:15], v[132:135], a[48:51]\n"
  "s_barrier\n"
  "s_nop 3\n"
  "v_mfma_f32_16x16x32_f16 a[48:51], v[8:11], v[128:131], a[48:51]\n"
  "ds_read_b128 v[156:159], v243 offset:15360\n"
  "v_mfma_f32_16x16x32_f16 a[80:83], v[16:19], v[128:131], a[80:83]\n"
  "s_nop 3\n"
  "v_mfma_f32_16x16x32_f16 a[80:83], v[20:23], v[132:135], a[80:83]\n"
  "buffer_load_dwordx4 v[180:183], v242, s[28:31], s43 offen nt\n"
  "s_nop 3\n"
  "v_mfma_f32_16x16x32_f16 a[112:115], v[28:31], v[132:135], a[112:115]\n"
  "s_nop 3\n"
  "v_mfma_f32_16x16x32_f16 a[112:115], v[24:27], v[128:131], a[112:115]\n"
  "v_add_u32 v243, 0x4000, v243\nv_and_b32 v243, 0x1ffff, v243\n"
  "s_waitcnt lgkmcnt(4)\n"
  "v_mfma_f32_16x16x32_f16 a[20:23], v[0:3], v[136:139], a[20:23]\n"
  "ds_read_b128 v[128:131], v243 offset:0\n"
  "s_nop 3\n"
  "v_mfma_f32_16x16x32_f16 a[20:23], v[4:7], v[140:143], a[20:23]\n"
  "buffer_load_dwordx4 v[184:187], v242, s[28:31], s44 offen nt\n"
  "s_nop 3\n"
  "v_mfma_f32_16x16x32_f16 a[52:55], v[12:15], v[140:143], a[52:55]\n"
  "v_mfma_f32_16x16x32_f16 a[52:55], v[8:11], v[136:139], a[52:55]\n"
  "ds_read_b128 v[132:135], v243 offset:8192\n"
  "s_nop 3\n"
  "v_mfma_f32_16x16x32_f16 a[84:87], v[16:19], v[136:139], a[84:87]\n"
  "s_nop 3\n"
  "v_mfma_f32_16x16x32_f16 a[84:87], v[20:23], v[140:143], a[84:87]\n"
  "buffer_load_dwordx4 v[188:191], v242, s[28:31], s45 offen nt\n"
  "v_mfma_f32_16x16x32_f16 a[116:119], v[28:31], v[140:143], a[116:119]\n"
  "s_waitcnt vmcnt(9)\n"
  "v_cvt_pk_f16_f32 v224, v192, v196\n"
  "s_nop 3\n"
  "v_mfma_f32_16x16x32_f16 a[116:119], v[24:27], v[136:139], a[116:119]\n"
  "v_cvt_pk_f16_f32 v225, v200, v204\n"
  "s_nop 3\n"
  "s_waitcnt lgkmcnt(4)\n"
  "v_mfma_f32_16x16x32_f16 a[24:27], v[0:3], v[144:147], a[24:27]\n"
  "ds_read_b128 v[136:139], v243 offset:1024\n"
  "v_mfma_f32_16x16x32_f16 a[24:27], v[4:7], v[148:151], a[24:27]\n"
  "v_cvt_pk_f16_f32 v226, v208, v212\n"
  "s_nop 3\n"
  "v_mfma_f32_16x16x32_f16 a[56:59], v[12:15], v[148:151], a[56:59]\n"
  "v_cvt_pk_f16_f32 v227, v216, v220\n"
  "s_nop 3\n"
  "v_mfma_f32_16x16x32_f16 a[56:59], v[8:11], v[144:147], a[56:59]\n"
  "ds_read_b128 v[140:143], v243 offset:9216\n"
  "s_nop 3\n"
  "v_mfma_f32_16x16x32_f16 a[88:91], v[16:19], v[144:147], a[88:91]\n"
  "ds_write_b128 v244, v[224:227] offset:0\n"
  "v_mfma_f32_16x16x32_f16 a[88:91], v[20:23], v[148:151], a[88:91]\n"
  "v_cvt_pk_f16_f32 v228, v193, v197\n"
  "s_nop 3\n"
  "v_mfma_f32_16x16x32_f16 a[120:123], v[28:31], v[148:151], a[120:123]\n"
  "v_cvt_pk_f16_f32 v229, v201, v205\n"
  "s_nop 3\n"
  "v_mfma_f32_16x16x32_f16 a[120:123], v[24:27], v[144:147], a[120:123]\n"
  "v_cvt_pk_f16_f32 v230, v209, v213\n"
  "s_waitcnt lgkmcnt(5)\n"
  "v_mfma_f32_16x16x32_f16 a[28:31], v[0:3], v[152:155], a[28:31]\n"
  "ds_read_b128 v[144:147], v243 offset:2048\n"
  "s_nop 3\n"
  "v_mfma_f32_16x16x32_f16 a[28:31], v[4:7], v[156:159], a[28:31]\n"
  "v_cvt_pk_f16_f32 v231, v217, v221\n"
  "s_nop 3\n"
  "v_mfma_f32_16x16x32_f16 a[60:63], v[12:15], v[156:159], a[60:63]\n"
  "ds_write_b128 v244, v[228:231] offset:1024\n"
  "v_mfma_f32_16x16x32_f16 a[60:63], v[8:11], v[152:155], a[60:63]\n"
  "ds_read_b128 v[148:151], v243 offset:10240\n"
  "s_nop 3\n"
  "v_mfma_f32_16x16x32_f16 a[92:95], v[16:19], v[152:155], a[92:95]\n"
  "v_cvt_pk_f16_f32 v224, v194, v198\n"
  "s_nop 3\n"
  "v_mfma_f32_16x16x32_f16 a[92:95], v[20:23], v[156:159], a[92:95]\n"
  "v_cvt_pk_f16_f32 v225, v202, v206\n"
  "v_mfma_f32_16x16x32_f16 a[124:127], v[28:31], v[156:159], a[124:127]\n"
  "v_readlane_b32 s80, v249, 31\n"
  "s_nop 3\n"
  "v_mfma_f32_16x16x32_f16 a[124:127], v[24:27], v[152:155], a[124:127]\n"
  "s_lshl_b32 s80, s80, 17\ns_add_u32 s80, s80, s83\n"
  "s_nop 3\n"
  "s_waitcnt lgkmcnt(6)\n"
  "v_mfma_f32_16x16x32_f16 a[0:3], v[32:35], v[128:131], a[0:3]\n"
  "ds_read_b128 v[152:155], v243 offset:3072\n"
  "s_add_u32 s46, s80, 0x0\ns_add_u32 s74, s80, 0x4000\n"
  "s_nop 3\n"
  "v_mfma_f32_16x16x32_f16 a[0:3], v[36:39], v[132:135], a[0:3]\n"
  "buffer_load_dwordx4 v[64:67], v240, s[20:23], s46 offen offset:0\n"
  "v_mfma_f32_16x16x32_f16 a[32:35], v[44:47], v[132:135], a[32:35]\n"
  "v_cvt_pk_f16_f32 v226, v210, v214\n"
  "s_nop 3\n"
  "v_mfma_f32_16x16x32_f16 a[32:35], v[40:43], v[128:131], a[32:35]\n"
  "ds_read_b128 v[156:159], v243 offset:11264\n"
  "s_nop 3\n"
  "v_mfma_f32_16x16x32_f16 a[64:67], v[48:51], v[128:131], a[64:67]\n"
  "v_cvt_pk_f16_f32 v227, v218, v222\n"
  "v_mfma_f32_16x16x32_f16 a[64:67], v[52:55], v[132:135], a[64:67]\n"
  "buffer_load_dwordx4 v[68:71], v241, s[24:27], s74 offen offset:0\n"
  "s_nop 3\n"
  "v_mfma_f32_16x16x32_f16 a[96:99], v[60:63], v[132:135], a[96:99]\n"
  "ds_write_b128 v244, v[224:227] offset:2048\n"
  "s_nop 3\n"
  "v_mfma_f32_16x16x32_f16 a[96:99], v[56:59], v[128:131], a[96:99]\n"
  "v_cvt_pk_f16_f32 v228, v195, v199\n"
  "s_waitcnt lgkmcnt(7)\n"
  "v_mfma_f32_16x16x32_f16 a[4:7], v[32:35], v[136:139], a[4:7]\n"
  "ds_read_b128 v[128:131], v243 offset:4096\n"
  "s_nop 3\n"
  "v_mfma_f32_16x16x32_f16 a[4:7], v[36:39], v[140:143], a[4:7]\n"
  "buffer_load_dwordx4 v[72:75], v241, s[24:27], s46 offen offset:512\n"
  "s_nop 3\n"
  "v_mfma_f32_16x16x32_f16 a[36:39], v[44:47], v[140:143], a[36:39]\n"
  "v_cvt_pk_f16_f32 v229, v203, v207\n"
  "v_mfma_f32_16x16x32_f16 a[36:39], v[40:43], v[136:139], a[36:39]\n"
  "ds_read_b128 v[132:135], v243 offset:12288\n"
  "s_nop 3\n"
  "v_mfma_f32_16x16x32_f16 a[68:71], v[48:51], v[136:139], a[68:71]\n"
  "v_cvt_pk_f16_f32 v230, v211, v215\n"
  "s_nop 3\n"
  "v_mfma_f32_16x16x32_f16 a[68:71], v[52:55], v[140:143], a[68:71]\n"
  "buffer_load_dwordx4 v[76:79], v240, s[20:23], s74 offen offset:512\n"
  "s_nop 3\n"
  "v_mfma_f32_16x16x32_f16 a[100:103], v[60:63], v[140:143], a[100:103]\n"
  "v_cvt_pk_f16_f32 v231, v219, v223\n"
  "v_mfma_f32_16x16x32_f16 a[100:103], v[56:59], v[136:139], a[100:103]\n"
  "ds_write_b128 v244, v[228:231] offset:3072\n"
  "s_nop 3\n"
  "s_waitcnt lgkmcnt(6)\n"
  "v_mfma_f32_16x16x32_f16 a[8:11], v[32:35], v[144:147], a[8:11]\n"
  "ds_read_b128 v[136:139], v243 offset:5120\n"
  "s_nop 3\n"
  "v_mfma_f32_16x16x32_f16 a[8:11], v[36:39], v[148:151], a[8:11]\n"
  "buffer_load_dwordx4 v[80:83], v241, s[24:27], s46 offen offset:1024\n"
  "v_mfma_f32_16x16x32_f16 a[40:43], v[44:47], v[148:151], a[40:43]\n"
  "v_add_u32 v244, 0x4000, v244\nv_and_b32 v244, 0x1ffff, v244\n"
  "s_nop 3\n"
  "v_mfma_f32_16x16x32_f16 a[40:43], v[40:43], v[144:147], a[40:43]\n"
  "ds_read_b128 v[140:143], v243 offset:13312\n"
  "s_nop 3\n"
  "v_mfma_f32_16x16x32_f16 a[72:75], v[48:51], v[144:147], a[72:75]\n"
  "v_mfma_f32_16x16x32_f16 a[72:75], v[52:55], v[148:151], a[72:75]\n"
  "buffer_load_dwordx4 v[84:87], v240, s[20:23], s74 offen offset:1024\n"
  "s_nop 3\n"
  "v_mfma_f32_16x16x32_f16 a[104:107], v[60:63], v[148:151], a[104:107]\n"
  "s_nop 3\n"
  "v_mfma_f32_16x16x32_f16 a[104:107], v[56:59], v[144:147], a[104:107]\n"
  "s_waitcnt lgkmcnt(6)\n"
  "v_mfma_f32_16x16x32_f16 a[12:15], v[32:35], v[152:155], a[12:15]\n"
  "ds_read_b128 v[144:147], v243 offset:6144\n"
  "s_nop 3\n"
  "v_mfma_f32_16x16x32_f16 a[12:15], v[36:39], v[156:159], a[12:15]\n"
  "buffer_load_dwordx4 v[88:91], v240, s[20:23], s46 offen offset:1536\n"
  "s_nop 3\n"
  "v_mfma_f32_16x16x32_f16 a[44:47], v[44:47], v[156:159], a[44:47]\n"
  "s_nop 3\n"
  "v_mfma_f32_16x16x32_f16 a[44:47], v[40:43], v[152:155], a[44:47]\n"
  "ds_read_b128 v[148:151], v243 offset:14336\n"
  "v_mfma_f32_16x16x32_f16 a[76:79], v[48:51], v[152:155], a[76:79]\n"
  "s_nop 3\n"
  "v_mfma_f32_16x16x32_f16 a[76:79], v[52:55], v[156:159], a[76:79]\n"
  "buffer_load_dwordx4 v[92:95], v241, s[24:27], s74 offen offset:1536\n"
  "s_nop 3\n"
  "v_mfma_f32_16x16x32_f16 a[108:111], v[60:63], v[156:159], a[108:111]\n"
  "v_readlane_b32 s80, v249, 31\n"
  "v_mfma_f32_16x16x32_f16 a[108:111], v[56:59], v[152:155], a[108:111]\n"
  "s_lshl_b32 s80, s80, 17\ns_add_u32 s80, s80, s83\n"
  "s_nop 3\n"
  "s_waitcnt lgkmcnt(5)\n"
  "v_mfma_f32_16x16x32_f16 a[16:19], v[32:35], v[128:131], a[16:19]\n"
  "ds_read_b128 v[152:155], v243 offset:7168\n"
  "s_add_u32 s46, s80, 0x10000\ns_add_u32 s74, s80, 0x14000\n"
  "s_nop 3\n"
  "v_mfma_f32_16x16x32_f16 a[16:19], v[36:39], v[132:135], a[16:19]\n"
  "buffer_load_dwordx4 v[96:99], v240, s[20:23], s46 offen offset:0\n"
  "v_mfma_f32_16x16x32_f16 a[48:51], v[44:47], v[132:135], a[48:51]\n"
  "s_nop 3\n"
  "v_mfma_f32_16x16x32_f16 a[48:51], v[40:43], v[128:131], a[48:51]\n"
  "ds_read_b128 v[156:159], v243 offset:15360\n"
  "s_nop 3\n"
  "v_mfma_f32_16x16x32_f16 a[80:83], v[48:51], v[128:131], a[80:83]\n"
  "v_mfma_f32_16x16x32_f16 a[80:83], v[52:55], v[132:135], a[80:83]\n"
  "buffer_load_dwordx4 v[100:103], v241, s[24:27], s74 offen offset:0\n"
  "s_nop 3\n"
  "v_mfma_f32_16x16x32_f16 a[112:115], v[60:63], v[132:135], a[112:115]\n"
  "s_waitcnt lgkmcnt(6)\n"
  "s_barrier\n"
  "s_nop 3\n"
  "v_mfma_f32_16x16x32_f16 a[112:115], v[56:59], v[128:131], a[112:115]\n"
  "v_add_u32 v243, 0x4000, v243\nv_and_b32 v243, 0x1ffff, v243\n"
  "s_nop 3\n"
  "s_waitcnt lgkmcnt(4)\n"
  "v_mfma_f32_16x16x32_f16 a[20:23], v[32:35], v[136:139], a[20:23]\n"
  "ds_read_b128 v[128:131], v243 offset:0\n"
  "v_mfma_f32_16x16x32_f16 a[20:23], v[36:39], v[140:143], a[20:23]\n"
  "buffer_load_dwordx4 v[104:107], v241, s[24:27], s46 offen offset:512\n"
  "s_nop 3\n"
  "v_mfma_f32_16x16x32_f16 a[52:55], v[44:47], v[140:143], a[52:55]\n"
  "s_nop 3\n"
  "v_mfma_f32_16x16x32_f16 a[52:55], v[40:43], v[136:139], a[52:55]\n"
  "ds_read_b128 v[132:135], v243 offset:8192\n"
  "v_mfma_f32_16x16x32_f16 a[84:87], v[48:51], v[136:139], a[84:87]\n"
  "s_nop 3\n"
  "v_mfma_f32_16x16x32_f16 a[84:87], v[52:55], v[140:143], a[84:87]\n"
  "buffer_load_dwordx4 v[108:111], v240, s[20:23], s74 offen offset:512\n"
  "s_waitcnt vmcnt(12)\n"
  "s_cmp_lg_u32 %7, 0\ns_cbranch_scc1 GB_%=\nv_readfirstlane_b32 s63, v253\ns_lshl_b32 s64, s61, 8\ns_add_u32 s63, s63, 1\ns_cmp_lg_u32 s63, s64\ns_cbranch_scc1 GB_%=\ns_memrealtime s[52:53]\nv_mov_b32 v232, s61\ns_waitcnt lgkmcnt(0)\nv_mov_b32 v233, s52\ns_mov_b64 exec, 1\nbuffer_store_dwordx2 v[232:233], off, s[32:35], s66 offset:128 sc1\ns_mov_b64 exec, -1\nGB_%=:\n"
  "s_nop 3\n"
  "v_mfma_f32_16x16x32_f16 a[116:119], v[60:63], v[140:143], a[116:119]\n"
  "v_mfma_f32_16x16x32_f16 a[116:119], v[56:59], v[136:139], a[116:119]\n"
  "s_nop 3\n"
  "s_waitcnt lgkmcnt(4)\n"
  "v_mfma_f32_16x16x32_f16 a[24:27], v[32:35], v[144:147], a[24:27]\n"
  "ds_read_b128 v[136:139], v243 offset:1024\n"
  "s_nop 3\n"
  "v_mfma_f32_16x16x32_f16 a[24:27], v[36:39], v[148:151], a[24:27]\n"
  "buffer_load_dwordx4 v[112:115], v241, s[24:27], s46 offen offset:1024\n"
  "v_mfma_f32_16x16x32_f16 a[56:59], v[44:47], v[148:151], a[56:59]\n"
  "s_nop 3\n"
  "v_mfma_f32_16x16x32_f16 a[56:59], v[40:43], v[144:147], a[56:59]\n"
  "ds_read_b128 v[140:143], v243 offset:9216\n"
  "s_nop 3\n"
  "v_mfma_f32_16x16x32_f16 a[88:91], v[48:51], v[144:147], a[88:91]\n"
  "s_nop 3\n"
  "v_mfma_f32_16x16x32_f16 a[88:91], v[52:55], v[148:151], a[88:91]\n"
  "buffer_load_dwordx4 v[116:119], v240, s[20:23], s74 offen offset:1024\n"
  "v_mfma_f32_16x16x32_f16 a[120:123], v[60:63], v[148:151], a[120:123]\n"
  "s_nop 3\n"
  "v_mfma_f32_16x16x32_f16 a[120:123], v[56:59], v[144:147], a[120:123]\n"
  "s_nop 3\n"
  "s_waitcnt lgkmcnt(4)\n"
  "v_mfma_f32_16x16x32_f16 a[28:31], v[32:35], v[152:155], a[28:31]\n"
  "ds_read_b128 v[144:147], v243 offset:2048\n"
  "v_mfma_f32_16x16x32_f16 a[28:31], v[36:39], v[156:159], a[28:31]\n"
  "buffer_load_dwordx4 v[120:123], v240, s[20:23], s46 offen offset:1536\n"
  "s_nop 3\n"
  "v_mfma_f32_16x16x32_f16 a[60:63], v[44:47], v[156:159], a[60:63]\n"
  "s_nop 3\n"
  "v_mfma_f32_16x16x32_f16 a[60:63], v[40:43], v[152:155], a[60:63]\n"
  "ds_read_b128 v[148:151], v243 offset:10240\n"
  "v_mfma_f32_16x16x32_f16 a[92:95], v[48:51], v[152:155], a[92:95]\n"
  "s_nop 3\n"
  "v_mfma_f32_16x16x32_f16 a[92:95], v[52:55], v[156:159], a[92:95]\n"
  "buffer_load_dwordx4 v[124:127], v241, s[24:27], s74 offen offset:1536\n"
  "s_nop 3\n"
  "v_mfma_f32_16x16x32_f16 a[124:127], v[60:63], v[156:159], a[124:127]\n"
  "v_mfma_f32_16x16x32_f16 a[124:127], v[56:59], v[152:155], a[124:127]\n"
  "v_cvt_pk_f16_f32 v224, v160, v164\n"
  "s_waitcnt vmcnt(8) lgkmcnt(4)\n"
  "v_mfma_f32_16x16x32_f16 a[0:3], v[64:67], v[128:131], a[0:3]\n"
  "ds_read_b128 v[152:155], v243 offset:3072\n"
  "v_cvt_pk_f16_f32 v225, v168, v172\n"
  "s_nop 3\n"
  "v_mfma_f32_16x16x32_f16 a[0:3], v[68:71], v[132:135], a[0:3]\n"
  "v_cvt_pk_f16_f32 v226, v176, v180\n"
  "s_nop 3\n"
  "v_mfma_f32_16x16x32_f16 a[32:35], v[76:79], v[132:135], a[32:35]\n"
  "v_readlane_b32 s80, v249, 30\n"
  "v_cvt_pk_f16_f32 v227, v184, v188\n"
  "v_mfma_f32_16x16x32_f16 a[32:35], v[72:75], v[128:131], a[32:35]\n"
  "ds_read_b128 v[156:159], v243 offset:11264\n"
  "s_lshl_b32 s80, s80, 17\ns_add_u32 s80, s80, s83\n"
  "s_nop 3\n"
  "v_mfma_f32_16x16x32_f16 a[64:67], v[80:83], v[128:131], a[64:67]\n"
  "s_add_u32 s46, s80, 0x0\ns_add_u32 s74, s80, 0x4000\n"
  "ds_write_b128 v244, v[224:227] offset:0\n"
  "v_mfma_f32_16x16x32_f16 a[64:67], v[84:87], v[132:135], a[64:67]\n"
  "buffer_load_dwordx4 v[0:3], v241, s[24:27], s46 offen offset:2048\n"
  "v_cvt_pk_f16_f32 v228, v161, v165\n"
  "s_nop 3\n"
  "v_mfma_f32_16x16x32_f16 a[96:99], v[92:95], v[132:135], a[96:99]\n"
  "v_cvt_pk_f16_f32 v229, v169, v173\n"
  "v_mfma_f32_16x16x32_f16 a[96:99], v[88:91], v[128:131], a[96:99]\n"
  "v_cvt_pk_f16_f32 v230, v177, v181\n"
  "s_nop 3\n"
  "s_waitcnt lgkmcnt(5)\n"
  "v_mfma_f32_16x16x32_f16 a[4:7], v[64:67], v[136:139], a[4:7]\n"
  "ds_read_b128 v[128:131], v243 offset:4096\n"
  "v_cvt_pk_f16_f32 v231, v185, v189\n"
  "v_mfma_f32_16x16x32_f16 a[4:7], v[68:71], v[140:143], a[4:7]\n"
  "ds_write_b128 v244, v[228:231] offset:1024\n"
  "s_nop 3\n"
  "v_mfma_f32_16x16x32_f16 a[36:39], v[76:79], v[140:143], a[36:39]\n"
  "buffer_load_dwordx4 v[4:7], v240, s[20:23], s74 offen offset:2048\n"
  "v_cvt_pk_f16_f32 v224, v162, v166\n"
  "v_mfma_f32_16x16x32_f16 a[36:39], v[72:75], v[136:139], a[36:39]\n"
  "ds_read_b128 v[132:135], v243 offset:12288\n"
  "v_cvt_pk_f16_f32 v225, v170, v174\n"
  "s_nop 3\n"
  "v_mfma_f32_16x16x32_f16 a[68:71], v[80:83], v[136:139], a[68:71]\n"
  "v_cvt_pk_f16_f32 v226, v178, v182\n"
  "v_mfma_f32_16x16x32_f16 a[68:71], v[84:87], v[140:143], a[68:71]\n"
  "v_cvt_pk_f16_f32 v227, v186, v190\n"
  "s_nop 3\n"
  "v_mfma_f32_16x16x32_f16 a[100:103], v[92:95], v[140:143], a[100:103]\n"
  "buffer_load_dwordx4 v[8:11], v240, s[20:23], s46 offen offset:2560\n"
  "ds_write_b128 v244, v[224:227] offset:2048\n"
  "v_mfma_f32_16x16x32_f16 a[100:103], v[88:91], v[136:139], a[100:103]\n"
  "v_cvt_pk_f16_f32 v228, v163, v167\n"
  "s_nop 3\n"
  "s_waitcnt lgkmcnt(7)\n"
  "v_mfma_f32_16x16x32_f16 a[8:11], v[64:67], v[144:147], a[8:11]\n"
  "ds_read_b128 v[136:139], v243 offset:5120\n"
  "v_cvt_pk_f16_f32 v229, v171, v175\n"
  "v_mfma_f32_16x16x32_f16 a[8:11], v[68:71], v[148:151], a[8:11]\n"
  "v_cvt_pk_f16_f32 v230, v179, v183\n"
  "s_nop 3\n"
  "v_mfma_f32_16x16x32_f16 a[40:43], v[76:79], v[148:151], a[40:43]\n"
  "v_cvt_pk_f16_f32 v231, v187, v191\n"
  "v_mfma_f32_16x16x32_f16 a[40:43], v[72:75], v[144:147], a[40:43]\n"
  "ds_read_b128 v[140:143], v243 offset:13312\n"
  "buffer_load_dwordx4 v[12:15], v241, s[24:27], s74 offen offset:2560\n"
  "s_nop 3\n"
  "v_mfma_f32_16x16x32_f16 a[72:75], v[80:83], v[144:147], a[72:75]\n"
  "ds_write_b128 v244, v[228:231] offset:3072\n"
  "v_mfma_f32_16x16x32_f16 a[72:75], v[84:87], v[148:151], a[72:75]\n"
  "v_add_u32 v244, 0x4000, v244\nv_and_b32 v244, 0x1ffff, v244\n"
  "s_nop 3\n"
  "v_mfma_f32_16x16x32_f16 a[104:107], v[92:95], v[148:151], a[104:107]\n"
  "v_mfma_f32_16x16x32_f16 a[104:107], v[88:91], v[144:147], a[104:107]\n"
  "buffer_load_dwordx4 v[16:19], v240, s[20:23], s46 offen offset:3072\n"
  "s_nop 3\n"
  "s_waitcnt lgkmcnt(8)\n"
  "v_mfma_f32_16x16x32_f16 a[12:15], v[64:67], v[152:155], a[12:15]\n"
  "ds_read_b128 v[144:147], v243 offset:6144\n"
  "v_mfma_f32_16x16x32_f16 a[12:15], v[68:71], v[156:159], a[12:15]\n"
  "s_nop 3\n"
  "v_mfma_f32_16x16x32_f16 a[44:47], v[76:79], v[156:159], a[44:47]\n"
  "v_mfma_f32_16x16x32_f16 a[44:47], v[72:75], v[152:155], a[44:47]\n"
  "ds_read_b128 v[148:151], v243 offset:14336\n"
  "s_nop 3\n"
  "v_mfma_f32_16x16x32_f16 a[76:79], v[80:83], v[152:155], a[76:79]\n"
  "buffer_load_dwordx4 v[20:23], v241, s[24:27], s74 offen offset:3072\n"
  "v_mfma_f32_16x16x32_f16 a[76:79], v[84:87], v[156:159], a[76:79]\n"
  "s_nop 3\n"
  "v_mfma_f32_16x16x32_f16 a[108:111], v[92:95], v[156:159], a[108:111]\n"
  "v_mfma_f32_16x16x32_f16 a[108:111], v[88:91], v[152:155], a[108:111]\n"
  "s_nop 3\n"
  "s_waitcnt lgkmcnt(6)\n"
  "v_mfma_f32_16x16x32_f16 a[16:19], v[64:67], v[128:131], a[16:19]\n"
  "ds_read_b128 v[152:155], v243 offset:7168\n"
  "buffer_load_dwordx4 v[24:27], v241, s[24:27], s46 offen offset:3584\n"
  "v_mfma_f32_16x16x32_f16 a[16:19], v[68:71], v[132:135], a[16:19]\n"
  "s_nop 3\n"
  "v_mfma_f32_16x16x32_f16 a[48:51], v[76:79], v[132:135], a[48:51]\n"
  "v_mfma_f32_16x16x32_f16 a[48:51], v[72:75], v[128:131], a[48:51]\n"
  "ds_read_b128 v[156:159], v243 offset:15360\n"
  "s_nop 3\n"
  "v_mfma_f32_16x16x32_f16 a[80:83], v[80:83], v[128:131], a[80:83]\n"
  "v_mfma_f32_16x16x32_f16 a[80:83], v[84:87], v[132:135], a[80:83]\n"
  "buffer_load_dwordx4 v[28:31], v240, s[20:23], s74 offen offset:3584\n"
  "s_nop 3\n"
  "v_mfma_f32_16x16x32_f16 a[112:115], v[92:95], v[132:135], a[112:115]\n"
  "s_waitcnt lgkmcnt(4)\n"
  "s_barrier\n"
  "v_mfma_f32_16x16x32_f16 a[112:115], v[88:91], v[128:131], a[112:115]\n"
  "v_add_u32 v243, 0x4000, v243\nv_and_b32 v243, 0x1ffff, v243\n"
  "s_nop 3\n"
  "v_mfma_f32_16x16x32_f16 a[20:23], v[64:67], v[136:139], a[20:23]\n"
  "ds_read_b128 v[128:131], v243 offset:0\n"
  "v_mfma_f32_16x16x32_f16 a[20:23], v[68:71], v[140:143], a[20:23]\n"
  "v_readlane_b32 s80, v249, 30\n"
  "s_nop 3\n"
  "v_mfma_f32_16x16x32_f16 a[52:55], v[76:79], v[140:143], a[52:55]\n"
  "s_lshl_b32 s80, s80, 17\ns_add_u32 s80, s80, s83\n"
  "v_mfma_f32_16x16x32_f16 a[52:55], v[72:75], v[136:139], a[52:55]\n"
  "ds_read_b128 v[132:135], v243 offset:8192\n"
  "s_add_u32 s46, s80, 0x10000\ns_add_u32 s74, s80, 0x14000\n"
  "s_nop 3\n"
  "v_mfma_f32_16x16x32_f16 a[84:87], v[80:83], v[136:139], a[84:87]\n"
  "buffer_load_dwordx4 v[32:35], v241, s[24:27], s46 offen offset:2048\n"
  "v_mfma_f32_16x16x32_f16 a[84:87], v[84:87], v[140:143], a[84:87]\n"
  "s_nop 3\n"
  "v_mfma_f32_16x16x32_f16 a[116:119], v[92:95], v[140:143], a[116:119]\n"
  "v_mfma_f32_16x16x32_f16 a[116:119], v[88:91], v[136:139], a[116:119]\n"
  "s_nop 3\n"
  "s_waitcnt lgkmcnt(4)\n"
  "v_mfma_f32_16x16x32_f16 a[24:27], v[64:67], v[144:147], a[24:27]\n"
  "ds_read_b128 v[136:139], v243 offset:1024\n"
  "v_mfma_f32_16x16x32_f16 a[24:27], v[68:71], v[148:151], a[24:27]\n"
  "buffer_load_dwordx4 v[36:39], v240, s[20:23], s74 offen offset:2048\n"
  "s_nop 3\n"
  "v_mfma_f32_16x16x32_f16 a[56:59], v[76:79], v[148:151], a[56:59]\n"
  "v_mbcnt_lo_u32_b32 v246, -1, 0\n"
  "v_mfma_f32_16x16x32_f16 a[56:59], v[72:75], v[144:147], a[56:59]\n"
  "ds_read_b128 v[140:143], v243 offset:9216\n"
  "s_nop 3\n"
  "v_mfma_f32_16x16x32_f16 a[88:91], v[80:83], v[144:147], a[88:91]\n"
  "v_mbcnt_hi_u32_b32 v246, -1, v246\n"
  "v_mfma_f32_16x16x32_f16 a[88:91], v[84:87], v[148:151], a[88:91]\n"
  "buffer_load_dwordx4 v[40:43], v240, s[20:23], s46 offen offset:2560\n"
  "s_nop 3\n"
  "v_mfma_f32_16x16x32_f16 a[120:123], v[92:95], v[148:151], a[120:123]\n"
  "s_mov_b64 s[28:29], %3\n"
  "v_mfma_f32_16x16x32_f16 a[120:123], v[88:91], v[144:147], a[120:123]\n"
  "s_add_u32 s28, s28, s78\ns_addc_u32 s29, s29, 0\n"
  "s_nop 3\n"
  "s_waitcnt lgkmcnt(4)\n"
  "v_mfma_f32_16x16x32_f16 a[28:31], v[64:67], v[152:155], a[28:31]\n"
  "ds_read_b128 v[144:147], v243 offset:2048\n"
  "v_mfma_f32_16x16x32_f16 a[28:31], v[68:71], v[156:159], a[28:31]\n"
  "s_sub_u32 s30, 0x4000000, s78\n"
  "s_nop 3\n"
  "v_mfma_f32_16x16x32_f16 a[60:63], v[76:79], v[156:159], a[60:63]\n"
  "buffer_load_dwordx4 v[44:47], v241, s[24:27], s74 offen offset:2560\n"
  "v_mfma_f32_16x16x32_f16 a[60:63], v[72:75], v[152:155], a[60:63]\n"
  "ds_read_b128 v[148:151], v243 offset:10240\n"
  "s_nop 3\n"
  "v_mfma_f32_16x16x32_f16 a[92:95], v[80:83], v[152:155], a[92:95]\n"
  "v_lshrrev_b32 v242, 4, v246\n"
  "v_mfma_f32_16x16x32_f16 a[92:95], v[84:87], v[156:159], a[92:95]\n"
  "v_lshlrev_b32 v242, 19, v242\n"
  "s_nop 3\n"
  "v_mfma_f32_16x16x32_f16 a[124:127], v[92:95], v[156:159], a[124:127]\n"
  "buffer_load_dwordx4 v[48:51], v240, s[20:23], s46 offen offset:3072\n"
  "v_mfma_f32_16x16x32_f16 a[124:127], v[88:91], v[152:155], a[124:127]\n"
  "v_and_b32 v246, 15, v246\n"
  "s_nop 3\n"
  "s_waitcnt vmcnt(13) lgkmcnt(4)\n"
  "v_mfma_f32_16x16x32_f16 a[0:3], v[96:99], v[128:131], a[0:3]\n"
  "ds_read_b128 v[152:155], v243 offset:3072\n"
  "v_mfma_f32_16x16x32_f16 a[0:3], v[100:103], v[132:135], a[0:3]\n"
  "v_lshl_add_u32 v242, v246, 4, v242\n"
  "s_nop 3\n"
  "v_mfma_f32_16x16x32_f16 a[32:35], v[108:111], v[132:135], a[32:35]\n"
  "s_lshl_b32 s63, %7, 24\n"
  "v_mfma_f32_16x16x32_f16 a[32:35], v[104:107], v[128:131], a[32:35]\n"
  "ds_read_b128 v[156:159], v243 offset:11264\n"
  "buffer_load_dwordx4 v[52:55], v241, s[24:27], s74 offen offset:3072\n"
  "s_nop 3\n"
  "v_mfma_f32_16x16x32_f16 a[64:67], v[112:115], v[128:131], a[64:67]\n"
  "v_add_u32 v242, s63, v242\n"
  "v_mfma_f32_16x16x32_f16 a[64:67], v[116:119], v[132:135], a[64:67]\n"
  "s_mov_b32 s64, 0xc2700000\n"
  "s_nop 3\n"
  "v_mfma_f32_16x16x32_f16 a[96:99], v[124:127], v[132:135], a[96:99]\n"
  "s_mov_b32 s65, 0xbfb8aa3b\n"
  "v_mfma_f32_16x16x32_f16 a[96:99], v[120:123], v[128:131], a[96:99]\n"
  "buffer_load_dwordx4 v[56:59], v241, s[24:27], s46 offen offset:3584\n"
  "s_nop 3\n"
  "s_waitcnt lgkmcnt(4)\n"
  "v_mfma_f32_16x16x32_f16 a[4:7], v[96:99], v[136:139], a[4:7]\n"
  "ds_read_b128 v[128:131], v243 offset:4096\n"
  "v_mfma_f32_16x16x32_f16 a[4:7], v[100:103], v[140:143], a[4:7]\n"
  "v_mov_b32 v245, 0x42700000\n"
  "s_nop 3\n"
  "v_mfma_f32_16x16x32_f16 a[36:39], v[108:111], v[140:143], a[36:39]\n"
  "v_mfma_f32_16x16x32_f16 a[36:39], v[104:107], v[136:139], a[36:39]\n"
  "ds_read_b128 v[132:135], v243 offset:12288\n"
  "s_nop 3\n"
  "v_mfma_f32_16x16x32_f16 a[68:71], v[112:115], v[136:139], a[68:71]\n"
  "buffer_load_dwordx4 v[60:63], v240, s[20:23], s74 offen offset:3584\n"
  "v_mfma_f32_16x16x32_f16 a[68:71], v[116:119], v[140:143], a[68:71]\n"
  "s_nop 3\n"
  "v_mfma_f32_16x16x32_f16 a[100:103], v[124:127], v[140:143], a[100:103]\n"
  "v_mfma_f32_16x16x32_f16 a[100:103], v[120:123], v[136:139], a[100:103]\n"
  "s_nop 3\n"
  "s_waitcnt lgkmcnt(4)\n"
  "v_mfma_f32_16x16x32_f16 a[8:11], v[96:99], v[144:147], a[8:11]\n"
  "ds_read_b128 v[136:139], v243 offset:5120\n"
  "v_mfma_f32_16x16x32_f16 a[8:11], v[100:103], v[148:151], a[8:11]\n"
  "v_readlane_b32 s80, v249, 31\n"
  "s_nop 3\n"
  "v_mfma_f32_16x16x32_f16 a[40:43], v[108:111], v[148:151], a[40:43]\n"
  "s_lshl_b32 s80, s80, 17\ns_add_u32 s80, s80, s83\n"
  "v_mfma_f32_16x16x32_f16 a[40:43], v[104:107], v[144:147], a[40:43]\n"
  "ds_read_b128 v[140:143], v243 offset:13312\n"
  "s_add_u32 s46, s80, 0x0\ns_add_u32 s74, s80, 0x4000\n"
  "s_nop 3\n"
  "v_mfma_f32_16x16x32_f16 a[72:75], v[112:115], v[144:147], a[72:75]\n"
  "buffer_load_dwordx4 v[64:67], v241, s[24:27], s46 offen offset:2048\n"
  "v_mfma_f32_16x16x32_f16 a[72:75], v[116:119], v[148:151], a[72:75]\n"
  "s_nop 3\n"
  "v_mfma_f32_16x16x32_f16 a[104:107], v[124:127], v[148:151], a[104:107]\n"
  "v_mfma_f32_16x16x32_f16 a[104:107], v[120:123], v[144:147], a[104:107]\n"
  "s_nop 3\n"
  "s_waitcnt lgkmcnt(4)\n"
  "v_mfma_f32_16x16x32_f16 a[12:15], v[96:99], v[152:155], a[12:15]\n"
  "ds_read_b128 v[144:147], v243 offset:6144\n"
  "v_mfma_f32_16x16x32_f16 a[12:15], v[100:103], v[156:159], a[12:15]\n"
  "buffer_load_dwordx4 v[68:71], v240, s[20:23], s74 offen offset:2048\n"
  "s_nop 3\n"
  "v_mfma_f32_16x16x32_f16 a[44:47], v[108:111], v[156:159], a[44:47]\n"
  "v_mfma_f32_16x16x32_f16 a[44:47], v[104:107], v[152:155], a[44:47]\n"
  "ds_read_b128 v[148:151], v243 offset:14336\n"
  "s_nop 3\n"
  "v_mfma_f32_16x16x32_f16 a[76:79], v[112:115], v[152:155], a[76:79]\n"
  "v_mfma_f32_16x16x32_f16 a[76:79], v[116:119], v[156:159], a[76:79]\n"
  "buffer_load_dwordx4 v[72:75], v240, s[20:23], s46 offen offset:2560\n"
  "s_nop 3\n"
  "v_mfma_f32_16x16x32_f16 a[108:111], v[124:127], v[156:159], a[108:111]\n"
  "v_mfma_f32_16x16x32_f16 a[108:111], v[120:123], v[152:155], a[108:111]\n"
  "s_nop 3\n"
  "s_waitcnt lgkmcnt(4)\n"
  "v_mfma_f32_16x16x32_f16 a[16:19], v[96:99], v[128:131], a[16:19]\n"
  "ds_read_b128 v[152:155], v243 offset:7168\n"
  "v_mfma_f32_16x16x32_f16 a[16:19], v[100:103], v[132:135], a[16:19]\n"
  "s_nop 3\n"
  "v_mfma_f32_16x16x32_f16 a[48:51], v[108:111], v[132:135], a[48:51]\n"
  "buffer_load_dwordx4 v[76:79], v241, s[24:27], s74 offen offset:2560\n"
  "v_mfma_f32_16x16x32_f16 a[48:51], v[104:107], v[128:131], a[48:51]\n"
  "ds_read_b128 v[156:159], v243 offset:15360\n"
  "s_nop 3\n"
  "v_mfma_f32_16x16x32_f16 a[80:83], v[112:115], v[128:131], a[80:83]\n"
  "v_mfma_f32_16x16x32_f16 a[80:83], v[116:119], v[132:135], a[80:83]\n"
  "s_nop 3\n"
  "v_mfma_f32_16x16x32_f16 a[112:115], v[124:127], v[132:135], a[112:115]\n"
  "v_mfma_f32_16x16x32_f16 a[112:115], v[120:123], v[128:131], a[112:115]\n"
  "v_add_u32 v243, 0x14000, v243\nv_and_b32 v243, 0x1ffff, v243\n"
  "buffer_load_dwordx4 v[80:83], v240, s[20:23], s46 offen offset:3072\n"
  "s_nop 3\n"
  "s_waitcnt lgkmcnt(4)\n"
  "v_mfma_f32_16x16x32_f16 a[20:23], v[96:99], v[136:139], a[20:23]\n"
  "ds_read_b128 v[128:131], v243 offset:0\n"
  "v_mfma_f32_16x16x32_f16 a[20:23], v[100:103], v[140:143], a[20:23]\n"
  "s_nop 3\n"
  "v_mfma_f32_16x16x32_f16 a[52:55], v[108:111], v[140:143], a[52:55]\n"
  "v_mfma_f32_16x16x32_f16 a[52:55], v[104:107], v[136:139], a[52:55]\n"
  "ds_read_b128 v[132:135], v243 offset:8192\n"
  "s_nop 3\n"
  "v_mfma_f32_16x16x32_f16 a[84:87], v[112:115], v[136:139], a[84:87]\n"
  "buffer_load_dwordx4 v[84:87], v241, s[24:27], s74 offen offset:3072\n"
  "v_mfma_f32_16x16x32_f16 a[84:87], v[116:119], v[140:143], a[84:87]\n"
  "s_nop 3\n"
  "v_mfma_f32_16x16x32_f16 a[116:119], v[124:127], v[140:143], a[116:119]\n"
  "v_mfma_f32_16x16x32_f16 a[116:119], v[120:123], v[136:139], a[116:119]\n"
  "s_nop 3\n"
  "s_waitcnt lgkmcnt(4)\n"
  "v_mfma_f32_16x16x32_f16 a[24:27], v[96:99], v[144:147], a[24:27]\n"
  "ds_read_b128 v[136:139], v243 offset:1024\n"
  "buffer_load_dwordx4 v[88:91], v241, s[24:27], s46 offen offset:3584\n"
  "v_mfma_f32_16x16x32_f16 a[24:27], v[100:103], v[148:151], a[24:27]\n"
  "s_nop 3\n"
  "v_mfma_f32_16x16x32_f16 a[56:59], v[108:111], v[148:151], a[56:59]\n"
  "v_mfma_f32_16x16x32_f16 a[56:59], v[104:107], v[144:147], a[56:59]\n"
  "ds_read_b128 v[140:143], v243 offset:9216\n"
  "s_nop 3\n"
  "v_mfma_f32_16x16x32_f16 a[88:91], v[112:115], v[144:147], a[88:91]\n"
  "v_mfma_f32_16x16x32_f16 a[88:91], v[116:119], v[148:151], a[88:91]\n"
  "buffer_load_dwordx4 v[92:95], v240, s[20:23], s74 offen offset:3584\n"
  "s_nop 3\n"
  "v_mfma_f32_16x16x32_f16 a[120:123], v[124:127], v[148:151], a[120:123]\n"
  "v_mfma_f32_16x16x32_f16 a[120:123], v[120:123], v[144:147], a[120:123]\n"
  "s_nop 3\n"
  "s_waitcnt lgkmcnt(4)\n"
  "v_mfma_f32_16x16x32_f16 a[28:31], v[96:99], v[152:155], a[28:31]\n"
  "ds_read_b128 v[144:147], v243 offset:2048\n"
  "v_mfma_f32_16x16x32_f16 a[28:31], v[100:103], v[156:159], a[28:31]\n"
  "s_nop 3\n"
  "v_mfma_f32_16x16x32_f16 a[60:63], v[108:111], v[156:159], a[60:63]\n"
  "v_mfma_f32_16x16x32_f16 a[60:63], v[104:107], v[152:155], a[60:63]\n"
  "ds_read_b128 v[148:151], v243 offset:10240\n"
  "s_nop 3\n"
  "v_mfma_f32_16x16x32_f16 a[92:95], v[112:115], v[152:155], a[92:95]\n"
  "v_mfma_f32_16x16x32_f16 a[92:95], v[116:119], v[156:159], a[92:95]\n"
  "s_nop 3\n"
  "v_mfma_f32_16x16x32_f16 a[124:127], v[124:127], v[156:159], a[124:127]\n"
  "v_mfma_f32_16x16x32_f16 a[124:127], v[120:123], v[152:155], a[124:127]\n"
  "v_readlane_b32 s80, v249, 31\n"
  "s_waitcnt vmcnt(16) lgkmcnt(4)\n"
  "v_mfma_f32_16x16x32_f16 a[128:131], v[0:3], v[128:131], a[128:131]\n"
  "ds_read_b128 v[152:155], v243 offset:3072\n"
  "s_lshl_b32 s80, s80, 17\ns_add_u32 s80, s80, s83\n"
  "s_nop 3\n"
  "v_mfma_f32_16x16x32_f16 a[128:131], v[4:7], v[132:135], a[128:131]\n"
  "s_add_u32 s46, s80, 0x10000\ns_add_u32 s74, s80, 0x14000\n"
  "v_mfma_f32_16x16x32_f16 a[160:163], v[12:15], v[132:135], a[160:163]\n"
  "buffer_load_dwordx4 v[96:99], v241, s[24:27], s46 offen offset:2048\n"
  "v_mfma_f32_16x16x32_f16 a[160:163], v[8:11], v[128:131], a[160:163]\n"
  "ds_read_b128 v[156:159], v243 offset:11264\n"
  "v_mfma_f32_16x16x32_f16 a[192:195], v[16:19], v[128:131], a[192:195]\n"
  "v_mfma_f32_16x16x32_f16 a[192:195], v[20:23], v[132:135], a[192:195]\n"
  "v_accvgpr_read_b32 v160, a0\n"
  "v_accvgpr_read_b32 v161, a4\n"
  "s_nop 3\n"
  "v_mfma_f32_16x16x32_f16 a[224:227], v[28:31], v[132:135], a[224:227]\n"
  "buffer_load_dwordx4 v[100:103], v240, s[20:23], s74 offen offset:2048\n"
  "v_accvgpr_read_b32 v162, a8\n"
  "v_mfma_f32_16x16x32_f16 a[224:227], v[24:27], v[128:131], a[224:227]\n"
  "v_accvgpr_read_b32 v163, a12\n"
  "v_med3_f32 v224, v160, s64, v245\n"
  "s_waitcnt lgkmcnt(4)\n"
  "v_mfma_f32_16x16x32_f16 a[132:135], v[0:3], v[136:139], a[132:135]\n"
  "ds_read_b128 v[128:131], v243 offset:4096\n"
  "v_med3_f32 v225, v161, s64, v245\n"
  "v_mfma_f32_16x16x32_f16 a[132:135], v[4:7], v[140:143], a[132:135]\n"
  "v_med3_f32 v226, v162, s64, v245\n"
  "v_med3_f32 v227, v163, s64, v245\n"
  "v_mfma_f32_16x16x32_f16 a[164:167], v[12:15], v[140:143], a[164:167]\n"
  "buffer_load_dwordx4 v[104:107], v240, s[20:23], s46 offen offset:2560\n"
  "v_mul_f32 v224, s65, v224\n"
  "s_nop 3\n"
  "v_mfma_f32_16x16x32_f16 a[164:167], v[8:11], v[136:139], a[164:167]\n"
  "ds_read_b128 v[132:135], v243 offset:12288\n"
  "v_mul_f32 v225, s65, v225\n"
  "v_mfma_f32_16x16x32_f16 a[196:199], v[16:19], v[136:139], a[196:199]\n"
  "v_mul_f32 v226, s65, v226\n"
  "v_mul_f32 v227, s65, v227\n"
  "v_mfma_f32_16x16x32_f16 a[196:199], v[20:23], v[140:143], a[196:199]\n"
  "v_exp_f32 v224, v224\n"
  "v_mfma_f32_16x16x32_f16 a[228:231], v[28:31], v[140:143], a[228:231]\n"
  "buffer_load_dwordx4 v[108:111], v241, s[24:27], s74 offen offset:2560\n"
  "v_exp_f32 v225, v225\n"
  "v_exp_f32 v226, v226\n"
  "v_mfma_f32_16x16x32_f16 a[228:231], v[24:27], v[136:139], a[228:231]\n"
  "v_exp_f32 v227, v227\n"
  "s_nop 3\n"
  "s_waitcnt lgkmcnt(4)\n"
  "v_mfma_f32_16x16x32_f16 a[136:139], v[0:3], v[144:147], a[136:139]\n"
  "ds_read_b128 v[136:139], v243 offset:5120\n"
  "v_add_f32 v224, s67, v224\n"
  "v_add_f32 v225, s67, v225\n"
  "v_mfma_f32_16x16x32_f16 a[136:139], v[4:7], v[148:151], a[136:139]\n"
  "buffer_load_dwordx4 v[112:115], v240, s[20:23], s46 offen offset:3072\n"
  "v_add_f32 v226, s67, v226\n"
  "v_mfma_f32_16x16x32_f16 a[168:171], v[12:15], v[148:151], a[168:171]\n"
  "v_add_f32 v227, s67, v227\n"
  "v_mfma_f32_16x16x32_f16 a[168:171], v[8:11], v[144:147], a[168:171]\n"
  "ds_read_b128 v[140:143], v243 offset:13312\n"
  "v_rcp_f32 v224, v224\n"
  "v_rcp_f32 v225, v225\n"
  "v_mfma_f32_16x16x32_f16 a[200:203], v[16:19], v[144:147], a[200:203]\n"
  "v_rcp_f32 v226, v226\n"
  "v_mfma_f32_16x16x32_f16 a[200:203], v[20:23], v[148:151], a[200:203]\n"
  "buffer_load_dwordx4 v[116:119], v241, s[24:27], s74 offen offset:3072\n"
  "v_rcp_f32 v227, v227\n"
  "v_mul_f32 v160, v160, v224\n"
  "s_nop 3\n"
  "v_mfma_f32_16x16x32_f16 a[232:235], v[28:31], v[148:151], a[232:235]\n"
  "v_mul_f32 v161, v161, v225\n"
  "v_mfma_f32_16x16x32_f16 a[232:235], v[24:27], v[144:147], a[232:235]\n"
  "v_mul_f32 v162, v162, v226\n"
  "v_mul_f32 v163, v163, v227\n"
  "s_waitcnt lgkmcnt(4)\n"
  "v_mfma_f32_16x16x32_f16 a[140:143], v[0:3], v[152:155], a[140:143]\n"
  "ds_read_b128 v[144:147], v243 offset:6144\n"
  "v_accvgpr_read_b32 v164, a16\n"
  "v_mfma_f32_16x16x32_f16 a[140:143], v[4:7], v[156:159], a[140:143]\n"
  "buffer_load_dwordx4 v[120:123], v241, s[24:27], s46 offen offset:3584\n"
  "v_accvgpr_read_b32 v165, a20\n"
  "v_mfma_f32_16x16x32_f16 a[172:175], v[12:15], v[156:159], a[172:175]\n"
  "v_accvgpr_read_b32 v166, a24\n"
  "v_accvgpr_read_b32 v167, a28\n"
  "s_nop 3\n"
  "v_mfma_f32_16x16x32_f16 a[172:175], v[8:11], v[152:155], a[172:175]\n"
  "ds_read_b128 v[148:151], v243 offset:14336\n"
  "v_med3_f32 v228, v164, s64, v245\n"
  "v_mfma_f32_16x16x32_f16 a[204:207], v[16:19], v[152:155], a[204:207]\n"
  "v_med3_f32 v229, v165, s64, v245\n"
  "v_med3_f32 v230, v166, s64, v245\n"
  "v_mfma_f32_16x16x32_f16 a[204:207], v[20:23], v[156:159], a[204:207]\n"
  "buffer_load_dwordx4 v[124:127], v240, s[20:23], s74 offen offset:3584\n"
  "v_med3_f32 v231, v167, s64, v245\n"
  "v_mfma_f32_16x16x32_f16 a[236:239], v[28:31], v[156:159], a[236:239]\n"
  "buffer_load_dwordx2 v[232:233], off, s[32:35], s66 offset:128 sc1\n"
  "v_mul_f32 v228, s65, v228\n"
  "v_mul_f32 v229, s65, v229\n"
  "v_mfma_f32_16x16x32_f16 a[236:239], v[24:27], v[152:155], a[236:239]\n"
  "v_mul_f32 v230, s65, v230\n"
  "s_nop 3\n"
  "s_waitcnt lgkmcnt(4)\n"
  "v_mfma_f32_16x16x32_f16 a[144:147], v[0:3], v[128:131], a[144:147]\n"
  "ds_read_b128 v[152:155], v243 offset:7168\n"
  "v_mul_f32 v231, s65, v231\n"
  "v_mfma_f32_16x16x32_f16 a[144:147], v[4:7], v[132:135], a[144:147]\n"
  "v_exp_f32 v228, v228\n"
  "v_exp_f32 v229, v229\n"
  "v_mfma_f32_16x16x32_f16 a[176:179], v[12:15], v[132:135], a[176:179]\n"
  "v_exp_f32 v230, v230\n"
  "v_mfma_f32_16x16x32_f16 a[176:179], v[8:11], v[128:131], a[176:179]\n"
  "ds_read_b128 v[156:159], v243 offset:15360\n"
  "v_exp_f32 v231, v231\n"
  "v_add_f32 v228, s67, v228\n"
  "v_mfma_f32_16x16x32_f16 a[208:211], v[16:19], v[128:131], a[208:211]\n"
  "v_add_f32 v229, s67, v229\n"
  "v_mfma_f32_16x16x32_f16 a[208:211], v[20:23], v[132:135], a[208:211]\n"
  "v_add_f32 v230, s67, v230\n"
  "v_add_f32 v231, s67, v231\n"
  "s_nop 3\n"
  "v_mfma_f32_16x16x32_f16 a[240:243], v[28:31], v[132:135], a[240:243]\n"
  "v_rcp_f32 v228, v228\n"
  "v_mfma_f32_16x16x32_f16 a[240:243], v[24:27], v[128:131], a[240:243]\n"
  "v_add_u32 v243, 0x4000, v243\nv_and_b32 v243, 0x1ffff, v243\n"
  "v_rcp_f32 v229, v229\n"
  "s_waitcnt lgkmcnt(4)\n"
  "v_mfma_f32_16x16x32_f16 a[148:151], v[0:3], v[136:139], a[148:151]\n"
  "ds_read_b128 v[128:131], v243 offset:0\n"
  "v_rcp_f32 v230, v230\n"
  "v_rcp_f32 v231, v231\n"
  "v_mfma_f32_16x16x32_f16 a[148:151], v[4:7], v[140:143], a[148:151]\n"
  "v_mul_f32 v164, v164, v228\n"
  "v_mfma_f32_16x16x32_f16 a[180:183], v[12:15], v[140:143], a[180:183]\n"
  "v_mul_f32 v165, v165, v229\n"
  "v_mul_f32 v166, v166, v230\n"
  "s_nop 3\n"
  "v_mfma_f32_16x16x32_f16 a[180:183], v[8:11], v[136:139], a[180:183]\n"
  "ds_read_b128 v[132:135], v243 offset:8192\n"
  "v_mul_f32 v167, v167, v231\n"
  "v_mfma_f32_16x16x32_f16 a[212:215], v[16:19], v[136:139], a[212:215]\n"
  "s_mov_b32 s63, 0x0\nbuffer_store_dwordx4 v[160:163], v242, s[28:31], s63 offen offset:0 nt\n"
  "s_mov_b32 s63, 0x0\nbuffer_store_dwordx4 v[164:167], v242, s[28:31], s63 offen offset:256 nt\n"
  "v_mfma_f32_16x16x32_f16 a[212:215], v[20:23], v[140:143], a[212:215]\n"
  "v_accvgpr_read_b32 v168, a1\n"
  "v_mfma_f32_16x16x32_f16 a[244:247], v[28:31], v[140:143], a[244:247]\n"
  "v_accvgpr_read_b32 v169, a5\n"
  "v_mfma_f32_16x16x32_f16 a[244:247], v[24:27], v[136:139], a[244:247]\n"
  "v_accvgpr_read_b32 v170, a9\n"
  "v_accvgpr_read_b32 v171, a13\n"
  "s_nop 3\n"
  "s_waitcnt lgkmcnt(4)\n"
  "v_mfma_f32_16x16x32_f16 a[152:155], v[0:3], v[144:147], a[152:155]\n"
  "ds_read_b128 v[136:139], v243 offset:1024\n"
  "v_med3_f32 v224, v168, s64, v245\n"
  "v_mfma_f32_16x16x32_f16 a[152:155], v[4:7], v[148:151], a[152:155]\n"
  "v_med3_f32 v225, v169, s64, v245\n"
  "v_med3_f32 v226, v170, s64, v245\n"
  "v_mfma_f32_16x16x32_f16 a[184:187], v[12:15], v[148:151], a[184:187]\n"
  "v_med3_f32 v227, v171, s64, v245\n"
  "v_mfma_f32_16x16x32_f16 a[184:187], v[8:11], v[144:147], a[184:187]\n"
  "ds_read_b128 v[140:143], v243 offset:9216\n"
  "v_mul_f32 v224, s65, v224\n"
  "v_mul_f32 v225, s65, v225\n"
  "v_mfma_f32_16x16x32_f16 a[216:219], v[16:19], v[144:147], a[216:219]\n"
  "v_mul_f32 v226, s65, v226\n"
  "s_nop 3\n"
  "v_mfma_f32_16x16x32_f16 a[216:219], v[20:23], v[148:151], a[216:219]\n"
  "v_mul_f32 v227, s65, v227\n"
  "v_mfma_f32_16x16x32_f16 a[248:251], v[28:31], v[148:151], a[248:251]\n"
  "v_exp_f32 v224, v224\n"
  "v_exp_f32 v225, v225\n"
  "v_mfma_f32_16x16x32_f16 a[248:251], v[24:27], v[144:147], a[248:251]\n"
  "v_exp_f32 v226, v226\n"
  "s_waitcnt lgkmcnt(4)\n"
  "v_mfma_f32_16x16x32_f16 a[156:159], v[0:3], v[152:155], a[156:159]\n"
  "ds_read_b128 v[144:147], v243 offset:2048\n"
  "v_exp_f32 v227, v227\n"
  "v_add_f32 v224, s67, v224\n"
  "v_mfma_f32_16x16x32_f16 a[156:159], v[4:7], v[156:159], a[156:159]\n"
  "v_add_f32 v225, s67, v225\n"
  "v_mfma_f32_16x16x32_f16 a[188:191], v[12:15], v[156:159], a[188:191]\n"
  "v_add_f32 v226, s67, v226\n"
  "v_add_f32 v227, s67, v227\n"
  "s_nop 3\n"
  "v_mfma_f32_16x16x32_f16 a[188:191], v[8:11], v[152:155], a[188:191]\n"
  "ds_read_b128 v[148:151], v243 offset:10240\n"
  "v_rcp_f32 v224, v224\n"
  "v_mfma_f32_16x16x32_f16 a[220:223], v[16:19], v[152:155], a[220:223]\n"
  "v_rcp_f32 v225, v225\n"
  "v_mfma_f32_16x16x32_f16 a[220:223], v[20:23], v[156:159], a[220:223]\n"
  "v_rcp_f32 v226, v226\n"
  "v_rcp_f32 v227, v227\n"
  "v_mfma_f32_16x16x32_f16 a[252:255], v[28:31], v[156:159], a[252:255]\n"
  "v_mul_f32 v168, v168, v224\n"
  "v_mfma_f32_16x16x32_f16 a[252:255], v[24:27], v[152:155], a[252:255]\n"
  "v_mul_f32 v169, v169, v225\n"
  "v_mul_f32 v170, v170, v226\n"
  "s_nop 3\n"
  "s_waitcnt vmcnt(19) lgkmcnt(4)\n"
  "v_mfma_f32_16x16x32_f16 a[128:131], v[32:35], v[128:131], a[128:131]\n"
  "ds_read_b128 v[152:155], v243 offset:3072\n"
  "v_mul_f32 v171, v171, v227\n"
  "v_mfma_f32_16x16x32_f16 a[128:131], v[36:39], v[132:135], a[128:131]\n"
  "s_mov_b32 s63, 0x20000\nbuffer_store_dwordx4 v[168:171], v242, s[28:31], s63 offen offset:0 nt\n"
  "v_mfma_f32_16x16x32_f16 a[160:163], v[44:47], v[132:135], a[160:163]\n"
  "v_accvgpr_read_b32 v172, a17\n"
  "v_accvgpr_read_b32 v173, a21\n"
  "v_mfma_f32_16x16x32_f16 a[160:163], v[40:43], v[128:131], a[160:163]\n"
  "ds_read_b128 v[156:159], v243 offset:11264\n"
  "v_accvgpr_read_b32 v174, a25\n"
  "v_mfma_f32_16x16x32_f16 a[192:195], v[48:51], v[128:131], a[192:195]\n"
  "v_accvgpr_read_b32 v175, a29\n"
  "v_med3_f32 v228, v172, s64, v245\n"
  "s_nop 3\n"
  "v_mfma_f32_16x16x32_f16 a[192:195], v[52:55], v[132:135], a[192:195]\n"
  "v_med3_f32 v229, v173, s64, v245\n"
  "v_mfma_f32_16x16x32_f16 a[224:227], v[60:63], v[132:135], a[224:227]\n"
  "v_med3_f32 v230, v174, s64, v245\n"
  "v_med3_f32 v231, v175, s64, v245\n"
  "v_mfma_f32_16x16x32_f16 a[224:227], v[56:59], v[128:131], a[224:227]\n"
  "v_mul_f32 v228, s65, v228\n"
  "s_waitcnt lgkmcnt(4)\n"
  "v_mfma_f32_16x16x32_f16 a[132:135], v[32:35], v[136:139], a[132:135]\n"
  "ds_read_b128 v[128:131], v243 offset:4096\n"
  "v_mul_f32 v229, s65, v229\n"
  "v_mfma_f32_16x16x32_f16 a[132:135], v[36:39], v[140:143], a[132:135]\n"
  "v_mul_f32 v230, s65, v230\n"
  "v_mul_f32 v231, s65, v231\n"
  "v_mfma_f32_16x16x32_f16 a[164:167], v[44:47], v[140:143], a[164:167]\n"
  "v_exp_f32 v228, v228\n"
  "s_nop 3\n"
  "v_mfma_f32_16x16x32_f16 a[164:167], v[40:43], v[136:139], a[164:167]\n"
  "ds_read_b128 v[132:135], v243 offset:12288\n"
  "v_exp_f32 v229, v229\n"
  "v_exp_f32 v230, v230\n"
  "v_mfma_f32_16x16x32_f16 a[196:199], v[48:51], v[136:139], a[196:199]\n"
  "v_exp_f32 v231, v231\n"
  "v_mfma_f32_16x16x32_f16 a[196:199], v[52:55], v[140:143], a[196:199]\n"
  "v_add_f32 v228, s67, v228\n"
  "v_add_f32 v229, s67, v229\n"
  "v_mfma_f32_16x16x32_f16 a[228:231], v[60:63], v[140:143], a[228:231]\n"
  "v_add_f32 v230, s67, v230\n"
  "v_mfma_f32_16x16x32_f16 a[228:231], v[56:59], v[136:139], a[228:231]\n"
  "v_add_f32 v231, s67, v231\n"
  "s_nop 3\n"
  "s_waitcnt lgkmcnt(4)\n"
  "v_mfma_f32_16x16x32_f16 a[136:139], v[32:35], v[144:147], a[136:139]\n"
  "ds_read_b128 v[136:139], v243 offset:5120\n"
  "v_rcp_f32 v228, v228\n"
  "v_rcp_f32 v229, v229\n"
  "v_mfma_f32_16x16x32_f16 a[136:139], v[36:39], v[148:151], a[136:139]\n"
  "v_rcp_f32 v230, v230\n"
  "v_mfma_f32_16x16x32_f16 a[168:171], v[44:47], v[148:151], a[168:171]\n"
  "v_rcp_f32 v231, v231\n"
  "v_mul_f32 v172, v172, v228\n"
  "v_mfma_f32_16x16x32_f16 a[168:171], v[40:43], v[144:147], a[168:171]\n"
  "ds_read_b128 v[140:143], v243 offset:13312\n"
  "v_mul_f32 v173, v173, v229\n"
  "v_mfma_f32_16x16x32_f16 a[200:203], v[48:51], v[144:147], a[200:203]\n"
  "v_mul_f32 v174, v174, v230\n"
  "v_mul_f32 v175, v175, v231\n"
  "s_nop 3\n"
  "v_mfma_f32_16x16x32_f16 a[200:203], v[52:55], v[148:151], a[200:203]\n"
  "s_mov_b32 s63, 0x20000\nbuffer_store_dwordx4 v[172:175], v242, s[28:31], s63 offen offset:256 nt\n"
  "v_mfma_f32_16x16x32_f16 a[232:235], v[60:63], v[148:151], a[232:235]\n"
  "v_accvgpr_read_b32 v176, a2\n"
  "v_mfma_f32_16x16x32_f16 a[232:235], v[56:59], v[144:147], a[232:235]\n"
  "v_accvgpr_read_b32 v177, a6\n"
  "v_accvgpr_read_b32 v178, a10\n"
  "s_waitcnt lgkmcnt(4)\n"
  "v_mfma_f32_16x16x32_f16 a[140:143], v[32:35], v[152:155], a[140:143]\n"
  "ds_read_b128 v[144:147], v243 offset:6144\n"
  "v_accvgpr_read_b32 v179, a14\n"
  "v_mfma_f32_16x16x32_f16 a[140:143], v[36:39], v[156:159], a[140:143]\n"
  "v_med3_f32 v224, v176, s64, v245\n"
  "v_med3_f32 v225, v177, s64, v245\n"
  "s_nop 3\n"
  "v_mfma_f32_16x16x32_f16 a[172:175], v[44:47], v[156:159], a[172:175]\n"
  "v_med3_f32 v226, v178, s64, v245\n"
  "v_mfma_f32_16x16x32_f16 a[172:175], v[40:43], v[152:155], a[172:175]\n"
  "ds_read_b128 v[148:151], v243 offset:14336\n"
  "v_med3_f32 v227, v179, s64, v245\n"
  "v_mul_f32 v224, s65, v224\n"
  "v_mfma_f32_16x16x32_f16 a[204:207], v[48:51], v[152:155], a[204:207]\n"
  "v_mul_f32 v225, s65, v225\n"
  "v_mfma_f32_16x16x32_f16 a[204:207], v[52:55], v[156:159], a[204:207]\n"
  "v_mul_f32 v226, s65, v226\n"
  "v_mfma_f32_16x16x32_f16 a[236:239], v[60:63], v[156:159], a[236:239]\n"
  "v_mul_f32 v227, s65, v227\n"
  "v_exp_f32 v224, v224\n"
  "v_mfma_f32_16x16x32_f16 a[236:239], v[56:59], v[152:155], a[236:239]\n"
  "v_exp_f32 v225, v225\n"
  "s_nop 3\n"
  "s_waitcnt lgkmcnt(4)\n"
  "v_mfma_f32_16x16x32_f16 a[144:147], v[32:35], v[128:131], a[144:147]\n"
  "ds_read_b128 v[152:155], v243 offset:7168\n"
  "v_exp_f32 v226, v226\n"
  "v_exp_f32 v227, v227\n"
  "v_mfma_f32_16x16x32_f16 a[144:147], v[36:39], v[132:135], a[144:147]\n"
  "v_add_f32 v224, s67, v224\n"
  "v_mfma_f32_16x16x32_f16 a[176:179], v[44:47], v[132:135], a[176:179]\n"
  "v_add_f32 v225, s67, v225\n"
  "v_add_f32 v226, s67, v226\n"
  "v_mfma_f32_16x16x32_f16 a[176:179], v[40:43], v[128:131], a[176:179]\n"
  "ds_read_b128 v[156:159], v243 offset:15360\n"
  "v_add_f32 v227, s67, v227\n"
  "v_mfma_f32_16x16x32_f16 a[208:211], v[48:51], v[128:131], a[208:211]\n"
  "v_rcp_f32 v224, v224\n"
  "s_nop 3\n"
  "v_mfma_f32_16x16x32_f16 a[208:211], v[52:55], v[132:135], a[208:211]\n"
  "v_rcp_f32 v225, v225\n"
  "v_rcp_f32 v226, v226\n"
  "v_mfma_f32_16x16x32_f16 a[240:243], v[60:63], v[132:135], a[240:243]\n"
  "v_rcp_f32 v227, v227\n"
  "v_mfma_f32_16x16x32_f16 a[240:243], v[56:59], v[128:131], a[240:243]\n"
  "v_add_u32 v243, 0x4000, v243\nv_and_b32 v243, 0x1ffff, v243\n"
  "v_mul_f32 v176, v176, v224\n"
  "v_mul_f32 v177, v177, v225\n"
  "s_waitcnt lgkmcnt(4)\n"
  "v_mfma_f32_16x16x32_f16 a[148:151], v[32:35], v[136:139], a[148:151]\n"
  "ds_read_b128 v[128:131], v243 offset:0\n"
  "v_mul_f32 v178, v178, v226\n"
  "v_mfma_f32_16x16x32_f16 a[148:151], v[36:39], v[140:143], a[148:151]\n"
  "v_mul_f32 v179, v179, v227\n"
  "s_mov_b32 s63, 0x40000\nbuffer_store_dwordx4 v[176:179], v242, s[28:31], s63 offen offset:0 nt\n"
  "s_nop 3\n"
  "v_mfma_f32_16x16x32_f16 a[180:183], v[44:47], v[140:143], a[180:183]\n"
  "v_accvgpr_read_b32 v180, a18\n"
  "v_mfma_f32_16x16x32_f16 a[180:183], v[40:43], v[136:139], a[180:183]\n"
  "ds_read_b128 v[132:135], v243 offset:8192\n"
  "v_accvgpr_read_b32 v181, a22\n"
  "v_mfma_f32_16x16x32_f16 a[212:215], v[48:51], v[136:139], a[212:215]\n"
  "v_accvgpr_read_b32 v182, a26\n"
  "v_accvgpr_read_b32 v183, a30\n"
  "v_mfma_f32_16x16x32_f16 a[212:215], v[52:55], v[140:143], a[212:215]\n"
  "v_med3_f32 v228, v180, s64, v245\n"
  "v_mfma_f32_16x16x32_f16 a[244:247], v[60:63], v[140:143], a[244:247]\n"
  "v_med3_f32 v229, v181, s64, v245\n"
  "v_med3_f32 v230, v182, s64, v245\n"
  "v_mfma_f32_16x16x32_f16 a[244:247], v[56:59], v[136:139], a[244:247]\n"
  "v_med3_f32 v231, v183, s64, v245\n"
  "s_nop 3\n"
  "s_waitcnt lgkmcnt(4)\n"
  "v_mfma_f32_16x16x32_f16 a[152:155], v[32:35], v[144:147], a[152:155]\n"
  "ds_read_b128 v[136:139], v243 offset:1024\n"
  "v_mul_f32 v228, s65, v228\n"
  "v_mul_f32 v229, s65, v229\n"
  "v_mfma_f32_16x16x32_f16 a[152:155], v[36:39], v[148:151], a[152:155]\n"
  "v_mul_f32 v230, s65, v230\n"
  "v_mfma_f32_16x16x32_f16 a[184:187], v[44:47], v[148:151], a[184:187]\n"
  "v_mul_f32 v231, s65, v231\n"
  "v_mfma_f32_16x16x32_f16 a[184:187], v[40:43], v[144:147], a[184:187]\n"
  "ds_read_b128 v[140:143], v243 offset:9216\n"
  "v_exp_f32 v228, v228\n"
  "v_exp_f32 v229, v229\n"
  "v_mfma_f32_16x16x32_f16 a[216:219], v[48:51], v[144:147], a[216:219]\n"
  "v_exp_f32 v230, v230\n"
  "s_nop 3\n"
  "v_mfma_f32_16x16x32_f16 a[216:219], v[52:55], v[148:151], a[216:219]\n"
  "v_exp_f32 v231, v231\n"
  "v_add_f32 v228, s67, v228\n"
  "v_mfma_f32_16x16x32_f16 a[248:251], v[60:63], v[148:151], a[248:251]\n"
  "v_add_f32 v229, s67, v229\n"
  "v_mfma_f32_16x16x32_f16 a[248:251], v[56:59], v[144:147], a[248:251]\n"
  "v_add_f32 v230, s67, v230\n"
  "v_add_f32 v231, s67, v231\n"
  "s_waitcnt lgkmcnt(4)\n"
  "v_mfma_f32_16x16x32_f16 a[156:159], v[32:35], v[152:155], a[156:159]\n"
  "ds_read_b128 v[144:147], v243 offset:2048\n"
  "v_rcp_f32 v228, v228\n"
  "v_mfma_f32_16x16x32_f16 a[156:159], v[36:39], v[156:159], a[156:159]\n"
  "v_rcp_f32 v229, v229\n"
  "s_nop 3\n"
  "v_mfma_f32_16x16x32_f16 a[188:191], v[44:47], v[156:159], a[188:191]\n"
  "v_rcp_f32 v230, v230\n"
  "v_rcp_f32 v231, v231\n"
  "v_mfma_f32_16x16x32_f16 a[188:191], v[40:43], v[152:155], a[188:191]\n"
  "ds_read_b128 v[148:151], v243 offset:10240\n"
  "v_mul_f32 v180, v180, v228\n"
  "v_mfma_f32_16x16x32_f16 a[220:223], v[48:51], v[152:155], a[220:223]\n"
  "v_mul_f32 v181, v181, v229\n"
  "v_mul_f32 v182, v182, v230\n"
  "v_mfma_f32_16x16x32_f16 a[220:223], v[52:55], v[156:159], a[220:223]\n"
  "v_mul_f32 v183, v183, v231\n"
  "v_mfma_f32_16x16x32_f16 a[252:255], v[60:63], v[156:159], a[252:255]\n"
  "s_mov_b32 s63, 0x40000\nbuffer_store_dwordx4 v[180:183], v242, s[28:31], s63 offen offset:256 nt\n"
  "v_mfma_f32_16x16x32_f16 a[252:255], v[56:59], v[152:155], a[252:255]\n"
  "v_accvgpr_read_b32 v184, a3\n"
  "v_accvgpr_read_b32 v185, a7\n"
  "s_waitcnt vmcnt(15) lgkmcnt(4)\n"
  "v_mfma_f32_16x16x32_f16 a[128:131], v[64:67], v[128:131], a[128:131]\n"
  "ds_read_b128 v[152:155], v243 offset:3072\n"
  "v_accvgpr_read_b32 v186, a11\n"
  "s_nop 3\n"
  "v_mfma_f32_16x16x32_f16 a[128:131], v[68:71], v[132:135], a[128:131]\n"
  "v_accvgpr_read_b32 v187, a15\n"
  "v_med3_f32 v224, v184, s64, v245\n"
  "v_mfma_f32_16x16x32_f16 a[160:163], v[76:79], v[132:135], a[160:163]\n"
  "v_med3_f32 v225, v185, s64, v245\n"
  "v_mfma_f32_16x16x32_f16 a[160:163], v[72:75], v[128:131], a[160:163]\n"
  "ds_read_b128 v[156:159], v243 offset:11264\n"
  "v_med3_f32 v226, v186, s64, v245\n"
  "v_med3_f32 v227, v187, s64, v245\n"
  "v_mfma_f32_16x16x32_f16 a[192:195], v[80:83], v[128:131], a[192:195]\n"
  "v_mul_f32 v224, s65, v224\n"
  "v_mfma_f32_16x16x32_f16 a[192:195], v[84:87], v[132:135], a[192:195]\n"
  "v_mul_f32 v225, s65, v225\n"
  "v_mfma_f32_16x16x32_f16 a[224:227], v[92:95], v[132:135], a[224:227]\n"
  "v_mul_f32 v226, s65, v226\n"
  "v_mul_f32 v227, s65, v227\n"
  "v_mfma_f32_16x16x32_f16 a[224:227], v[88:91], v[128:131], a[224:227]\n"
  "v_exp_f32 v224, v224\n"
  "s_nop 3\n"
  "s_waitcnt lgkmcnt(4)\n"
  "v_mfma_f32_16x16x32_f16 a[132:135], v[64:67], v[136:139], a[132:135]\n"
  "ds_read_b128 v[128:131], v243 offset:4096\n"
  "v_exp_f32 v225, v225\n"
  "v_exp_f32 v226, v226\n"
  "v_mfma_f32_16x16x32_f16 a[132:135], v[68:71], v[140:143], a[132:135]\n"
  "v_exp_f32 v227, v227\n"
  "v_mfma_f32_16x16x32_f16 a[164:167], v[76:79], v[140:143], a[164:167]\n"
  "v_add_f32 v224, s67, v224\n"
  "v_add_f32 v225, s67, v225\n"
  "v_mfma_f32_16x16x32_f16 a[164:167], v[72:75], v[136:139], a[164:167]\n"
  "ds_read_b128 v[132:135], v243 offset:12288\n"
  "v_add_f32 v226, s67, v226\n"
  "v_mfma_f32_16x16x32_f16 a[196:199], v[80:83], v[136:139], a[196:199]\n"
  "v_add_f32 v227, s67, v227\n"
  "v_mfma_f32_16x16x32_f16 a[196:199], v[84:87], v[140:143], a[196:199]\n"
  "v_rcp_f32 v224, v224\n"
  "v_rcp_f32 v225, v225\n"
  "v_mfma_f32_16x16x32_f16 a[228:231], v[92:95], v[140:143], a[228:231]\n"
  "v_rcp_f32 v226, v226\n"
  "v_mfma_f32_16x16x32_f16 a[228:231], v[88:91], v[136:139], a[228:231]\n"
  "v_rcp_f32 v227, v227\n"
  "v_mul_f32 v184, v184, v224\n"
  "s_nop 3\n"
  "s_waitcnt lgkmcnt(4)\n"
  "v_mfma_f32_16x16x32_f16 a[136:139], v[64:67], v[144:147], a[136:139]\n"
  "ds_read_b128 v[136:139], v243 offset:5120\n"
  "v_mul_f32 v185, v185, v225\n"
  "v_mfma_f32_16x16x32_f16 a[136:139], v[68:71], v[148:151], a[136:139]\n"
  "v_mul_f32 v186, v186, v226\n"
  "v_mul_f32 v187, v187, v227\n"
  "v_mfma_f32_16x16x32_f16 a[168:171], v[76:79], v[148:151], a[168:171]\n"
  "s_mov_b32 s63, 0x60000\nbuffer_store_dwordx4 v[184:187], v242, s[28:31], s63 offen offset:0 nt\n"
  "v_mfma_f32_16x16x32_f16 a[168:171], v[72:75], v[144:147], a[168:171]\n"
  "ds_read_b128 v[140:143], v243 offset:13312\n"
  "v_accvgpr_read_b32 v188, a19\n"
  "v_mfma_f32_16x16x32_f16 a[200:203], v[80:83], v[144:147], a[200:203]\n"
  "v_accvgpr_read_b32 v189, a23\n"
  "v_accvgpr_read_b32 v190, a27\n"
  "v_mfma_f32_16x16x32_f16 a[200:203], v[84:87], v[148:151], a[200:203]\n"
  "v_accvgpr_read_b32 v191, a31\n"
  "v_mfma_f32_16x16x32_f16 a[232:235], v[92:95], v[148:151], a[232:235]\n"
  "v_med3_f32 v228, v188, s64, v245\n"
  "v_med3_f32 v229, v189, s64, v245\n"
  "v_mfma_f32_16x16x32_f16 a[232:235], v[88:91], v[144:147], a[232:235]\n"
  "v_med3_f32 v230, v190, s64, v245\n"
  "s_nop 3\n"
  "s_waitcnt lgkmcnt(4)\n"
  "v_mfma_f32_16x16x32_f16 a[140:143], v[64:67], v[152:155], a[140:143]\n"
  "ds_read_b128 v[144:147], v243 offset:6144\n"
  "v_med3_f32 v231, v191, s64, v245\n"
  "v_mul_f32 v228, s65, v228\n"
  "v_mfma_f32_16x16x32_f16 a[140:143], v[68:71], v[156:159], a[140:143]\n"
  "v_mul_f32 v229, s65, v229\n"
  "v_mfma_f32_16x16x32_f16 a[172:175], v[76:79], v[156:159], a[172:175]\n"
  "v_mul_f32 v230, s65, v230\n"
  "v_mfma_f32_16x16x32_f16 a[172:175], v[72:75], v[152:155], a[172:175]\n"
  "ds_read_b128 v[148:151], v243 offset:14336\n"
  "v_mul_f32 v231, s65, v231\n"
  "v_exp_f32 v228, v228\n"
  "v_mfma_f32_16x16x32_f16 a[204:207], v[80:83], v[152:155], a[204:207]\n"
  "v_exp_f32 v229, v229\n"
  "v_mfma_f32_16x16x32_f16 a[204:207], v[84:87], v[156:159], a[204:207]\n"
  "v_exp_f32 v230, v230\n"
  "v_exp_f32 v231, v231\n"
  "v_mfma_f32_16x16x32_f16 a[236:239], v[92:95], v[156:159], a[236:239]\n"
  "v_add_f32 v228, s67, v228\n"
  "v_mfma_f32_16x16x32_f16 a[236:239], v[88:91], v[152:155], a[236:239]\n"
  "v_add_f32 v229, s67, v229\n"
  "v_add_f32 v230, s67, v230\n"
  "s_nop 3\n"
  "s_waitcnt lgkmcnt(4)\n"
  "v_mfma_f32_16x16x32_f16 a[144:147], v[64:67], v[128:131], a[144:147]\n"
  "ds_read_b128 v[152:155], v243 offset:7168\n"
  "v_add_f32 v231, s67, v231\n"
  "v_mfma_f32_16x16x32_f16 a[144:147], v[68:71], v[132:135], a[144:147]\n"
  "v_rcp_f32 v228, v228\n"
  "v_mfma_f32_16x16x32_f16 a[176:179], v[76:79], v[132:135], a[176:179]\n"
  "v_rcp_f32 v229, v229\n"
  "v_rcp_f32 v230, v230\n"
  "v_mfma_f32_16x16x32_f16 a[176:179], v[72:75], v[128:131], a[176:179]\n"
  "ds_read_b128 v[156:159], v243 offset:15360\n"
  "v_rcp_f32 v231, v231\n"
  "v_mfma_f32_16x16x32_f16 a[208:211], v[80:83], v[128:131], a[208:211]\n"
  "v_mul_f32 v188, v188, v228\n"
  "v_mul_f32 v189, v189, v229\n"
  "v_mfma_f32_16x16x32_f16 a[208:211], v[84:87], v[132:135], a[208:211]\n"
  "v_mul_f32 v190, v190, v230\n"
  "v_mfma_f32_16x16x32_f16 a[240:243], v[92:95], v[132:135], a[240:243]\n"
  "v_mul_f32 v191, v191, v231\n"
  "s_mov_b32 s63, 0x60000\nbuffer_store_dwordx4 v[188:191], v242, s[28:31], s63 offen offset:256 nt\n"
  "v_mfma_f32_16x16x32_f16 a[240:243], v[88:91], v[128:131], a[240:243]\n"
  "v_add_u32 v243, 0x4000, v243\nv_and_b32 v243, 0x1ffff, v243\n"
  "v_accvgpr_read_b32 v192, a32\n"
  "s_nop 3\n"
  "s_waitcnt lgkmcnt(4)\n"
  "v_mfma_f32_16x16x32_f16 a[148:151], v[64:67], v[136:139], a[148:151]\n"
  "ds_read_b128 v[128:131], v243 offset:0\n"
  "v_accvgpr_read_b32 v193, a36\n"
  "v_mfma_f32_16x16x32_f16 a[148:151], v[68:71], v[140:143], a[148:151]\n"
  "v_accvgpr_read_b32 v194, a40\n"
  "v_accvgpr_read_b32 v195, a44\n"
  "v_mfma_f32_16x16x32_f16 a[180:183], v[76:79], v[140:143], a[180:183]\n"
  "v_med3_f32 v224, v192, s64, v245\n"
  "v_mfma_f32_16x16x32_f16 a[180:183], v[72:75], v[136:139], a[180:183]\n"
  "ds_read_b128 v[132:135], v243 offset:8192\n"
  "v_med3_f32 v225, v193, s64, v245\n"
  "v_med3_f32 v226, v194, s64, v245\n"
  "v_mfma_f32_16x16x32_f16 a[212:215], v[80:83], v[136:139], a[212:215]\n"
  "v_med3_f32 v227, v195, s64, v245\n"
  "v_mfma_f32_16x16x32_f16 a[212:215], v[84:87], v[140:143], a[212:215]\n"
  "v_mul_f32 v224, s65, v224\n"
  "v_mul_f32 v225, s65, v225\n"
  "v_mfma_f32_16x16x32_f16 a[244:247], v[92:95], v[140:143], a[244:247]\n"
  "v_mul_f32 v226, s65, v226\n"
  "v_mfma_f32_16x16x32_f16 a[244:247], v[88:91], v[136:139], a[244:247]\n"
  "v_mul_f32 v227, s65, v227\n"
  "s_nop 3\n"
  "s_waitcnt lgkmcnt(4)\n"
  "v_mfma_f32_16x16x32_f16 a[152:155], v[64:67], v[144:147], a[152:155]\n"
  "ds_read_b128 v[136:139], v243 offset:1024\n"
  "v_exp_f32 v224, v224\n"
  "v_exp_f32 v225, v225\n"
  "v_mfma_f32_16x16x32_f16 a[152:155], v[68:71], v[148:151], a[152:155]\n"
  "v_exp_f32 v226, v226\n"
  "v_mfma_f32_16x16x32_f16 a[184:187], v[76:79], v[148:151], a[184:187]\n"
  "v_exp_f32 v227, v227\n"
  "v_add_f32 v224, s67, v224\n"
  "v_mfma_f32_16x16x32_f16 a[184:187], v[72:75], v[144:147], a[184:187]\n"
  "ds_read_b128 v[140:143], v243 offset:9216\n"
  "v_add_f32 v225, s67, v225\n"
  "v_mfma_f32_16x16x32_f16 a[216:219], v[80:83], v[144:147], a[216:219]\n"
  "v_add_f32 v226, s67, v226\n"
  "v_add_f32 v227, s67, v227\n"
  "v_mfma_f32_16x16x32_f16 a[216:219], v[84:87], v[148:151], a[216:219]\n"
  "v_rcp_f32 v224, v224\n"
  "v_mfma_f32_16x16x32_f16 a[248:251], v[92:95], v[148:151], a[248:251]\n"
  "v_rcp_f32 v225, v225\n"
  "v_mfma_f32_16x16x32_f16 a[248:251], v[88:91], v[144:147], a[248:251]\n"
  "v_rcp_f32 v226, v226\n"
  "v_rcp_f32 v227, v227\n"
  "s_nop 3\n"
  "s_waitcnt lgkmcnt(4)\n"
  "v_mfma_f32_16x16x32_f16 a[156:159], v[64:67], v[152:155], a[156:159]\n"
  "ds_read_b128 v[144:147], v243 offset:2048\n"
  "v_mul_f32 v192, v192, v224\n"
  "v_mfma_f32_16x16x32_f16 a[156:159], v[68:71], v[156:159], a[156:159]\n"
  "v_mul_f32 v193, v193, v225\n"
  "v_mul_f32 v194, v194, v226\n"
  "v_mfma_f32_16x16x32_f16 a[188:191], v[76:79], v[156:159], a[188:191]\n"
  "v_mul_f32 v195, v195, v227\n"
  "v_mfma_f32_16x16x32_f16 a[188:191], v[72:75], v[152:155], a[188:191]\n"
  "ds_read_b128 v[148:151], v243 offset:10240\n"
  "s_mov_b32 s63, 0x200000\nbuffer_store_dwordx4 v[192:195], v242, s[28:31], s63 offen offset:0 nt\n"
  "v_mfma_f32_16x16x32_f16 a[220:223], v[80:83], v[152:155], a[220:223]\n"
  "v_accvgpr_read_b32 v196, a48\n"
  "v_accvgpr_read_b32 v197, a52\n"
  "v_mfma_f32_16x16x32_f16 a[220:223], v[84:87], v[156:159], a[220:223]\n"
  "v_accvgpr_read_b32 v198, a56\n"
  "v_mfma_f32_16x16x32_f16 a[252:255], v[92:95], v[156:159], a[252:255]\n"
  "v_accvgpr_read_b32 v199, a60\n"
  "v_med3_f32 v228, v196, s64, v245\n"
  "v_mfma_f32_16x16x32_f16 a[252:255], v[88:91], v[152:155], a[252:255]\n"
  "v_med3_f32 v229, v197, s64, v245\n"
  "s_nop 3\n"
  "s_waitcnt vmcnt(10) lgkmcnt(4)\n"
  "v_mfma_f32_16x16x32_f16 a[128:131], v[96:99], v[128:131], a[128:131]\n"
  "ds_read_b128 v[152:155], v243 offset:3072\n"
  "v_med3_f32 v230, v198, s64, v245\n"
  "v_med3_f32 v231, v199, s64, v245\n"
  "v_mfma_f32_16x16x32_f16 a[128:131], v[100:103], v[132:135], a[128:131]\n"
  "v_mul_f32 v228, s65, v228\n"
  "v_mfma_f32_16x16x32_f16 a[160:163], v[108:111], v[132:135], a[160:163]\n"
  "v_mul_f32 v229, s65, v229\n"
  "v_mfma_f32_16x16x32_f16 a[160:163], v[104:107], v[128:131], a[160:163]\n"
  "ds_read_b128 v[156:159], v243 offset:11264\n"
  "v_mul_f32 v230, s65, v230\n"
  "v_mul_f32 v231, s65, v231\n"
  "v_mfma_f32_16x16x32_f16 a[192:195], v[112:115], v[128:131], a[192:195]\n"
  "v_exp_f32 v228, v228\n"
  "v_mfma_f32_16x16x32_f16 a[192:195], v[116:119], v[132:135], a[192:195]\n"
  "s_waitcnt vmcnt(9)\n"
  "v_readfirstlane_b32 s88, v232\nv_readfirstlane_b32 s86, v233\n"
  "v_exp_f32 v229, v229\n"
  "v_exp_f32 v230, v230\n"
  "v_mfma_f32_16x16x32_f16 a[224:227], v[124:127], v[132:135], a[224:227]\n"
  "v_exp_f32 v231, v231\n"
  "v_mfma_f32_16x16x32_f16 a[224:227], v[120:123], v[128:131], a[224:227]\n"
  "v_add_f32 v228, s67, v228\n"
  "v_add_f32 v229, s67, v229\n"
  "s_nop 3\n"
  "s_waitcnt lgkmcnt(4)\n"
  "v_mfma_f32_16x16x32_f16 a[132:135], v[96:99], v[136:139], a[132:135]\n"
  "ds_read_b128 v[128:131], v243 offset:4096\n"
  "v_add_f32 v230, s67, v230\n"
  "v_mfma_f32_16x16x32_f16 a[132:135], v[100:103], v[140:143], a[132:135]\n"
  "v_add_f32 v231, s67, v231\n"
  "v_mfma_f32_16x16x32_f16 a[164:167], v[108:111], v[140:143], a[164:167]\n"
  "v_rcp_f32 v228, v228\n"
  "v_rcp_f32 v229, v229\n"
  "v_mfma_f32_16x16x32_f16 a[164:167], v[104:107], v[136:139], a[164:167]\n"
  "ds_read_b128 v[132:135], v243 offset:12288\n"
  "v_rcp_f32 v230, v230\n"
  "v_mfma_f32_16x16x32_f16 a[196:199], v[112:115], v[136:139], a[196:199]\n"
  "v_rcp_f32 v231, v231\n"
  "v_mul_f32 v196, v196, v228\n"
  "v_mfma_f32_16x16x32_f16 a[196:199], v[116:119], v[140:143], a[196:199]\n"
  "v_mul_f32 v197, v197, v229\n"
  "v_mfma_f32_16x16x32_f16 a[228:231], v[124:127], v[140:143], a[228:231]\n"
  "v_mul_f32 v198, v198, v230\n"
  "v_mul_f32 v199, v199, v231\n"
  "v_mfma_f32_16x16x32_f16 a[228:231], v[120:123], v[136:139], a[228:231]\n"
  "s_mov_b32 s63, 0x200000\nbuffer_store_dwordx4 v[196:199], v242, s[28:31], s63 offen offset:256 nt\n"
  "s_nop 3\n"
  "s_waitcnt lgkmcnt(4)\n"
  "v_mfma_f32_16x16x32_f16 a[136:139], v[96:99], v[144:147], a[136:139]\n"
  "ds_read_b128 v[136:139], v243 offset:5120\n"
  "v_accvgpr_read_b32 v200, a33\n"
  "v_mfma_f32_16x16x32_f16 a[136:139], v[100:103], v[148:151], a[136:139]\n"
  "v_accvgpr_read_b32 v201, a37\n"
  "v_accvgpr_read_b32 v202, a41\n"
  "v_mfma_f32_16x16x32_f16 a[168:171], v[108:111], v[148:151], a[168:171]\n"
  "v_accvgpr_read_b32 v203, a45\n"
  "v_mfma_f32_16x16x32_f16 a[168:171], v[104:107], v[144:147], a[168:171]\n"
  "ds_read_b128 v[140:143], v243 offset:13312\n"
  "v_med3_f32 v224, v200, s64, v245\n"
  "v_med3_f32 v225, v201, s64, v245\n"
  "v_mfma_f32_16x16x32_f16 a[200:203], v[112:115], v[144:147], a[200:203]\n"
  "v_med3_f32 v226, v202, s64, v245\n"
  "v_mfma_f32_16x16x32_f16 a[200:203], v[116:119], v[148:151], a[200:203]\n"
  "v_med3_f32 v227, v203, s64, v245\n"
  "v_mul_f32 v224, s65, v224\n"
  "v_mfma_f32_16x16x32_f16 a[232:235], v[124:127], v[148:151], a[232:235]\n"
  "v_mul_f32 v225, s65, v225\n"
  "v_mfma_f32_16x16x32_f16 a[232:235], v[120:123], v[144:147], a[232:235]\n"
  "v_mul_f32 v226, s65, v226\n"
  "s_nop 3\n"
  "s_waitcnt lgkmcnt(4)\n"
  "v_mfma_f32_16x16x32_f16 a[140:143], v[96:99], v[152:155], a[140:143]\n"
  "ds_read_b128 v[144:147], v243 offset:6144\n"
  "v_mul_f32 v227, s65, v227\n"
  "v_exp_f32 v224, v224\n"
  "v_mfma_f32_16x16x32_f16 a[140:143], v[100:103], v[156:159], a[140:143]\n"
  "v_exp_f32 v225, v225\n"
  "v_mfma_f32_16x16x32_f16 a[172:175], v[108:111], v[156:159], a[172:175]\n"
  "v_exp_f32 v226, v226\n"
  "v_exp_f32 v227, v227\n"
  "v_mfma_f32_16x16x32_f16 a[172:175], v[104:107], v[152:155], a[172:175]\n"
  "ds_read_b128 v[148:151], v243 offset:14336\n"
  "v_add_f32 v224, s67, v224\n"
  "v_mfma_f32_16x16x32_f16 a[204:207], v[112:115], v[152:155], a[204:207]\n"
  "v_add_f32 v225, s67, v225\n"
  "v_add_f32 v226, s67, v226\n"
  "v_mfma_f32_16x16x32_f16 a[204:207], v[116:119], v[156:159], a[204:207]\n"
  "v_add_f32 v227, s67, v227\n"
  "v_mfma_f32_16x16x32_f16 a[236:239], v[124:127], v[156:159], a[236:239]\n"
  "v_rcp_f32 v224, v224\n"
  "v_mfma_f32_16x16x32_f16 a[236:239], v[120:123], v[152:155], a[236:239]\n"
  "v_rcp_f32 v225, v225\n"
  "v_rcp_f32 v226, v226\n"
  "s_nop 3\n"
  "s_waitcnt lgkmcnt(4)\n"
  "v_mfma_f32_16x16x32_f16 a[144:147], v[96:99], v[128:131], a[144:147]\n"
  "ds_read_b128 v[152:155], v243 offset:7168\n"
  "v_rcp_f32 v227, v227\n"
  "v_mfma_f32_16x16x32_f16 a[144:147], v[100:103], v[132:135], a[144:147]\n"
  "v_mul_f32 v200, v200, v224\n"
  "v_mul_f32 v201, v201, v225\n"
  "v_mfma_f32_16x16x32_f16 a[176:179], v[108:111], v[132:135], a[176:179]\n"
  "v_mul_f32 v202, v202, v226\n"
  "v_mfma_f32_16x16x32_f16 a[176:179], v[104:107], v[128:131], a[176:179]\n"
  "ds_read_b128 v[156:159], v243 offset:15360\n"
  "v_mul_f32 v203, v203, v227\n"
  "s_mov_b32 s63, 0x220000\nbuffer_store_dwordx4 v[200:203], v242, s[28:31], s63 offen offset:0 nt\n"
  "v_mfma_f32_16x16x32_f16 a[208:211], v[112:115], v[128:131], a[208:211]\n"
  "v_accvgpr_read_b32 v204, a49\n"
  "v_mfma_f32_16x16x32_f16 a[208:211], v[116:119], v[132:135], a[208:211]\n"
  "v_accvgpr_read_b32 v205, a53\n"
  "v_mfma_f32_16x16x32_f16 a[240:243], v[124:127], v[132:135], a[240:243]\n"
  "v_accvgpr_read_b32 v206, a57\n"
  "v_accvgpr_read_b32 v207, a61\n"
  "v_mfma_f32_16x16x32_f16 a[240:243], v[120:123], v[128:131], a[240:243]\n"
  "v_med3_f32 v228, v204, s64, v245\n"
  "s_nop 3\n"
  "s_waitcnt lgkmcnt(4)\n"
  "v_mfma_f32_16x16x32_f16 a[148:151], v[96:99], v[136:139], a[148:151]\n"
  "v_med3_f32 v229, v205, s64, v245\n"
  "v_med3_f32 v230, v206, s64, v245\n"
  "v_mfma_f32_16x16x32_f16 a[148:151], v[100:103], v[140:143], a[148:151]\n"
  "v_med3_f32 v231, v207, s64, v245\n"
  "v_mfma_f32_16x16x32_f16 a[180:183], v[108:111], v[140:143], a[180:183]\n"
  "v_mul_f32 v228, s65, v228\n"
  "v_mul_f32 v229, s65, v229\n"
  "v_mfma_f32_16x16x32_f16 a[180:183], v[104:107], v[136:139], a[180:183]\n"
  "v_mul_f32 v230, s65, v230\n"
  "v_mfma_f32_16x16x32_f16 a[212:215], v[112:115], v[136:139], a[212:215]\n"
  "v_mul_f32 v231, s65, v231\n"
  "v_mfma_f32_16x16x32_f16 a[212:215], v[116:119], v[140:143], a[212:215]\n"
  "v_exp_f32 v228, v228\n"
  "v_exp_f32 v229, v229\n"
  "v_mfma_f32_16x16x32_f16 a[244:247], v[124:127], v[140:143], a[244:247]\n"
  "v_exp_f32 v230, v230\n"
  "v_mfma_f32_16x16x32_f16 a[244:247], v[120:123], v[136:139], a[244:247]\n"
  "v_exp_f32 v231, v231\n"
  "v_add_f32 v228, s67, v228\n"
  "s_nop 3\n"
  "s_waitcnt lgkmcnt(2)\n"
  "v_mfma_f32_16x16x32_f16 a[152:155], v[96:99], v[144:147], a[152:155]\n"
  "v_add_f32 v229, s67, v229\n"
  "v_mfma_f32_16x16x32_f16 a[152:155], v[100:103], v[148:151], a[152:155]\n"
  "v_add_f32 v230, s67, v230\n"
  "v_add_f32 v231, s67, v231\n"
  "v_mfma_f32_16x16x32_f16 a[184:187], v[108:111], v[148:151], a[184:187]\n"
  "v_rcp_f32 v228, v228\n"
  "v_mfma_f32_16x16x32_f16 a[184:187], v[104:107], v[144:147], a[184:187]\n"
  "v_rcp_f32 v229, v229\n"
  "v_mfma_f32_16x16x32_f16 a[216:219], v[112:115], v[144:147], a[216:219]\n"
  "v_rcp_f32 v230, v230\n"
  "v_rcp_f32 v231, v231\n"
  "v_mfma_f32_16x16x32_f16 a[216:219], v[116:119], v[148:151], a[216:219]\n"
  "v_mul_f32 v204, v204, v228\n"
  "v_mfma_f32_16x16x32_f16 a[248:251], v[124:127], v[148:151], a[248:251]\n"
  "v_mul_f32 v205, v205, v229\n"
  "v_mul_f32 v206, v206, v230\n"
  "v_mfma_f32_16x16x32_f16 a[248:251], v[120:123], v[144:147], a[248:251]\n"
  "v_mul_f32 v207, v207, v231\n"
  "s_nop 3\n"
  "s_waitcnt lgkmcnt(0)\n"
  "v_mfma_f32_16x16x32_f16 a[156:159], v[96:99], v[152:155], a[156:159]\n"
  "s_mov_b32 s63, 0x220000\nbuffer_store_dwordx4 v[204:207], v242, s[28:31], s63 offen offset:256 nt\n"
  "v_mfma_f32_16x16x32_f16 a[156:159], v[100:103], v[156:159], a[156:159]\n"
  "v_mfma_f32_16x16x32_f16 a[188:191], v[108:111], v[156:159], a[188:191]\n"
  "v_mfma_f32_16x16x32_f16 a[188:191], v[104:107], v[152:155], a[188:191]\n"
  "v_mfma_f32_16x16x32_f16 a[220:223], v[112:115], v[152:155], a[220:223]\n"
  "v_mfma_f32_16x16x32_f16 a[220:223], v[116:119], v[156:159], a[220:223]\n"
  "v_mfma_f32_16x16x32_f16 a[252:255], v[124:127], v[156:159], a[252:255]\n"
  "v_mfma_f32_16x16x32_f16 a[252:255], v[120:123], v[152:155], a[252:255]\n"
  "s_memrealtime s[52:53]\n"
  "s_mov_b64 s[32:33], %1\n"
  "s_mov_b32 s34, 0x900000\n"
  "s_lshl_b32 s81, %6, 7\n"
  "s_waitcnt lgkmcnt(0)\n"
  "v_mov_b32 v253, s52\n"
  "s_mov_b64 exec, 1\n"
  "buffer_store_dword v253, off, s[32:35], s81 offset:4\n"
  "s_mov_b64 exec, -1\n"
  "s_nop 7\n"
  "s_mov_b32 s56, s65\n"
  "s_mov_b32 s57, s65\n"
  "s_mov_b32 s50, s67\n"
  "s_mov_b32 s51, s67\n"
  "s_cmp_eq_u32 s88, s61\n"
  "s_cbranch_scc1 GW3_%=\n"
  "GW1_%=:\n"
  "s_sleep 2\n"
  "buffer_load_dwordx2 v[232:233], off, s[32:35], s66 offset:128 sc1\n"
  "GW2_%=:\n"
  "s_memrealtime s[54:55]\n"
  "s_waitcnt vmcnt(0) lgkmcnt(0)\n"
  "v_readfirstlane_b32 s88, v232\n"
  "v_readfirstlane_b32 s86, v233\n"
  "s_sub_u32 s82, s54, s52\n"
  "s_cmp_eq_u32 s88, s61\n"
  "s_cbranch_scc1 GW3_%=\n"
  "s_cmp_lt_u32 s82, 800\n"
  "s_cbranch_scc1 GW1_%=\n"
  "s_mov_b32 s86, s54\n"
  "GW3_%=:\n"
  "s_memrealtime s[52:53]\n"
  "v_accvgpr_read_b32 v56, a34\n"
  "v_accvgpr_read_b32 v57, a38\n"
  "v_accvgpr_read_b32 v58, a42\n"
  "v_accvgpr_read_b32 v59, a46\n"
  "v_med3_f32 v0, v56, s64, v245\n"
  "v_med3_f32 v1, v57, s64, v245\n"
  "v_med3_f32 v2, v58, s64, v245\n"
  "v_med3_f32 v3, v59, s64, v245\n"
  "v_pk_mul_f32 v[0:1], v[0:1], s[56:57]\n"
  "v_pk_mul_f32 v[2:3], v[2:3], s[56:57]\n"
  "v_exp_f32 v0, v0\n"
  "v_exp_f32 v1, v1\n"
  "v_exp_f32 v2, v2\n"
  "v_exp_f32 v3, v3\n"
  "v_pk_add_f32 v[0:1], v[0:1], s[50:51]\n"
  "v_pk_add_f32 v[2:3], v[2:3], s[50:51]\n"
  "v_rcp_f32 v0, v0\n"
  "v_rcp_f32 v1, v1\n"
  "v_rcp_f32 v2, v2\n"
  "v_rcp_f32 v3, v3\n"
  "v_pk_mul_f32 v[56:57], v[56:57], v[0:1]\n"
  "v_pk_mul_f32 v[58:59], v[58:59], v[2:3]\n"
  "s_add_u32 s88, s86, 0\n"
  "s_branch EQ12_%=\n"
  "EW12_%=:\n"
  "s_sleep 1\n"
  "s_memrealtime s[52:53]\n"
  "EQ12_%=:\n"
  "s_waitcnt lgkmcnt(0)\n"
  "s_sub_u32 s82, s52, s88\n"
  "s_cmp_lt_i32 s82, 0\n"
  "s_cbranch_scc1 EW12_%=\n"
  "s_mov_b32 s63, 0x240000\n"
  "buffer_store_dwordx4 v[56:59], v242, s[28:31], s63 offen offset:0 nt\n"
  "v_accvgpr_read_b32 v60, a50\n"
  "v_accvgpr_read_b32 v61, a54\n"
  "v_accvgpr_read_b32 v62, a58\n"
  "v_accvgpr_read_b32 v63, a62\n"
  "v_med3_f32 v4, v60, s64, v245\n"
  "v_med3_f32 v5, v61, s64, v245\n"
  "v_med3_f32 v6, v62, s64, v245\n"
  "v_med3_f32 v7, v63, s64, v245\n"
  "v_pk_mul_f32 v[4:5], v[4:5], s[56:57]\n"
  "v_pk_mul_f32 v[6:7], v[6:7], s[56:57]\n"
  "v_exp_f32 v4, v4\n"
  "v_exp_f32 v5, v5\n"
  "v_exp_f32 v6, v6\n"
  "v_exp_f32 v7, v7\n"
  "v_pk_add_f32 v[4:5], v[4:5], s[50:51]\n"
  "v_pk_add_f32 v[6:7], v[6:7], s[50:51]\n"
  "v_rcp_f32 v4, v4\n"
  "v_rcp_f32 v5, v5\n"
  "v_rcp_f32 v6, v6\n"
  "v_rcp_f32 v7, v7\n"
  "v_pk_mul_f32 v[60:61], v[60:61], v[4:5]\n"
  "v_pk_mul_f32 v[62:63], v[62:63], v[6:7]\n"
  "s_mov_b32 s63, 0x240000\n"
  "buffer_store_dwordx4 v[60:63], v242, s[28:31], s63 offen offset:256 nt\n"
  "v_accvgpr_read_b32 v64, a35\n"
  "v_accvgpr_read_b32 v65, a39\n"
  "v_accvgpr_read_b32 v66, a43\n"
  "v_accvgpr_read_b32 v67, a47\n"
  "v_med3_f32 v0, v64, s64, v245\n"
  "v_med3_f32 v1, v65, s64, v245\n"
  "v_med3_f32 v2, v66, s64, v245\n"
  "v_med3_f32 v3, v67, s64, v245\n"
  "v_pk_mul_f32 v[0:1], v[0:1], s[56:57]\n"
  "v_pk_mul_f32 v[2:3], v[2:3], s[56:57]\n"
  "v_exp_f32 v0, v0\n"
  "v_exp_f32 v1, v1\n"
  "v_exp_f32 v2, v2\n"
  "v_exp_f32 v3, v3\n"
  "v_pk_add_f32 v[0:1], v[0:1], s[50:51]\n"
  "v_pk_add_f32 v[2:3], v[2:3], s[50:51]\n"
  "v_rcp_f32 v0, v0\n"
  "v_rcp_f32 v1, v1\n"
  "v_rcp_f32 v2, v2\n"
  "v_rcp_f32 v3, v3\n"
  "v_pk_mul_f32 v[64:65], v[64:65], v[0:1]\n"
  "v_pk_mul_f32 v[66:67], v[66:67], v[2:3]\n"
  "s_mov_b32 s63, 0x260000\n"
  "buffer_store_dwordx4 v[64:67], v242, s[28:31], s63 offen offset:0 nt\n"
  "v_accvgpr_read_b32 v68, a51\n"
  "v_accvgpr_read_b32 v69, a55\n"
  "v_accvgpr_read_b32 v70, a59\n"
  "v_accvgpr_read_b32 v71, a63\n"
  "v_med3_f32 v4, v68, s64, v245\n"
  "v_med3_f32 v5, v69, s64, v245\n"
  "v_med3_f32 v6, v70, s64, v245\n"
  "v_med3_f32 v7, v71, s64, v245\n"
  "v_pk_mul_f32 v[4:5], v[4:5], s[56:57]\n"
  "v_pk_mul_f32 v[6:7], v[6:7], s[56:57]\n"
  "v_exp_f32 v4, v4\n"
  "v_exp_f32 v5, v5\n"
  "v_exp_f32 v6, v6\n"
  "v_exp_f32 v7, v7\n"
  "v_pk_add_f32 v[4:5], v[4:5], s[50:51]\n"
  "v_pk_add_f32 v[6:7], v[6:7], s[50:51]\n"
  "v_rcp_f32 v4, v4\n"
  "v_rcp_f32 v5, v5\n"
  "v_rcp_f32 v6, v6\n"
  "v_rcp_f32 v7, v7\n"
  "v_pk_mul_f32 v[68:69], v[68:69], v[4:5]\n"
  "v_pk_mul_f32 v[70:71], v[70:71], v[6:7]\n"
  "s_mov_b32 s63, 0x260000\n"
  "buffer_store_dwordx4 v[68:71], v242, s[28:31], s63 offen offset:256 nt\n"
  "s_memrealtime s[52:53]\n"
  "v_accvgpr_read_b32 v72, a64\n"
  "v_accvgpr_read_b32 v73, a68\n"
  "v_accvgpr_read_b32 v74, a72\n"
  "v_accvgpr_read_b32 v75, a76\n"
  "v_med3_f32 v0, v72, s64, v245\n"
  "v_med3_f32 v1, v73, s64, v245\n"
  "v_med3_f32 v2, v74, s64, v245\n"
  "v_med3_f32 v3, v75, s64, v245\n"
  "v_pk_mul_f32 v[0:1], v[0:1], s[56:57]\n"
  "v_pk_mul_f32 v[2:3], v[2:3], s[56:57]\n"
  "v_exp_f32 v0, v0\n"
  "v_exp_f32 v1, v1\n"
  "v_exp_f32 v2, v2\n"
  "v_exp_f32 v3, v3\n"
  "v_pk_add_f32 v[0:1], v[0:1], s[50:51]\n"
  "v_pk_add_f32 v[2:3], v[2:3], s[50:51]\n"
  "v_rcp_f32 v0, v0\n"
  "v_rcp_f32 v1, v1\n"
  "v_rcp_f32 v2, v2\n"
  "v_rcp_f32 v3, v3\n"
  "v_pk_mul_f32 v[72:73], v[72:73], v[0:1]\n"
  "v_pk_mul_f32 v[74:75], v[74:75], v[2:3]\n"
  "s_add_u32 s88, s86, 65\n"
  "s_branch EQ16_%=\n"
  "EW16_%=:\n"
  "s_sleep 1\n"
  "s_memrealtime s[52:53]\n"
  "EQ16_%=:\n"
  "s_waitcnt lgkmcnt(0)\n"
  "s_sub_u32 s82, s52, s88\n"
  "s_cmp_lt_i32 s82, 0\n"
  "s_cbranch_scc1 EW16_%=\n"
  "s_mov_b32 s63, 0x400000\n"
  "buffer_store_dwordx4 v[72:75], v242, s[28:31], s63 offen offset:0 nt\n"
  "v_accvgpr_read_b32 v76, a80\n"
  "v_accvgpr_read_b32 v77, a84\n"
  "v_accvgpr_read_b32 v78, a88\n"
  "v_accvgpr_read_b32 v79, a92\n"
  "v_med3_f32 v4, v76, s64, v245\n"
  "v_med3_f32 v5, v77, s64, v245\n"
  "v_med3_f32 v6, v78, s64, v245\n"
  "v_med3_f32 v7, v79, s64, v245\n"
  "v_pk_mul_f32 v[4:5], v[4:5], s[56:57]\n"
  "v_pk_mul_f32 v[6:7], v[6:7], s[56:57]\n"
  "v_exp_f32 v4, v4\n"
  "v_exp_f32 v5, v5\n"
  "v_exp_f32 v6, v6\n"
  "v_exp_f32 v7, v7\n"
  "v_pk_add_f32 v[4:5], v[4:5], s[50:51]\n"
  "v_pk_add_f32 v[6:7], v[6:7], s[50:51]\n"
  "v_rcp_f32 v4, v4\n"
  "v_rcp_f32 v5, v5\n"
  "v_rcp_f32 v6, v6\n"
  "v_rcp_f32 v7, v7\n"
  "v_pk_mul_f32 v[76:77], v[76:77], v[4:5]\n"
  "v_pk_mul_f32 v[78:79], v[78:79], v[6:7]\n"
  "s_mov_b32 s63, 0x400000\n"
  "buffer_store_dwordx4 v[76:79], v242, s[28:31], s63 offen offset:256 nt\n"
  "v_accvgpr_read_b32 v80, a65\n"
  "v_accvgpr_read_b32 v81, a69\n"
  "v_accvgpr_read_b32 v82, a73\n"
  "v_accvgpr_read_b32 v83, a77\n"
  "v_med3_f32 v0, v80, s64, v245\n"
  "v_med3_f32 v1, v81, s64, v245\n"
  "v_med3_f32 v2, v82, s64, v245\n"
  "v_med3_f32 v3, v83, s64, v245\n"
  "v_pk_mul_f32 v[0:1], v[0:1], s[56:57]\n"
  "v_pk_mul_f32 v[2:3], v[2:3], s[56:57]\n"
  "v_exp_f32 v0, v0\n"
  "v_exp_f32 v1, v1\n"
  "v_exp_f32 v2, v2\n"
  "v_exp_f32 v3, v3\n"
  "v_pk_add_f32 v[0:1], v[0:1], s[50:51]\n"
  "v_pk_add_f32 v[2:3], v[2:3], s[50:51]\n"
  "v_rcp_f32 v0, v0\n"
  "v_rcp_f32 v1, v1\n"
  "v_rcp_f32 v2, v2\n"
  "v_rcp_f32 v3, v3\n"
  "v_pk_mul_f32 v[80:81], v[80:81], v[0:1]\n"
  "v_pk_mul_f32 v[82:83], v[82:83], v[2:3]\n"
  "s_mov_b32 s63, 0x420000\n"
  "buffer_store_dwordx4 v[80:83], v242, s[28:31], s63 offen offset:0 nt\n"
  "v_accvgpr_read_b32 v84, a81\n"
  "v_accvgpr_read_b32 v85, a85\n"
  "v_accvgpr_read_b32 v86, a89\n"
  "v_accvgpr_read_b32 v87, a93\n"
  "v_med3_f32 v4, v84, s64, v245\n"
  "v_med3_f32 v5, v85, s64, v245\n"
  "v_med3_f32 v6, v86, s64, v245\n"
  "v_med3_f32 v7, v87, s64, v245\n"
  "v_pk_mul_f32 v[4:5], v[4:5], s[56:57]\n"
  "v_pk_mul_f32 v[6:7], v[6:7], s[56:57]\n"
  "v_exp_f32 v4, v4\n"
  "v_exp_f32 v5, v5\n"
  "v_exp_f32 v6, v6\n"
  "v_exp_f32 v7, v7\n"
  "v_pk_add_f32 v[4:5], v[4:5], s[50:51]\n"
  "v_pk_add_f32 v[6:7], v[6:7], s[50:51]\n"
  "v_rcp_f32 v4, v4\n"
  "v_rcp_f32 v5, v5\n"
  "v_rcp_f32 v6, v6\n"
  "v_rcp_f32 v7, v7\n"
  "v_pk_mul_f32 v[84:85], v[84:85], v[4:5]\n"
  "v_pk_mul_f32 v[86:87], v[86:87], v[6:7]\n"
  "s_mov_b32 s63, 0x420000\n"
  "buffer_store_dwordx4 v[84:87], v242, s[28:31], s63 offen offset:256 nt\n"
  "s_memrealtime s[52:53]\n"
  "v_accvgpr_read_b32 v88, a66\n"
  "v_accvgpr_read_b32 v89, a70\n"
  "v_accvgpr_read_b32 v90, a74\n"
  "v_accvgpr_read_b32 v91, a78\n"
  "v_med3_f32 v0, v88, s64, v245\n"
  "v_med3_f32 v1, v89, s64, v245\n"
  "v_med3_f32 v2, v90, s64, v245\n"
  "v_med3_f32 v3, v91, s64, v245\n"
  "v_pk_mul_f32 v[0:1], v[0:1], s[56:57]\n"
  "v_pk_mul_f32 v[2:3], v[2:3], s[56:57]\n"
  "v_exp_f32 v0, v0\n"
  "v_exp_f32 v1, v1\n"
  "v_exp_f32 v2, v2\n"
  "v_exp_f32 v3, v3\n"
  "v_pk_add_f32 v[0:1], v[0:1], s[50:51]\n"
  "v_pk_add_f32 v[2:3], v[2:3], s[50:51]\n"
  "v_rcp_f32 v0, v0\n"
  "v_rcp_f32 v1, v1\n"
  "v_rcp_f32 v2, v2\n"
  "v_rcp_f32 v3, v3\n"
  "v_pk_mul_f32 v[88:89], v[88:89], v[0:1]\n"
  "v_pk_mul_f32 v[90:91], v[90:91], v[2:3]\n"
  "s_add_u32 s88, s86, 130\n"
  "s_branch EQ20_%=\n"
  "EW20_%=:\n"
  "s_sleep 1\n"
  "s_memrealtime s[52:53]\n"
  "EQ20_%=:\n"
  "s_waitcnt lgkmcnt(0)\n"
  "s_sub_u32 s82, s52, s88\n"
  "s_cmp_lt_i32 s82, 0\n"
  "s_cbranch_scc1 EW20_%=\n"
  "s_mov_b32 s63, 0x440000\n"
  "buffer_store_dwordx4 v[88:91], v242, s[28:31], s63 offen offset:0 nt\n"
  "v_accvgpr_read_b32 v92, a82\n"
  "v_accvgpr_read_b32 v93, a86\n"
  "v_accvgpr_read_b32 v94, a90\n"
  "v_accvgpr_read_b32 v95, a94\n"
  "v_med3_f32 v4, v92, s64, v245\n"
  "v_med3_f32 v5, v93, s64, v245\n"
  "v_med3_f32 v6, v94, s64, v245\n"
  "v_med3_f32 v7, v95, s64, v245\n"
  "v_pk_mul_f32 v[4:5], v[4:5], s[56:57]\n"
  "v_pk_mul_f32 v[6:7], v[6:7], s[56:57]\n"
  "v_exp_f32 v4, v4\n"
  "v_exp_f32 v5, v5\n"
  "v_exp_f32 v6, v6\n"
  "v_exp_f32 v7, v7\n"
  "v_pk_add_f32 v[4:5], v[4:5], s[50:51]\n"
  "v_pk_add_f32 v[6:7], v[6:7], s[50:51]\n"
  "v_rcp_f32 v4, v4\n"
  "v_rcp_f32 v5, v5\n"
  "v_rcp_f32 v6, v6\n"
  "v_rcp_f32 v7, v7\n"
  "v_pk_mul_f32 v[92:93], v[92:93], v[4:5]\n"
  "v_pk_mul_f32 v[94:95], v[94:95], v[6:7]\n"
  "s_mov_b32 s63, 0x440000\n"
  "buffer_store_dwordx4 v[92:95], v242, s[28:31], s63 offen offset:256 nt\n"
  "v_accvgpr_read_b32 v96, a67\n"
  "v_accvgpr_read_b32 v97, a71\n"
  "v_accvgpr_read_b32 v98, a75\n"
  "v_accvgpr_read_b32 v99, a79\n"
  "v_med3_f32 v0, v96, s64, v245\n"
  "v_med3_f32 v1, v97, s64, v245\n"
  "v_med3_f32 v2, v98, s64, v245\n"
  "v_med3_f32 v3, v99, s64, v245\n"
  "v_pk_mul_f32 v[0:1], v[0:1], s[56:57]\n"
  "v_pk_mul_f32 v[2:3], v[2:3], s[56:57]\n"
  "v_exp_f32 v0, v0\n"
  "v_exp_f32 v1, v1\n"
  "v_exp_f32 v2, v2\n"
  "v_exp_f32 v3, v3\n"
  "v_pk_add_f32 v[0:1], v[0:1], s[50:51]\n"
  "v_pk_add_f32 v[2:3], v[2:3], s[50:51]\n"
  "v_rcp_f32 v0, v0\n"
  "v_rcp_f32 v1, v1\n"
  "v_rcp_f32 v2, v2\n"
  "v_rcp_f32 v3, v3\n"
  "v_pk_mul_f32 v[96:97], v[96:97], v[0:1]\n"
  "v_pk_mul_f32 v[98:99], v[98:99], v[2:3]\n"
  "s_mov_b32 s63, 0x460000\n"
  "buffer_store_dwordx4 v[96:99], v242, s[28:31], s63 offen offset:0 nt\n"
  "v_accvgpr_read_b32 v100, a83\n"
  "v_accvgpr_read_b32 v101, a87\n"
  "v_accvgpr_read_b32 v102, a91\n"
  "v_accvgpr_read_b32 v103, a95\n"
  "v_med3_f32 v4, v100, s64, v245\n"
  "v_med3_f32 v5, v101, s64, v245\n"
  "v_med3_f32 v6, v102, s64, v245\n"
  "v_med3_f32 v7, v103, s64, v245\n"
  "v_pk_mul_f32 v[4:5], v[4:5], s[56:57]\n"
  "v_pk_mul_f32 v[6:7], v[6:7], s[56:57]\n"
  "v_exp_f32 v4, v4\n"
  "v_exp_f32 v5, v5\n"
  "v_exp_f32 v6, v6\n"
  "v_exp_f32 v7, v7\n"
  "v_pk_add_f32 v[4:5], v[4:5], s[50:51]\n"
  "v_pk_add_f32 v[6:7], v[6:7], s[50:51]\n"
  "v_rcp_f32 v4, v4\n"
  "v_rcp_f32 v5, v5\n"
  "v_rcp_f32 v6, v6\n"
  "v_rcp_f32 v7, v7\n"
  "v_pk_mul_f32 v[100:101], v[100:101], v[4:5]\n"
  "v_pk_mul_f32 v[102:103], v[102:103], v[6:7]\n"
  "s_mov_b32 s63, 0x460000\n"
  "buffer_store_dwordx4 v[100:103], v242, s[28:31], s63 offen offset:256 nt\n"
  "s_memrealtime s[52:53]\n"
  "v_accvgpr_read_b32 v104, a96\n"
  "v_accvgpr_read_b32 v105, a100\n"
  "v_accvgpr_read_b32 v106, a104\n"
  "v_accvgpr_read_b32 v107, a108\n"
  "v_med3_f32 v0, v104, s64, v245\n"
  "v_med3_f32 v1, v105, s64, v245\n"
  "v_med3_f32 v2, v106, s64, v245\n"
  "v_med3_f32 v3, v107, s64, v245\n"
  "v_pk_mul_f32 v[0:1], v[0:1], s[56:57]\n"
  "v_pk_mul_f32 v[2:3], v[2:3], s[56:57]\n"
  "v_exp_f32 v0, v0\n"
  "v_exp_f32 v1, v1\n"
  "v_exp_f32 v2, v2\n"
  "v_exp_f32 v3, v3\n"
  "v_pk_add_f32 v[0:1], v[0:1], s[50:51]\n"
  "v_pk_add_f32 v[2:3], v[2:3], s[50:51]\n"
  "v_rcp_f32 v0, v0\n"
  "v_rcp_f32 v1, v1\n"
  "v_rcp_f32 v2, v2\n"
  "v_rcp_f32 v3, v3\n"
  "v_pk_mul_f32 v[104:105], v[104:105], v[0:1]\n"
  "v_pk_mul_f32 v[106:107], v[106:107], v[2:3]\n"
  "s_add_u32 s88, s86, 195\n"
  "s_branch EQ24_%=\n"
  "EW24_%=:\n"
  "s_sleep 1\n"
  "s_memrealtime s[52:53]\n"
  "EQ24_%=:\n"
  "s_waitcnt lgkmcnt(0)\n"
  "s_sub_u32 s82, s52, s88\n"
  "s_cmp_lt_i32 s82, 0\n"
  "s_cbranch_scc1 EW24_%=\n"
  "s_mov_b32 s63, 0x600000\n"
  "buffer_store_dwordx4 v[104:107], v242, s[28:31], s63 offen offset:0 nt\n"
  "v_accvgpr_read_b32 v108, a112\n"
  "v_accvgpr_read_b32 v109, a116\n"
  "v_accvgpr_read_b32 v110, a120\n"
  "v_accvgpr_read_b32 v111, a124\n"
  "v_med3_f32 v4, v108, s64, v245\n"
  "v_med3_f32 v5, v109, s64, v245\n"
  "v_med3_f32 v6, v110, s64, v245\n"
  "v_med3_f32 v7, v111, s64, v245\n"
  "v_pk_mul_f32 v[4:5], v[4:5], s[56:57]\n"
  "v_pk_mul_f32 v[6:7], v[6:7], s[56:57]\n"
  "v_exp_f32 v4, v4\n"
  "v_exp_f32 v5, v5\n"
  "v_exp_f32 v6, v6\n"
  "v_exp_f32 v7, v7\n"
  "v_pk_add_f32 v[4:5], v[4:5], s[50:51]\n"
  "v_pk_add_f32 v[6:7], v[6:7], s[50:51]\n"
  "v_rcp_f32 v4, v4\n"
  "v_rcp_f32 v5, v5\n"
  "v_rcp_f32 v6, v6\n"
  "v_rcp_f32 v7, v7\n"
  "v_pk_mul_f32 v[108:109], v[108:109], v[4:5]\n"
  "v_pk_mul_f32 v[110:111], v[110:111], v[6:7]\n"
  "s_mov_b32 s63, 0x600000\n"
  "buffer_store_dwordx4 v[108:111], v242, s[28:31], s63 offen offset:256 nt\n"
  "v_accvgpr_read_b32 v112, a97\n"
  "v_accvgpr_read_b32 v113, a101\n"
  "v_accvgpr_read_b32 v114, a105\n"
  "v_accvgpr_read_b32 v115, a109\n"
  "v_med3_f32 v0, v112, s64, v245\n"
  "v_med3_f32 v1, v113, s64, v245\n"
  "v_med3_f32 v2, v114, s64, v245\n"
  "v_med3_f32 v3, v115, s64, v245\n"
  "v_pk_mul_f32 v[0:1], v[0:1], s[56:57]\n"
  "v_pk_mul_f32 v[2:3], v[2:3], s[56:57]\n"
  "v_exp_f32 v0, v0\n"
  "v_exp_f32 v1, v1\n"
  "v_exp_f32 v2, v2\n"
  "v_exp_f32 v3, v3\n"
  "v_pk_add_f32 v[0:1], v[0:1], s[50:51]\n"
  "v_pk_add_f32 v[2:3], v[2:3], s[50:51]\n"
  "v_rcp_f32 v0, v0\n"
  "v_rcp_f32 v1, v1\n"
  "v_rcp_f32 v2, v2\n"
  "v_rcp_f32 v3, v3\n"
  "v_pk_mul_f32 v[112:113], v[112:113], v[0:1]\n"
  "v_pk_mul_f32 v[114:115], v[114:115], v[2:3]\n"
  "s_mov_b32 s63, 0x620000\n"
  "buffer_store_dwordx4 v[112:115], v242, s[28:31], s63 offen offset:0 nt\n"
  "v_accvgpr_read_b32 v116, a113\n"
  "v_accvgpr_read_b32 v117, a117\n"
  "v_accvgpr_read_b32 v118, a121\n"
  "v_accvgpr_read_b32 v119, a125\n"
  "v_med3_f32 v4, v116, s64, v245\n"
  "v_med3_f32 v5, v117, s64, v245\n"
  "v_med3_f32 v6, v118, s64, v245\n"
  "v_med3_f32 v7, v119, s64, v245\n"
  "v_pk_mul_f32 v[4:5], v[4:5], s[56:57]\n"
  "v_pk_mul_f32 v[6:7], v[6:7], s[56:57]\n"
  "v_exp_f32 v4, v4\n"
  "v_exp_f32 v5, v5\n"
  "v_exp_f32 v6, v6\n"
  "v_exp_f32 v7, v7\n"
  "v_pk_add_f32 v[4:5], v[4:5], s[50:51]\n"
  "v_pk_add_f32 v[6:7], v[6:7], s[50:51]\n"
  "v_rcp_f32 v4, v4\n"
  "v_rcp_f32 v5, v5\n"
  "v_rcp_f32 v6, v6\n"
  "v_rcp_f32 v7, v7\n"
  "v_pk_mul_f32 v[116:117], v[116:117], v[4:5]\n"
  "v_pk_mul_f32 v[118:119], v[118:119], v[6:7]\n"
  "s_mov_b32 s63, 0x620000\n"
  "buffer_store_dwordx4 v[116:119], v242, s[28:31], s63 offen offset:256 nt\n"
  "s_memrealtime s[52:53]\n"
  "v_accvgpr_read_b32 v8, a98\n"
  "v_accvgpr_read_b32 v9, a102\n"
  "v_accvgpr_read_b32 v10, a106\n"
  "v_accvgpr_read_b32 v11, a110\n"
  "v_med3_f32 v0, v8, s64, v245\n"
  "v_med3_f32 v1, v9, s64, v245\n"
  "v_med3_f32 v2, v10, s64, v245\n"
  "v_med3_f32 v3, v11, s64, v245\n"
  "v_pk_mul_f32 v[0:1], v[0:1], s[56:57]\n"
  "v_pk_mul_f32 v[2:3], v[2:3], s[56:57]\n"
  "v_exp_f32 v0, v0\n"
  "v_exp_f32 v1, v1\n"
  "v_exp_f32 v2, v2\n"
  "v_exp_f32 v3, v3\n"
  "v_pk_add_f32 v[0:1], v[0:1], s[50:51]\n"
  "v_pk_add_f32 v[2:3], v[2:3], s[50:51]\n"
  "v_rcp_f32 v0, v0\n"
  "v_rcp_f32 v1, v1\n"
  "v_rcp_f32 v2, v2\n"
  "v_rcp_f32 v3, v3\n"
  "v_pk_mul_f32 v[8:9], v[8:9], v[0:1]\n"
  "v_pk_mul_f32 v[10:11], v[10:11], v[2:3]\n"
  "s_add_u32 s88, s86, 260\n"
  "s_branch EQ28_%=\n"
  "EW28_%=:\n"
  "s_sleep 1\n"
  "s_memrealtime s[52:53]\n"
  "EQ28_%=:\n"
  "s_waitcnt lgkmcnt(0)\n"
  "s_sub_u32 s82, s52, s88\n"
  "s_cmp_lt_i32 s82, 0\n"
  "s_cbranch_scc1 EW28_%=\n"
  "s_mov_b32 s63, 0x640000\n"
  "buffer_store_dwordx4 v[8:11], v242, s[28:31], s63 offen offset:0 nt\n"
  "v_accvgpr_read_b32 v12, a114\n"
  "v_accvgpr_read_b32 v13, a118\n"
  "v_accvgpr_read_b32 v14, a122\n"
  "v_accvgpr_read_b32 v15, a126\n"
  "v_med3_f32 v4, v12, s64, v245\n"
  "v_med3_f32 v5, v13, s64, v245\n"
  "v_med3_f32 v6, v14, s64, v245\n"
  "v_med3_f32 v7, v15, s64, v245\n"
  "v_pk_mul_f32 v[4:5], v[4:5], s[56:57]\n"
  "v_pk_mul_f32 v[6:7], v[6:7], s[56:57]\n"
  "v_exp_f32 v4, v4\n"
  "v_exp_f32 v5, v5\n"
  "v_exp_f32 v6, v6\n"
  "v_exp_f32 v7, v7\n"
  "v_pk_add_f32 v[4:5], v[4:5], s[50:51]\n"
  "v_pk_add_f32 v[6:7], v[6:7], s[50:51]\n"
  "v_rcp_f32 v4, v4\n"
  "v_rcp_f32 v5, v5\n"
  "v_rcp_f32 v6, v6\n"
  "v_rcp_f32 v7, v7\n"
  "v_pk_mul_f32 v[12:13], v[12:13], v[4:5]\n"
  "v_pk_mul_f32 v[14:15], v[14:15], v[6:7]\n"
  "s_mov_b32 s63, 0x640000\n"
  "buffer_store_dwordx4 v[12:15], v242, s[28:31], s63 offen offset:256 nt\n"
  "v_accvgpr_read_b32 v16, a99\n"
  "v_accvgpr_read_b32 v17, a103\n"
  "v_accvgpr_read_b32 v18, a107\n"
  "v_accvgpr_read_b32 v19, a111\n"
  "v_med3_f32 v0, v16, s64, v245\n"
  "v_med3_f32 v1, v17, s64, v245\n"
  "v_med3_f32 v2, v18, s64, v245\n"
  "v_med3_f32 v3, v19, s64, v245\n"
  "v_pk_mul_f32 v[0:1], v[0:1], s[56:57]\n"
  "v_pk_mul_f32 v[2:3], v[2:3], s[56:57]\n"
  "v_exp_f32 v0, v0\n"
  "v_exp_f32 v1, v1\n"
  "v_exp_f32 v2, v2\n"
  "v_exp_f32 v3, v3\n"
  "v_pk_add_f32 v[0:1], v[0:1], s[50:51]\n"
  "v_pk_add_f32 v[2:3], v[2:3], s[50:51]\n"
  "v_rcp_f32 v0, v0\n"
  "v_rcp_f32 v1, v1\n"
  "v_rcp_f32 v2, v2\n"
  "v_rcp_f32 v3, v3\n"
  "v_pk_mul_f32 v[16:17], v[16:17], v[0:1]\n"
  "v_pk_mul_f32 v[18:19], v[18:19], v[2:3]\n"
  "s_mov_b32 s63, 0x660000\n"
  "buffer_store_dwordx4 v[16:19], v242, s[28:31], s63 offen offset:0 nt\n"
  "v_accvgpr_read_b32 v20, a115\n"
  "v_accvgpr_read_b32 v21, a119\n"
  "v_accvgpr_read_b32 v22, a123\n"
  "v_accvgpr_read_b32 v23, a127\n"
  "v_med3_f32 v4, v20, s64, v245\n"
  "v_med3_f32 v5, v21, s64, v245\n"
  "v_med3_f32 v6, v22, s64, v245\n"
  "v_med3_f32 v7, v23, s64, v245\n"
  "v_pk_mul_f32 v[4:5], v[4:5], s[56:57]\n"
  "v_pk_mul_f32 v[6:7], v[6:7], s[56:57]\n"
  "v_exp_f32 v4, v4\n"
  "v_exp_f32 v5, v5\n"
  "v_exp_f32 v6, v6\n"
  "v_exp_f32 v7, v7\n"
  "v_pk_add_f32 v[4:5], v[4:5], s[50:51]\n"
  "v_pk_add_f32 v[6:7], v[6:7], s[50:51]\n"
  "v_rcp_f32 v4, v4\n"
  "v_rcp_f32 v5, v5\n"
  "v_rcp_f32 v6, v6\n"
  "v_rcp_f32 v7, v7\n"
  "v_pk_mul_f32 v[20:21], v[20:21], v[4:5]\n"
  "v_pk_mul_f32 v[22:23], v[22:23], v[6:7]\n"
  "s_mov_b32 s63, 0x660000\n"
  "buffer_store_dwordx4 v[20:23], v242, s[28:31], s63 offen offset:256 nt\n"
  "s_memrealtime s[52:53]\n"
  "v_accvgpr_read_b32 v24, a128\n"
  "v_accvgpr_read_b32 v25, a132\n"
  "v_accvgpr_read_b32 v26, a136\n"
  "v_accvgpr_read_b32 v27, a140\n"
  "v_med3_f32 v0, v24, s64, v245\n"
  "v_med3_f32 v1, v25, s64, v245\n"
  "v_med3_f32 v2, v26, s64, v245\n"
  "v_med3_f32 v3, v27, s64, v245\n"
  "v_pk_mul_f32 v[0:1], v[0:1], s[56:57]\n"
  "v_pk_mul_f32 v[2:3], v[2:3], s[56:57]\n"
  "v_exp_f32 v0, v0\n"
  "v_exp_f32 v1, v1\n"
  "v_exp_f32 v2, v2\n"
  "v_exp_f32 v3, v3\n"
  "v_pk_add_f32 v[0:1], v[0:1], s[50:51]\n"
  "v_pk_add_f32 v[2:3], v[2:3], s[50:51]\n"
  "v_rcp_f32 v0, v0\n"
  "v_rcp_f32 v1, v1\n"
  "v_rcp_f32 v2, v2\n"
  "v_rcp_f32 v3, v3\n"
  "v_pk_mul_f32 v[24:25], v[24:25], v[0:1]\n"
  "v_pk_mul_f32 v[26:27], v[26:27], v[2:3]\n"
  "s_add_u32 s88, s86, 325\n"
  "s_branch EQ32_%=\n"
  "EW32_%=:\n"
  "s_sleep 1\n"
  "s_memrealtime s[52:53]\n"
  "EQ32_%=:\n"
  "s_waitcnt lgkmcnt(0)\n"
  "s_sub_u32 s82, s52, s88\n"
  "s_cmp_lt_i32 s82, 0\n"
  "s_cbranch_scc1 EW32_%=\n"
  "s_mov_b32 s63, 0x800000\n"
  "buffer_store_dwordx4 v[24:27], v242, s[28:31], s63 offen offset:0 nt\n"
  "v_accvgpr_read_b32 v28, a144\n"
  "v_accvgpr_read_b32 v29, a148\n"
  "v_accvgpr_read_b32 v30, a152\n"
  "v_accvgpr_read_b32 v31, a156\n"
  "v_med3_f32 v4, v28, s64, v245\n"
  "v_med3_f32 v5, v29, s64, v245\n"
  "v_med3_f32 v6, v30, s64, v245\n"
  "v_med3_f32 v7, v31, s64, v245\n"
  "v_pk_mul_f32 v[4:5], v[4:5], s[56:57]\n"
  "v_pk_mul_f32 v[6:7], v[6:7], s[56:57]\n"
  "v_exp_f32 v4, v4\n"
  "v_exp_f32 v5, v5\n"
  "v_exp_f32 v6, v6\n"
  "v_exp_f32 v7, v7\n"
  "v_pk_add_f32 v[4:5], v[4:5], s[50:51]\n"
  "v_pk_add_f32 v[6:7], v[6:7], s[50:51]\n"
  "v_rcp_f32 v4, v4\n"
  "v_rcp_f32 v5, v5\n"
  "v_rcp_f32 v6, v6\n"
  "v_rcp_f32 v7, v7\n"
  "v_pk_mul_f32 v[28:29], v[28:29], v[4:5]\n"
  "v_pk_mul_f32 v[30:31], v[30:31], v[6:7]\n"
  "s_mov_b32 s63, 0x800000\n"
  "buffer_store_dwordx4 v[28:31], v242, s[28:31], s63 offen offset:256 nt\n"
  "v_accvgpr_read_b32 v32, a129\n"
  "v_accvgpr_read_b32 v33, a133\n"
  "v_accvgpr_read_b32 v34, a137\n"
  "v_accvgpr_read_b32 v35, a141\n"
  "v_med3_f32 v0, v32, s64, v245\n"
  "v_med3_f32 v1, v33, s64, v245\n"
  "v_med3_f32 v2, v34, s64, v245\n"
  "v_med3_f32 v3, v35, s64, v245\n"
  "v_pk_mul_f32 v[0:1], v[0:1], s[56:57]\n"
  "v_pk_mul_f32 v[2:3], v[2:3], s[56:57]\n"
  "v_exp_f32 v0, v0\n"
  "v_exp_f32 v1, v1\n"
  "v_exp_f32 v2, v2\n"
  "v_exp_f32 v3, v3\n"
  "v_pk_add_f32 v[0:1], v[0:1], s[50:51]\n"
  "v_pk_add_f32 v[2:3], v[2:3], s[50:51]\n"
  "v_rcp_f32 v0, v0\n"
  "v_rcp_f32 v1, v1\n"
  "v_rcp_f32 v2, v2\n"
  "v_rcp_f32 v3, v3\n"
  "v_pk_mul_f32 v[32:33], v[32:33], v[0:1]\n"
  "v_pk_mul_f32 v[34:35], v[34:35], v[2:3]\n"
  "s_mov_b32 s63, 0x820000\n"
  "buffer_store_dwordx4 v[32:35], v242, s[28:31], s63 offen offset:0 nt\n"
  "v_accvgpr_read_b32 v36, a145\n"
  "v_accvgpr_read_b32 v37, a149\n"
  "v_accvgpr_read_b32 v38, a153\n"
  "v_accvgpr_read_b32 v39, a157\n"
  "v_med3_f32 v4, v36, s64, v245\n"
  "v_med3_f32 v5, v37, s64, v245\n"
  "v_med3_f32 v6, v38, s64, v245\n"
  "v_med3_f32 v7, v39, s64, v245\n"
  "v_pk_mul_f32 v[4:5], v[4:5], s[56:57]\n"
  "v_pk_mul_f32 v[6:7], v[6:7], s[56:57]\n"
  "v_exp_f32 v4, v4\n"
  "v_exp_f32 v5, v5\n"
  "v_exp_f32 v6, v6\n"
  "v_exp_f32 v7, v7\n"
  "v_pk_add_f32 v[4:5], v[4:5], s[50:51]\n"
  "v_pk_add_f32 v[6:7], v[6:7], s[50:51]\n"
  "v_rcp_f32 v4, v4\n"
  "v_rcp_f32 v5, v5\n"
  "v_rcp_f32 v6, v6\n"
  "v_rcp_f32 v7, v7\n"
  "v_pk_mul_f32 v[36:37], v[36:37], v[4:5]\n"
  "v_pk_mul_f32 v[38:39], v[38:39], v[6:7]\n"
  "s_mov_b32 s63, 0x820000\n"
  "buffer_store_dwordx4 v[36:39], v242, s[28:31], s63 offen offset:256 nt\n"
  "s_memrealtime s[52:53]\n"
  "v_accvgpr_read_b32 v40, a130\n"
  "v_accvgpr_read_b32 v41, a134\n"
  "v_accvgpr_read_b32 v42, a138\n"
  "v_accvgpr_read_b32 v43, a142\n"
  "v_med3_f32 v0, v40, s64, v245\n"
  "v_med3_f32 v1, v41, s64, v245\n"
  "v_med3_f32 v2, v42, s64, v245\n"
  "v_med3_f32 v3, v43, s64, v245\n"
  "v_pk_mul_f32 v[0:1], v[0:1], s[56:57]\n"
  "v_pk_mul_f32 v[2:3], v[2:3], s[56:57]\n"
  "v_exp_f32 v0, v0\n"
  "v_exp_f32 v1, v1\n"
  "v_exp_f32 v2, v2\n"
  "v_exp_f32 v3, v3\n"
  "v_pk_add_f32 v[0:1], v[0:1], s[50:51]\n"
  "v_pk_add_f32 v[2:3], v[2:3], s[50:51]\n"
  "v_rcp_f32 v0, v0\n"
  "v_rcp_f32 v1, v1\n"
  "v_rcp_f32 v2, v2\n"
  "v_rcp_f32 v3, v3\n"
  "v_pk_mul_f32 v[40:41], v[40:41], v[0:1]\n"
  "v_pk_mul_f32 v[42:43], v[42:43], v[2:3]\n"
  "s_add_u32 s88, s86, 390\n"
  "s_branch EQ36_%=\n"
  "EW36_%=:\n"
  "s_sleep 1\n"
  "s_memrealtime s[52:53]\n"
  "EQ36_%=:\n"
  "s_waitcnt lgkmcnt(0)\n"
  "s_sub_u32 s82, s52, s88\n"
  "s_cmp_lt_i32 s82, 0\n"
  "s_cbranch_scc1 EW36_%=\n"
  "s_mov_b32 s63, 0x840000\n"
  "buffer_store_dwordx4 v[40:43], v242, s[28:31], s63 offen offset:0 nt\n"
  "v_accvgpr_read_b32 v44, a146\n"
  "v_accvgpr_read_b32 v45, a150\n"
  "v_accvgpr_read_b32 v46, a154\n"
  "v_accvgpr_read_b32 v47, a158\n"
  "v_med3_f32 v4, v44, s64, v245\n"
  "v_med3_f32 v5, v45, s64, v245\n"
  "v_med3_f32 v6, v46, s64, v245\n"
  "v_med3_f32 v7, v47, s64, v245\n"
  "v_pk_mul_f32 v[4:5], v[4:5], s[56:57]\n"
  "v_pk_mul_f32 v[6:7], v[6:7], s[56:57]\n"
  "v_exp_f32 v4, v4\n"
  "v_exp_f32 v5, v5\n"
  "v_exp_f32 v6, v6\n"
  "v_exp_f32 v7, v7\n"
  "v_pk_add_f32 v[4:5], v[4:5], s[50:51]\n"
  "v_pk_add_f32 v[6:7], v[6:7], s[50:51]\n"
  "v_rcp_f32 v4, v4\n"
  "v_rcp_f32 v5, v5\n"
  "v_rcp_f32 v6, v6\n"
  "v_rcp_f32 v7, v7\n"
  "v_pk_mul_f32 v[44:45], v[44:45], v[4:5]\n"
  "v_pk_mul_f32 v[46:47], v[46:47], v[6:7]\n"
  "s_mov_b32 s63, 0x840000\n"
  "buffer_store_dwordx4 v[44:47], v242, s[28:31], s63 offen offset:256 nt\n"
  "v_accvgpr_read_b32 v48, a131\n"
  "v_accvgpr_read_b32 v49, a135\n"
  "v_accvgpr_read_b32 v50, a139\n"
  "v_accvgpr_read_b32 v51, a143\n"
  "v_med3_f32 v0, v48, s64, v245\n"
  "v_med3_f32 v1, v49, s64, v245\n"
  "v_med3_f32 v2, v50, s64, v245\n"
  "v_med3_f32 v3, v51, s64, v245\n"
  "v_pk_mul_f32 v[0:1], v[0:1], s[56:57]\n"
  "v_pk_mul_f32 v[2:3], v[2:3], s[56:57]\n"
  "v_exp_f32 v0, v0\n"
  "v_exp_f32 v1, v1\n"
  "v_exp_f32 v2, v2\n"
  "v_exp_f32 v3, v3\n"
  "v_pk_add_f32 v[0:1], v[0:1], s[50:51]\n"
  "v_pk_add_f32 v[2:3], v[2:3], s[50:51]\n"
  "v_rcp_f32 v0, v0\n"
  "v_rcp_f32 v1, v1\n"
  "v_rcp_f32 v2, v2\n"
  "v_rcp_f32 v3, v3\n"
  "v_pk_mul_f32 v[48:49], v[48:49], v[0:1]\n"
  "v_pk_mul_f32 v[50:51], v[50:51], v[2:3]\n"
  "s_mov_b32 s63, 0x860000\n"
  "buffer_store_dwordx4 v[48:51], v242, s[28:31], s63 offen offset:0 nt\n"
  "v_accvgpr_read_b32 v52, a147\n"
  "v_accvgpr_read_b32 v53, a151\n"
  "v_accvgpr_read_b32 v54, a155\n"
  "v_accvgpr_read_b32 v55, a159\n"
  "v_med3_f32 v4, v52, s64, v245\n"
  "v_med3_f32 v5, v53, s64, v245\n"
  "v_med3_f32 v6, v54, s64, v245\n"
  "v_med3_f32 v7, v55, s64, v245\n"
  "v_pk_mul_f32 v[4:5], v[4:5], s[56:57]\n"
  "v_pk_mul_f32 v[6:7], v[6:7], s[56:57]\n"
  "v_exp_f32 v4, v4\n"
  "v_exp_f32 v5, v5\n"
  "v_exp_f32 v6, v6\n"
  "v_exp_f32 v7, v7\n"
  "v_pk_add_f32 v[4:5], v[4:5], s[50:51]\n"
  "v_pk_add_f32 v[6:7], v[6:7], s[50:51]\n"
  "v_rcp_f32 v4, v4\n"
  "v_rcp_f32 v5, v5\n"
  "v_rcp_f32 v6, v6\n"
  "v_rcp_f32 v7, v7\n"
  "v_pk_mul_f32 v[52:53], v[52:53], v[4:5]\n"
  "v_pk_mul_f32 v[54:55], v[54:55], v[6:7]\n"
  "s_mov_b32 s63, 0x860000\n"
  "buffer_store_dwordx4 v[52:55], v242, s[28:31], s63 offen offset:256 nt\n"
  "s_memrealtime s[52:53]\n"
  "v_accvgpr_read_b32 v56, a160\n"
  "v_accvgpr_read_b32 v57, a164\n"
  "v_accvgpr_read_b32 v58, a168\n"
  "v_accvgpr_read_b32 v59, a172\n"
  "v_med3_f32 v0, v56, s64, v245\n"
  "v_med3_f32 v1, v57, s64, v245\n"
  "v_med3_f32 v2, v58, s64, v245\n"
  "v_med3_f32 v3, v59, s64, v245\n"
  "v_pk_mul_f32 v[0:1], v[0:1], s[56:57]\n"
  "v_pk_mul_f32 v[2:3], v[2:3], s[56:57]\n"
  "v_exp_f32 v0, v0\n"
  "v_exp_f32 v1, v1\n"
  "v_exp_f32 v2, v2\n"
  "v_exp_f32 v3, v3\n"
  "v_pk_add_f32 v[0:1], v[0:1], s[50:51]\n"
  "v_pk_add_f32 v[2:3], v[2:3], s[50:51]\n"
  "v_rcp_f32 v0, v0\n"
  "v_rcp_f32 v1, v1\n"
  "v_rcp_f32 v2, v2\n"
  "v_rcp_f32 v3, v3\n"
  "v_pk_mul_f32 v[56:57], v[56:57], v[0:1]\n"
  "v_pk_mul_f32 v[58:59], v[58:59], v[2:3]\n"
  "s_add_u32 s88, s86, 455\n"
  "s_branch EQ40_%=\n"
  "EW40_%=:\n"
  "s_sleep 1\n"
  "s_memrealtime s[52:53]\n"
  "EQ40_%=:\n"
  "s_waitcnt lgkmcnt(0)\n"
  "s_sub_u32 s82, s52, s88\n"
  "s_cmp_lt_i32 s82, 0\n"
  "s_cbranch_scc1 EW40_%=\n"
  "s_mov_b32 s63, 0xa00000\n"
  "buffer_store_dwordx4 v[56:59], v242, s[28:31], s63 offen offset:0 nt\n"
  "v_accvgpr_read_b32 v60, a176\n"
  "v_accvgpr_read_b32 v61, a180\n"
  "v_accvgpr_read_b32 v62, a184\n"
  "v_accvgpr_read_b32 v63, a188\n"
  "v_med3_f32 v4, v60, s64, v245\n"
  "v_med3_f32 v5, v61, s64, v245\n"
  "v_med3_f32 v6, v62, s64, v245\n"
  "v_med3_f32 v7, v63, s64, v245\n"
  "v_pk_mul_f32 v[4:5], v[4:5], s[56:57]\n"
  "v_pk_mul_f32 v[6:7], v[6:7], s[56:57]\n"
  "v_exp_f32 v4, v4\n"
  "v_exp_f32 v5, v5\n"
  "v_exp_f32 v6, v6\n"
  "v_exp_f32 v7, v7\n"
  "v_pk_add_f32 v[4:5], v[4:5], s[50:51]\n"
  "v_pk_add_f32 v[6:7], v[6:7], s[50:51]\n"
  "v_rcp_f32 v4, v4\n"
  "v_rcp_f32 v5, v5\n"
  "v_rcp_f32 v6, v6\n"
  "v_rcp_f32 v7, v7\n"
  "v_pk_mul_f32 v[60:61], v[60:61], v[4:5]\n"
  "v_pk_mul_f32 v[62:63], v[62:63], v[6:7]\n"
  "s_mov_b32 s63, 0xa00000\n"
  "buffer_store_dwordx4 v[60:63], v242, s[28:31], s63 offen offset:256 nt\n"
  "v_accvgpr_read_b32 v64, a161\n"
  "v_accvgpr_read_b32 v65, a165\n"
  "v_accvgpr_read_b32 v66, a169\n"
  "v_accvgpr_read_b32 v67, a173\n"
  "v_med3_f32 v0, v64, s64, v245\n"
  "v_med3_f32 v1, v65, s64, v245\n"
  "v_med3_f32 v2, v66, s64, v245\n"
  "v_med3_f32 v3, v67, s64, v245\n"
  "v_pk_mul_f32 v[0:1], v[0:1], s[56:57]\n"
  "v_pk_mul_f32 v[2:3], v[2:3], s[56:57]\n"
  "v_exp_f32 v0, v0\n"
  "v_exp_f32 v1, v1\n"
  "v_exp_f32 v2, v2\n"
  "v_exp_f32 v3, v3\n"
  "v_pk_add_f32 v[0:1], v[0:1], s[50:51]\n"
  "v_pk_add_f32 v[2:3], v[2:3], s[50:51]\n"
  "v_rcp_f32 v0, v0\n"
  "v_rcp_f32 v1, v1\n"
  "v_rcp_f32 v2, v2\n"
  "v_rcp_f32 v3, v3\n"
  "v_pk_mul_f32 v[64:65], v[64:65], v[0:1]\n"
  "v_pk_mul_f32 v[66:67], v[66:67], v[2:3]\n"
  "s_mov_b32 s63, 0xa20000\n"
  "buffer_store_dwordx4 v[64:67], v242, s[28:31], s63 offen offset:0 nt\n"
  "v_accvgpr_read_b32 v68, a177\n"
  "v_accvgpr_read_b32 v69, a181\n"
  "v_accvgpr_read_b32 v70, a185\n"
  "v_accvgpr_read_b32 v71, a189\n"
  "v_med3_f32 v4, v68, s64, v245\n"
  "v_med3_f32 v5, v69, s64, v245\n"
  "v_med3_f32 v6, v70, s64, v245\n"
  "v_med3_f32 v7, v71, s64, v245\n"
  "v_pk_mul_f32 v[4:5], v[4:5], s[56:57]\n"
  "v_pk_mul_f32 v[6:7], v[6:7], s[56:57]\n"
  "v_exp_f32 v4, v4\n"
  "v_exp_f32 v5, v5\n"
  "v_exp_f32 v6, v6\n"
  "v_exp_f32 v7, v7\n"
  "v_pk_add_f32 v[4:5], v[4:5], s[50:51]\n"
  "v_pk_add_f32 v[6:7], v[6:7], s[50:51]\n"
  "v_rcp_f32 v4, v4\n"
  "v_rcp_f32 v5, v5\n"
  "v_rcp_f32 v6, v6\n"
  "v_rcp_f32 v7, v7\n"
  "v_pk_mul_f32 v[68:69], v[68:69], v[4:5]\n"
  "v_pk_mul_f32 v[70:71], v[70:71], v[6:7]\n"
  "s_mov_b32 s63, 0xa20000\n"
  "buffer_store_dwordx4 v[68:71], v242, s[28:31], s63 offen offset:256 nt\n"
  "s_memrealtime s[52:53]\n"
  "v_accvgpr_read_b32 v72, a162\n"
  "v_accvgpr_read_b32 v73, a166\n"
  "v_accvgpr_read_b32 v74, a170\n"
  "v_accvgpr_read_b32 v75, a174\n"
  "v_med3_f32 v0, v72, s64, v245\n"
  "v_med3_f32 v1, v73, s64, v245\n"
  "v_med3_f32 v2, v74, s64, v245\n"
  "v_med3_f32 v3, v75, s64, v245\n"
  "v_pk_mul_f32 v[0:1], v[0:1], s[56:57]\n"
  "v_pk_mul_f32 v[2:3], v[2:3], s[56:57]\n"
  "v_exp_f32 v0, v0\n"
  "v_exp_f32 v1, v1\n"
  "v_exp_f32 v2, v2\n"
  "v_exp_f32 v3, v3\n"
  "v_pk_add_f32 v[0:1], v[0:1], s[50:51]\n"
  "v_pk_add_f32 v[2:3], v[2:3], s[50:51]\n"
  "v_rcp_f32 v0, v0\n"
  "v_rcp_f32 v1, v1\n"
  "v_rcp_f32 v2, v2\n"
  "v_rcp_f32 v3, v3\n"
  "v_pk_mul_f32 v[72:73], v[72:73], v[0:1]\n"
  "v_pk_mul_f32 v[74:75], v[74:75], v[2:3]\n"
  "s_add_u32 s88, s86, 520\n"
  "s_branch EQ44_%=\n"
  "EW44_%=:\n"
  "s_sleep 1\n"
  "s_memrealtime s[52:53]\n"
  "EQ44_%=:\n"
  "s_waitcnt lgkmcnt(0)\n"
  "s_sub_u32 s82, s52, s88\n"
  "s_cmp_lt_i32 s82, 0\n"
  "s_cbranch_scc1 EW44_%=\n"
  "s_mov_b32 s63, 0xa40000\n"
  "buffer_store_dwordx4 v[72:75], v242, s[28:31], s63 offen offset:0 nt\n"
  "v_accvgpr_read_b32 v76, a178\n"
  "v_accvgpr_read_b32 v77, a182\n"
  "v_accvgpr_read_b32 v78, a186\n"
  "v_accvgpr_read_b32 v79, a190\n"
  "v_med3_f32 v4, v76, s64, v245\n"
  "v_med3_f32 v5, v77, s64, v245\n"
  "v_med3_f32 v6, v78, s64, v245\n"
  "v_med3_f32 v7, v79, s64, v245\n"
  "v_pk_mul_f32 v[4:5], v[4:5], s[56:57]\n"
  "v_pk_mul_f32 v[6:7], v[6:7], s[56:57]\n"
  "v_exp_f32 v4, v4\n"
  "v_exp_f32 v5, v5\n"
  "v_exp_f32 v6, v6\n"
  "v_exp_f32 v7, v7\n"
  "v_pk_add_f32 v[4:5], v[4:5], s[50:51]\n"
  "v_pk_add_f32 v[6:7], v[6:7], s[50:51]\n"
  "v_rcp_f32 v4, v4\n"
  "v_rcp_f32 v5, v5\n"
  "v_rcp_f32 v6, v6\n"
  "v_rcp_f32 v7, v7\n"
  "v_pk_mul_f32 v[76:77], v[76:77], v[4:5]\n"
  "v_pk_mul_f32 v[78:79], v[78:79], v[6:7]\n"
  "s_mov_b32 s63, 0xa40000\n"
  "buffer_store_dwordx4 v[76:79], v242, s[28:31], s63 offen offset:256 nt\n"
  "v_accvgpr_read_b32 v80, a163\n"
  "v_accvgpr_read_b32 v81, a167\n"
  "v_accvgpr_read_b32 v82, a171\n"
  "v_accvgpr_read_b32 v83, a175\n"
  "v_med3_f32 v0, v80, s64, v245\n"
  "v_med3_f32 v1, v81, s64, v245\n"
  "v_med3_f32 v2, v82, s64, v245\n"
  "v_med3_f32 v3, v83, s64, v245\n"
  "v_pk_mul_f32 v[0:1], v[0:1], s[56:57]\n"
  "v_pk_mul_f32 v[2:3], v[2:3], s[56:57]\n"
  "v_exp_f32 v0, v0\n"
  "v_exp_f32 v1, v1\n"
  "v_exp_f32 v2, v2\n"
  "v_exp_f32 v3, v3\n"
  "v_pk_add_f32 v[0:1], v[0:1], s[50:51]\n"
  "v_pk_add_f32 v[2:3], v[2:3], s[50:51]\n"
  "v_rcp_f32 v0, v0\n"
  "v_rcp_f32 v1, v1\n"
  "v_rcp_f32 v2, v2\n"
  "v_rcp_f32 v3, v3\n"
  "v_pk_mul_f32 v[80:81], v[80:81], v[0:1]\n"
  "v_pk_mul_f32 v[82:83], v[82:83], v[2:3]\n"
  "s_mov_b32 s63, 0xa60000\n"
  "buffer_store_dwordx4 v[80:83], v242, s[28:31], s63 offen offset:0 nt\n"
  "v_accvgpr_read_b32 v84, a179\n"
  "v_accvgpr_read_b32 v85, a183\n"
  "v_accvgpr_read_b32 v86, a187\n"
  "v_accvgpr_read_b32 v87, a191\n"
  "v_med3_f32 v4, v84, s64, v245\n"
  "v_med3_f32 v5, v85, s64, v245\n"
  "v_med3_f32 v6, v86, s64, v245\n"
  "v_med3_f32 v7, v87, s64, v245\n"
  "v_pk_mul_f32 v[4:5], v[4:5], s[56:57]\n"
  "v_pk_mul_f32 v[6:7], v[6:7], s[56:57]\n"
  "v_exp_f32 v4, v4\n"
  "v_exp_f32 v5, v5\n"
  "v_exp_f32 v6, v6\n"
  "v_exp_f32 v7, v7\n"
  "v_pk_add_f32 v[4:5], v[4:5], s[50:51]\n"
  "v_pk_add_f32 v[6:7], v[6:7], s[50:51]\n"
  "v_rcp_f32 v4, v4\n"
  "v_rcp_f32 v5, v5\n"
  "v_rcp_f32 v6, v6\n"
  "v_rcp_f32 v7, v7\n"
  "v_pk_mul_f32 v[84:85], v[84:85], v[4:5]\n"
  "v_pk_mul_f32 v[86:87], v[86:87], v[6:7]\n"
  "s_mov_b32 s63, 0xa60000\n"
  "buffer_store_dwordx4 v[84:87], v242, s[28:31], s63 offen offset:256 nt\n"
  "s_memrealtime s[52:53]\n"
  "v_accvgpr_read_b32 v88, a192\n"
  "v_accvgpr_read_b32 v89, a196\n"
  "v_accvgpr_read_b32 v90, a200\n"
  "v_accvgpr_read_b32 v91, a204\n"
  "v_med3_f32 v0, v88, s64, v245\n"
  "v_med3_f32 v1, v89, s64, v245\n"
  "v_med3_f32 v2, v90, s64, v245\n"
  "v_med3_f32 v3, v91, s64, v245\n"
  "v_pk_mul_f32 v[0:1], v[0:1], s[56:57]\n"
  "v_pk_mul_f32 v[2:3], v[2:3], s[56:57]\n"
  "v_exp_f32 v0, v0\n"
  "v_exp_f32 v1, v1\n"
  "v_exp_f32 v2, v2\n"
  "v_exp_f32 v3, v3\n"
  "v_pk_add_f32 v[0:1], v[0:1], s[50:51]\n"
  "v_pk_add_f32 v[2:3], v[2:3], s[50:51]\n"
  "v_rcp_f32 v0, v0\n"
  "v_rcp_f32 v1, v1\n"
  "v_rcp_f32 v2, v2\n"
  "v_rcp_f32 v3, v3\n"
  "v_pk_mul_f32 v[88:89], v[88:89], v[0:1]\n"
  "v_pk_mul_f32 v[90:91], v[90:91], v[2:3]\n"
  "s_add_u32 s88, s86, 585\n"
  "s_branch EQ48_%=\n"
  "EW48_%=:\n"
  "s_sleep 1\n"
  "s_memrealtime s[52:53]\n"
  "EQ48_%=:\n"
  "s_waitcnt lgkmcnt(0)\n"
  "s_sub_u32 s82, s52, s88\n"
  "s_cmp_lt_i32 s82, 0\n"
  "s_cbranch_scc1 EW48_%=\n"
  "s_mov_b32 s63, 0xc00000\n"
  "buffer_store_dwordx4 v[88:91], v242, s[28:31], s63 offen offset:0 nt\n"
  "v_accvgpr_read_b32 v92, a208\n"
  "v_accvgpr_read_b32 v93, a212\n"
  "v_accvgpr_read_b32 v94, a216\n"
  "v_accvgpr_read_b32 v95, a220\n"
  "v_med3_f32 v4, v92, s64, v245\n"
  "v_med3_f32 v5, v93, s64, v245\n"
  "v_med3_f32 v6, v94, s64, v245\n"
  "v_med3_f32 v7, v95, s64, v245\n"
  "v_pk_mul_f32 v[4:5], v[4:5], s[56:57]\n"
  "v_pk_mul_f32 v[6:7], v[6:7], s[56:57]\n"
  "v_exp_f32 v4, v4\n"
  "v_exp_f32 v5, v5\n"
  "v_exp_f32 v6, v6\n"
  "v_exp_f32 v7, v7\n"
  "v_pk_add_f32 v[4:5], v[4:5], s[50:51]\n"
  "v_pk_add_f32 v[6:7], v[6:7], s[50:51]\n"
  "v_rcp_f32 v4, v4\n"
  "v_rcp_f32 v5, v5\n"
  "v_rcp_f32 v6, v6\n"
  "v_rcp_f32 v7, v7\n"
  "v_pk_mul_f32 v[92:93], v[92:93], v[4:5]\n"
  "v_pk_mul_f32 v[94:95], v[94:95], v[6:7]\n"
  "s_mov_b32 s63, 0xc00000\n"
  "buffer_store_dwordx4 v[92:95], v242, s[28:31], s63 offen offset:256 nt\n"
  "v_accvgpr_read_b32 v96, a193\n"
  "v_accvgpr_read_b32 v97, a197\n"
  "v_accvgpr_read_b32 v98, a201\n"
  "v_accvgpr_read_b32 v99, a205\n"
  "v_med3_f32 v0, v96, s64, v245\n"
  "v_med3_f32 v1, v97, s64, v245\n"
  "v_med3_f32 v2, v98, s64, v245\n"
  "v_med3_f32 v3, v99, s64, v245\n"
  "v_pk_mul_f32 v[0:1], v[0:1], s[56:57]\n"
  "v_pk_mul_f32 v[2:3], v[2:3], s[56:57]\n"
  "v_exp_f32 v0, v0\n"
  "v_exp_f32 v1, v1\n"
  "v_exp_f32 v2, v2\n"
  "v_exp_f32 v3, v3\n"
  "v_pk_add_f32 v[0:1], v[0:1], s[50:51]\n"
  "v_pk_add_f32 v[2:3], v[2:3], s[50:51]\n"
  "v_rcp_f32 v0, v0\n"
  "v_rcp_f32 v1, v1\n"
  "v_rcp_f32 v2, v2\n"
  "v_rcp_f32 v3, v3\n"
  "v_pk_mul_f32 v[96:97], v[96:97], v[0:1]\n"
  "v_pk_mul_f32 v[98:99], v[98:99], v[2:3]\n"
  "s_mov_b32 s63, 0xc20000\n"
  "buffer_store_dwordx4 v[96:99], v242, s[28:31], s63 offen offset:0 nt\n"
  "v_accvgpr_read_b32 v100, a209\n"
  "v_accvgpr_read_b32 v101, a213\n"
  "v_accvgpr_read_b32 v102, a217\n"
  "v_accvgpr_read_b32 v103, a221\n"
  "v_med3_f32 v4, v100, s64, v245\n"
  "v_med3_f32 v5, v101, s64, v245\n"
  "v_med3_f32 v6, v102, s64, v245\n"
  "v_med3_f32 v7, v103, s64, v245\n"
  "v_pk_mul_f32 v[4:5], v[4:5], s[56:57]\n"
  "v_pk_mul_f32 v[6:7], v[6:7], s[56:57]\n"
  "v_exp_f32 v4, v4\n"
  "v_exp_f32 v5, v5\n"
  "v_exp_f32 v6, v6\n"
  "v_exp_f32 v7, v7\n"
  "v_pk_add_f32 v[4:5], v[4:5], s[50:51]\n"
  "v_pk_add_f32 v[6:7], v[6:7], s[50:51]\n"
  "v_rcp_f32 v4, v4\n"
  "v_rcp_f32 v5, v5\n"
  "v_rcp_f32 v6, v6\n"
  "v_rcp_f32 v7, v7\n"
  "v_pk_mul_f32 v[100:101], v[100:101], v[4:5]\n"
  "v_pk_mul_f32 v[102:103], v[102:103], v[6:7]\n"
  "s_mov_b32 s63, 0xc20000\n"
  "buffer_store_dwordx4 v[100:103], v242, s[28:31], s63 offen offset:256 nt\n"
  "s_memrealtime s[52:53]\n"
  "v_accvgpr_read_b32 v104, a194\n"
  "v_accvgpr_read_b32 v105, a198\n"
  "v_accvgpr_read_b32 v106, a202\n"
  "v_accvgpr_read_b32 v107, a206\n"
  "v_med3_f32 v0, v104, s64, v245\n"
  "v_med3_f32 v1, v105, s64, v245\n"
  "v_med3_f32 v2, v106, s64, v245\n"
  "v_med3_f32 v3, v107, s64, v245\n"
  "v_pk_mul_f32 v[0:1], v[0:1], s[56:57]\n"
  "v_pk_mul_f32 v[2:3], v[2:3], s[56:57]\n"
  "v_exp_f32 v0, v0\n"
  "v_exp_f32 v1, v1\n"
  "v_exp_f32 v2, v2\n"
  "v_exp_f32 v3, v3\n"
  "v_pk_add_f32 v[0:1], v[0:1], s[50:51]\n"
  "v_pk_add_f32 v[2:3], v[2:3], s[50:51]\n"
  "v_rcp_f32 v0, v0\n"
  "v_rcp_f32 v1, v1\n"
  "v_rcp_f32 v2, v2\n"
  "v_rcp_f32 v3, v3\n"
  "v_pk_mul_f32 v[104:105], v[104:105], v[0:1]\n"
  "v_pk_mul_f32 v[106:107], v[106:107], v[2:3]\n"
  "s_add_u32 s88, s86, 650\n"
  "s_branch EQ52_%=\n"
  "EW52_%=:\n"
  "s_sleep 1\n"
  "s_memrealtime s[52:53]\n"
  "EQ52_%=:\n"
  "s_waitcnt lgkmcnt(0)\n"
  "s_sub_u32 s82, s52, s88\n"
  "s_cmp_lt_i32 s82, 0\n"
  "s_cbranch_scc1 EW52_%=\n"
  "s_mov_b32 s63, 0xc40000\n"
  "buffer_store_dwordx4 v[104:107], v242, s[28:31], s63 offen offset:0 nt\n"
  "v_accvgpr_read_b32 v108, a210\n"
  "v_accvgpr_read_b32 v109, a214\n"
  "v_accvgpr_read_b32 v110, a218\n"
  "v_accvgpr_read_b32 v111, a222\n"
  "v_med3_f32 v4, v108, s64, v245\n"
  "v_med3_f32 v5, v109, s64, v245\n"
  "v_med3_f32 v6, v110, s64, v245\n"
  "v_med3_f32 v7, v111, s64, v245\n"
  "v_pk_mul_f32 v[4:5], v[4:5], s[56:57]\n"
  "v_pk_mul_f32 v[6:7], v[6:7], s[56:57]\n"
  "v_exp_f32 v4, v4\n"
  "v_exp_f32 v5, v5\n"
  "v_exp_f32 v6, v6\n"
  "v_exp_f32 v7, v7\n"
  "v_pk_add_f32 v[4:5], v[4:5], s[50:51]\n"
  "v_pk_add_f32 v[6:7], v[6:7], s[50:51]\n"
  "v_rcp_f32 v4, v4\n"
  "v_rcp_f32 v5, v5\n"
  "v_rcp_f32 v6, v6\n"
  "v_rcp_f32 v7, v7\n"
  "v_pk_mul_f32 v[108:109], v[108:109], v[4:5]\n"
  "v_pk_mul_f32 v[110:111], v[110:111], v[6:7]\n"
  "s_mov_b32 s63, 0xc40000\n"
  "buffer_store_dwordx4 v[108:111], v242, s[28:31], s63 offen offset:256 nt\n"
  "v_accvgpr_read_b32 v112, a195\n"
  "v_accvgpr_read_b32 v113, a199\n"
  "v_accvgpr_read_b32 v114, a203\n"
  "v_accvgpr_read_b32 v115, a207\n"
  "v_med3_f32 v0, v112, s64, v245\n"
  "v_med3_f32 v1, v113, s64, v245\n"
  "v_med3_f32 v2, v114, s64, v245\n"
  "v_med3_f32 v3, v115, s64, v245\n"
  "v_pk_mul_f32 v[0:1], v[0:1], s[56:57]\n"
  "v_pk_mul_f32 v[2:3], v[2:3], s[56:57]\n"
  "v_exp_f32 v0, v0\n"
  "v_exp_f32 v1, v1\n"
  "v_exp_f32 v2, v2\n"
  "v_exp_f32 v3, v3\n"
  "v_pk_add_f32 v[0:1], v[0:1], s[50:51]\n"
  "v_pk_add_f32 v[2:3], v[2:3], s[50:51]\n"
  "v_rcp_f32 v0, v0\n"
  "v_rcp_f32 v1, v1\n"
  "v_rcp_f32 v2, v2\n"
  "v_rcp_f32 v3, v3\n"
  "v_pk_mul_f32 v[112:113], v[112:113], v[0:1]\n"
  "v_pk_mul_f32 v[114:115], v[114:115], v[2:3]\n"
  "s_mov_b32 s63, 0xc60000\n"
  "buffer_store_dwordx4 v[112:115], v242, s[28:31], s63 offen offset:0 nt\n"
  "v_accvgpr_read_b32 v116, a211\n"
  "v_accvgpr_read_b32 v117, a215\n"
  "v_accvgpr_read_b32 v118, a219\n"
  "v_accvgpr_read_b32 v119, a223\n"
  "v_med3_f32 v4, v116, s64, v245\n"
  "v_med3_f32 v5, v117, s64, v245\n"
  "v_med3_f32 v6, v118, s64, v245\n"
  "v_med3_f32 v7, v119, s64, v245\n"
  "v_pk_mul_f32 v[4:5], v[4:5], s[56:57]\n"
  "v_pk_mul_f32 v[6:7], v[6:7], s[56:57]\n"
  "v_exp_f32 v4, v4\n"
  "v_exp_f32 v5, v5\n"
  "v_exp_f32 v6, v6\n"
  "v_exp_f32 v7, v7\n"
  "v_pk_add_f32 v[4:5], v[4:5], s[50:51]\n"
  "v_pk_add_f32 v[6:7], v[6:7], s[50:51]\n"
  "v_rcp_f32 v4, v4\n"
  "v_rcp_f32 v5, v5\n"
  "v_rcp_f32 v6, v6\n"
  "v_rcp_f32 v7, v7\n"
  "v_pk_mul_f32 v[116:117], v[116:117], v[4:5]\n"
  "v_pk_mul_f32 v[118:119], v[118:119], v[6:7]\n"
  "s_mov_b32 s63, 0xc60000\n"
  "buffer_store_dwordx4 v[116:119], v242, s[28:31], s63 offen offset:256 nt\n"
  "s_memrealtime s[52:53]\n"
  "v_accvgpr_read_b32 v8, a224\n"
  "v_accvgpr_read_b32 v9, a228\n"
  "v_accvgpr_read_b32 v10, a232\n"
  "v_accvgpr_read_b32 v11, a236\n"
  "v_med3_f32 v0, v8, s64, v245\n"
  "v_med3_f32 v1, v9, s64, v245\n"
  "v_med3_f32 v2, v10, s64, v245\n"
  "v_med3_f32 v3, v11, s64, v245\n"
  "v_pk_mul_f32 v[0:1], v[0:1], s[56:57]\n"
  "v_pk_mul_f32 v[2:3], v[2:3], s[56:57]\n"
  "v_exp_f32 v0, v0\n"
  "v_exp_f32 v1, v1\n"
  "v_exp_f32 v2, v2\n"
  "v_exp_f32 v3, v3\n"
  "v_pk_add_f32 v[0:1], v[0:1], s[50:51]\n"
  "v_pk_add_f32 v[2:3], v[2:3], s[50:51]\n"
  "v_rcp_f32 v0, v0\n"
  "v_rcp_f32 v1, v1\n"
  "v_rcp_f32 v2, v2\n"
  "v_rcp_f32 v3, v3\n"
  "v_pk_mul_f32 v[8:9], v[8:9], v[0:1]\n"
  "v_pk_mul_f32 v[10:11], v[10:11], v[2:3]\n"
  "s_add_u32 s88, s86, 715\n"
  "s_branch EQ56_%=\n"
  "EW56_%=:\n"
  "s_sleep 1\n"
  "s_memrealtime s[52:53]\n"
  "EQ56_%=:\n"
  "s_waitcnt lgkmcnt(0)\n"
  "s_sub_u32 s82, s52, s88\n"
  "s_cmp_lt_i32 s82, 0\n"
  "s_cbranch_scc1 EW56_%=\n"
  "s_mov_b32 s63, 0xe00000\n"
  "buffer_store_dwordx4 v[8:11], v242, s[28:31], s63 offen offset:0 nt\n"
  "v_accvgpr_read_b32 v12, a240\n"
  "v_accvgpr_read_b32 v13, a244\n"
  "v_accvgpr_read_b32 v14, a248\n"
  "v_accvgpr_read_b32 v15, a252\n"
  "v_med3_f32 v4, v12, s64, v245\n"
  "v_med3_f32 v5, v13, s64, v245\n"
  "v_med3_f32 v6, v14, s64, v245\n"
  "v_med3_f32 v7, v15, s64, v245\n"
  "v_pk_mul_f32 v[4:5], v[4:5], s[56:57]\n"
  "v_pk_mul_f32 v[6:7], v[6:7], s[56:57]\n"
  "v_exp_f32 v4, v4\n"
  "v_exp_f32 v5, v5\n"
  "v_exp_f32 v6, v6\n"
  "v_exp_f32 v7, v7\n"
  "v_pk_add_f32 v[4:5], v[4:5], s[50:51]\n"
  "v_pk_add_f32 v[6:7], v[6:7], s[50:51]\n"
  "v_rcp_f32 v4, v4\n"
  "v_rcp_f32 v5, v5\n"
  "v_rcp_f32 v6, v6\n"
  "v_rcp_f32 v7, v7\n"
  "v_pk_mul_f32 v[12:13], v[12:13], v[4:5]\n"
  "v_pk_mul_f32 v[14:15], v[14:15], v[6:7]\n"
  "s_mov_b32 s63, 0xe00000\n"
  "buffer_store_dwordx4 v[12:15], v242, s[28:31], s63 offen offset:256 nt\n"
  "v_accvgpr_read_b32 v16, a225\n"
  "v_accvgpr_read_b32 v17, a229\n"
  "v_accvgpr_read_b32 v18, a233\n"
  "v_accvgpr_read_b32 v19, a237\n"
  "v_med3_f32 v0, v16, s64, v245\n"
  "v_med3_f32 v1, v17, s64, v245\n"
  "v_med3_f32 v2, v18, s64, v245\n"
  "v_med3_f32 v3, v19, s64, v245\n"
  "v_pk_mul_f32 v[0:1], v[0:1], s[56:57]\n"
  "v_pk_mul_f32 v[2:3], v[2:3], s[56:57]\n"
  "v_exp_f32 v0, v0\n"
  "v_exp_f32 v1, v1\n"
  "v_exp_f32 v2, v2\n"
  "v_exp_f32 v3, v3\n"
  "v_pk_add_f32 v[0:1], v[0:1], s[50:51]\n"
  "v_pk_add_f32 v[2:3], v[2:3], s[50:51]\n"
  "v_rcp_f32 v0, v0\n"
  "v_rcp_f32 v1, v1\n"
  "v_rcp_f32 v2, v2\n"
  "v_rcp_f32 v3, v3\n"
  "v_pk_mul_f32 v[16:17], v[16:17], v[0:1]\n"
  "v_pk_mul_f32 v[18:19], v[18:19], v[2:3]\n"
  "s_mov_b32 s63, 0xe20000\n"
  "buffer_store_dwordx4 v[16:19], v242, s[28:31], s63 offen offset:0 nt\n"
  "v_accvgpr_read_b32 v20, a241\n"
  "v_accvgpr_read_b32 v21, a245\n"
  "v_accvgpr_read_b32 v22, a249\n"
  "v_accvgpr_read_b32 v23, a253\n"
  "v_med3_f32 v4, v20, s64, v245\n"
  "v_med3_f32 v5, v21, s64, v245\n"
  "v_med3_f32 v6, v22, s64, v245\n"
  "v_med3_f32 v7, v23, s64, v245\n"
  "v_pk_mul_f32 v[4:5], v[4:5], s[56:57]\n"
  "v_pk_mul_f32 v[6:7], v[6:7], s[56:57]\n"
  "v_exp_f32 v4, v4\n"
  "v_exp_f32 v5, v5\n"
  "v_exp_f32 v6, v6\n"
  "v_exp_f32 v7, v7\n"
  "v_pk_add_f32 v[4:5], v[4:5], s[50:51]\n"
  "v_pk_add_f32 v[6:7], v[6:7], s[50:51]\n"
  "v_rcp_f32 v4, v4\n"
  "v_rcp_f32 v5, v5\n"
  "v_rcp_f32 v6, v6\n"
  "v_rcp_f32 v7, v7\n"
  "v_pk_mul_f32 v[20:21], v[20:21], v[4:5]\n"
  "v_pk_mul_f32 v[22:23], v[22:23], v[6:7]\n"
  "s_mov_b32 s63, 0xe20000\n"
  "buffer_store_dwordx4 v[20:23], v242, s[28:31], s63 offen offset:256 nt\n"
  "s_memrealtime s[52:53]\n"
  "v_accvgpr_read_b32 v24, a226\n"
  "v_accvgpr_read_b32 v25, a230\n"
  "v_accvgpr_read_b32 v26, a234\n"
  "v_accvgpr_read_b32 v27, a238\n"
  "v_med3_f32 v0, v24, s64, v245\n"
  "v_med3_f32 v1, v25, s64, v245\n"
  "v_med3_f32 v2, v26, s64, v245\n"
  "v_med3_f32 v3, v27, s64, v245\n"
  "v_pk_mul_f32 v[0:1], v[0:1], s[56:57]\n"
  "v_pk_mul_f32 v[2:3], v[2:3], s[56:57]\n"
  "v_exp_f32 v0, v0\n"
  "v_exp_f32 v1, v1\n"
  "v_exp_f32 v2, v2\n"
  "v_exp_f32 v3, v3\n"
  "v_pk_add_f32 v[0:1], v[0:1], s[50:51]\n"
  "v_pk_add_f32 v[2:3], v[2:3], s[50:51]\n"
  "v_rcp_f32 v0, v0\n"
  "v_rcp_f32 v1, v1\n"
  "v_rcp_f32 v2, v2\n"
  "v_rcp_f32 v3, v3\n"
  "v_pk_mul_f32 v[24:25], v[24:25], v[0:1]\n"
  "v_pk_mul_f32 v[26:27], v[26:27], v[2:3]\n"
  "s_add_u32 s88, s86, 780\n"
  "s_branch EQ60_%=\n"
  "EW60_%=:\n"
  "s_sleep 1\n"
  "s_memrealtime s[52:53]\n"
  "EQ60_%=:\n"
  "s_waitcnt lgkmcnt(0)\n"
  "s_sub_u32 s82, s52, s88\n"
  "s_cmp_lt_i32 s82, 0\n"
  "s_cbranch_scc1 EW60_%=\n"
  "s_mov_b32 s63, 0xe40000\n"
  "buffer_store_dwordx4 v[24:27], v242, s[28:31], s63 offen offset:0 nt\n"
  "v_accvgpr_read_b32 v28, a242\n"
  "v_accvgpr_read_b32 v29, a246\n"
  "v_accvgpr_read_b32 v30, a250\n"
  "v_accvgpr_read_b32 v31, a254\n"
  "v_med3_f32 v4, v28, s64, v245\n"
  "v_med3_f32 v5, v29, s64, v245\n"
  "v_med3_f32 v6, v30, s64, v245\n"
  "v_med3_f32 v7, v31, s64, v245\n"
  "v_pk_mul_f32 v[4:5], v[4:5], s[56:57]\n"
  "v_pk_mul_f32 v[6:7], v[6:7], s[56:57]\n"
  "v_exp_f32 v4, v4\n"
  "v_exp_f32 v5, v5\n"
  "v_exp_f32 v6, v6\n"
  "v_exp_f32 v7, v7\n"
  "v_pk_add_f32 v[4:5], v[4:5], s[50:51]\n"
  "v_pk_add_f32 v[6:7], v[6:7], s[50:51]\n"
  "v_rcp_f32 v4, v4\n"
  "v_rcp_f32 v5, v5\n"
  "v_rcp_f32 v6, v6\n"
  "v_rcp_f32 v7, v7\n"
  "v_pk_mul_f32 v[28:29], v[28:29], v[4:5]\n"
  "v_pk_mul_f32 v[30:31], v[30:31], v[6:7]\n"
  "s_mov_b32 s63, 0xe40000\n"
  "buffer_store_dwordx4 v[28:31], v242, s[28:31], s63 offen offset:256 nt\n"
  "v_accvgpr_read_b32 v32, a227\n"
  "v_accvgpr_read_b32 v33, a231\n"
  "v_accvgpr_read_b32 v34, a235\n"
  "v_accvgpr_read_b32 v35, a239\n"
  "v_med3_f32 v0, v32, s64, v245\n"
  "v_med3_f32 v1, v33, s64, v245\n"
  "v_med3_f32 v2, v34, s64, v245\n"
  "v_med3_f32 v3, v35, s64, v245\n"
  "v_pk_mul_f32 v[0:1], v[0:1], s[56:57]\n"
  "v_pk_mul_f32 v[2:3], v[2:3], s[56:57]\n"
  "v_exp_f32 v0, v0\n"
  "v_exp_f32 v1, v1\n"
  "v_exp_f32 v2, v2\n"
  "v_exp_f32 v3, v3\n"
  "v_pk_add_f32 v[0:1], v[0:1], s[50:51]\n"
  "v_pk_add_f32 v[2:3], v[2:3], s[50:51]\n"
  "v_rcp_f32 v0, v0\n"
  "v_rcp_f32 v1, v1\n"
  "v_rcp_f32 v2, v2\n"
  "v_rcp_f32 v3, v3\n"
  "v_pk_mul_f32 v[32:33], v[32:33], v[0:1]\n"
  "v_pk_mul_f32 v[34:35], v[34:35], v[2:3]\n"
  "s_mov_b32 s63, 0xe60000\n"
  "buffer_store_dwordx4 v[32:35], v242, s[28:31], s63 offen offset:0 nt\n"
  "v_accvgpr_read_b32 v36, a243\n"
  "v_accvgpr_read_b32 v37, a247\n"
  "v_accvgpr_read_b32 v38, a251\n"
  "v_accvgpr_read_b32 v39, a255\n"
  "v_med3_f32 v4, v36, s64, v245\n"
  "v_med3_f32 v5, v37, s64, v245\n"
  "v_med3_f32 v6, v38, s64, v245\n"
  "v_med3_f32 v7, v39, s64, v245\n"
  "v_pk_mul_f32 v[4:5], v[4:5], s[56:57]\n"
  "v_pk_mul_f32 v[6:7], v[6:7], s[56:57]\n"
  "v_exp_f32 v4, v4\n"
  "v_exp_f32 v5, v5\n"
  "v_exp_f32 v6, v6\n"
  "v_exp_f32 v7, v7\n"
  "v_pk_add_f32 v[4:5], v[4:5], s[50:51]\n"
  "v_pk_add_f32 v[6:7], v[6:7], s[50:51]\n"
  "v_rcp_f32 v4, v4\n"
  "v_rcp_f32 v5, v5\n"
  "v_rcp_f32 v6, v6\n"
  "v_rcp_f32 v7, v7\n"
  "v_pk_mul_f32 v[36:37], v[36:37], v[4:5]\n"
  "v_pk_mul_f32 v[38:39], v[38:39], v[6:7]\n"
  "s_mov_b32 s63, 0xe60000\n"
  "buffer_store_dwordx4 v[36:39], v242, s[28:31], s63 offen offset:256 nt\n"
  "s_endpgm\n"
  "KE_%=:\n"
  :: "s"(__builtin_amdgcn_kernarg_segment_ptr()), "s"(ws), "s"(w), "s"(out), "s"(x), "s"(rank), "s"(b), "s"(wave),
     "s"(dbg + (b * 4u + wave) * 136u), "s"((unsigned)(size_t)lds)
  : "v0","v1","v2","v3","v4","v5","v6","v7","v8","v9","v10","v11","v12","v13","v14","v15","v16","v17","v18","v19","v20","v21","v22","v23","v24","v25","v26","v27","v28","v29","v30","v31","v32","v33","v34","v35","v36","v37","v38","v39","v40","v41","v42","v43","v44","v45","v46","v47","v48","v49","v50","v51","v52","v53","v54","v55","v56","v57","v58","v59","v60","v61","v62","v63","v64","v65","v66","v67","v68","v69","v70","v71","v72","v73","v74","v75","v76","v77","v78","v79","v80","v81","v82","v83","v84","v85","v86","v87","v88","v89","v90","v91","v92","v93","v94","v95","v96","v97","v98","v99","v100","v101","v102","v103","v104","v105","v106","v107","v108","v109","v110","v111","v112","v113","v114","v115","v116","v117","v118","v119","v120","v121","v122","v123","v124","v125","v126","v127","v128","v129","v130","v131","v132","v133","v134","v135","v136","v137","v138","v139","v140","v141","v142","v143","v144","v145","v146","v147","v148","v149","v150","v151","v152","v153","v154","v155","v156","v157","v158","v159","v160","v161","v162","v163","v164","v165","v166","v167","v168","v169","v170","v171","v172","v173","v174","v175","v176","v177","v178","v179","v180","v181","v182","v183","v184","v185","v186","v187","v188","v189","v190","v191","v192","v193","v194","v195","v196","v197","v198","v199","v200","v201","v202","v203","v204","v205","v206","v207","v208","v209","v210","v211","v212","v213","v214","v215","v216","v217","v218","v219","v220","v221","v222","v223","v224","v225","v226","v227","v228","v229","v230","v231","v232","v233","v234","v235","v236","v237","v238","v239","v240","v241","v242","v243","v244","v245","v246","v247","v248","v249","v250","v251","v252","v253","a0","a1","a2","a3","a4","a5","a6","a7","a8","a9","a10","a11","a12","a13","a14","a15","a16","a17","a18","a19","a20","a21","a22","a23","a24","a25","a26","a27","a28","a29","a30","a31","a32","a33","a34","a35","a36","a37","a38","a39","a40","a41","a42","a43","a44","a45","a46","a47","a48","a49","a50","a51","a52","a53","a54","a55","a56","a57","a58","a59","a60","a61","a62","a63","a64","a65","a66","a67","a68","a69","a70","a71","a72","a73","a74","a75","a76","a77","a78","a79","a80","a81","a82","a83","a84","a85","a86","a87","a88","a89","a90","a91","a92","a93","a94","a95","a96","a97","a98","a99","a100","a101","a102","a103","a104","a105","a106","a107","a108","a109","a110","a111","a112","a113","a114","a115","a116","a117","a118","a119","a120","a121","a122","a123","a124","a125","a126","a127","a128","a129","a130","a131","a132","a133","a134","a135","a136","a137","a138","a139","a140","a141","a142","a143","a144","a145","a146","a147","a148","a149","a150","a151","a152","a153","a154","a155","a156","a157","a158","a159","a160","a161","a162","a163","a164","a165","a166","a167","a168","a169","a170","a171","a172","a173","a174","a175","a176","a177","a178","a179","a180","a181","a182","a183","a184","a185","a186","a187","a188","a189","a190","a191","a192","a193","a194","a195","a196","a197","a198","a199","a200","a201","a202","a203","a204","a205","a206","a207","a208","a209","a210","a211","a212","a213","a214","a215","a216","a217","a218","a219","a220","a221","a222","a223","a224","a225","a226","a227","a228","a229","a230","a231","a232","a233","a234","a235","a236","a237","a238","a239","a240","a241","a242","a243","a244","a245","a246","a247","a248","a249","a250","a251","a252","a253","a254","a255","s20","s21","s22","s23","s24","s25","s26","s27","s28","s29","s30","s31","s33","s34","s35","s36","s37","s38","s39","s40","s41","s42","s43","s44","s45","s46","s47","s48","s49","s50","s51","s52","s53","s54","s55","s56","s57","s58","s59","s60","s61","s62","s63","s64","s65","s66","s67","s68","s69","s70","s71","s72","s73","s74","s75","s76","s77","s78","s79","s80","s81","s82","s83","s84","s85","s86","s87","s88","s89","s90","s91","s92","s93","s94","s95","s96","s97","s98","s99","vcc","scc","memory");
}

extern "C" void kernel_launch(int rank, int nranks, ncclComm_t comm, hipStream_t stream,
                              void* const* d_in, const int* in_sizes, int n_in, void* d_out, int out_size,
                              const int* in_dtypes, int out_dtype, void* const* d_out_all,
                              void* const* d_ws_all, size_t ws_bytes) {
  if (nranks != NR || (unsigned)rank >= (unsigned)NR || ws_bytes < (size_t)9437184 || n_in != 2 ||
      in_sizes[0] != 4096 * 512 || in_sizes[1] != 4096 * 32768 || out_size != 512 * 32768) {
    fprintf(stderr, "unexpected problem geometry\n");
    return;
  }
  Ptrs P;
  for (int p = 0; p < NR; ++p) P.ws[p] = (char*)d_ws_all[p];
  a2a_gemm_silu<<<256, 256, 0, stream>>>(P, P.ws[rank], (const float*)d_in[0], (const float*)d_in[1], (float*)d_out, nullptr, rank);
}
```
